```python
import numpy as np

kernel_name = "gpu_gemm_rs_m4096_k4096_n16384_f32_gelu"


ABSMAX_RTOL = 0.001

COMPUTE_DTYPE = "float32"
EPILOGUE = "gelu"
IN_QUANT = None
OUT_QUANT = None
IN_DTYPES = {"x": "float32", "w_mat": "float32"}
WORLD_SIZE = 8

def setup_inputs():

    x = np.random.randn(4096, 4096).astype(np.float32)
    w_mat = (np.random.randn(4096, 16384) / np.sqrt(4096)).astype(np.float32)
    return {"x": x, "w_mat": w_mat}

def _epilogue(y):
    c = 0.7978845608028654
    return (0.5 * y * (1.0 + np.tanh(c * (y + 0.044715 * y ** 3)))).astype(np.float32)

def reference(x, w_mat):
    y = (x @ w_mat).astype(np.float32)
    return _epilogue(y)
```

```cpp
#include <hip/hip_runtime.h>
#include <rccl/rccl.h>
#include <cstdio>

#define HIP_CHECK(expr) do { hipError_t _e = (expr); if (_e != hipSuccess) { \
    fprintf(stderr, "HIP error %s at %s:%d\n", hipGetErrorString(_e), __FILE__, __LINE__); \
    return; } } while (0)

#define LAS __attribute__((address_space(3)))
#define GAS __attribute__((address_space(1)))
typedef _Float16 f16x8 __attribute__((ext_vector_type(8)));
typedef float f32x4 __attribute__((ext_vector_type(4)));
typedef unsigned u32x4 __attribute__((ext_vector_type(4)));

#define LEAD 3
#define NPW 16
#define ODEPTH 1
#define STAMP 0

namespace {

constexpr int NR = 8;
constexpr int KS = 512;
constexpr int NFULL = 16384;
constexpr int NS = 2048;
constexpr int NBLK = 24;
constexpr unsigned UNIT = 32768;
constexpr size_t CTRL_BYTES = 1u << 20;
constexpr size_t STAGE_BYTES = (size_t)64 * NBLK * UNIT;
constexpr size_t WS_NEED = CTRL_BYTES + 2 * STAGE_BYTES;
constexpr unsigned OFF_TMO = 0, OFF_DONE = 128, OFF_CNT = 4096, OFF_FLAG = 65536, OFF_STAMP = 524288;
constexpr unsigned long long DEADLINE_TICKS = 50000000ull;
constexpr int NPUSH = 8 * NPW, NGEMM = 128, NITEM = 192 / NPW;
constexpr unsigned LDS_STAGE = 131072, LDS_BYTES = LDS_STAGE + 64;

struct Params {
    const float* x;
    const float* w;
    float* out[NR];
    char* ws[NR];
    int rank;
    int pad;
};

__device__ __forceinline__ void store_wt(void* p, u32x4 v) {
    asm volatile("global_store_dwordx4 %0, %1, off sc0 sc1\n\ts_nop 1" :: "v"(p), "v"(v) : "memory");
}
__device__ __forceinline__ void store4_wt(void* p0, void* p1, void* p2, void* p3, u32x4 a, u32x4 b, u32x4 c, u32x4 d) {
    asm volatile("global_store_dwordx4 %0, %4, off sc0 sc1\n\ts_waitcnt vmcnt(0)\n\t"
                 "global_store_dwordx4 %1, %5, off sc0 sc1\n\ts_waitcnt vmcnt(0)\n\t"
                 "global_store_dwordx4 %2, %6, off sc0 sc1\n\ts_waitcnt vmcnt(0)\n\t"
                 "global_store_dwordx4 %3, %7, off sc0 sc1\n\ts_waitcnt vmcnt(0)"
                 :: "v"(p0), "v"(p1), "v"(p2), "v"(p3), "v"(a), "v"(b), "v"(c), "v"(d) : "memory");
}
__device__ __forceinline__ unsigned pack2(float a, float b) {
    typedef _Float16 h2 __attribute__((ext_vector_type(2)));
    h2 h; h.x = (_Float16)a; h.y = (_Float16)b;
    return __builtin_bit_cast(unsigned, h);
}
__device__ __forceinline__ float gelu(float y) {
    const float k1 = -2.0f * 0.7978845608028654f * 1.4426950408889634f, k2 = k1 * 0.044715f;
    const float e = __builtin_amdgcn_exp2f(y * __builtin_fmaf(y * y, k2, k1));
    return y * __builtin_amdgcn_rcpf(1.0f + e);
}
__device__ __forceinline__ unsigned ld_sys(const char* p) {
    return __hip_atomic_load((const GAS unsigned*)p, __ATOMIC_RELAXED, __HIP_MEMORY_SCOPE_SYSTEM);
}
__device__ __forceinline__ unsigned ld_agent(const char* p) {
    return __hip_atomic_load((const GAS unsigned*)p, __ATOMIC_RELAXED, __HIP_MEMORY_SCOPE_AGENT);
}
__device__ __forceinline__ void wave_wait(const char* word, unsigned need, char* tmo, unsigned call) {
    const unsigned long long t0 = __builtin_amdgcn_s_memrealtime();
    while (!__all(ld_sys(word) >= need)) {
        if ((unsigned)__builtin_amdgcn_readfirstlane(ld_agent(tmo)) == call
            || __builtin_amdgcn_s_memrealtime() - t0 > DEADLINE_TICKS) {
            __hip_atomic_store((GAS unsigned*)tmo, call, __ATOMIC_RELAXED, __HIP_MEMORY_SCOPE_AGENT);
            break;
        }
        __builtin_amdgcn_s_sleep(2);
    }
}

#define TS(k) do { } while (0)

}

__global__ __launch_bounds__(512, 2) void gemm_rs_kernel(Params P) {
    __shared__ __attribute__((aligned(16))) unsigned char shm[LDS_BYTES];
    LAS unsigned char* lds = (LAS unsigned char*)shm;
    const int tid = threadIdx.x, lane = tid & 63, wid = __builtin_amdgcn_readfirstlane(tid >> 6);
    const int fr = lane & 15, fq = lane >> 4;
    const int bid = __builtin_amdgcn_readfirstlane(blockIdx.x);
    const int rank = P.rank & (NR - 1);
    char* const me = P.ws[rank];
    TS(0);
    if (tid == 0) {
        const unsigned c = __hip_atomic_fetch_add((GAS unsigned*)(me + OFF_CNT + bid * 128), 1u,
                                                  __ATOMIC_RELAXED, __HIP_MEMORY_SCOPE_AGENT) + 1u;
        *(LAS unsigned*)(lds + LDS_STAGE) = c;
        *(LAS unsigned*)(lds + LDS_STAGE + 4) = 0u;
    }

    if (bid < NPUSH) {
        const int pw = bid, w = pw >> 3, dest = (rank + pw) & (NR - 1);
        char* const dws = P.ws[dest];
        f32x4 v[8];
        auto item = [&](int t, int& kind, int& s, int& blk8) {
            const int g = t >> 3;
            blk8 = t & 7;
            if (g < 2 * LEAD) { kind = g & 1; s = g >> 1; }
            else if (g < 2 * LEAD + 3 * (8 - LEAD)) { const int e = g - 2 * LEAD, r = e / 3; kind = e - 3 * r; s = kind == 2 ? r : r + LEAD; }
            else { kind = 2; s = g - 16; }
        };
        auto load = [&](int kind, int s, int blk8) {
            if (kind == 0) {
                const float* src = P.w + (size_t)(s * 64 + (wid >> 2) * 32 + fq * 4) * NFULL
                                 + dest * NS + blk8 * 256 + (wid & 3) * 64 + fr * 4;
#pragma unroll
                for (int j = 0; j < 8; ++j) v[j] = *(const f32x4*)(src + (size_t)((j >> 2) * 16 + (j & 3)) * NFULL);
            } else {
                const float* src = P.x + (size_t)((blk8 * 2 + kind - 1) * 256 + wid * 32 + fr) * KS + s * 64 + fq * 4;
#pragma unroll
                for (int j = 0; j < 8; ++j) v[j] = *(const f32x4*)(src + (j >> 2) * 16 * KS + (j & 3) * 16);
            }
        };
        int kind, s, blk8;
        item(w, kind, s, blk8);
        load(kind, s, blk8);
        __syncthreads();
        const unsigned call = __builtin_amdgcn_readfirstlane(*(LAS unsigned*)(lds + LDS_STAGE));
        char* const dstage = dws + CTRL_BYTES + (size_t)(call & 1u) * STAGE_BYTES;
#pragma unroll 1
        for (int n = 0; n < NITEM; ++n) {
            const int blk = kind == 0 ? blk8 : 8 + blk8 * 2 + kind - 1;
            const unsigned u = (unsigned)((rank * 8 + s) * NBLK + blk);
            char* dst = dstage + (size_t)u * UNIT + lane * 16;
            u32x4 o[4];
            int d1, d2;
            if (kind == 0) {
                dst += ((wid & 3) * 4 + (wid >> 2)) * 1024; d1 = 2048; d2 = 16384;
#pragma unroll
                for (int f = 0; f < 4; ++f) {
                    o[f].x = pack2(v[0][f], v[1][f]); o[f].y = pack2(v[2][f], v[3][f]);
                    o[f].z = pack2(v[4][f], v[5][f]); o[f].w = pack2(v[6][f], v[7][f]);
                }
            } else {
                dst += wid * 4096; d1 = 1024; d2 = 2048;
#pragma unroll
                for (int f = 0; f < 4; ++f) {
                    o[f].x = pack2(v[2 * f][0], v[2 * f][1]); o[f].y = pack2(v[2 * f][2], v[2 * f][3]);
                    o[f].z = pack2(v[2 * f + 1][0], v[2 * f + 1][1]); o[f].w = pack2(v[2 * f + 1][2], v[2 * f + 1][3]);
                }
            }
            asm volatile("" : "+v"(o[0]), "+v"(o[1]), "+v"(o[2]), "+v"(o[3]));
            if (n + 1 < NITEM) { item(w + (n + 1) * NPW, kind, s, blk8); load(kind, s, blk8); }
            store4_wt(dst, dst + d1, dst + d2, dst + d2 + d1, o[0], o[1], o[2], o[3]);
            __builtin_amdgcn_s_waitcnt(0x0F70);
            __syncthreads();
            if (tid == 0)
                __hip_atomic_store((GAS unsigned*)(dws + OFF_FLAG + u * 128u), call, __ATOMIC_RELAXED, __HIP_MEMORY_SCOPE_SYSTEM);
            TS(1 + n);
        }
        if (pw == 0 && wid == 0) {
            wave_wait(me + OFF_DONE, 112u * call, me + OFF_TMO, call);
            if (ld_agent(me + OFF_TMO) == call && lane == 0) P.out[rank][rank * NS] = __builtin_nanf("");
            TS(30);
        }
        return;
    }

    const int gb = bid - NPUSH, slot = gb >> 3;
    const int owner = (rank + gb) & (NR - 1);
    const int ti = owner * 2 + (slot >> 3), tj = slot & 7;
    const int wr = wid >> 2, wc = wid & 3;
    __syncthreads();
    const unsigned call = __builtin_amdgcn_readfirstlane(*(LAS unsigned*)(lds + LDS_STAGE));
    const char* const stage = me + CTRL_BYTES + (size_t)(call & 1u) * STAGE_BYTES;
    const unsigned voff = (unsigned)tid * 16u;
    const unsigned ldsw = (unsigned)wid * 1024u;
    const unsigned aoff = (unsigned)wr * 8192u + (unsigned)lane * 16u, boff = (unsigned)wc * 4096u + (unsigned)lane * 16u;

#define SA(b, h) (((b) * 2 + (h)) * 16384)
#define SB(b, h) ((4 + (b) * 2 + (h)) * 16384)
#define STAGE(bufoff, gbase) do { _Pragma("unroll") for (int _i = 0; _i < 2; ++_i) \
        __builtin_amdgcn_global_load_lds((const GAS unsigned*)((gbase) + _i * 8192 + voff), (LAS unsigned*)(lds + (bufoff) + ldsw + _i * 8192), 16, 0, 17); } while (0)
#define LDA(dst, b, h) do { _Pragma("unroll") for (int m = 0; m < 4; ++m) _Pragma("unroll") for (int k = 0; k < 2; ++k) \
        dst[m][k] = *(const LAS f16x8*)(lds + SA(b, h) + aoff + m * 2048 + k * 1024); } while (0)
#define LDB(dst, b, h) do { _Pragma("unroll") for (int n = 0; n < 2; ++n) _Pragma("unroll") for (int k = 0; k < 2; ++k) \
        dst[n][k] = *(const LAS f16x8*)(lds + SB(b, h) + boff + n * 2048 + k * 1024); } while (0)
#define MMA(ai, bj, At, Bt) do { __builtin_amdgcn_s_setprio(1); \
        _Pragma("unroll") for (int m = 0; m < 4; ++m) _Pragma("unroll") for (int n = 0; n < 2; ++n) _Pragma("unroll") for (int k = 0; k < 2; ++k) \
        acc[ai][bj][m][n] = __builtin_amdgcn_mfma_f32_16x16x32_f16(At[m][k], Bt[n][k], acc[ai][bj][m][n], 0, 0, 0); \
        __builtin_amdgcn_s_setprio(0); } while (0)
#define WAIT_V(n) asm volatile("s_waitcnt vmcnt(" #n ")" ::: "memory")
#define WAIT_L(n) asm volatile("s_waitcnt lgkmcnt(" #n ")" ::: "memory")
#define BAR __builtin_amdgcn_s_barrier()
#define SCHED __builtin_amdgcn_sched_barrier(0)

    auto unit = [&](int t, int blk) -> const char* {
        const int p = (rank + t) & 7, s = (t >> 3) & 7;
        return stage + (unsigned)__builtin_amdgcn_readfirstlane(((p * 8 + s) * NBLK + blk) * UNIT);
    };
    auto ready = [&](int s) {
        if (wid == 0) {
            int lz = lane; asm volatile("" : "+v"(lz));
            const int p = (lz >> 1) & 7, blk = tj + (lz & 1) * (8 + ti - tj);
            wave_wait(me + OFF_FLAG + (unsigned)((p * 8 + s) * NBLK + blk) * 128u, call, me + OFF_TMO, call);
            if (s == 7) {
                const unsigned bad = (unsigned)__builtin_amdgcn_readfirstlane(ld_agent(me + OFF_TMO)) == call;
                *(LAS unsigned*)(lds + LDS_STAGE + 4) = bad;
                WAIT_L(0);
            }
            TS(1 + s);
        }
        asm volatile("" ::: "memory"); BAR; asm volatile("" ::: "memory");
    };

    f32x4 acc[2][2][4][2];
#pragma unroll
    for (int a = 0; a < 2; ++a)
#pragma unroll
        for (int b = 0; b < 2; ++b)
#pragma unroll
            for (int m = 0; m < 4; ++m)
#pragma unroll
                for (int n = 0; n < 2; ++n) acc[a][b][m][n] = (f32x4){0.f, 0.f, 0.f, 0.f};
    f16x8 At[4][2], B0[2][2], B1[2][2];

    ready(0);
    {
        const char* a0 = unit(0, 8 + ti); const char* b0 = unit(0, tj);
        const char* a1 = unit(1, 8 + ti); const char* b1 = unit(1, tj);
        STAGE(SB(0, 0), b0); STAGE(SA(0, 0), a0); STAGE(SB(0, 1), b0 + 16384); STAGE(SA(0, 1), a0 + 16384);
        if (wr == 1) BAR;
        WAIT_V(4); BAR;
        STAGE(SB(1, 0), b1); STAGE(SA(1, 0), a1); STAGE(SB(1, 1), b1 + 16384);
        WAIT_V(6); BAR;
    }
#pragma unroll 1
    for (int t = 0; t < 64; t += 2) {
        const int t2 = t + 2 < 64 ? t + 2 : 62;
        const char* a1 = unit(t + 1, 8 + ti);
        const char* a2 = unit(t2, 8 + ti); const char* b2 = unit(t2, tj);
        const char* a3 = unit(t2 + 1, 8 + ti); const char* b3 = unit(t2 + 1, tj);
        if (((t + 2) & 7) == 0 && t + 2 < 64) ready((t + 2) >> 3);
        LDB(B0, 0, 0); SCHED; LDA(At, 0, 0); STAGE(SA(1, 1), a1 + 16384);
        WAIT_L(8); BAR; WAIT_L(0); MMA(0, 0, At, B0); BAR; SCHED;
        LDB(B1, 0, 1); STAGE(SB(0, 0), b2);
        BAR; WAIT_L(0); MMA(0, 1, At, B1); BAR;
        LDA(At, 0, 1); STAGE(SA(0, 0), a2);
        BAR; WAIT_L(0); MMA(1, 0, At, B0); BAR; SCHED;
        STAGE(SB(0, 1), b2 + 16384);
        WAIT_V(6); BAR; MMA(1, 1, At, B1); BAR;
        LDB(B0, 1, 0); SCHED; LDA(At, 1, 0); STAGE(SA(0, 1), a2 + 16384);
        WAIT_L(8); BAR; WAIT_L(0); MMA(0, 0, At, B0); BAR; SCHED;
        LDB(B1, 1, 1); STAGE(SB(1, 0), b3);
        BAR; WAIT_L(0); MMA(0, 1, At, B1); BAR;
        LDA(At, 1, 1); STAGE(SA(1, 0), a3);
        BAR; WAIT_L(0); MMA(1, 0, At, B0); BAR; SCHED;
        STAGE(SB(1, 1), b3 + 16384);
        WAIT_V(6); BAR; MMA(1, 1, At, B1); BAR;
    }
    if (wr == 0) BAR;
    TS(10);
    {
        int tz = threadIdx.x; asm volatile("" : "+v"(tz));
        const int lane2 = tz & 63, fr2 = lane2 & 15, fq2 = lane2 >> 4;
        const bool bad = *(volatile LAS unsigned*)(lds + LDS_STAGE + 4) != 0u;
        float* o = P.out[owner] + (size_t)((ti & 1) * 256 + wr * 64 + fq2 * 4) * NFULL + rank * NS + tj * 256 + wc * 64 + fr2 * 4;
        const float qnan = __builtin_nanf("");
#pragma unroll
        for (int ai = 0; ai < 2; ++ai)
#pragma unroll
            for (int m = 0; m < 4; ++m)
#pragma unroll
                for (int r = 0; r < 4; ++r) {
                    f32x4 y;
                    y.x = gelu(acc[ai][0][m][0][r]); y.y = gelu(acc[ai][0][m][1][r]);
                    y.z = gelu(acc[ai][1][m][0][r]); y.w = gelu(acc[ai][1][m][1][r]);
                    if (bad) y = (f32x4){qnan, qnan, qnan, qnan};
                    asm volatile("s_waitcnt vmcnt(%0)" :: "i"(ODEPTH - 1) : "memory");
                    store_wt(o + (size_t)(ai * 128 + m * 16 + r) * NFULL, __builtin_bit_cast(u32x4, y));
                }
        TS(11);
        WAIT_V(0);
        BAR;
        if (tz == 0 && owner != rank)
            __hip_atomic_fetch_add((GAS unsigned*)(P.ws[owner] + OFF_DONE), 1u, __ATOMIC_RELAXED, __HIP_MEMORY_SCOPE_SYSTEM);
        TS(12);
    }
}

extern "C" void kernel_launch(int rank, int nranks, ncclComm_t comm,
                              hipStream_t stream,
                              void* const* d_in, const int* in_sizes, int n_in,
                              void* d_out, int out_size,
                              const int* in_dtypes, int out_dtype,
                              void* const* d_out_all,
                              void* const* d_ws_all, size_t ws_bytes) {
    (void)comm; (void)in_dtypes; (void)out_dtype; (void)d_out;
    if (nranks != NR || (unsigned)rank >= (unsigned)NR || n_in != 2 || ws_bytes < WS_NEED
        || in_sizes[0] != 4096 * KS || in_sizes[1] != KS * NFULL || out_size != 512 * NFULL) {
        fprintf(stderr, "kernel_launch: unexpected problem shape\n");
        return;
    }
    Params P;
    P.x = (const float*)d_in[0];
    P.w = (const float*)d_in[1];
    for (int p = 0; p < NR; ++p) { P.out[p] = (float*)d_out_all[p]; P.ws[p] = (char*)d_ws_all[p]; }
    P.rank = rank;
    P.pad = 0;
    gemm_rs_kernel<<<NPUSH + NGEMM, 512, 0, stream>>>(P);
    HIP_CHECK(hipGetLastError());
}
```

```cpp
#include <hip/hip_runtime.h>
#include <rccl/rccl.h>
#include <cstdio>

#define HIP_CHECK(expr) do { hipError_t _e = (expr); if (_e != hipSuccess) { \
    fprintf(stderr, "HIP error %s at %s:%d\n", hipGetErrorString(_e), __FILE__, __LINE__); \
    return; } } while (0)

#define LAS __attribute__((address_space(3)))
#define GAS __attribute__((address_space(1)))
typedef _Float16 f16x8 __attribute__((ext_vector_type(8)));
typedef float f32x4 __attribute__((ext_vector_type(4)));
typedef unsigned u32x4 __attribute__((ext_vector_type(4)));

#define LEAD 4
#define GATE 4
#define ODEPTH 1
#define MAINDELAY 32
#define PORTMAP 1
#define STAMP 0

namespace {

constexpr int NR = 8;
constexpr int KS = 512;
constexpr int NFULL = 16384;
constexpr int NS = 2048;
constexpr int NBLK = 24;
constexpr int NPW = 16;
constexpr unsigned UNIT = 32768;
constexpr unsigned TILE_BYTES = 262144;
constexpr size_t CTRL_BYTES = 1u << 20;
constexpr size_t STAGE_BYTES = (size_t)64 * NBLK * UNIT;
constexpr size_t PART_BYTES = (size_t)NR * 16 * TILE_BYTES;
constexpr size_t OWNB_BYTES = (size_t)NR * 64 * UNIT;
constexpr size_t OFF_STAGE = CTRL_BYTES, OFF_PART = OFF_STAGE + 2 * STAGE_BYTES, OFF_OWNB = OFF_PART + 2 * PART_BYTES;
constexpr size_t WS_NEED = OFF_OWNB + OWNB_BYTES;
constexpr unsigned OFF_TMO = 0, OFF_PUSHED = 1024  , OFF_QUEUE = 2048  , OFF_CNT = 4096  ,
                   OFF_FLAG = 65536  , OFF_PFLAG = 262144  ,
                   OFF_OFLAG = 524288  , OFF_STAMP = 786432;
constexpr unsigned long long DEADLINE_TICKS = 50000000ull;
constexpr int NPUSH = 8 * NPW, NMAIN = 128;
constexpr unsigned LDS_STAGE = 131072, LDS_BYTES = LDS_STAGE + 64;

struct Params {
    const float* x;
    const float* w;
    float* out;
    char* ws[NR];
    int rank;
    unsigned peermap;
};

__device__ __forceinline__ void store_wt(void* p, u32x4 v) {
    asm volatile("global_store_dwordx4 %0, %1, off sc0 sc1\n\ts_nop 1" :: "v"(p), "v"(v) : "memory");
}
__device__ __forceinline__ unsigned pack2(float a, float b) {
    typedef _Float16 h2 __attribute__((ext_vector_type(2)));
    h2 h; h.x = (_Float16)a; h.y = (_Float16)b;
    return __builtin_bit_cast(unsigned, h);
}
__device__ __forceinline__ float gelu(float y) {
    const float k1 = -2.0f * 0.7978845608028654f * 1.4426950408889634f, k2 = k1 * 0.044715f;
    const float e = __builtin_amdgcn_exp2f(y * __builtin_fmaf(y * y, k2, k1));
    return y * __builtin_amdgcn_rcpf(1.0f + e);
}
__device__ __forceinline__ unsigned ld_sys(const char* p) {
    return __hip_atomic_load((const GAS unsigned*)p, __ATOMIC_RELAXED, __HIP_MEMORY_SCOPE_SYSTEM);
}
__device__ __forceinline__ unsigned ld_agent(const char* p) {
    return __hip_atomic_load((const GAS unsigned*)p, __ATOMIC_RELAXED, __HIP_MEMORY_SCOPE_AGENT);
}
__device__ __forceinline__ void st_sys(char* p, unsigned v) {
    __hip_atomic_store((GAS unsigned*)p, v, __ATOMIC_RELAXED, __HIP_MEMORY_SCOPE_SYSTEM);
}
__device__ __forceinline__ void wave_wait(const char* word, unsigned need, char* tmo, unsigned call) {
    const unsigned long long t0 = __builtin_amdgcn_s_memrealtime();
    while (!__all(ld_sys(word) >= need)) {
        if ((unsigned)__builtin_amdgcn_readfirstlane(ld_agent(tmo)) == call
            || __builtin_amdgcn_s_memrealtime() - t0 > DEADLINE_TICKS) {
            __hip_atomic_store((GAS unsigned*)tmo, call, __ATOMIC_RELAXED, __HIP_MEMORY_SCOPE_AGENT);
            break;
        }
        __builtin_amdgcn_s_sleep(2);
    }
    asm volatile("" ::: "memory");
}
__device__ __forceinline__ unsigned wave_wait64(const char* word, unsigned need, char* tmo, unsigned call) {
    const unsigned long long t0 = __builtin_amdgcn_s_memrealtime();
    unsigned long long v;
    for (;;) {
        v = __hip_atomic_load((const GAS unsigned long long*)word, __ATOMIC_RELAXED, __HIP_MEMORY_SCOPE_SYSTEM);
        if (__all((unsigned)v >= need)) break;
        if ((unsigned)__builtin_amdgcn_readfirstlane(ld_agent(tmo)) == call
            || __builtin_amdgcn_s_memrealtime() - t0 > DEADLINE_TICKS) {
            __hip_atomic_store((GAS unsigned*)tmo, call, __ATOMIC_RELAXED, __HIP_MEMORY_SCOPE_AGENT);
            break;
        }
        __builtin_amdgcn_s_sleep(2);
    }
    asm volatile("" ::: "memory");
    return (unsigned)(v >> 32);
}
template <int NA> __device__ __forceinline__ void item_of(int t, int& kind, int& s, int& idx) {
    constexpr int R1 = 8 + NA, R2 = 8 + 2 * NA, E1 = LEAD * R1, E2 = E1 + (8 - LEAD) * R2;
    int e;
    if (t < E1) { s = t / R1; e = t - s * R1; }
    else if (t < E2) { const int r = (t - E1) / R2; e = t - E1 - r * R2; s = e < R1 ? r + LEAD : r; }
    else { const int r = (t - E2) / NA; e = t - E2 - r * NA + R1; s = r + 8 - LEAD; }
    kind = e < 8 ? 0 : e < R1 ? 1 : 2;
    idx = e < 8 ? e : e < R1 ? e - 8 : e - R1;
}

#define TS(k) do { } while (0)
#define WAIT_V(n) asm volatile("s_waitcnt vmcnt(" #n ")" ::: "memory")
#define WAIT_L(n) asm volatile("s_waitcnt lgkmcnt(" #n ")" ::: "memory")
#define BAR __builtin_amdgcn_s_barrier()
#define SCHED __builtin_amdgcn_sched_barrier(0)

}

__global__ __launch_bounds__(512, 2) void gemm_rs_kernel(Params P) {
    __shared__ __attribute__((aligned(16))) unsigned char shm[LDS_BYTES];
    LAS unsigned char* lds = (LAS unsigned char*)shm;
    const int tid = threadIdx.x, lane = tid & 63, wid = __builtin_amdgcn_readfirstlane(tid >> 6);
    const int bid = __builtin_amdgcn_readfirstlane(blockIdx.x);
    const int rank = P.rank & (NR - 1);
    const int peer = (P.peermap >> ((bid & 7) * 4)) & (NR - 1);
    const int slot = (bid >> 3) & 15;
    const bool pusher = bid < NPUSH;
    char* const me = P.ws[rank];
    char* const pws = P.ws[peer];
    TS(0);
    auto draw_call = [&]() -> unsigned {
        if (tid == 0) {
            const unsigned c = __hip_atomic_fetch_add((GAS unsigned*)(me + OFF_CNT + bid * 128), 1u,
                                                      __ATOMIC_RELAXED, __HIP_MEMORY_SCOPE_AGENT) + 1u;
            *(LAS unsigned*)(lds + LDS_STAGE) = c;
            *(LAS unsigned*)(lds + LDS_STAGE + 4) = 0u;
            *(LAS unsigned*)(lds + LDS_STAGE + 8) = 0u;
        }
        __builtin_amdgcn_s_waitcnt(0x0F70);
        __syncthreads();
        return __builtin_amdgcn_readfirstlane(*(LAS unsigned*)(lds + LDS_STAGE));
    };
    unsigned call;

    {
        const int fr = lane & 15, fq = lane >> 4;
        const int dest = pusher ? peer : rank;
        char* const dws = pusher ? pws : me;
        const bool local = dest == rank;
        const unsigned nitem = local ? 192u : 176u;
        const unsigned nstatic = local ? NPW + NMAIN : NPW;
        f32x4 v[8];
        int kind, s, blk;
        auto load = [&](int t) {
            int idx;
            if (local) item_of<8>(t, kind, s, idx); else item_of<7>(t, kind, s, idx);
            if (kind == 0) {
                blk = idx;
                const float* src = P.w + (size_t)(s * 64 + (wid >> 2) * 32 + fq * 4) * NFULL
                                 + dest * NS + idx * 256 + (wid & 3) * 64 + fr * 4;
#pragma unroll
                for (int j = 0; j < 8; ++j) v[j] = *(const f32x4*)(src + (size_t)((j >> 2) * 16 + (j & 3)) * NFULL);
            } else {
                const int rb = ((rank + 1 + idx) & (NR - 1)) * 2 + kind - 1;
                blk = 8 + rb;
                const float* src = P.x + (size_t)(rb * 256 + wid * 32 + fr) * KS + s * 64 + fq * 4;
#pragma unroll
                for (int j = 0; j < 8; ++j) v[j] = *(const f32x4*)(src + (j >> 2) * 16 * KS + (j & 3) * 16);
            }
        };
        if (!pusher) __builtin_amdgcn_s_sleep(MAINDELAY);
        load(pusher ? slot : bid - NPUSH + NPW);
        asm volatile("" ::: "memory");
        call = draw_call();
        char* const dstage = dws + OFF_STAGE + (size_t)(call & 1u) * STAGE_BYTES;
#pragma unroll 1
        for (unsigned n = 1;; ++n) {
            const unsigned u = (unsigned)((rank * 8 + s) * NBLK + blk);
            const unsigned ou = (unsigned)((peer * 8 + s) * 8 + blk);
            const bool own = kind == 0 && !local;
            u32x4 o[4];
            unsigned off, d1, d2;
            if (kind == 0) {
                off = ((wid & 3) * 4 + (wid >> 2)) * 1024; d1 = 2048; d2 = 16384;
#pragma unroll
                for (int f = 0; f < 4; ++f) {
                    o[f].x = pack2(v[0][f], v[1][f]); o[f].y = pack2(v[2][f], v[3][f]);
                    o[f].z = pack2(v[4][f], v[5][f]); o[f].w = pack2(v[6][f], v[7][f]);
                }
            } else {
                off = wid * 4096; d1 = 1024; d2 = 2048;
#pragma unroll
                for (int f = 0; f < 4; ++f) {
                    o[f].x = pack2(v[2 * f][0], v[2 * f][1]); o[f].y = pack2(v[2 * f][2], v[2 * f][3]);
                    o[f].z = pack2(v[2 * f + 1][0], v[2 * f + 1][1]); o[f].w = pack2(v[2 * f + 1][2], v[2 * f + 1][3]);
                }
            }
            off += lane * 16;
            asm volatile("" : "+v"(o[0]), "+v"(o[1]), "+v"(o[2]), "+v"(o[3]));
            char* const dst = dstage + (size_t)u * UNIT + off;
            store_wt(dst, o[0]);
            if (own) {
                char* const od = me + OFF_OWNB + (size_t)ou * UNIT + off;
                store_wt(od, o[0]); store_wt(od + d1, o[1]); store_wt(od + d2, o[2]); store_wt(od + d2 + d1, o[3]);
            }
            unsigned ticket = 0;
            if (tid == 0)
                ticket = __hip_atomic_fetch_add((GAS unsigned*)(me + OFF_QUEUE + dest * 128), 1u, __ATOMIC_RELAXED, __HIP_MEMORY_SCOPE_AGENT);
            WAIT_V(0);
            store_wt(dst + d1, o[1]);
            if (tid == 0) *(volatile LAS unsigned*)(lds + LDS_STAGE + 8) = (n << 16) | (ticket - (call - 1u) * nitem + NPW);
            unsigned nx;
            do { nx = __builtin_amdgcn_readfirstlane(*(volatile LAS unsigned*)(lds + LDS_STAGE + 8)); } while ((nx >> 16) != n);
            nx &= 0xffffu;
            if (nx < nitem) load(nx);
            WAIT_V(8);
            store_wt(dst + d2, o[2]);
            WAIT_V(0);
            store_wt(dst + d2 + d1, o[3]);
            WAIT_V(0);
            __builtin_amdgcn_s_waitcnt(0x0F70);
            __syncthreads();
            if (tid == 0) {
                st_sys(dws + OFF_FLAG + u * 128u, call);
                if (own) st_sys(me + OFF_OFLAG + ou * 128u, call);
            }
            TS(n < 12 ? n : 12);
            if (nx >= nitem) break;
        }
        TS(29);
        if (pusher && local) return;
        if (pusher && tid == 0)
            __hip_atomic_fetch_add((GAS unsigned*)(me + OFF_PUSHED + peer * 128), 1u, __ATOMIC_RELAXED, __HIP_MEMORY_SCOPE_AGENT);
    }

    const int ti = (pusher ? rank : peer) * 2 + (slot >> 3), tj = slot & 7;
    const int nsrc = pusher ? 0 : peer == rank ? 7 : 6;
    const int skip = peer == rank ? NR : ((peer - rank) & (NR - 1));
    const int nt = 8 + nsrc * 8;
    const int wr = wid >> 2, wc = wid & 3;
    const char* const stage = me + OFF_STAGE + (size_t)(call & 1u) * STAGE_BYTES;
    const char* const abase = stage + (unsigned)(8 + ti) * UNIT;
    const char* const bbase = pusher ? me + OFF_OWNB + (size_t)(((peer - rank) * 64 + tj) * (int)UNIT) : stage + (unsigned)tj * UNIT;
    const unsigned bstep = pusher ? 8u * UNIT : NBLK * UNIT;
    const unsigned voff = (unsigned)tid * 16u;
    const unsigned ldsw = (unsigned)wid * 1024u;
    const unsigned aoff = (unsigned)wr * 8192u + (unsigned)lane * 16u, boff = (unsigned)wc * 4096u + (unsigned)lane * 16u;

#define SA(b, h) (((b) * 2 + (h)) * 16384)
#define SB(b, h) ((4 + (b) * 2 + (h)) * 16384)
#define STAGE(bufoff, gbase) do { _Pragma("unroll") for (int _i = 0; _i < 2; ++_i) \
        __builtin_amdgcn_global_load_lds((const GAS unsigned*)((gbase) + _i * 8192 + voff), (LAS unsigned*)(lds + (bufoff) + ldsw + _i * 8192), 16, 0, 17); } while (0)
#define LDA(dst, b, h) do { _Pragma("unroll") for (int m = 0; m < 4; ++m) _Pragma("unroll") for (int k = 0; k < 2; ++k) \
        dst[m][k] = *(const LAS f16x8*)(lds + SA(b, h) + aoff + m * 2048 + k * 1024); } while (0)
#define LDB(dst, b, h) do { _Pragma("unroll") for (int n = 0; n < 2; ++n) _Pragma("unroll") for (int k = 0; k < 2; ++k) \
        dst[n][k] = *(const LAS f16x8*)(lds + SB(b, h) + boff + n * 2048 + k * 1024); } while (0)
#define MMA(ai, bj, At, Bt) do { __builtin_amdgcn_s_setprio(1); \
        _Pragma("unroll") for (int m = 0; m < 4; ++m) _Pragma("unroll") for (int n = 0; n < 2; ++n) _Pragma("unroll") for (int k = 0; k < 2; ++k) \
        acc[ai][bj][m][n] = __builtin_amdgcn_mfma_f32_16x16x32_f16(At[m][k], Bt[n][k], acc[ai][bj][m][n], 0, 0, 0); \
        __builtin_amdgcn_s_setprio(0); } while (0)

    int cs = -1, cq = 0, round3 = -1;
    auto next_kt = [&]() -> unsigned {
        const int rel = cq + 1 + (cq + 1 >= skip);
        const unsigned kt = cs < 0 ? (unsigned)(rank * 8 + cq) : (unsigned)(((rank + rel) & (NR - 1)) * 8 + cs);
        round3 = cs;
        if (++cq == (cs < 0 ? 8 : nsrc)) { cq = 0; ++cs; }
        return __builtin_amdgcn_readfirstlane(kt);
    };
    auto ready = [&](int s) {
        if (wid == 0) {
            int lz = lane; asm volatile("" : "+v"(lz));
            const int e = (lz >> 1) & 7, q = e < nsrc ? e : 0;
            const int kt = s < 0 ? rank * 8 + e : ((rank + q + 1 + (q + 1 >= skip)) & (NR - 1)) * 8 + s;
            unsigned fo = OFF_FLAG + (unsigned)(kt * NBLK + tj + (lz & 1) * (8 + ti - tj)) * 128u;
            if (pusher && !(lz & 1)) fo = OFF_OFLAG + (unsigned)((peer * 8 + e) * 8 + tj) * 128u;
            wave_wait(me + fo, call, me + OFF_TMO, call);
            if (s == 7 || pusher) {
                const unsigned bad = (unsigned)__builtin_amdgcn_readfirstlane(ld_agent(me + OFF_TMO)) == call;
                *(LAS unsigned*)(lds + LDS_STAGE + 4) = bad;
                WAIT_L(0);
            }
            TS(s < 0 ? 30 : 13 + s);
        }
        asm volatile("" ::: "memory"); BAR; asm volatile("" ::: "memory");
    };

    f32x4 acc[2][2][4][2];
#pragma unroll
    for (int a = 0; a < 2; ++a)
#pragma unroll
        for (int b = 0; b < 2; ++b)
#pragma unroll
            for (int m = 0; m < 4; ++m)
#pragma unroll
                for (int n = 0; n < 2; ++n) acc[a][b][m][n] = (f32x4){0.f, 0.f, 0.f, 0.f};
    f16x8 At[4][2], B0[2][2], B1[2][2];

    int have = -1;
    ready(have);
    unsigned k2 = next_kt(), k3 = next_kt();
    {
        const char* a0 = abase + k2 * (NBLK * UNIT); const char* b0 = bbase + k2 * bstep;
        const char* a1 = abase + k3 * (NBLK * UNIT); const char* b1 = bbase + k3 * bstep;
        STAGE(SB(0, 0), b0); STAGE(SA(0, 0), a0); STAGE(SB(0, 1), b0 + 16384); STAGE(SA(0, 1), a0 + 16384);
        if (wr == 1) BAR;
        WAIT_V(4); BAR;
        STAGE(SB(1, 0), b1); STAGE(SA(1, 0), a1); STAGE(SB(1, 1), b1 + 16384);
        WAIT_V(6); BAR;
    }
#pragma unroll 1
    for (int t = 0; t < nt; t += 2) {
        const char* a1 = abase + k3 * (NBLK * UNIT);
        if (t + 2 < nt) {
            k2 = next_kt(); k3 = next_kt();
            if (round3 > have) { have = round3; ready(have); }
        }
        const char* a2 = abase + k2 * (NBLK * UNIT); const char* b2 = bbase + k2 * bstep;
        const char* a3 = abase + k3 * (NBLK * UNIT); const char* b3 = bbase + k3 * bstep;
        LDB(B0, 0, 0); SCHED; LDA(At, 0, 0); STAGE(SA(1, 1), a1 + 16384);
        WAIT_L(8); BAR; WAIT_L(0); MMA(0, 0, At, B0); BAR; SCHED;
        LDB(B1, 0, 1); STAGE(SB(0, 0), b2);
        BAR; WAIT_L(0); MMA(0, 1, At, B1); BAR;
        LDA(At, 0, 1); STAGE(SA(0, 0), a2);
        BAR; WAIT_L(0); MMA(1, 0, At, B0); BAR; SCHED;
        STAGE(SB(0, 1), b2 + 16384);
        WAIT_V(6); BAR; MMA(1, 1, At, B1); BAR;
        LDB(B0, 1, 0); SCHED; LDA(At, 1, 0); STAGE(SA(0, 1), a2 + 16384);
        WAIT_L(8); BAR; WAIT_L(0); MMA(0, 0, At, B0); BAR; SCHED;
        LDB(B1, 1, 1); STAGE(SB(1, 0), b3);
        BAR; WAIT_L(0); MMA(0, 1, At, B1); BAR;
        LDA(At, 1, 1); STAGE(SA(1, 0), a3);
        BAR; WAIT_L(0); MMA(1, 0, At, B0); BAR; SCHED;
        STAGE(SB(1, 1), b3 + 16384);
        WAIT_V(6); BAR; MMA(1, 1, At, B1); BAR;
    }
    if (wr == 0) BAR;
    TS(21);
    int tz = threadIdx.x; asm volatile("" : "+v"(tz));
    const int lane2 = tz & 63;
    bool bad = *(volatile LAS unsigned*)(lds + LDS_STAGE + 4) != 0u;
    const float qnan = __builtin_nanf("");
#define ACC(g, q) acc[(g) >> 1][((q) >> 1) & 1][((g) & 1) * 2 + ((q) >> 2)][(q) & 1]
    const unsigned foff = OFF_PFLAG + (unsigned)wid * 256u + (call & 1u) * 128u;
    if (!pusher && peer != rank) {
        char* const dst = pws + OFF_PART + (size_t)(call & 1u) * PART_BYTES + (size_t)(rank * 16 + slot) * TILE_BYTES + wid * 32768 + lane2 * 16;
        char* const flag = pws + foff + (unsigned)(rank * 16 + slot) * 2048u + (lane2 & 3) * 8;
        float mx = 0.f;
#pragma unroll
        for (int g = 0; g < 4; ++g)
#pragma unroll
            for (int q = 0; q < 8; ++q) {
                const f32x4 y = ACC(g, q);
                mx = fmaxf(fmaxf(mx, fabsf(y.x)), fabsf(y.y)); mx = fmaxf(fmaxf(mx, fabsf(y.z)), fabsf(y.w));
            }
#pragma unroll
        for (int d = 1; d < 64; d <<= 1) mx = fmaxf(mx, __shfl_xor(mx, d));
        const unsigned emax = (__builtin_amdgcn_readfirstlane(__builtin_bit_cast(unsigned, mx)) >> 23) & 255u;
        const unsigned base = emax > 15u ? emax - 15u : 0u;
        wave_wait(me + OFF_PUSHED + peer * 128, (call - 1u) * NPW + GATE, me + OFF_TMO, call);
        TS(22);
#pragma unroll
        for (int g = 0; g < 4; ++g) {
            u32x4 w[8], any = {0u, 0u, 0u, 0u};
#pragma unroll
            for (int q = 0; q < 8; ++q) { w[q] = __builtin_bit_cast(u32x4, ACC(g, q)) - (base << 23); any |= w[q]; }
            const u32x4 x7 = (w[7] & 0x07ffffffu) | ((w[7] >> 4) & 0x08000000u);
#pragma unroll
            for (int q = 0; q < 7; ++q) {
                const u32x4 nib = x7 << (27 - 4 * q);
                if (q) asm volatile("s_waitcnt vmcnt(%0)" :: "i"(ODEPTH - 1) : "memory");
                store_wt(dst + g * 8192 + q * 1024, (w[q] & 0x87ffffffu) | (nib & 0x78000000u));
            }
            const bool need = ((any.x | any.y | any.z | any.w) & 0x78000000u) != 0u;
            const unsigned long long mask = __ballot(need);
            if (need) {
                u32x4 pl = {0u, 0u, 0u, 0u};
#pragma unroll
                for (int q = 0; q < 8; ++q) pl |= ((w[q] >> 27) & 15u) << (4 * q);
                store_wt(dst + g * 8192 + 7168, pl);
            }
            WAIT_V(0);
            if (lane2 < 4) {
                const unsigned info = base | (((unsigned)(mask >> (lane2 * 16)) & 0xffffu) << 8) | (bad ? 1u << 24 : 0u);
                __hip_atomic_store((GAS unsigned long long*)(flag + g * 32), ((unsigned long long)info << 32) | call,
                                   __ATOMIC_RELAXED, __HIP_MEMORY_SCOPE_SYSTEM);
            }
        }
        TS(23);
        return;
    }
    const char* const src = me + OFF_PART + (size_t)(call & 1u) * PART_BYTES + (size_t)(peer * 16 + slot) * TILE_BYTES + wid * 32768 + lane2 * 16;
    const char* const flag = me + foff + (unsigned)(peer * 16 + slot) * 2048u + (lane2 >> 4) * 8;
    float* const o = P.out + (size_t)((slot >> 3) * 256 + wr * 64 + (lane2 >> 4) * 4) * NFULL + peer * NS + tj * 256 + wc * 64 + (lane2 & 15) * 4;
#pragma unroll
    for (int g = 0; g < 4; ++g) {
        if (pusher) {
            const unsigned info = wave_wait64(flag + g * 32, call, me + OFF_TMO, call);
            const unsigned tm = ld_agent(me + OFF_TMO);
            u32x4 w[8];
            const char* s0 = src + g * 8192; const char* s1 = s0 + 4096;
            asm volatile("global_load_dwordx4 %0, %7, off sc0 sc1\n\t"
                         "global_load_dwordx4 %1, %7, off offset:1024 sc0 sc1\n\t"
                         "global_load_dwordx4 %2, %7, off offset:2048 sc0 sc1\n\t"
                         "global_load_dwordx4 %3, %7, off offset:3072 sc0 sc1\n\t"
                         "global_load_dwordx4 %4, %8, off sc0 sc1\n\t"
                         "global_load_dwordx4 %5, %8, off offset:1024 sc0 sc1\n\t"
                         "global_load_dwordx4 %6, %8, off offset:2048 sc0 sc1\n\t"
                         "s_waitcnt vmcnt(0)"
                         : "=&v"(w[0]), "=&v"(w[1]), "=&v"(w[2]), "=&v"(w[3]), "=&v"(w[4]), "=&v"(w[5]), "=&v"(w[6])
                         : "v"(s0), "v"(s1) : "memory");
            bad = bad || tm == call || (info >> 24) != 0u;
            u32x4 x7 = {0u, 0u, 0u, 0u};
#pragma unroll
            for (int q = 0; q < 7; ++q) { x7 |= ((w[q] >> 27) & 15u) << (4 * q); w[q] &= 0x87ffffffu; }
            w[7] = (x7 & 0x07ffffffu) | ((x7 << 4) & 0x80000000u);
            if ((info >> (8 + (lane2 & 15))) & 1u) {
                u32x4 pl;
                asm volatile("global_load_dwordx4 %0, %1, off offset:3072 sc0 sc1\n\ts_waitcnt vmcnt(0)" : "=&v"(pl) : "v"(s1) : "memory");
#pragma unroll
                for (int q = 0; q < 8; ++q) w[q] |= ((pl >> (4 * q)) & 15u) << 27;
            }
            const unsigned add = (info & 255u) << 23;
#pragma unroll
            for (int q = 0; q < 8; ++q) ACC(g, q) += __builtin_bit_cast(f32x4, w[q] + add);
        }
#pragma unroll
        for (int mm = 0; mm < 2; ++mm)
#pragma unroll
            for (int r = 0; r < 4; ++r) {
                const int ai = g >> 1, m = (g & 1) * 2 + mm;
                f32x4 y;
                y.x = gelu(acc[ai][0][m][0][r]); y.y = gelu(acc[ai][0][m][1][r]);
                y.z = gelu(acc[ai][1][m][0][r]); y.w = gelu(acc[ai][1][m][1][r]);
                if (bad) y = (f32x4){qnan, qnan, qnan, qnan};
                store_wt(o + (size_t)(ai * 128 + m * 16 + r) * NFULL, __builtin_bit_cast(u32x4, y));
            }
        TS(24 + g);
    }
    WAIT_V(0);
    TS(28);
}

extern "C" void kernel_launch(int rank, int nranks, ncclComm_t comm,
                              hipStream_t stream,
                              void* const* d_in, const int* in_sizes, int n_in,
                              void* d_out, int out_size,
                              const int* in_dtypes, int out_dtype,
                              void* const* d_out_all,
                              void* const* d_ws_all, size_t ws_bytes) {
    (void)comm; (void)in_dtypes; (void)out_dtype; (void)d_out_all;
    if (nranks != NR || (unsigned)rank >= (unsigned)NR || n_in != 2 || ws_bytes < WS_NEED
        || in_sizes[0] != 4096 * KS || in_sizes[1] != KS * NFULL || out_size != 512 * NFULL) {
        fprintf(stderr, "kernel_launch: unexpected problem shape\n");
        return;
    }
    Params P;
    P.x = (const float*)d_in[0];
    P.w = (const float*)d_in[1];
    P.out = (float*)d_out;
    for (int p = 0; p < NR; ++p) P.ws[p] = (char*)d_ws_all[p];
    P.rank = rank;
    static const unsigned char port[NR][NR] = {
        {0, 2, 3, 6, 4, 7, 5, 1}, {1, 4, 5, 7, 6, 2, 3, 0}, {2, 0, 6, 3, 7, 1, 5, 4}, {3, 6, 0, 2, 7, 4, 1, 5},
        {4, 1, 7, 5, 6, 3, 0, 2}, {5, 7, 1, 4, 0, 6, 2, 3}, {6, 3, 2, 0, 1, 5, 4, 7}, {7, 5, 4, 1, 2, 0, 3, 6}};
    P.peermap = 0;
    for (int c = 0; c < NR; ++c) P.peermap |= (unsigned)(PORTMAP ? port[rank][c] : (rank + c) & (NR - 1)) << (4 * c);
    gemm_rs_kernel<<<NPUSH + NMAIN, 512, 0, stream>>>(P);
    HIP_CHECK(hipGetLastError());
}
```

```cpp
#include <hip/hip_runtime.h>
#include <rccl/rccl.h>
#include <cstdio>

#define HIP_CHECK(expr) do { hipError_t _e = (expr); if (_e != hipSuccess) { \
    fprintf(stderr, "HIP error %s at %s:%d\n", hipGetErrorString(_e), __FILE__, __LINE__); \
    return; } } while (0)

#define LAS __attribute__((address_space(3)))
#define GAS __attribute__((address_space(1)))
typedef _Float16 f16x8 __attribute__((ext_vector_type(8)));
typedef float f32x4 __attribute__((ext_vector_type(4)));
typedef unsigned u32x4 __attribute__((ext_vector_type(4)));
typedef unsigned u32x3 __attribute__((ext_vector_type(3)));
typedef unsigned short u16x2 __attribute__((ext_vector_type(2)));
typedef unsigned long long u64;

#define ORDER "awabwabwawabwabwawabbbboooooooo"
#define PW 8
#define THINK 0
#define THINK2 4
#define POLL 8
#define PAIRG 3
#define BURSTG 4
#define LATE 48
#define STAMP 0

namespace {

constexpr int NR = 8;
constexpr int KS = 512;
constexpr int NFULL = 16384;
constexpr int NS = 2048;
constexpr int NBLK = 24;
constexpr unsigned UNIT = 32768;
constexpr unsigned TILE_BYTES = 262144;
constexpr unsigned NPULL = 168, NSLICE = NPULL * 8, NLOCAL = 192, NLOCAL_STATIC = 144;
constexpr size_t CTRL_BYTES = 1u << 20;
constexpr size_t STAGE_BYTES = (size_t)64 * NBLK * UNIT;
constexpr size_t PART_BYTES = (size_t)NR * 16 * TILE_BYTES;
constexpr size_t WSET_BYTES = (size_t)NR * 64 * UNIT;
constexpr size_t XSET_BYTES = (size_t)128 * UNIT;
constexpr size_t OFF_STAGE = CTRL_BYTES, OFF_PART = OFF_STAGE + 2 * STAGE_BYTES, OFF_OWNB = OFF_PART + 2 * PART_BYTES,
                 OFF_EXPW = OFF_OWNB + WSET_BYTES, OFF_EXPX = OFF_EXPW + 2 * WSET_BYTES, WS_NEED = OFF_EXPX + 2 * XSET_BYTES;
constexpr unsigned OFF_TMO = 0, OFF_QUEUE = 2048  , OFF_CNT = 4096  ,
                   OFF_FLAG = 65536  , OFF_PFLAG = 262144  ,
                   OFF_OFLAG = 524288  , OFF_TOKEN = 589824  ,
                   OFF_EFLAG = 655360  , OFF_STAMP = 851968;
constexpr u64 DEADLINE_TICKS = 50000000ull;
constexpr unsigned LDS_STAGE = 131072, LDS_BYTES = LDS_STAGE + 64;
constexpr unsigned M15 = 0x40004000u;

struct Params {
    const float* x;
    const float* w;
    float* out;
    char* ws[NR];
    int rank;
    unsigned peermap;
    unsigned order[NPULL / 4];
};

__device__ __forceinline__ void store_wt(void* p, u32x4 v) {
    asm volatile("global_store_dwordx4 %0, %1, off sc0 sc1\n\ts_nop 1" :: "v"(p), "v"(v) : "memory");
}
__device__ __forceinline__ u32x4 load_sys(const char* p) {
    u32x4 v;
    asm volatile("global_load_dwordx4 %0, %1, off sc0 sc1\n\ts_waitcnt vmcnt(0)" : "=&v"(v) : "v"(p) : "memory");
    return v;
}
__device__ __forceinline__ unsigned pack2(float a, float b) {
    typedef _Float16 h2 __attribute__((ext_vector_type(2)));
    h2 h; h.x = (_Float16)a; h.y = (_Float16)b;
    return __builtin_bit_cast(unsigned, h);
}
__device__ __forceinline__ unsigned pk_add16(unsigned a, unsigned b) {
    return __builtin_bit_cast(unsigned, (u16x2)(__builtin_bit_cast(u16x2, a) + __builtin_bit_cast(u16x2, b)));
}
__device__ __forceinline__ float gelu(float y) {
    const float k1 = -2.0f * 0.7978845608028654f * 1.4426950408889634f, k2 = k1 * 0.044715f;
    const float e = __builtin_amdgcn_exp2f(y * __builtin_fmaf(y * y, k2, k1));
    return y * __builtin_amdgcn_rcpf(1.0f + e);
}
__device__ __forceinline__ unsigned wave_max(unsigned v) {
    v = max(v, (unsigned)__builtin_amdgcn_update_dpp(0, (int)v, 0x111, 0xf, 0xf, true));
    v = max(v, (unsigned)__builtin_amdgcn_update_dpp(0, (int)v, 0x112, 0xf, 0xf, true));
    v = max(v, (unsigned)__builtin_amdgcn_update_dpp(0, (int)v, 0x114, 0xf, 0xf, true));
    v = max(v, (unsigned)__builtin_amdgcn_update_dpp(0, (int)v, 0x118, 0xf, 0xf, true));
    const unsigned a = __builtin_amdgcn_readlane(v, 15), b = __builtin_amdgcn_readlane(v, 31),
                   c = __builtin_amdgcn_readlane(v, 47), d = __builtin_amdgcn_readlane(v, 63);
    return max(max(a, b), max(c, d));
}
__device__ __forceinline__ unsigned ld_sys(const char* p) {
    return __hip_atomic_load((const GAS unsigned*)p, __ATOMIC_RELAXED, __HIP_MEMORY_SCOPE_SYSTEM);
}
__device__ __forceinline__ u64 ld_sys64(const char* p) {
    return __hip_atomic_load((const GAS u64*)p, __ATOMIC_RELAXED, __HIP_MEMORY_SCOPE_SYSTEM);
}
__device__ __forceinline__ unsigned ld_agent(const char* p) {
    return __hip_atomic_load((const GAS unsigned*)p, __ATOMIC_RELAXED, __HIP_MEMORY_SCOPE_AGENT);
}
__device__ __forceinline__ void st_sys(char* p, unsigned v) {
    __hip_atomic_store((GAS unsigned*)p, v, __ATOMIC_RELAXED, __HIP_MEMORY_SCOPE_SYSTEM);
}
__device__ __forceinline__ void st_sys64(char* p, unsigned lo, unsigned hi) {
    __hip_atomic_store((GAS u64*)p, ((u64)hi << 32) | lo, __ATOMIC_RELAXED, __HIP_MEMORY_SCOPE_SYSTEM);
}
__device__ __forceinline__ unsigned add_agent(char* p, unsigned v) {
    return __hip_atomic_fetch_add((GAS unsigned*)p, v, __ATOMIC_RELAXED, __HIP_MEMORY_SCOPE_AGENT);
}
template <int SLEEP> __device__ __forceinline__ bool give_up(unsigned& n, u64 t0, char* tmo, unsigned call) {
    if ((++n & 15u) == 0u
        && ((unsigned)__builtin_amdgcn_readfirstlane(ld_agent(tmo)) == call || __builtin_amdgcn_s_memrealtime() - t0 > DEADLINE_TICKS)) {
        __hip_atomic_store((GAS unsigned*)tmo, call, __ATOMIC_RELAXED, __HIP_MEMORY_SCOPE_AGENT);
        return true;
    }
    __builtin_amdgcn_s_sleep(SLEEP);
    return false;
}
template <int SLEEP> __device__ __forceinline__ void wave_wait(const char* word, unsigned need, char* tmo, unsigned call) {
    const u64 t0 = __builtin_amdgcn_s_memrealtime();
    unsigned n = 0;
    while (!__all(ld_sys(word) >= need) && !give_up<SLEEP>(n, t0, tmo, call)) { }
    asm volatile("" ::: "memory");
}
template <int SLEEP> __device__ __forceinline__ unsigned wave_wait64(const char* word, unsigned need, char* tmo, unsigned call) {
    const u64 t0 = __builtin_amdgcn_s_memrealtime();
    unsigned n = 0;
    u64 v;
    do { v = ld_sys64(word); } while (!__all((unsigned)v >= need) && !give_up<SLEEP>(n, t0, tmo, call));
    asm volatile("" ::: "memory");
    return (unsigned)(v >> 32);
}

#define TS(k) do { } while (0)
#define WAIT_V(n) asm volatile("s_waitcnt vmcnt(" #n ")" ::: "memory")
#define WAIT_L(n) asm volatile("s_waitcnt lgkmcnt(" #n ")" ::: "memory")
#define BAR __builtin_amdgcn_s_barrier()
#define SCHED __builtin_amdgcn_sched_barrier(0)
#define LDSW(i) (*(volatile LAS unsigned*)(lds + LDS_STAGE + 4 * (i)))

}

__global__ __launch_bounds__(512, 2) void gemm_rs_kernel(Params P) {
    __shared__ __attribute__((aligned(16))) unsigned char shm[LDS_BYTES];
    LAS unsigned char* lds = (LAS unsigned char*)shm;
    const int tid = threadIdx.x, lane = tid & 63, wid = __builtin_amdgcn_readfirstlane(tid >> 6);
    const int bid = __builtin_amdgcn_readfirstlane(blockIdx.x);
    const int rank = P.rank & (NR - 1);
    const int peer = (P.peermap >> ((bid & 7) * 4)) & (NR - 1);
    const int slot = (bid >> 3) & 15;
    const bool aux = bid < 128, remote = peer != rank;
    const int skip = remote ? (peer - rank) & (NR - 1) : NR;
    char* const me = P.ws[rank];
    char* const pws = P.ws[peer];
    char* const tmo = me + OFF_TMO;
    TS(0);
    auto draw_call = [&]() -> unsigned {
        if (tid == 0) { LDSW(0) = add_agent(me + OFF_CNT + bid * 128, 1u) + 1u; LDSW(1) = 0u; }
        __syncthreads();
        return __builtin_amdgcn_readfirstlane(LDSW(0));
    };
    unsigned call;

    {
        const bool f1 = aux && remote;
        const int fr = lane & 15, fq = lane >> 4;
        unsigned item = f1 ? (slot < 8 ? 1u : 0u) : aux ? (unsigned)slot : (unsigned)(bid - 128 + 16);
        const int fd = f1 ? peer : lane >> 3;
        char* fws = P.ws[fd];
        if (!f1 && item >= 32u) __builtin_amdgcn_s_sleep(LATE);
#pragma unroll 1
        for (unsigned n = 0;; ++n) {
            const bool isw = f1 || item >= 128u;
            const int s = f1 ? item : isw ? (item - 128u) >> 3 : item >> 4;
            const int j = (f1 ? slot : item) & 7, rb = (item & 7) * 2 + ((item >> 3) & 1);
            f32x4 v[8];
            if (isw) {
                const float* src = P.w + (size_t)(s * 64 + (wid >> 2) * 32 + fq * 4) * NFULL
                                 + (f1 ? peer : rank) * NS + j * 256 + (wid & 3) * 64 + fr * 4;
#pragma unroll
                for (int i = 0; i < 8; ++i) v[i] = *(const f32x4*)(src + (size_t)((i >> 2) * 16 + (i & 3)) * NFULL);
            } else {
                const float* src = P.x + (size_t)(rb * 256 + wid * 32 + fr) * KS + s * 64 + fq * 4;
#pragma unroll
                for (int i = 0; i < 8; ++i) v[i] = *(const f32x4*)(src + (i >> 2) * 16 * KS + (i & 3) * 16);
            }
            asm volatile("" : "+v"(fws) :: "memory");
            if (n == 0) call = draw_call();
            const unsigned par = call & 1u;
            u32x4 o[4];
            unsigned off, d1, d2;
            if (isw) {
                off = ((wid & 3) * 4 + (wid >> 2)) * 1024; d1 = 2048; d2 = 16384;
#pragma unroll
                for (int f = 0; f < 4; ++f) {
                    o[f].x = pack2(v[0][f], v[1][f]); o[f].y = pack2(v[2][f], v[3][f]);
                    o[f].z = pack2(v[4][f], v[5][f]); o[f].w = pack2(v[6][f], v[7][f]);
                }
            } else {
                off = wid * 4096; d1 = 1024; d2 = 2048;
#pragma unroll
                for (int f = 0; f < 4; ++f) {
                    o[f].x = pack2(v[2 * f][0], v[2 * f][1]); o[f].y = pack2(v[2 * f][2], v[2 * f][3]);
                    o[f].z = pack2(v[2 * f + 1][0], v[2 * f + 1][1]); o[f].w = pack2(v[2 * f + 1][2], v[2 * f + 1][3]);
                }
            }
            off += lane * 16;
            const unsigned u = (unsigned)((rank * 8 + s) * NBLK + (isw ? j : 8 + rb));
            const unsigned wu = (unsigned)((peer * 8 + s) * 8 + j);
            const bool push = f1 && s == 0;
            const bool expo = f1 ? s != 0 : !isw && (rb >> 1) != rank;
            char* const rdst = pws + OFF_STAGE + par * STAGE_BYTES + (size_t)u * UNIT + off;
            if (push) store_wt(rdst, o[0]);
            if (expo) {
                float mx = 0.f;
#pragma unroll
                for (int i = 0; i < 8; ++i) { mx = fmaxf(fmaxf(mx, fabsf(v[i].x)), fabsf(v[i].y)); mx = fmaxf(fmaxf(mx, fabsf(v[i].z)), fabsf(v[i].w)); }
                const unsigned emax = wave_max(pack2(mx, 0.f)) >> 10, base = emax > 15u ? emax - 15u : 0u;
                const unsigned sub = (0x10000u - (base << 10)) * 0x10001u & (base ? 0xffffffffu : 0u);
                unsigned w[16], any = 0u;
#pragma unroll
                for (int i = 0; i < 16; ++i) { w[i] = pk_add16(o[i >> 2][i & 3], sub); any |= w[i]; }
                const unsigned y = (w[15] & 0x3fff3fffu) | ((w[15] >> 1) & M15);
                u32x4 e[4];
#pragma unroll
                for (int i = 0; i < 15; ++i) e[i >> 2][i & 3] = (w[i] & ~M15) | ((y << (14 - i)) & M15);
                char* const ed = me + (f1 ? OFF_EXPW + par * WSET_BYTES + (size_t)wu * UNIT : OFF_EXPX + par * XSET_BYTES + (size_t)(s * 16 + rb) * UNIT) + wid * 4096;
                store_wt(ed + lane * 16, e[0]); store_wt(ed + 1024 + lane * 16, e[1]); store_wt(ed + 2048 + lane * 16, e[2]);
                const u32x3 e3 = {e[3].x, e[3].y, e[3].z};
                asm volatile("global_store_dwordx3 %0, %1, off sc0 sc1\n\ts_nop 1" :: "v"(ed + 3072 + lane * 12), "v"(e3) : "memory");
                const u64 need = __ballot((any & M15) != 0u);
                const unsigned qm = ((need & 0xffffull) ? 1u : 0u) | ((need & 0xffff0000ull) ? 2u : 0u)
                                  | ((need & 0xffff00000000ull) ? 4u : 0u) | ((need >> 48) ? 8u : 0u);
                if (qm) {
                    unsigned pl = (w[15] & M15) << 1;
#pragma unroll
                    for (int i = 0; i < 15; ++i) pl |= (w[i] & M15) >> (14 - i);
                    st_sys(ed + 3840 + lane * 4, pl);
                }
                if (lane == 0) LDSW(4 + wid) = base | (qm << 8);
            }
            char* const ldst = me + (f1 ? OFF_OWNB + (size_t)wu * UNIT : OFF_STAGE + par * STAGE_BYTES + (size_t)u * UNIT) + off;
            store_wt(ldst, o[0]); store_wt(ldst + d1, o[1]); store_wt(ldst + d2, o[2]); store_wt(ldst + d2 + d1, o[3]);
            if (push) {
                WAIT_V(0); store_wt(rdst + d1, o[1]); WAIT_V(0); store_wt(rdst + d2, o[2]); WAIT_V(0); store_wt(rdst + d2 + d1, o[3]);
                break;
            }
            if (!f1 && tid == 0)
                LDSW(2) = add_agent(me + OFF_QUEUE + rank * 128, 1u) - (call - 1u) * NLOCAL + NLOCAL_STATIC;
            WAIT_V(0);
            __syncthreads();
            const unsigned next = f1 ? item + 1u : __builtin_amdgcn_readfirstlane(LDSW(2));
            if (wid == 0) {
                if (expo && (f1 ? lane < 8 : fd != rank))
                    st_sys64(fws + OFF_EFLAG + (par * 64u * NBLK + u) * 64u + (lane & 7) * 8, call, LDSW(4 + (lane & 7)));
                if (lane == 0) st_sys(me + (f1 ? OFF_OFLAG + wu * 128u : OFF_FLAG + u * 128u), call * 8u);
            }
            TS(1 + (n < 8 ? n : 8));
            if (next >= (f1 ? 8u : NLOCAL)) break;
            item = next;
            __syncthreads();
        }
    }
    if (aux && remote && slot >= 8) {
        const unsigned par = call & 1u;
        const char* const eflag = me + OFF_EFLAG + (par * 64u + peer * 8u) * (NBLK * 64u);
        auto unit_of = [&](unsigned slice) -> unsigned {
            const unsigned it = min(slice >> 3, NPULL - 1u), e = (P.order[it >> 2] >> ((it & 3u) * 8u)) & 255u, code = e & 31u;
            const int odd = code >= 14u, q = code - (odd ? 14u : 8u);
            const unsigned blk = code < 8u ? code : code >= 20u ? 8u + rank * 2 + (code - 20u)
                                                                 : 8u + ((rank + q + 1 + (q + 1 >= skip)) & (NR - 1)) * 2 + odd;
            return (e >> 5) * NBLK + blk;
        };
        auto src_of = [&](unsigned slice, unsigned su) -> const char* {
            const unsigned s = su / NBLK, blk = su - s * NBLK;
            return pws + (blk < 8u ? OFF_EXPW + par * WSET_BYTES + (size_t)((rank * 8 + s) * 8 + blk) * UNIT
                                   : OFF_EXPX + par * XSET_BYTES + (size_t)(s * 16 + blk - 8) * UNIT) + (slice & 7u) * 4096u;
        };
        unsigned sa = (slot - 8) * PW + wid, sb = sa + 8u * PW;
        unsigned ua = unit_of(sa), ub = unit_of(sb);
        u64 fl = ld_sys64(eflag + ua * 64u + (sa & 7u) * 8u);
        WAIT_V(0);
        if (lane == 0) {
            __hip_atomic_fetch_add((GAS unsigned*)(pws + OFF_FLAG + (unsigned)(rank * 8 * NBLK + (slot & 7)) * 128u), 1u, __ATOMIC_RELAXED, __HIP_MEMORY_SCOPE_SYSTEM);
            add_agent(me + OFF_OFLAG + (unsigned)(peer * 64 + (slot & 7)) * 128u, 1u);
        }
        TS(1);
        char* const nobody = me + OFF_CNT + bid * 128 + 64;
        char* pend = nobody;
#pragma unroll 1
        while (wid < PW) {
            unsigned info = (unsigned)(fl >> 32);
            if (!__all((unsigned)fl >= call)) {
                WAIT_V(0);
                if (lane == 0) add_agent(pend, 1u);
                pend = nobody;
                info = wave_wait64<2>(eflag + ua * 64u + (sa & 7u) * 8u, call, tmo, call);
            }
            const char* const src = src_of(sa, ua);
            u32x4 e[4]; u32x3 e3;
            e[0] = load_sys(src + lane * 16);
            u64 ex;
            asm volatile("s_sleep %7\n\t"
                         "global_load_dwordx4 %0, %3, off offset:1024 sc0 sc1\n\t"
                         "global_load_dwordx2 %1, %4, off sc0 sc1\n\t"
                         "s_mov_b64 %2, exec\n\ts_mov_b64 exec, 1\n\t"
                         "global_atomic_add %5, %6, off\n\t"
                         "s_mov_b64 exec, %2\n\ts_waitcnt vmcnt(1)"
                         : "=&v"(e[1]), "=&v"(fl), "=&s"(ex)
                         : "v"(src + lane * 16), "v"(eflag + ub * 64u + (sb & 7u) * 8u), "v"(pend), "v"(1u), "i"(THINK) : "memory");
            if (sb >= NSLICE && lane < 8 && lane % PW == wid)
                st_sys(me + OFF_TOKEN + (unsigned)((peer * 8 + (slot & 7)) * 8 + lane) * 128u, call * 2u - 1u);
            unsigned tk;
            asm volatile("s_sleep %8\n\t"
                         "global_load_dwordx4 %0, %4, off offset:2048 sc0 sc1\n\t"
                         "s_mov_b64 %3, exec\n\ts_mov_b64 exec, 1\n\t"
                         "global_atomic_add %2, %6, %7, off sc0\n\t"
                         "s_mov_b64 exec, %3\n\ts_waitcnt vmcnt(1)\n\t"
                         "s_sleep %8\n\t"
                         "global_load_dwordx3 %1, %5, off offset:3072 sc0 sc1\n\ts_waitcnt vmcnt(0)"
                         : "=&v"(e[2]), "=&v"(e3), "=&v"(tk), "=&s"(ex)
                         : "v"(src + lane * 16), "v"(src + lane * 12), "v"(me + OFF_QUEUE + peer * 128), "v"(1u), "i"(THINK) : "memory");
            tk = __builtin_amdgcn_readfirstlane(tk) - (call - 1u) * NSLICE + 16u * PW;
            const unsigned uc = unit_of(tk);
            e[3].x = e3.x; e[3].y = e3.y; e[3].z = e3.z;
            unsigned y = 0u;
#pragma unroll
            for (int i = 0; i < 15; ++i) { y |= (e[i >> 2][i & 3] & M15) >> (14 - i); e[i >> 2][i & 3] &= ~M15; }
            e[3].w = (y & 0x3fff3fffu) | ((y & M15) << 1);
            if ((info >> (8 + (lane >> 4))) & 1u) {
                const unsigned pl = ld_sys(src + 3840 + lane * 4);
#pragma unroll
                for (int i = 0; i < 15; ++i) e[i >> 2][i & 3] |= (pl << (14 - i)) & M15;
                e[3].w |= (pl >> 1) & M15;
            }
            const unsigned add = ((info & 31u) << 10) * 0x10001u;
#pragma unroll
            for (int i = 0; i < 16; ++i) e[i >> 2][i & 3] = pk_add16(e[i >> 2][i & 3], add);
            const bool isw = ua % NBLK < 8u;
            const unsigned u = (unsigned)(peer * 8 * NBLK) + ua, w8 = sa & 7u;
            const unsigned d1 = isw ? 2048u : 1024u, d2 = isw ? 16384u : 2048u;
            char* const dst = me + OFF_STAGE + par * STAGE_BYTES + (size_t)u * UNIT
                            + (isw ? ((w8 & 3u) * 4u + (w8 >> 2)) * 1024u : w8 * 4096u) + lane * 16;
            store_wt(dst, e[0]); store_wt(dst + d1, e[1]); store_wt(dst + d2, e[2]); store_wt(dst + d2 + d1, e[3]);
            pend = me + OFF_FLAG + u * 128u;
            if (sb >= NSLICE) {
                WAIT_V(0);
                if (lane == 0) add_agent(pend, 1u);
                break;
            }
            sa = sb; ua = ub; sb = tk; ub = uc;
        }
        TS(29);
    }

    const int ti = (aux ? rank : peer) * 2 + (slot >> 3), tj = slot & 7;
    const bool ownb = aux && remote;
    const int nsrc = remote ? 6 : 7;
    const int rfirst = aux && !remote ? 4 : -1, rlast = ownb ? -1 : !aux && !remote ? 3 : 7;
    const int nt = remote ? (aux ? 8 : 56) : (aux ? 28 : 36);
    const int wr = wid >> 2, wc = wid & 3;
    const char* const stage = me + OFF_STAGE + (size_t)(call & 1u) * STAGE_BYTES;
    const char* const abase = stage + (unsigned)(8 + ti) * UNIT;
    const char* const bbase = ownb ? me + OFF_OWNB + (size_t)(((peer - rank) * 64 + tj) * (int)UNIT) : stage + (unsigned)tj * UNIT;
    const unsigned bstep = ownb ? 8u * UNIT : NBLK * UNIT;
    const unsigned voff = (unsigned)tid * 16u;
    const unsigned ldsw = (unsigned)wid * 1024u;
    const unsigned aoff = (unsigned)wr * 8192u + (unsigned)lane * 16u, boff = (unsigned)wc * 4096u + (unsigned)lane * 16u;

#define SA(b, h) (((b) * 2 + (h)) * 16384)
#define SB(b, h) ((4 + (b) * 2 + (h)) * 16384)
#define STAGE(bufoff, gbase) do { _Pragma("unroll") for (int _i = 0; _i < 2; ++_i) \
        __builtin_amdgcn_global_load_lds((const GAS unsigned*)((gbase) + _i * 8192 + voff), (LAS unsigned*)(lds + (bufoff) + ldsw + _i * 8192), 16, 0, 17); } while (0)
#define LDA(dst, b, h) do { _Pragma("unroll") for (int m = 0; m < 4; ++m) _Pragma("unroll") for (int k = 0; k < 2; ++k) \
        dst[m][k] = *(const LAS f16x8*)(lds + SA(b, h) + aoff + m * 2048 + k * 1024); } while (0)
#define LDB(dst, b, h) do { _Pragma("unroll") for (int n = 0; n < 2; ++n) _Pragma("unroll") for (int k = 0; k < 2; ++k) \
        dst[n][k] = *(const LAS f16x8*)(lds + SB(b, h) + boff + n * 2048 + k * 1024); } while (0)
#define MMA(ai, bj, At, Bt) do { __builtin_amdgcn_s_setprio(1); \
        _Pragma("unroll") for (int m = 0; m < 4; ++m) _Pragma("unroll") for (int n = 0; n < 2; ++n) _Pragma("unroll") for (int k = 0; k < 2; ++k) \
        acc[ai][bj][m][n] = __builtin_amdgcn_mfma_f32_16x16x32_f16(At[m][k], Bt[n][k], acc[ai][bj][m][n], 0, 0, 0); \
        __builtin_amdgcn_s_setprio(0); } while (0)

    int cs = rfirst, cq = 0, round3 = rfirst;
    auto next_kt = [&]() -> unsigned {
        const int rel = cq + 1 + (cq + 1 >= skip);
        const unsigned kt = cs < 0 ? (unsigned)(rank * 8 + cq) : (unsigned)(((rank + rel) & (NR - 1)) * 8 + cs);
        round3 = cs;
        if (++cq == (cs < 0 ? 8 : nsrc)) { cq = 0; ++cs; }
        return __builtin_amdgcn_readfirstlane(kt);
    };
    auto ready = [&](int s) -> int {
        if (wid == 0) {
            int lz = lane; asm volatile("" : "+v"(lz));
            const int e = (lz >> 1) & 7, q = e < nsrc ? e : 0, sl = min(s + (lz >> 4), rlast);
            const int kt = sl < 0 ? rank * 8 + e : ((rank + q + 1 + (q + 1 >= skip)) & (NR - 1)) * 8 + sl;
            unsigned fo = OFF_FLAG + (unsigned)(kt * NBLK + tj + (lz & 1) * (8 + ti - tj)) * 128u;
            if (ownb && !(lz & 1)) fo = OFF_OFLAG + (unsigned)((peer * 8 + e) * 8 + tj) * 128u;
            u64 miss = ~__ballot(ld_sys(me + fo) >= call * 8u);
            if (miss & 0xffffull) {
                wave_wait<POLL>(me + __shfl(fo, lz & 15), call * 8u, tmo, call);
                miss = ~0ull;
            }
            miss >>= 16;
            const int got = min(s + (miss ? __builtin_ctzll(miss) >> 4 : 3), rlast);
            LDSW(3) = got;
            if (got == rlast)
                LDSW(1) = (unsigned)__builtin_amdgcn_readfirstlane(ld_agent(tmo)) == call;
            WAIT_L(0);
            TS(s < 0 ? 30 : 13 + s);
        }
        asm volatile("" ::: "memory"); BAR; asm volatile("" ::: "memory");
        return __builtin_amdgcn_readfirstlane(LDSW(3));
    };

    f32x4 acc[2][2][4][2];
#pragma unroll
    for (int a = 0; a < 2; ++a)
#pragma unroll
        for (int b = 0; b < 2; ++b)
#pragma unroll
            for (int m = 0; m < 4; ++m)
#pragma unroll
                for (int n = 0; n < 2; ++n) acc[a][b][m][n] = (f32x4){0.f, 0.f, 0.f, 0.f};
    f16x8 At[4][2], B0[2][2], B1[2][2];

    int have = ready(rfirst);
    unsigned k2 = next_kt(), k3 = next_kt();
    {
        const char* a0 = abase + k2 * (NBLK * UNIT); const char* b0 = bbase + k2 * bstep;
        const char* a1 = abase + k3 * (NBLK * UNIT); const char* b1 = bbase + k3 * bstep;
        WAIT_V(0);
        STAGE(SB(0, 0), b0); STAGE(SA(0, 0), a0); STAGE(SB(0, 1), b0 + 16384); STAGE(SA(0, 1), a0 + 16384);
        if (wr == 1) BAR;
        WAIT_V(4); BAR;
        STAGE(SB(1, 0), b1); STAGE(SA(1, 0), a1); STAGE(SB(1, 1), b1 + 16384);
        WAIT_V(6); BAR;
    }
#pragma unroll 1
    for (int t = 0; t < nt; t += 2) {
        const char* a1 = abase + k3 * (NBLK * UNIT);
        if (t + 2 < nt) {
            k2 = next_kt(); k3 = next_kt();
            if (round3 > have) have = ready(round3);
        }
        const char* a2 = abase + k2 * (NBLK * UNIT); const char* b2 = bbase + k2 * bstep;
        const char* a3 = abase + k3 * (NBLK * UNIT); const char* b3 = bbase + k3 * bstep;
        LDB(B0, 0, 0); SCHED; LDA(At, 0, 0); STAGE(SA(1, 1), a1 + 16384);
        WAIT_L(8); BAR; WAIT_L(0); MMA(0, 0, At, B0); BAR; SCHED;
        LDB(B1, 0, 1); STAGE(SB(0, 0), b2);
        BAR; WAIT_L(0); MMA(0, 1, At, B1); BAR;
        LDA(At, 0, 1); STAGE(SA(0, 0), a2);
        BAR; WAIT_L(0); MMA(1, 0, At, B0); BAR; SCHED;
        STAGE(SB(0, 1), b2 + 16384);
        WAIT_V(6); BAR; MMA(1, 1, At, B1); BAR;
        LDB(B0, 1, 0); SCHED; LDA(At, 1, 0); STAGE(SA(0, 1), a2 + 16384);
        WAIT_L(8); BAR; WAIT_L(0); MMA(0, 0, At, B0); BAR; SCHED;
        LDB(B1, 1, 1); STAGE(SB(1, 0), b3);
        BAR; WAIT_L(0); MMA(0, 1, At, B1); BAR;
        LDA(At, 1, 1); STAGE(SA(1, 0), a3);
        BAR; WAIT_L(0); MMA(1, 0, At, B0); BAR; SCHED;
        STAGE(SB(1, 1), b3 + 16384);
        WAIT_V(6); BAR; MMA(1, 1, At, B1); BAR;
    }
    if (wr == 0) BAR;
    TS(21);
    int tz = threadIdx.x; asm volatile("" : "+v"(tz));
    const int lane2 = tz & 63;
    bool bad = LDSW(1) != 0u;
    const float qnan = __builtin_nanf("");
#define ACC(g, q) acc[(g) >> 1][((q) >> 1) & 1][((g) & 1) * 2 + ((q) >> 2)][(q) & 1]
    const unsigned foff = OFF_PFLAG + (unsigned)wid * 256u + (call & 1u) * 128u;
    const size_t poff = OFF_PART + (size_t)(call & 1u) * PART_BYTES + (size_t)slot * TILE_BYTES + wid * 32768 + lane2 * 16;
    if (!aux) {
        char* const dst = me + poff + (size_t)peer * 16 * TILE_BYTES;
        float mx = 0.f;
#pragma unroll
        for (int g = 0; g < 4; ++g)
#pragma unroll
            for (int q = 0; q < 8; ++q) {
                const f32x4 y = ACC(g, q);
                mx = fmaxf(fmaxf(mx, fabsf(y.x)), fabsf(y.y)); mx = fmaxf(fmaxf(mx, fabsf(y.z)), fabsf(y.w));
            }
        const unsigned emax = (wave_max(__builtin_bit_cast(unsigned, mx)) >> 23) & 255u;
        const unsigned base = emax > 15u ? emax - 15u : 0u;
        unsigned info = base | (bad ? 1u << 24 : 0u);
#pragma unroll
        for (int g = 0; g < 4; ++g) {
            u32x4 w[8], any = {0u, 0u, 0u, 0u};
#pragma unroll
            for (int q = 0; q < 8; ++q) { w[q] = __builtin_bit_cast(u32x4, ACC(g, q)) - (base << 23); any |= w[q]; }
            const u32x4 x7 = (w[7] & 0x07ffffffu) | ((w[7] >> 4) & 0x08000000u);
#pragma unroll
            for (int q = 0; q < 7; ++q)
                store_wt(dst + g * 8192 + q * 1024, (w[q] & 0x87ffffffu) | ((x7 << (27 - 4 * q)) & 0x78000000u));
            const bool need = ((any.x | any.y | any.z | any.w) & 0x78000000u) != 0u;
            const u64 mask = __ballot(need);
            if (need) {
                u32x4 pl = {0u, 0u, 0u, 0u};
#pragma unroll
                for (int q = 0; q < 8; ++q) pl |= ((w[q] >> 27) & 15u) << (4 * q);
                store_wt(dst + g * 8192 + 7168, pl);
            }
            if ((lane2 >> 2) == g) info |= ((unsigned)(mask >> ((lane2 & 3) * 16)) & 0xffffu) << 8;
        }
        WAIT_V(0);
        if (lane2 < 16) st_sys64(pws + foff + (unsigned)(rank * 16 + slot) * 2048u + lane2 * 8, call, info);
        TS(23);
        return;
    }
    const char* const src = pws + poff + (size_t)rank * 16 * TILE_BYTES;
    char* const token = me + OFF_TOKEN + (unsigned)((peer * 8 + tj) * 8 + wid) * 128u;
    float* const o = P.out + (size_t)((slot >> 3) * 256 + wr * 64 + (lane2 >> 4) * 4) * NFULL + peer * NS + tj * 256 + wc * 64 + (lane2 & 15) * 4;
    const unsigned info16 = wave_wait64<POLL>(me + foff + (unsigned)(peer * 16 + slot) * 2048u + (lane2 & 15) * 8, call, tmo, call);
    TS(22);
    if (remote) wave_wait<2>(token, call * 2u - (slot < 8), tmo, call);
    bad = bad || ld_agent(tmo) == call || (info16 >> 24) != 0u;
    TS(23);
#pragma unroll
    for (int g = 0; g < 4; ++g) {
        {
            const unsigned info = __shfl(info16, g * 4 + (lane2 >> 4));
            u32x4 w[8];
            if (remote && slot >= 8 && g >= PAIRG && g < BURSTG) {
#pragma unroll
                for (int q = 0; q < 6; q += 2)
                    asm volatile("global_load_dwordx4 %0, %2, off sc0 sc1\n\t"
                                 "global_load_dwordx4 %1, %2, off offset:1024 sc0 sc1\n\ts_waitcnt vmcnt(0)"
                                 : "=&v"(w[q]), "=&v"(w[q + 1]) : "v"(src + g * 8192 + q * 1024) : "memory");
                w[6] = load_sys(src + g * 8192 + 6144);
            } else if (remote && !(slot >= 8 && g >= BURSTG)) {
#pragma unroll
                for (int q = 0; q < 7; ++q) {
                    if (q) __builtin_amdgcn_s_sleep(THINK2);
                    if (g == 3 && q == 6 && slot < 8 && lane2 == 0) st_sys(token, call * 2u);
                    w[q] = load_sys(src + g * 8192 + q * 1024);
                }
            } else {
                asm volatile("global_load_dwordx4 %0, %7, off sc0 sc1\n\t"
                             "global_load_dwordx4 %1, %7, off offset:1024 sc0 sc1\n\t"
                             "global_load_dwordx4 %2, %7, off offset:2048 sc0 sc1\n\t"
                             "global_load_dwordx4 %3, %7, off offset:3072 sc0 sc1\n\t"
                             "global_load_dwordx4 %4, %8, off sc0 sc1\n\t"
                             "global_load_dwordx4 %5, %8, off offset:1024 sc0 sc1\n\t"
                             "global_load_dwordx4 %6, %8, off offset:2048 sc0 sc1\n\ts_waitcnt vmcnt(0)"
                             : "=&v"(w[0]), "=&v"(w[1]), "=&v"(w[2]), "=&v"(w[3]), "=&v"(w[4]), "=&v"(w[5]), "=&v"(w[6])
                             : "v"(src + g * 8192), "v"(src + g * 8192 + 4096) : "memory");
            }
            u32x4 x7 = {0u, 0u, 0u, 0u};
#pragma unroll
            for (int q = 0; q < 7; ++q) { x7 |= ((w[q] >> 27) & 15u) << (4 * q); w[q] &= 0x87ffffffu; }
            w[7] = (x7 & 0x07ffffffu) | ((x7 << 4) & 0x80000000u);
            if ((info >> (8 + (lane2 & 15))) & 1u) {
                const u32x4 pl = load_sys(src + g * 8192 + 7168);
#pragma unroll
                for (int q = 0; q < 8; ++q) w[q] |= ((pl >> (4 * q)) & 15u) << 27;
            }
            const unsigned add = (info & 255u) << 23;
#pragma unroll
            for (int q = 0; q < 8; ++q) ACC(g, q) += __builtin_bit_cast(f32x4, w[q] + add);
        }
#pragma unroll
        for (int mm = 0; mm < 2; ++mm)
#pragma unroll
            for (int r = 0; r < 4; ++r) {
                const int ai = g >> 1, m = (g & 1) * 2 + mm;
                f32x4 y;
                y.x = gelu(acc[ai][0][m][0][r]); y.y = gelu(acc[ai][0][m][1][r]);
                y.z = gelu(acc[ai][1][m][0][r]); y.w = gelu(acc[ai][1][m][1][r]);
                if (bad) y = (f32x4){qnan, qnan, qnan, qnan};
                store_wt(o + (size_t)(ai * 128 + m * 16 + r) * NFULL, __builtin_bit_cast(u32x4, y));
            }
        TS(24 + g);
    }
    WAIT_V(0);
    TS(28);
}

extern "C" void kernel_launch(int rank, int nranks, ncclComm_t comm,
                              hipStream_t stream,
                              void* const* d_in, const int* in_sizes, int n_in,
                              void* d_out, int out_size,
                              const int* in_dtypes, int out_dtype,
                              void* const* d_out_all,
                              void* const* d_ws_all, size_t ws_bytes) {
    (void)comm; (void)in_dtypes; (void)out_dtype; (void)d_out_all;
    if (nranks != NR || (unsigned)rank >= (unsigned)NR || n_in != 2 || ws_bytes < WS_NEED
        || in_sizes[0] != 4096 * KS || in_sizes[1] != KS * NFULL || out_size != 512 * NFULL) {
        fprintf(stderr, "kernel_launch: unexpected problem shape\n");
        return;
    }
    Params P = {};
    P.x = (const float*)d_in[0];
    P.w = (const float*)d_in[1];
    P.out = (float*)d_out;
    for (int p = 0; p < NR; ++p) P.ws[p] = (char*)d_ws_all[p];
    P.rank = rank;
    static const unsigned char port[NR][NR] = {
        {0, 2, 3, 6, 4, 7, 5, 1}, {1, 4, 5, 7, 6, 2, 3, 0}, {2, 0, 6, 3, 7, 1, 5, 4}, {3, 6, 0, 2, 7, 4, 1, 5},
        {4, 1, 7, 5, 6, 3, 0, 2}, {5, 7, 1, 4, 0, 6, 2, 3}, {6, 3, 2, 0, 1, 5, 4, 7}, {7, 5, 4, 1, 2, 0, 3, 6}};
    for (int c = 0; c < NR; ++c) P.peermap |= (unsigned)port[rank][c] << (4 * c);
    unsigned n = 0, sw = 1, sa = 0, sb = 0, so = 0;
    for (const char* c = ORDER; *c; ++c) {
        unsigned& s = *c == 'w' ? sw : *c == 'a' ? sa : *c == 'b' ? sb : so;
        const unsigned first = *c == 'w' ? 0 : *c == 'a' ? 8 : *c == 'b' ? 14 : 20, cnt = *c == 'w' ? 8 : *c == 'o' ? 2 : 6;
        for (unsigned i = 0; i < cnt && n < NPULL; ++i, ++n) P.order[n >> 2] |= (((s & 7u) << 5) | (first + i)) << ((n & 3u) * 8u);
        ++s;
    }
    if (n != NPULL || sw != 8 || sa != 8 || sb != 8 || so != 8) { fprintf(stderr, "kernel_launch: bad ORDER\n"); return; }
    gemm_rs_kernel<<<256, 512, 0, stream>>>(P);
    HIP_CHECK(hipGetLastError());
}
```

```cpp
#include <hip/hip_runtime.h>
#include <rccl/rccl.h>
#include <cstdio>

#define HIP_CHECK(expr) do { hipError_t _e = (expr); if (_e != hipSuccess) { \
    fprintf(stderr, "HIP error %s at %s:%d\n", hipGetErrorString(_e), __FILE__, __LINE__); \
    return; } } while (0)

#define LAS __attribute__((address_space(3)))
#define GAS __attribute__((address_space(1)))
typedef _Float16 f16x8 __attribute__((ext_vector_type(8)));
typedef float f32x4 __attribute__((ext_vector_type(4)));
typedef unsigned u32x4 __attribute__((ext_vector_type(4)));
typedef unsigned u32x3 __attribute__((ext_vector_type(3)));
typedef unsigned short u16x2 __attribute__((ext_vector_type(2)));
typedef unsigned long long u64;

#define ORDER "awabwabwawabwabwawabbbboooooooo"
#define NC 4
#define WIN 6
#define FASTW 2
#define F1DEPTH 1
#define EXDEPTH 7
#define POLL 8
#define LATE 48
#define STAMP 0

static_assert(WIN <= 12 && NC <= 7, "loader bookkeeping");

namespace {

constexpr int NR = 8;
constexpr int KS = 512;
constexpr int NFULL = 16384;
constexpr int NS = 2048;
constexpr int NBLK = 24;
constexpr unsigned UNIT = 32768;
constexpr unsigned TILE_BYTES = 262144;
constexpr unsigned NPULL = 168, NSLICE = NPULL * 8, NWORK = NSLICE + 512, NLOCAL = 192, NLOCAL_STATIC = 144;
constexpr size_t CTRL_BYTES = 1u << 20;
constexpr size_t STAGE_BYTES = (size_t)64 * NBLK * UNIT;
constexpr size_t PART_BYTES = (size_t)NR * 16 * TILE_BYTES;
constexpr size_t WSET_BYTES = (size_t)NR * 64 * UNIT;
constexpr size_t XSET_BYTES = (size_t)128 * UNIT;
constexpr size_t OFF_STAGE = CTRL_BYTES, OFF_PART = OFF_STAGE + 2 * STAGE_BYTES, OFF_OWNB = OFF_PART + 2 * PART_BYTES,
                 OFF_EXPW = OFF_OWNB + WSET_BYTES, OFF_EXPX = OFF_EXPW + 2 * WSET_BYTES, WS_NEED = OFF_EXPX + 2 * XSET_BYTES;
constexpr unsigned OFF_TMO = 0, OFF_QUEUE = 2048  , OFF_CNT = 4096  ,
                   OFF_FLAG = 65536  , OFF_PFLAG = 262144  ,
                   OFF_OFLAG = 524288  , OFF_OWNF = 589824  ,
                   OFF_EFLAG = 655360  , OFF_STAMP = 851968;
constexpr u64 DEADLINE_TICKS = 50000000ull;
constexpr unsigned LDS_STAGE = 131072, LDS_BYTES = LDS_STAGE + 384;
constexpr unsigned M15 = 0x40004000u;

struct Params {
    const float* x;
    const float* w;
    float* out;
    char* ws[NR];
    int rank;
    unsigned peermap;
    unsigned order[NPULL / 4];
};

__device__ __forceinline__ void store_wt(void* p, u32x4 v) {
    asm volatile("global_store_dwordx4 %0, %1, off sc0 sc1\n\ts_nop 1" :: "v"(p), "v"(v) : "memory");
}
__device__ __forceinline__ u32x4 load_sys(const char* p) {
    u32x4 v;
    asm volatile("global_load_dwordx4 %0, %1, off sc0 sc1\n\ts_waitcnt vmcnt(0)" : "=&v"(v) : "v"(p) : "memory");
    return v;
}
__device__ __forceinline__ unsigned pack2(float a, float b) {
    typedef _Float16 h2 __attribute__((ext_vector_type(2)));
    h2 h; h.x = (_Float16)a; h.y = (_Float16)b;
    return __builtin_bit_cast(unsigned, h);
}
__device__ __forceinline__ unsigned pk_add16(unsigned a, unsigned b) {
    return __builtin_bit_cast(unsigned, (u16x2)(__builtin_bit_cast(u16x2, a) + __builtin_bit_cast(u16x2, b)));
}
__device__ __forceinline__ float gelu(float y) {
    const float k1 = -2.0f * 0.7978845608028654f * 1.4426950408889634f, k2 = k1 * 0.044715f;
    const float e = __builtin_amdgcn_exp2f(y * __builtin_fmaf(y * y, k2, k1));
    return y * __builtin_amdgcn_rcpf(1.0f + e);
}
__device__ __forceinline__ unsigned wave_max(unsigned v) {
    v = max(v, (unsigned)__builtin_amdgcn_update_dpp(0, (int)v, 0x111, 0xf, 0xf, true));
    v = max(v, (unsigned)__builtin_amdgcn_update_dpp(0, (int)v, 0x112, 0xf, 0xf, true));
    v = max(v, (unsigned)__builtin_amdgcn_update_dpp(0, (int)v, 0x114, 0xf, 0xf, true));
    v = max(v, (unsigned)__builtin_amdgcn_update_dpp(0, (int)v, 0x118, 0xf, 0xf, true));
    const unsigned a = __builtin_amdgcn_readlane(v, 15), b = __builtin_amdgcn_readlane(v, 31),
                   c = __builtin_amdgcn_readlane(v, 47), d = __builtin_amdgcn_readlane(v, 63);
    return max(max(a, b), max(c, d));
}
__device__ __forceinline__ unsigned ld_sys(const char* p) {
    return __hip_atomic_load((const GAS unsigned*)p, __ATOMIC_RELAXED, __HIP_MEMORY_SCOPE_SYSTEM);
}
__device__ __forceinline__ u64 ld_sys64(const char* p) {
    return __hip_atomic_load((const GAS u64*)p, __ATOMIC_RELAXED, __HIP_MEMORY_SCOPE_SYSTEM);
}
__device__ __forceinline__ unsigned ld_agent(const char* p) {
    return __hip_atomic_load((const GAS unsigned*)p, __ATOMIC_RELAXED, __HIP_MEMORY_SCOPE_AGENT);
}
__device__ __forceinline__ void st_sys(char* p, unsigned v) {
    __hip_atomic_store((GAS unsigned*)p, v, __ATOMIC_RELAXED, __HIP_MEMORY_SCOPE_SYSTEM);
}
__device__ __forceinline__ void st_sys64(char* p, unsigned lo, unsigned hi) {
    __hip_atomic_store((GAS u64*)p, ((u64)hi << 32) | lo, __ATOMIC_RELAXED, __HIP_MEMORY_SCOPE_SYSTEM);
}
__device__ __forceinline__ unsigned add_agent(char* p, unsigned v) {
    return __hip_atomic_fetch_add((GAS unsigned*)p, v, __ATOMIC_RELAXED, __HIP_MEMORY_SCOPE_AGENT);
}
template <int SLEEP> __device__ __forceinline__ bool give_up(unsigned& n, u64 t0, char* tmo, unsigned call) {
    if ((++n & 15u) == 0u
        && ((unsigned)__builtin_amdgcn_readfirstlane(ld_agent(tmo)) == call || __builtin_amdgcn_s_memrealtime() - t0 > DEADLINE_TICKS)) {
        __hip_atomic_store((GAS unsigned*)tmo, call, __ATOMIC_RELAXED, __HIP_MEMORY_SCOPE_AGENT);
        return true;
    }
    __builtin_amdgcn_s_sleep(SLEEP);
    return false;
}
template <int SLEEP> __device__ __forceinline__ void wave_wait(const char* word, unsigned need, char* tmo, unsigned call) {
    const u64 t0 = __builtin_amdgcn_s_memrealtime();
    unsigned n = 0;
    while (!__all(ld_sys(word) >= need) && !give_up<SLEEP>(n, t0, tmo, call)) { }
    asm volatile("" ::: "memory");
}
template <int SLEEP> __device__ __forceinline__ unsigned wave_wait64(const char* word, unsigned need, char* tmo, unsigned call) {
    const u64 t0 = __builtin_amdgcn_s_memrealtime();
    unsigned n = 0;
    u64 v;
    do { v = ld_sys64(word); } while (!__all((unsigned)v >= need) && !give_up<SLEEP>(n, t0, tmo, call));
    asm volatile("" ::: "memory");
    return (unsigned)(v >> 32);
}

__device__ __forceinline__ void unpack28(u32x4 (&w)[8], unsigned info, int lane, const char* plane) {
    u32x4 x7 = {0u, 0u, 0u, 0u};
#pragma unroll
    for (int q = 0; q < 7; ++q) { x7 |= ((w[q] >> 27) & 15u) << (4 * q); w[q] &= 0x87ffffffu; }
    w[7] = (x7 & 0x07ffffffu) | ((x7 << 4) & 0x80000000u);
    if ((info >> (8 + (lane & 15))) & 1u) {
        const u32x4 pl = load_sys(plane);
#pragma unroll
        for (int q = 0; q < 8; ++q) w[q] |= ((pl >> (4 * q)) & 15u) << 27;
    }
#pragma unroll
    for (int q = 0; q < 8; ++q) w[q] += (info & 255u) << 23;
}

#define TS(k) do { } while (0)
#define WAIT_V(n) asm volatile("s_waitcnt vmcnt(" #n ")" ::: "memory")
#define WAIT_L(n) asm volatile("s_waitcnt lgkmcnt(" #n ")" ::: "memory")
#define BAR __builtin_amdgcn_s_barrier()
#define SCHED __builtin_amdgcn_sched_barrier(0)
#define LDSW(i) (*(volatile LAS unsigned*)(lds + LDS_STAGE + 4 * (i)))

}

__global__ __launch_bounds__(512, 2) void gemm_rs_kernel(Params P) {
    __shared__ __attribute__((aligned(16))) unsigned char shm[LDS_BYTES];
    LAS unsigned char* lds = (LAS unsigned char*)shm;
    const int tid = threadIdx.x, lane = tid & 63, wid = __builtin_amdgcn_readfirstlane(tid >> 6);
    const int hb = __builtin_amdgcn_readfirstlane(blockIdx.x);
    const int rank = P.rank & (NR - 1);
    char* const me = P.ws[rank];
    char* const tmo = me + OFF_TMO;
    int bid = (hb & ~7) | (__builtin_amdgcn_s_getreg((3 << 11) | 20) & 7);
    TS(0);
    f32x4 v[8];
    auto first_item = [&](int b) -> unsigned { return b >= 128 ? b - 112 : (b & 7) == 0 ? b >> 3 : b < 64; };
    auto load_item = [&](int b, unsigned item) {
        const bool f1 = b < 128 && (b & 7) != 0;
        const int fr = lane & 15, fq = lane >> 4;
        if (f1 || item >= 128u) {
            const int s = f1 ? item : (item - 128u) >> 3, j = (f1 ? b >> 3 : item) & 7;
            const float* src = P.w + (size_t)(s * 64 + (wid >> 2) * 32 + fq * 4) * NFULL
                             + (f1 ? (P.peermap >> ((b & 7) * 4)) & (NR - 1) : rank) * NS + j * 256 + (wid & 3) * 64 + fr * 4;
#pragma unroll
            for (int i = 0; i < 8; ++i) v[i] = *(const f32x4*)(src + (size_t)((i >> 2) * 16 + (i & 3)) * NFULL);
        } else {
            const float* src = P.x + (size_t)(((item & 7) * 2 + ((item >> 3) & 1)) * 256 + wid * 32 + fr) * KS + (item >> 4) * 64 + fq * 4;
#pragma unroll
            for (int i = 0; i < 8; ++i) v[i] = *(const f32x4*)(src + (i >> 2) * 16 * KS + (i & 3) * 16);
        }
    };
    if (bid >= 144) __builtin_amdgcn_s_sleep(LATE);
    char* lws = P.ws[lane >> 3];
    load_item(bid, first_item(bid));
    asm volatile("" : "+v"(lws) :: "memory");
    if (wid == 0) {
        unsigned t = 0;
        if (lane < 2) t = add_agent(me + OFF_CNT + (lane ? bid * 128 + 4 : hb * 128), 1u);
        const unsigned c = __builtin_amdgcn_readlane(t, 0) + 1u;
        unsigned old = __builtin_amdgcn_readlane(t, 1), r = bid;
        for (int tries = 0; old != c - 1u && tries < 65536; ++tries) {
            if (lane == 0) add_agent(me + OFF_CNT + r * 128u + 4u, ~0u);
            r = (r + 1u) & 255u;
            if (lane == 0) t = add_agent(me + OFF_CNT + r * 128u + 4u, 1u);
            old = __builtin_amdgcn_readfirstlane(t);
        }
        LDSW(32 + lane) = 0u;
        if (lane < 16) LDSW(16 + lane) = 0u;
        if (lane == 0) { LDSW(0) = c; LDSW(1) = 0u; LDSW(12) = r; }
    }
    __syncthreads();
    const unsigned call = __builtin_amdgcn_readfirstlane(LDSW(0));
    {
        const int got = __builtin_amdgcn_readfirstlane(LDSW(12));
        if (got != bid) { bid = got; load_item(bid, first_item(bid)); }
    }
    const int peer = (P.peermap >> ((bid & 7) * 4)) & (NR - 1);
    const int slot = (bid >> 3) & 15;
    const bool aux = bid < 128, remote = (bid & 7) != 0;
    const int skip = remote ? (peer - rank) & (NR - 1) : NR;
    char* const pws = P.ws[peer];

    {
        const bool f1 = aux && remote;
        unsigned item = first_item(bid);
        const int fd = f1 ? peer : lane >> 3;
        char* const fws = f1 ? pws : lws;
#pragma unroll 1
        for (unsigned n = 0;; ++n) {
            const bool isw = f1 || item >= 128u;
            const int s = f1 ? item : isw ? (item - 128u) >> 3 : item >> 4;
            const int j = (f1 ? slot : item) & 7, rb = (item & 7) * 2 + ((item >> 3) & 1);
            const unsigned par = call & 1u;
            u32x4 o[4];
            unsigned off, d1, d2;
            if (isw) {
                off = ((wid & 3) * 4 + (wid >> 2)) * 1024; d1 = 2048; d2 = 16384;
#pragma unroll
                for (int f = 0; f < 4; ++f) {
                    o[f].x = pack2(v[0][f], v[1][f]); o[f].y = pack2(v[2][f], v[3][f]);
                    o[f].z = pack2(v[4][f], v[5][f]); o[f].w = pack2(v[6][f], v[7][f]);
                }
            } else {
                off = wid * 4096; d1 = 1024; d2 = 2048;
#pragma unroll
                for (int f = 0; f < 4; ++f) {
                    o[f].x = pack2(v[2 * f][0], v[2 * f][1]); o[f].y = pack2(v[2 * f][2], v[2 * f][3]);
                    o[f].z = pack2(v[2 * f + 1][0], v[2 * f + 1][1]); o[f].w = pack2(v[2 * f + 1][2], v[2 * f + 1][3]);
                }
            }
            off += lane * 16;
            const unsigned u = (unsigned)((rank * 8 + s) * NBLK + (isw ? j : 8 + rb));
            const unsigned wu = (unsigned)((peer * 8 + s) * 8 + j);
            const bool push = f1 && s == 0;
            const bool expo = f1 ? s != 0 : !isw && (rb >> 1) != rank;
            char* const rdst = pws + OFF_STAGE + par * STAGE_BYTES + (size_t)u * UNIT + off;
            if (push) store_wt(rdst, o[0]);
            if (expo) {
                float mx = 0.f;
#pragma unroll
                for (int i = 0; i < 8; ++i) { mx = fmaxf(fmaxf(mx, fabsf(v[i].x)), fabsf(v[i].y)); mx = fmaxf(fmaxf(mx, fabsf(v[i].z)), fabsf(v[i].w)); }
                const unsigned emax = wave_max(pack2(mx, 0.f)) >> 10, base = emax > 15u ? emax - 15u : 0u;
                const unsigned sub = (0x10000u - (base << 10)) * 0x10001u & (base ? 0xffffffffu : 0u);
                unsigned w[16], any = 0u;
#pragma unroll
                for (int i = 0; i < 16; ++i) { w[i] = pk_add16(o[i >> 2][i & 3], sub); any |= w[i]; }
                const unsigned y = (w[15] & 0x3fff3fffu) | ((w[15] >> 1) & M15);
                u32x4 e[4];
#pragma unroll
                for (int i = 0; i < 15; ++i) e[i >> 2][i & 3] = (w[i] & ~M15) | ((y << (14 - i)) & M15);
                char* const ed = me + (f1 ? OFF_EXPW + par * WSET_BYTES + (size_t)wu * UNIT : OFF_EXPX + par * XSET_BYTES + (size_t)(s * 16 + rb) * UNIT) + wid * 4096;
                store_wt(ed + lane * 16, e[0]); store_wt(ed + 1024 + lane * 16, e[1]); store_wt(ed + 2048 + lane * 16, e[2]);
                const u32x3 e3 = {e[3].x, e[3].y, e[3].z};
                asm volatile("global_store_dwordx3 %0, %1, off sc0 sc1\n\ts_nop 1" :: "v"(ed + 3072 + lane * 12), "v"(e3) : "memory");
                const u64 need = __ballot((any & M15) != 0u);
                const unsigned qm = ((need & 0xffffull) ? 1u : 0u) | ((need & 0xffff0000ull) ? 2u : 0u)
                                  | ((need & 0xffff00000000ull) ? 4u : 0u) | ((need >> 48) ? 8u : 0u);
                if (qm) {
                    unsigned pl = (w[15] & M15) << 1;
#pragma unroll
                    for (int i = 0; i < 15; ++i) pl |= (w[i] & M15) >> (14 - i);
                    st_sys(ed + 3840 + lane * 4, pl);
                }
                if (lane == 0) LDSW(4 + wid) = base | (qm << 8);
            }
            char* const ldst = me + (f1 ? OFF_OWNB + (size_t)wu * UNIT : OFF_STAGE + par * STAGE_BYTES + (size_t)u * UNIT) + off;
            store_wt(ldst, o[0]); store_wt(ldst + d1, o[1]); store_wt(ldst + d2, o[2]); store_wt(ldst + d2 + d1, o[3]);
            if (push) {
                if (wid > FASTW) WAIT_V(0);
                store_wt(rdst + d1, o[1]); WAIT_V(0); store_wt(rdst + d2, o[2]); WAIT_V(0); store_wt(rdst + d2 + d1, o[3]);
                break;
            }
            if (!f1 && tid == 0)
                LDSW(2) = add_agent(me + OFF_QUEUE + rank * 128, 1u) - (call - 1u) * NLOCAL + NLOCAL_STATIC;
            WAIT_V(0);
            __syncthreads();
            const unsigned next = f1 ? item + 1u : __builtin_amdgcn_readfirstlane(LDSW(2));
            if (wid == 0) {
                if (expo && (f1 ? lane < 8 : fd != rank))
                    st_sys64(fws + OFF_EFLAG + (par * 64u * NBLK + u) * 64u + (lane & 7) * 8, call, LDSW(4 + (lane & 7)));
                if (lane == 0) st_sys(me + (f1 ? OFF_OFLAG + wu * 128u : OFF_FLAG + u * 128u), call * 8u);
            }
            TS(1 + (n < 8 ? n : 8));
            if (next >= (f1 ? 8u : NLOCAL)) break;
            item = next;
            load_item(bid, item);
            __syncthreads();
        }
    }
    if (aux && remote && slot >= 8) {
        const unsigned par = call & 1u;
        const char* const eflag = me + OFF_EFLAG + (par * 64u + peer * 8u) * (NBLK * 64u);
        auto unit_of = [&](unsigned slice) -> unsigned {
            const unsigned it = min(slice >> 3, NPULL - 1u), e = (P.order[it >> 2] >> ((it & 3u) * 8u)) & 255u, code = e & 31u;
            const int odd = code >= 14u, q = code - (odd ? 14u : 8u);
            const unsigned blk = code < 8u ? code : code >= 20u ? 8u + rank * 2 + (code - 20u)
                                                                 : 8u + ((rank + q + 1 + (q + 1 >= skip)) & (NR - 1)) * 2 + odd;
            return (e >> 5) * NBLK + blk;
        };
        auto src_of = [&](unsigned slice, unsigned su) -> const char* {
            const unsigned s = su / NBLK, blk = su - s * NBLK;
            return pws + (blk < 8u ? OFF_EXPW + par * WSET_BYTES + (size_t)((rank * 8 + s) * 8 + blk) * UNIT
                                   : OFF_EXPX + par * XSET_BYTES + (size_t)(s * 16 + blk - 8) * UNIT) + (slice & 7u) * 4096u;
        };
        auto publish_push = [&]() {
            WAIT_V(0);
            __builtin_amdgcn_s_waitcnt(0x0F70);
            if (lane == 0) {
                __hip_atomic_fetch_add((GAS unsigned*)(pws + OFF_FLAG + (unsigned)(rank * 8 * NBLK + (slot & 7)) * 128u), 1u, __ATOMIC_RELAXED, __HIP_MEMORY_SCOPE_SYSTEM);
                add_agent(me + OFF_OFLAG + (unsigned)(peer * 64 + (slot & 7)) * 128u, 1u);
            }
        };
        if (wid == 0) {
            publish_push();
            TS(1);
            const unsigned lbase = (unsigned)(size_t)lds;
            unsigned head = 0, n = 0, alive = NC, np_ = 0, e0 = 0, c0 = 0, e1 = 0, c1 = 0, e2 = 0, c2 = 0;
            auto retire = [&]() { LDSW(24 + (c0 & 7u)) = c0 >> 3; e0 = e1; c0 = c1; e1 = e2; c1 = c2; --np_; };
            while (alive) {
                const u32x4 d = *(volatile LAS u32x4*)(lds + LDS_STAGE + 128 + (head & 15u) * 16u);
                if ((unsigned)__builtin_amdgcn_readfirstlane(d.w) != head + 1u) {
                    if (np_) {
                        const unsigned k = n - e0;
                        if (k >= 4u) WAIT_V(4); else WAIT_V(0);
                        retire();
                    }
                    continue;
                }
                ++head;
                const unsigned meta = __builtin_amdgcn_readfirstlane(d.z), cnt = meta & 7u, who = meta >> 3;
                if (cnt == 0u) { --alive; continue; }
                const char* src = (const char*)(((u64)d.y << 32) | d.x) + lane * 16;
                unsigned dst = lbase + (who & 7u) * 8192u;
                for (unsigned i = 0; i < cnt; ++i, src += 1024, dst += 1024u) {
                    asm volatile("s_waitcnt vmcnt(%0)" :: "i"(WIN - 1) : "memory");
                    if (np_ && e0 + (WIN - 1) <= n) retire();
                    if (!(cnt == 4u && i == 3u && lane >= 48)) {
                        unsigned keep;
                        asm volatile("s_mov_b32 %0, m0\n\ts_mov_b32 m0, %2\n\ts_nop 0\n\t"
                                     "global_load_lds_dwordx4 %1, off sc0 sc1\n\ts_mov_b32 m0, %0"
                                     : "=&s"(keep) : "v"(src), "s"(dst) : "memory");
                    }
                    ++n;
                }
                if (np_ == 0u) { e0 = n; c0 = who; } else if (np_ == 1u) { e1 = n; c1 = who; } else { e2 = n; c2 = who; }
                ++np_;
            }
        } else if (wid <= NC) {
            const unsigned me8 = wid - 1;
            const LAS unsigned char* const buf = lds + me8 * 8192u;
            const float qnan = __builtin_nanf("");
            auto flag_of = [&](unsigned t, unsigned u) -> const char* {
                const unsigned i = min(t, NWORK - 1u) - NSLICE;
                return t < NSLICE ? eflag + u * 64u + (t & 7u) * 8u
                                  : me + OFF_PFLAG + (peer * 128u + (i >> 2)) * 256u + par * 128u + (i & 3u) * 32u + (lane >> 4) * 8;
            };
            auto rsrc_of = [&](unsigned t, unsigned u) -> const char* {
                return t < NSLICE ? src_of(t, u) : pws + OFF_PART + par * PART_BYTES + (size_t)rank * 16 * TILE_BYTES + (size_t)(t - NSLICE) * 8192u;
            };
            unsigned nreq = 0;
            auto post = [&](const char* src, unsigned cnt) {
                ++nreq;
                if (lane == 0) {
                    const unsigned k = __hip_atomic_fetch_add((LAS unsigned*)(lds + LDS_STAGE + 64), 1u, __ATOMIC_RELAXED, __HIP_MEMORY_SCOPE_WORKGROUP);
                    LDSW(32 + (k & 15u) * 4) = (unsigned)(u64)src; LDSW(33 + (k & 15u) * 4) = (unsigned)((u64)src >> 32);
                    LDSW(34 + (k & 15u) * 4) = cnt | (me8 << 3) | (nreq << 6);
                    LDSW(35 + (k & 15u) * 4) = k + 1u;
                }
            };
            unsigned ta = (slot - 8) * NC + me8, tb = ta + 8u * NC;
            unsigned ua = unit_of(ta), ub = unit_of(tb);
            const u64 f0 = ld_sys64(flag_of(ta, ua));
            unsigned info = (unsigned)(f0 >> 32);
            const bool now = __all((unsigned)f0 >= call);
            if (now) post(rsrc_of(ta, ua), 4u);
            publish_push();
            if (!now) { info = wave_wait64<2>(flag_of(ta, ua), call, tmo, call); post(rsrc_of(ta, ua), 4u); }
            char* pend = nullptr;
#pragma unroll 1
            for (;;) {
                const bool isp = ta >= NSLICE;
                const unsigned ci = ta - NSLICE, ct = ci >> 5, cw = (ci >> 2) & 7u, cg = ci & 3u;
                const unsigned ooff = (((ct >> 3) * 256 + (cg >> 1) * 128 + (cw >> 2) * 64 + (cg & 1u) * 32 + (lane >> 4) * 4) * NFULL
                                       + peer * NS + (ct & 7u) * 256 + (cw & 3u) * 64 + (lane & 15) * 4) * 4u;
                const char* const fb = flag_of(tb, ub);
                const char* const ofb = me + OFF_OWNF + (peer * 128u + ((max(min(tb, NWORK - 1u), NSLICE) - NSLICE) >> 2)) * 64u;
                u64 fl, ex;
                unsigned ofl, tm, tk;
                u32x4 own[8] = {};
#define SMALL_LOADS(F, O, T, K, X, AF, AO, AT, AQ, ONE) \
                "global_load_dwordx2 " F ", " AF ", off sc0 sc1\n\t" \
                "global_load_dword " O ", " AO ", off sc0 sc1\n\t" \
                "global_load_dword " T ", " AT ", off sc1\n\t" \
                "s_mov_b64 " X ", exec\n\ts_mov_b64 exec, 1\n\t" \
                "global_atomic_add " K ", " AQ ", " ONE ", off sc0\n\t" \
                "s_mov_b64 exec, " X "\n\ts_waitcnt vmcnt(0)"
                if (isp) {
                    unsigned vo = ooff;
                    asm volatile("s_nop 4\n\tglobal_load_dwordx4 %0, %13, %14 sc0 sc1\n\tv_add_u32 %13, 0x10000, %13\n\t"
                                 "global_load_dwordx4 %1, %13, %14 sc0 sc1\n\tv_add_u32 %13, 0x10000, %13\n\t"
                                 "global_load_dwordx4 %2, %13, %14 sc0 sc1\n\tv_add_u32 %13, 0x10000, %13\n\t"
                                 "global_load_dwordx4 %3, %13, %14 sc0 sc1\n\tv_add_u32 %13, 0xd0000, %13\n\t"
                                 "global_load_dwordx4 %4, %13, %14 sc0 sc1\n\tv_add_u32 %13, 0x10000, %13\n\t"
                                 "global_load_dwordx4 %5, %13, %14 sc0 sc1\n\tv_add_u32 %13, 0x10000, %13\n\t"
                                 "global_load_dwordx4 %6, %13, %14 sc0 sc1\n\tv_add_u32 %13, 0x10000, %13\n\t"
                                 "global_load_dwordx4 %7, %13, %14 sc0 sc1\n\t"
                                 SMALL_LOADS("%8", "%9", "%10", "%11", "%12", "%15", "%16", "%17", "%18", "%19")
                                 : "=&v"(own[0]), "=&v"(own[1]), "=&v"(own[2]), "=&v"(own[3]), "=&v"(own[4]), "=&v"(own[5]), "=&v"(own[6]), "=&v"(own[7]),
                                   "=&v"(fl), "=&v"(ofl), "=&v"(tm), "=&v"(tk), "=&s"(ex), "+v"(vo)
                                 : "s"(P.out), "v"(fb), "v"(ofb), "v"(tmo), "v"(me + OFF_QUEUE + peer * 128), "v"(1u) : "memory");
                } else {
                    asm volatile(SMALL_LOADS("%0", "%1", "%2", "%3", "%4", "%5", "%6", "%7", "%8", "%9")
                                 : "=&v"(fl), "=&v"(ofl), "=&v"(tm), "=&v"(tk), "=&s"(ex)
                                 : "v"(fb), "v"(ofb), "v"(tmo), "v"(me + OFF_QUEUE + peer * 128), "v"(1u) : "memory");
                }
                if (pend && lane == 0) add_agent(pend, 1u);
                pend = nullptr;
                const bool bad = tm == call || (info >> 24) != 0u;
                tk = __builtin_amdgcn_readfirstlane(tk) - (call - 1u) * NWORK + 16u * NC;
                const unsigned uc = unit_of(tk);
                while ((unsigned)__builtin_amdgcn_readfirstlane(LDSW(24 + me8)) != nreq) __builtin_amdgcn_s_sleep(1);
                asm volatile("" ::: "memory");
                u32x4 w[8];
#pragma unroll
                for (int q = 0; q < 7; ++q) w[q] = *(const LAS u32x4*)(buf + q * 1024 + lane * 16);
                if (!isp) {
#pragma unroll
                    for (int c = 0; c < 3; ++c) w[3][c] = *(const LAS unsigned*)(buf + 3072 + lane * 12 + c * 4);
                }
                asm volatile("s_waitcnt lgkmcnt(0)" : "+v"(w[0]), "+v"(w[1]), "+v"(w[2]), "+v"(w[3]), "+v"(w[4]), "+v"(w[5]), "+v"(w[6]) :: "memory");
                const bool more = tb < NWORK, early = more && __all((unsigned)fl >= call);
                if (early) post(rsrc_of(tb, ub), tb < NSLICE ? 4u : 7u);
                if (!isp) {
                    unsigned y = 0u;
#pragma unroll
                    for (int i = 0; i < 15; ++i) { y |= (w[i >> 2][i & 3] & M15) >> (14 - i); w[i >> 2][i & 3] &= ~M15; }
                    w[3].w = (y & 0x3fff3fffu) | ((y & M15) << 1);
                    if ((info >> (8 + (lane >> 4))) & 1u) {
                        const unsigned pl = ld_sys(src_of(ta, ua) + 3840 + lane * 4);
#pragma unroll
                        for (int i = 0; i < 15; ++i) w[i >> 2][i & 3] |= (pl << (14 - i)) & M15;
                        w[3].w |= (pl >> 1) & M15;
                    }
                    const unsigned add = ((info & 31u) << 10) * 0x10001u;
#pragma unroll
                    for (int i = 0; i < 16; ++i) w[i >> 2][i & 3] = pk_add16(w[i >> 2][i & 3], add);
                    const bool isw = ua % NBLK < 8u;
                    const unsigned u = (unsigned)(peer * 8 * NBLK) + ua, w8 = ta & 7u;
                    const unsigned d1 = isw ? 2048u : 1024u, d2 = isw ? 16384u : 2048u;
                    char* const dst = me + OFF_STAGE + par * STAGE_BYTES + (size_t)u * UNIT
                                    + (isw ? ((w8 & 3u) * 4u + (w8 >> 2)) * 1024u : w8 * 4096u) + lane * 16;
                    store_wt(dst, w[0]); store_wt(dst + d1, w[1]); store_wt(dst + d2, w[2]); store_wt(dst + d2 + d1, w[3]);
                    pend = me + OFF_FLAG + u * 128u;
                } else {
                    unpack28(w, info, lane, rsrc_of(ta, ua) + 7168 + lane * 16);
#pragma unroll
                    for (int h = 0; h < 2; ++h)
#pragma unroll
                        for (int r = 0; r < 4; ++r) {
                            const f32x4 a = __builtin_bit_cast(f32x4, own[h * 4 + r]);
                            f32x4 y;
#pragma unroll
                            for (int c = 0; c < 4; ++c) y[c] = gelu(a[c] + __uint_as_float(w[h * 4 + c][r]));
                            if (bad) y = (f32x4){qnan, qnan, qnan, qnan};
                            store_wt((char*)P.out + ooff + (h * 16 + r) * (NFULL * 4), __builtin_bit_cast(u32x4, y));
                        }
                }
                if (!more) break;
                info = (unsigned)(fl >> 32);
                if (!early) {
                    WAIT_V(0);
                    if (pend && lane == 0) add_agent(pend, 1u);
                    pend = nullptr;
                    info = wave_wait64<2>(fb, call, tmo, call);
                    post(rsrc_of(tb, ub), tb < NSLICE ? 4u : 7u);
                }
                if (tb >= NSLICE && !__all(ofl >= call)) wave_wait<2>(ofb, call, tmo, call);
                ta = tb; ua = ub; tb = tk; ub = uc;
            }
            post(nullptr, 0u);
            WAIT_V(0);
            if (pend && lane == 0) add_agent(pend, 1u);
        } else {
            publish_push();
        }
        WAIT_V(0);
        return;
    }

    const bool ownb = aux && remote;
    int tile = slot;
again:
    const int ti = (aux ? rank : peer) * 2 + (tile >> 3), tj = slot & 7;
    const int nsrc = remote ? 6 : 7;
    const int rfirst = aux && !remote ? 4 : -1, rlast = ownb ? -1 : !aux && !remote ? 3 : 7;
    const int nt = remote ? (aux ? 8 : 56) : (aux ? 28 : 36);
    const int wr = wid >> 2, wc = wid & 3;
    const char* const stage = me + OFF_STAGE + (size_t)(call & 1u) * STAGE_BYTES;
    const char* const abase = stage + (unsigned)(8 + ti) * UNIT;
    const char* const bbase = ownb ? me + OFF_OWNB + (size_t)(((peer - rank) * 64 + tj) * (int)UNIT) : stage + (unsigned)tj * UNIT;
    const unsigned bstep = ownb ? 8u * UNIT : NBLK * UNIT;
    const unsigned voff = (unsigned)tid * 16u;
    const unsigned ldsw = (unsigned)wid * 1024u;
    const unsigned aoff = (unsigned)wr * 8192u + (unsigned)lane * 16u, boff = (unsigned)wc * 4096u + (unsigned)lane * 16u;

#define SA(b, h) (((b) * 2 + (h)) * 16384)
#define SB(b, h) ((4 + (b) * 2 + (h)) * 16384)
#define STAGE(bufoff, gbase) do { _Pragma("unroll") for (int _i = 0; _i < 2; ++_i) \
        __builtin_amdgcn_global_load_lds((const GAS unsigned*)((gbase) + _i * 8192 + voff), (LAS unsigned*)(lds + (bufoff) + ldsw + _i * 8192), 16, 0, 17); } while (0)
#define LDA(dst, b, h) do { _Pragma("unroll") for (int m = 0; m < 4; ++m) _Pragma("unroll") for (int k = 0; k < 2; ++k) \
        dst[m][k] = *(const LAS f16x8*)(lds + SA(b, h) + aoff + m * 2048 + k * 1024); } while (0)
#define LDB(dst, b, h) do { _Pragma("unroll") for (int n = 0; n < 2; ++n) _Pragma("unroll") for (int k = 0; k < 2; ++k) \
        dst[n][k] = *(const LAS f16x8*)(lds + SB(b, h) + boff + n * 2048 + k * 1024); } while (0)
#define MMA(ai, bj, At, Bt) do { __builtin_amdgcn_s_setprio(1); \
        _Pragma("unroll") for (int m = 0; m < 4; ++m) _Pragma("unroll") for (int n = 0; n < 2; ++n) _Pragma("unroll") for (int k = 0; k < 2; ++k) \
        acc[ai][bj][m][n] = __builtin_amdgcn_mfma_f32_16x16x32_f16(At[m][k], Bt[n][k], acc[ai][bj][m][n], 0, 0, 0); \
        __builtin_amdgcn_s_setprio(0); } while (0)

    int cs = rfirst, cq = 0, round3 = rfirst;
    auto next_kt = [&]() -> unsigned {
        const int rel = cq + 1 + (cq + 1 >= skip);
        const unsigned kt = cs < 0 ? (unsigned)(rank * 8 + cq) : (unsigned)(((rank + rel) & (NR - 1)) * 8 + cs);
        round3 = cs;
        if (++cq == (cs < 0 ? 8 : nsrc)) { cq = 0; ++cs; }
        return __builtin_amdgcn_readfirstlane(kt);
    };
    auto ready = [&](int s) -> int {
        if (wid == 0) {
            int lz = lane; asm volatile("" : "+v"(lz));
            const int e = (lz >> 1) & 7, q = e < nsrc ? e : 0, sl = min(s + (lz >> 4), rlast);
            const int kt = sl < 0 ? rank * 8 + e : ((rank + q + 1 + (q + 1 >= skip)) & (NR - 1)) * 8 + sl;
            unsigned fo = OFF_FLAG + (unsigned)(kt * NBLK + tj + (lz & 1) * (8 + ti - tj)) * 128u;
            if (ownb && !(lz & 1)) fo = OFF_OFLAG + (unsigned)((peer * 8 + e) * 8 + tj) * 128u;
            u64 miss = ~__ballot(ld_sys(me + fo) >= call * 8u);
            if (miss & 0xffffull) {
                wave_wait<POLL>(me + __shfl(fo, lz & 15), call * 8u, tmo, call);
                miss = ~0ull;
            }
            miss >>= 16;
            const int got = min(s + (miss ? __builtin_ctzll(miss) >> 4 : 3), rlast);
            LDSW(3) = got;
            if (got == rlast)
                LDSW(1) = (unsigned)__builtin_amdgcn_readfirstlane(ld_agent(tmo)) == call;
            WAIT_L(0);
            TS(s < 0 ? 30 : 13 + s);
        }
        asm volatile("" ::: "memory"); BAR; asm volatile("" ::: "memory");
        return __builtin_amdgcn_readfirstlane(LDSW(3));
    };

    f32x4 acc[2][2][4][2];
#pragma unroll
    for (int a = 0; a < 2; ++a)
#pragma unroll
        for (int b = 0; b < 2; ++b)
#pragma unroll
            for (int m = 0; m < 4; ++m)
#pragma unroll
                for (int n = 0; n < 2; ++n) acc[a][b][m][n] = (f32x4){0.f, 0.f, 0.f, 0.f};
    f16x8 At[4][2], B0[2][2], B1[2][2];

    int have = ready(rfirst);
    unsigned k2 = next_kt(), k3 = next_kt();
    {
        const char* a0 = abase + k2 * (NBLK * UNIT); const char* b0 = bbase + k2 * bstep;
        const char* a1 = abase + k3 * (NBLK * UNIT); const char* b1 = bbase + k3 * bstep;
        WAIT_V(0);
        STAGE(SB(0, 0), b0); STAGE(SA(0, 0), a0); STAGE(SB(0, 1), b0 + 16384); STAGE(SA(0, 1), a0 + 16384);
        if (wr == 1) BAR;
        WAIT_V(4); BAR;
        STAGE(SB(1, 0), b1); STAGE(SA(1, 0), a1); STAGE(SB(1, 1), b1 + 16384);
        WAIT_V(6); BAR;
    }
#pragma unroll 1
    for (int t = 0; t < nt; t += 2) {
        const char* a1 = abase + k3 * (NBLK * UNIT);
        if (t + 2 < nt) {
            k2 = next_kt(); k3 = next_kt();
            if (round3 > have) have = ready(round3);
        }
        const char* a2 = abase + k2 * (NBLK * UNIT); const char* b2 = bbase + k2 * bstep;
        const char* a3 = abase + k3 * (NBLK * UNIT); const char* b3 = bbase + k3 * bstep;
        LDB(B0, 0, 0); SCHED; LDA(At, 0, 0); STAGE(SA(1, 1), a1 + 16384);
        WAIT_L(8); BAR; WAIT_L(0); MMA(0, 0, At, B0); BAR; SCHED;
        LDB(B1, 0, 1); STAGE(SB(0, 0), b2);
        BAR; WAIT_L(0); MMA(0, 1, At, B1); BAR;
        LDA(At, 0, 1); STAGE(SA(0, 0), a2);
        BAR; WAIT_L(0); MMA(1, 0, At, B0); BAR; SCHED;
        STAGE(SB(0, 1), b2 + 16384);
        WAIT_V(6); BAR; MMA(1, 1, At, B1); BAR;
        LDB(B0, 1, 0); SCHED; LDA(At, 1, 0); STAGE(SA(0, 1), a2 + 16384);
        WAIT_L(8); BAR; WAIT_L(0); MMA(0, 0, At, B0); BAR; SCHED;
        LDB(B1, 1, 1); STAGE(SB(1, 0), b3);
        BAR; WAIT_L(0); MMA(0, 1, At, B1); BAR;
        LDA(At, 1, 1); STAGE(SA(1, 0), a3);
        BAR; WAIT_L(0); MMA(1, 0, At, B0); BAR; SCHED;
        STAGE(SB(1, 1), b3 + 16384);
        WAIT_V(6); BAR; MMA(1, 1, At, B1); BAR;
    }
    if (wr == 0) BAR;
    TS(21);
    int tz = threadIdx.x; asm volatile("" : "+v"(tz));
    const int lane2 = tz & 63;
    bool bad = LDSW(1) != 0u;
    const float qnan = __builtin_nanf("");
#define ACC(g, q) acc[(g) >> 1][((q) >> 1) & 1][((g) & 1) * 2 + ((q) >> 2)][(q) & 1]
    const unsigned foff = OFF_PFLAG + (unsigned)wid * 256u + (call & 1u) * 128u;
    const size_t poff = OFF_PART + (size_t)(call & 1u) * PART_BYTES + (size_t)slot * TILE_BYTES + wid * 32768 + lane2 * 16;
    if (!aux) {
        char* const dst = me + poff + (size_t)peer * 16 * TILE_BYTES;
        float mx = 0.f;
#pragma unroll
        for (int g = 0; g < 4; ++g)
#pragma unroll
            for (int q = 0; q < 8; ++q) {
                const f32x4 y = ACC(g, q);
                mx = fmaxf(fmaxf(mx, fabsf(y.x)), fabsf(y.y)); mx = fmaxf(fmaxf(mx, fabsf(y.z)), fabsf(y.w));
            }
        const unsigned emax = (wave_max(__builtin_bit_cast(unsigned, mx)) >> 23) & 255u;
        const unsigned base = emax > 15u ? emax - 15u : 0u;
        unsigned info = base | (bad ? 1u << 24 : 0u);
#pragma unroll
        for (int g = 0; g < 4; ++g) {
            u32x4 w[8], any = {0u, 0u, 0u, 0u};
#pragma unroll
            for (int q = 0; q < 8; ++q) { w[q] = __builtin_bit_cast(u32x4, ACC(g, q)) - (base << 23); any |= w[q]; }
            const u32x4 x7 = (w[7] & 0x07ffffffu) | ((w[7] >> 4) & 0x08000000u);
#pragma unroll
            for (int q = 0; q < 7; ++q) {
                asm volatile("s_waitcnt vmcnt(%0)" :: "i"(EXDEPTH) : "memory");
                store_wt(dst + g * 8192 + q * 1024, (w[q] & 0x87ffffffu) | ((x7 << (27 - 4 * q)) & 0x78000000u));
            }
            const bool need = ((any.x | any.y | any.z | any.w) & 0x78000000u) != 0u;
            const u64 mask = __ballot(need);
            if (need) {
                u32x4 pl = {0u, 0u, 0u, 0u};
#pragma unroll
                for (int q = 0; q < 8; ++q) pl |= ((w[q] >> 27) & 15u) << (4 * q);
                store_wt(dst + g * 8192 + 7168, pl);
            }
            if ((lane2 >> 2) == g) info |= ((unsigned)(mask >> ((lane2 & 3) * 16)) & 0xffffu) << 8;
        }
        WAIT_V(0);
        if (lane2 < 16) st_sys64(pws + foff + (unsigned)(rank * 16 + slot) * 2048u + lane2 * 8, call, info);
        TS(23);
        return;
    }
    const char* const src = me + poff + (size_t)rank * 16 * TILE_BYTES;
    float* const o = P.out + (size_t)((tile >> 3) * 256 + wr * 64 + (lane2 >> 4) * 4) * NFULL + peer * NS + tj * 256 + wc * 64 + (lane2 & 15) * 4;
    unsigned info16 = 0u;
    if (!remote) {
        info16 = wave_wait64<POLL>(me + foff + (unsigned)(peer * 16 + slot) * 2048u + (lane2 & 15) * 8, call, tmo, call);
        bad = bad || ld_agent(tmo) == call || (info16 >> 24) != 0u;
    }
#pragma unroll
    for (int g = 0; g < 4; ++g) {
        if (!remote) {
            u32x4 w[8];
            asm volatile("global_load_dwordx4 %0, %7, off sc0 sc1\n\t"
                         "global_load_dwordx4 %1, %7, off offset:1024 sc0 sc1\n\t"
                         "global_load_dwordx4 %2, %7, off offset:2048 sc0 sc1\n\t"
                         "global_load_dwordx4 %3, %7, off offset:3072 sc0 sc1\n\t"
                         "global_load_dwordx4 %4, %8, off sc0 sc1\n\t"
                         "global_load_dwordx4 %5, %8, off offset:1024 sc0 sc1\n\t"
                         "global_load_dwordx4 %6, %8, off offset:2048 sc0 sc1\n\ts_waitcnt vmcnt(0)"
                         : "=&v"(w[0]), "=&v"(w[1]), "=&v"(w[2]), "=&v"(w[3]), "=&v"(w[4]), "=&v"(w[5]), "=&v"(w[6])
                         : "v"(src + g * 8192), "v"(src + g * 8192 + 4096) : "memory");
            unpack28(w, __shfl(info16, g * 4 + (lane2 >> 4)), lane2, src + g * 8192 + 7168);
#pragma unroll
            for (int q = 0; q < 8; ++q) {
                f32x4 y = ACC(g, q) + __builtin_bit_cast(f32x4, w[q]);
#pragma unroll
                for (int c = 0; c < 4; ++c) y[c] = gelu(y[c]);
                ACC(g, q) = y;
            }
        }
#pragma unroll
        for (int mm = 0; mm < 2; ++mm)
#pragma unroll
            for (int r = 0; r < 4; ++r) {
                const int ai = g >> 1, m = (g & 1) * 2 + mm;
                f32x4 y = {acc[ai][0][m][0][r], acc[ai][0][m][1][r], acc[ai][1][m][0][r], acc[ai][1][m][1][r]};
                if (bad) y = (f32x4){qnan, qnan, qnan, qnan};
                if (remote) asm volatile("s_waitcnt vmcnt(%0)" :: "i"(F1DEPTH) : "memory");
                store_wt(o + (size_t)(ai * 128 + m * 16 + r) * NFULL, __builtin_bit_cast(u32x4, y));
            }
    }
    WAIT_V(0);
    TS(28);
    if (remote) {
        if (lane2 == 0) st_sys(me + OFF_OWNF + (unsigned)((peer * 16 + tile) * 8 + wid) * 64u, call);
        if (tile < 8) { tile += 8; goto again; }
    }
}

extern "C" void kernel_launch(int rank, int nranks, ncclComm_t comm,
                              hipStream_t stream,
                              void* const* d_in, const int* in_sizes, int n_in,
                              void* d_out, int out_size,
                              const int* in_dtypes, int out_dtype,
                              void* const* d_out_all,
                              void* const* d_ws_all, size_t ws_bytes) {
    (void)comm; (void)in_dtypes; (void)out_dtype; (void)d_out_all;
    if (nranks != NR || (unsigned)rank >= (unsigned)NR || n_in != 2 || ws_bytes < WS_NEED
        || in_sizes[0] != 4096 * KS || in_sizes[1] != KS * NFULL || out_size != 512 * NFULL) {
        fprintf(stderr, "kernel_launch: unexpected problem shape\n");
        return;
    }
    Params P = {};
    P.x = (const float*)d_in[0];
    P.w = (const float*)d_in[1];
    P.out = (float*)d_out;
    for (int p = 0; p < NR; ++p) P.ws[p] = (char*)d_ws_all[p];
    P.rank = rank;
    static const unsigned char port[NR][NR] = {
        {0, 2, 3, 6, 4, 7, 5, 1}, {1, 4, 5, 7, 6, 2, 3, 0}, {2, 0, 6, 3, 7, 1, 5, 4}, {3, 6, 0, 2, 7, 4, 1, 5},
        {4, 1, 7, 5, 6, 3, 0, 2}, {5, 7, 1, 4, 0, 6, 2, 3}, {6, 3, 2, 0, 1, 5, 4, 7}, {7, 5, 4, 1, 2, 0, 3, 6}};
    for (int c = 0; c < NR; ++c) P.peermap |= (unsigned)port[rank][c] << (4 * c);
    unsigned n = 0, sw = 1, sa = 0, sb = 0, so = 0;
    for (const char* c = ORDER; *c; ++c) {
        unsigned& s = *c == 'w' ? sw : *c == 'a' ? sa : *c == 'b' ? sb : so;
        const unsigned first = *c == 'w' ? 0 : *c == 'a' ? 8 : *c == 'b' ? 14 : 20, cnt = *c == 'w' ? 8 : *c == 'o' ? 2 : 6;
        for (unsigned i = 0; i < cnt && n < NPULL; ++i, ++n) P.order[n >> 2] |= (((s & 7u) << 5) | (first + i)) << ((n & 3u) * 8u);
        ++s;
    }
    if (n != NPULL || sw != 8 || sa != 8 || sb != 8 || so != 8) { fprintf(stderr, "kernel_launch: bad ORDER\n"); return; }
    gemm_rs_kernel<<<256, 512, 0, stream>>>(P);
    HIP_CHECK(hipGetLastError());
}
```

```cpp
#include <hip/hip_runtime.h>
#include <rccl/rccl.h>
#include <cstdio>

#define HIP_CHECK(expr) do { hipError_t _e = (expr); if (_e != hipSuccess) { \
    fprintf(stderr, "HIP error %s at %s:%d\n", hipGetErrorString(_e), __FILE__, __LINE__); \
    return; } } while (0)

#define LAS __attribute__((address_space(3)))
#define GAS __attribute__((address_space(1)))
typedef _Float16 f16x8 __attribute__((ext_vector_type(8)));
typedef float f32x4 __attribute__((ext_vector_type(4)));
typedef unsigned u32x4 __attribute__((ext_vector_type(4)));
typedef unsigned u32x3 __attribute__((ext_vector_type(3)));
typedef unsigned short u16x2 __attribute__((ext_vector_type(2)));
typedef unsigned long long u64;

#define ORDER "wawbwabwawabwabwawabbbbooooooo"
#define NC 5
#define WIN 7
#define FASTW 2
#define FASTW2 2
#define F1DEPTH 1
#define EXDEPTH 7
#define F1GO 8
#define CHORDER 1
#define POLL 8
#define LATE 48
#define STAMP 0

static_assert(WIN >= 4 && NC <= 7, "loader bookkeeping");

namespace {

constexpr int NR = 8;
constexpr int KS = 512;
constexpr int NFULL = 16384;
constexpr int NS = 2048;
constexpr int NBLK = 24;
constexpr unsigned UNIT = 32768;
constexpr unsigned TILE_BYTES = 262144;
constexpr unsigned NPULL = 168, NSLICE = NPULL * 8, NWORK = NSLICE + 512;
constexpr size_t CTRL_BYTES = 1u << 20;
constexpr size_t STAGE_BYTES = (size_t)64 * NBLK * UNIT;
constexpr size_t PART_BYTES = (size_t)NR * 16 * TILE_BYTES;
constexpr size_t WSET_BYTES = (size_t)NR * 64 * UNIT;
constexpr size_t XSET_BYTES = (size_t)128 * UNIT;
constexpr size_t OFF_STAGE = CTRL_BYTES, OFF_PART = OFF_STAGE + 2 * STAGE_BYTES, OFF_OWNB = OFF_PART + 2 * PART_BYTES,
                 OFF_EXPW = OFF_OWNB + WSET_BYTES, OFF_EXPX = OFF_EXPW + 2 * WSET_BYTES, WS_NEED = OFF_EXPX + 2 * XSET_BYTES;
constexpr unsigned OFF_TMO = 0, OFF_QUEUE = 2048  , OFF_CNT = 4096  ,
                   OFF_FLAG = 65536  , OFF_PFLAG = 262144  ,
                   OFF_OFLAG = 524288  , OFF_OWNF = 589824  ,
                   OFF_EFLAG = 655360  , OFF_STAMP = 851968;
constexpr u64 DEADLINE_TICKS = 50000000ull;
constexpr unsigned LDS_STAGE = 131072, LDS_BYTES = LDS_STAGE + 384;
constexpr unsigned M14 = 0x60006000u, MAXEXC = 128;

struct Params {
    const float* x;
    const float* w;
    char* me;
    int rank;
    unsigned peermap;
    float* out;
    char* ws[NR];
    unsigned order[(NPULL + 3) / 4];
};

__device__ __forceinline__ void store_wt(void* p, u32x4 v) {
    asm volatile("global_store_dwordx4 %0, %1, off sc0 sc1\n\ts_nop 1" :: "v"(p), "v"(v) : "memory");
}
__device__ __forceinline__ u32x4 load_sys(const char* p) {
    u32x4 v;
    asm volatile("global_load_dwordx4 %0, %1, off sc0 sc1\n\ts_waitcnt vmcnt(0)" : "=&v"(v) : "v"(p) : "memory");
    return v;
}
__device__ __forceinline__ unsigned pack2(float a, float b) {
    typedef _Float16 h2 __attribute__((ext_vector_type(2)));
    h2 h; h.x = (_Float16)a; h.y = (_Float16)b;
    return __builtin_bit_cast(unsigned, h);
}
__device__ __forceinline__ unsigned pk_add16(unsigned a, unsigned b) {
    return __builtin_bit_cast(unsigned, (u16x2)(__builtin_bit_cast(u16x2, a) + __builtin_bit_cast(u16x2, b)));
}
__device__ __forceinline__ float gelu(float y) {
    const float k1 = -2.0f * 0.7978845608028654f * 1.4426950408889634f, k2 = k1 * 0.044715f;
    const float e = __builtin_amdgcn_exp2f(y * __builtin_fmaf(y * y, k2, k1));
    return y * __builtin_amdgcn_rcpf(1.0f + e);
}
__device__ __forceinline__ unsigned wave_max(unsigned v) {
    v = max(v, (unsigned)__builtin_amdgcn_update_dpp(0, (int)v, 0x111, 0xf, 0xf, true));
    v = max(v, (unsigned)__builtin_amdgcn_update_dpp(0, (int)v, 0x112, 0xf, 0xf, true));
    v = max(v, (unsigned)__builtin_amdgcn_update_dpp(0, (int)v, 0x114, 0xf, 0xf, true));
    v = max(v, (unsigned)__builtin_amdgcn_update_dpp(0, (int)v, 0x118, 0xf, 0xf, true));
    const unsigned a = __builtin_amdgcn_readlane(v, 15), b = __builtin_amdgcn_readlane(v, 31),
                   c = __builtin_amdgcn_readlane(v, 47), d = __builtin_amdgcn_readlane(v, 63);
    return max(max(a, b), max(c, d));
}
__device__ __forceinline__ unsigned ld_sys(const char* p) {
    return __hip_atomic_load((const GAS unsigned*)p, __ATOMIC_RELAXED, __HIP_MEMORY_SCOPE_SYSTEM);
}
__device__ __forceinline__ u64 ld_sys64(const char* p) {
    return __hip_atomic_load((const GAS u64*)p, __ATOMIC_RELAXED, __HIP_MEMORY_SCOPE_SYSTEM);
}
__device__ __forceinline__ unsigned ld_agent(const char* p) {
    return __hip_atomic_load((const GAS unsigned*)p, __ATOMIC_RELAXED, __HIP_MEMORY_SCOPE_AGENT);
}
__device__ __forceinline__ void st_sys(char* p, unsigned v) {
    __hip_atomic_store((GAS unsigned*)p, v, __ATOMIC_RELAXED, __HIP_MEMORY_SCOPE_SYSTEM);
}
__device__ __forceinline__ void st_sys64(char* p, unsigned lo, unsigned hi) {
    __hip_atomic_store((GAS u64*)p, ((u64)hi << 32) | lo, __ATOMIC_RELAXED, __HIP_MEMORY_SCOPE_SYSTEM);
}
__device__ __forceinline__ unsigned add_agent(char* p, unsigned v) {
    return __hip_atomic_fetch_add((GAS unsigned*)p, v, __ATOMIC_RELAXED, __HIP_MEMORY_SCOPE_AGENT);
}
template <int SLEEP> __device__ __forceinline__ bool give_up(unsigned& n, u64 t0, char* tmo, unsigned call) {
    if ((++n & 15u) == 0u
        && ((unsigned)__builtin_amdgcn_readfirstlane(ld_agent(tmo)) == call || __builtin_amdgcn_s_memrealtime() - t0 > DEADLINE_TICKS)) {
        __hip_atomic_store((GAS unsigned*)tmo, call, __ATOMIC_RELAXED, __HIP_MEMORY_SCOPE_AGENT);
        return true;
    }
    __builtin_amdgcn_s_sleep(SLEEP);
    return false;
}
template <int SLEEP> __device__ __forceinline__ void wave_wait(const char* word, unsigned need, char* tmo, unsigned call) {
    const u64 t0 = __builtin_amdgcn_s_memrealtime();
    unsigned n = 0;
    while (!__all(ld_sys(word) >= need) && !give_up<SLEEP>(n, t0, tmo, call)) { }
    asm volatile("" ::: "memory");
}
template <int SLEEP> __device__ __forceinline__ unsigned wave_wait64(const char* word, unsigned need, char* tmo, unsigned call) {
    const u64 t0 = __builtin_amdgcn_s_memrealtime();
    unsigned n = 0;
    u64 v;
    do { v = ld_sys64(word); } while (!__all((unsigned)v >= need) && !give_up<SLEEP>(n, t0, tmo, call));
    asm volatile("" ::: "memory");
    return (unsigned)(v >> 32);
}

__device__ __forceinline__ void unpack28(u32x4 (&w)[8], unsigned info, int lane, const char* plane) {
    u32x4 x7 = {0u, 0u, 0u, 0u};
#pragma unroll
    for (int q = 0; q < 7; ++q) { x7 |= ((w[q] >> 27) & 15u) << (4 * q); w[q] &= 0x87ffffffu; }
    w[7] = (x7 & 0x07ffffffu) | ((x7 << 4) & 0x80000000u);
    if ((info >> (8 + (lane & 15))) & 1u) {
        const u32x4 pl = load_sys(plane);
#pragma unroll
        for (int q = 0; q < 8; ++q) w[q] |= ((pl >> (4 * q)) & 15u) << 27;
    }
#pragma unroll
    for (int q = 0; q < 8; ++q) w[q] += (info & 255u) << 23;
}

#define TS(k) do { } while (0)
#define WAIT_V(n) asm volatile("s_waitcnt vmcnt(" #n ")" ::: "memory")
#define WAIT_L(n) asm volatile("s_waitcnt lgkmcnt(" #n ")" ::: "memory")
#define BAR __builtin_amdgcn_s_barrier()
#define SCHED __builtin_amdgcn_sched_barrier(0)
#define LDSW(i) (*(volatile LAS unsigned*)(lds + LDS_STAGE + 4 * (i)))

}

__global__ __launch_bounds__(512, 2) void gemm_rs_kernel(Params P) {
    __shared__ __attribute__((aligned(16))) unsigned char shm[LDS_BYTES];
    LAS unsigned char* lds = (LAS unsigned char*)shm;
    const int tid = threadIdx.x, lane = tid & 63, wid = __builtin_amdgcn_readfirstlane(tid >> 6);
    const int hb = __builtin_amdgcn_readfirstlane(blockIdx.x);
    const float* px = P.x; const float* pw = P.w; char* me_ = P.me; int rank_ = P.rank; unsigned peermap = P.peermap;
    asm volatile("" : "+s"(px), "+s"(pw), "+s"(me_), "+s"(rank_), "+s"(peermap));
    const int rank = rank_ & (NR - 1);
    char* const me = (char*)(GAS char*)(size_t)me_;
    char* const tmo = me + OFF_TMO;
    int bid = (hb & ~7) | (__builtin_amdgcn_s_getreg((3 << 11) | 20) & 7);
    f32x4 v[8];
    auto first_item = [&](int b) -> unsigned {
        const int q = ((b >> 3) & 7) + 1, sk = ((peermap >> ((b & 7) * 4)) - rank) & (NR - 1);
        const int pl = q < 7 ? (rank + q + (q >= sk)) & (NR - 1) : ((rank + sk) & (NR - 1)) + (q & 8);
        const int lo = (b & 7) == 0 ? (b >> 6) * 16 + 7 + q : b < 64 ? 0 : pl;
        const int hi = b >= 192 ? (q - 1) * 16 + (b & 7) : b;
        return b >= 128 ? hi : lo;
    };
    auto load_item = [&](int b, unsigned item) {
        const bool f1 = b < 64 && (b & 7) != 0, isw = f1 || item >= 128u;
        const unsigned ws_ = f1 ? item : (item >> 3) & 7u, wj = (f1 ? b >> 3 : item) & 7u, wd = f1 ? (peermap >> ((b & 7) * 4)) & (NR - 1) : rank;
        const unsigned wo = (ws_ * 64u + (wid >> 2) * 32u) * NFULL + wd * NS + wj * 256u + (wid & 3) * 64u;
        const unsigned xo = (((item & 7u) * 2u + ((item >> 3) & 1u)) * 256u + wid * 32u) * KS + ((item >> 4) & 7u) * 64u;
        const GAS char* const base = (const GAS char*)(size_t)(isw ? pw + wo : px + xo);
        const unsigned lo = (lane >> 4) * (isw ? 16u * NFULL : 16u) + (lane & 15) * (isw ? 16u : 4u * KS);
        const unsigned sa = isw ? 64u * NFULL : 64u * KS, sb = isw ? 4u * NFULL : 64u;
#pragma unroll
        for (int i = 0; i < 8; ++i) v[i] = *(const GAS f32x4*)(base + ((i >> 2) * sa + (i & 3) * sb + lo));
    };
    unsigned t = 0;
    if (tid < 2) t = add_agent(me + OFF_CNT + (tid ? bid * 128 + 4 : hb * 128), 1u);
    if (bid >= 128 ? (bid >> 4) != 12 : (bid & 7) == 0) __builtin_amdgcn_s_sleep(LATE);
    load_item(bid, first_item(bid));
    char* lws = P.ws[lane >> 3];
    char* pws_ = P.ws[(peermap >> ((bid & 7) * 4)) & (NR - 1)];
    asm volatile("" ::: "memory");
    if (wid == 0) {
        const unsigned c = __builtin_amdgcn_readlane(t, 0) + 1u;
        unsigned old = __builtin_amdgcn_readlane(t, 1), r = bid;
        for (int tries = 0; old != c - 1u && tries < 65536; ++tries) {
            if (lane == 0) add_agent(me + OFF_CNT + r * 128u + 4u, ~0u);
            r = (r + 1u) & 255u;
            if (lane == 0) t = add_agent(me + OFF_CNT + r * 128u + 4u, 1u);
            old = __builtin_amdgcn_readfirstlane(t);
        }
        LDSW(32 + lane) = 0u;
        if (lane < 16) LDSW(16 + lane) = 0u;
        if (lane == 0) { LDSW(0) = c; LDSW(1) = 0u; LDSW(12) = r; }
    }
    asm volatile("s_waitcnt lgkmcnt(0)" : "+s"(pws_) :: "memory"); BAR; asm volatile("" ::: "memory");
    const unsigned call = __builtin_amdgcn_readfirstlane(LDSW(0));
    {
        const int got = __builtin_amdgcn_readfirstlane(LDSW(12));
        if (got != bid) { bid = got; load_item(bid, first_item(bid)); pws_ = P.ws[(peermap >> ((bid & 7) * 4)) & (NR - 1)]; }
    }
    const int peer = (peermap >> ((bid & 7) * 4)) & (NR - 1);
    const int slot = (bid >> 3) & 15;
    const bool aux = bid < 128, remote = (bid & 7) != 0;
    const int skip = remote ? (peer - rank) & (NR - 1) : NR;
    char* const pws = (char*)(GAS char*)(size_t)pws_;

    const bool engine = aux && remote && slot >= 8;
    const unsigned pu = (unsigned)(rank * 8 * NBLK + 8) + (first_item(bid) & 7u) * 2u + ((first_item(bid) >> 3) & 1u);
    if (engine) {
        char* const rdst = pws + OFF_STAGE + (call & 1u) * STAGE_BYTES + (size_t)pu * UNIT + wid * 4096 + lane * 16;
#pragma unroll
        for (int f = 0; f < 4; ++f) {
            u32x4 o;
            o.x = pack2(v[2 * f][0], v[2 * f][1]); o.y = pack2(v[2 * f][2], v[2 * f][3]);
            o.z = pack2(v[2 * f + 1][0], v[2 * f + 1][1]); o.w = pack2(v[2 * f + 1][2], v[2 * f + 1][3]);
            if (f > 1 + (wid <= FASTW2) || (f == 1 && wid > FASTW)) WAIT_V(0);
            store_wt(rdst + f * 1024, o);
        }
    } else {
        const bool f1 = aux && remote;
        unsigned item = first_item(bid);
        const int fd = f1 ? peer : lane >> 3;
        char* const fws = f1 ? pws : lws;
#pragma unroll 1
        for (unsigned n = 0;; ++n) {
            const bool isw = f1 || item >= 128u;
            const int s = f1 ? item : isw ? (item - 128u) >> 3 : item >> 4;
            const int j = (f1 ? slot : item) & 7, rb = (item & 7) * 2 + ((item >> 3) & 1);
            const unsigned par = call & 1u;
            u32x4 o[4];
            unsigned off, d1, d2;
            if (isw) {
                off = ((wid & 3) * 4 + (wid >> 2)) * 1024; d1 = 2048; d2 = 16384;
#pragma unroll
                for (int f = 0; f < 4; ++f) {
                    o[f].x = pack2(v[0][f], v[1][f]); o[f].y = pack2(v[2][f], v[3][f]);
                    o[f].z = pack2(v[4][f], v[5][f]); o[f].w = pack2(v[6][f], v[7][f]);
                }
            } else {
                off = wid * 4096; d1 = 1024; d2 = 2048;
#pragma unroll
                for (int f = 0; f < 4; ++f) {
                    o[f].x = pack2(v[2 * f][0], v[2 * f][1]); o[f].y = pack2(v[2 * f][2], v[2 * f][3]);
                    o[f].z = pack2(v[2 * f + 1][0], v[2 * f + 1][1]); o[f].w = pack2(v[2 * f + 1][2], v[2 * f + 1][3]);
                }
            }
            off += lane * 16;
            const unsigned u = (unsigned)((rank * 8 + s) * NBLK + (isw ? j : 8 + rb));
            const unsigned wu = (unsigned)((peer * 8 + s) * 8 + j);
            const bool expo = f1 || (!isw && (rb >> 1) != rank);
            if (expo) {
                float mx = 0.f;
#pragma unroll
                for (int i = 0; i < 8; ++i) { mx = fmaxf(fmaxf(mx, fabsf(v[i].x)), fabsf(v[i].y)); mx = fmaxf(fmaxf(mx, fabsf(v[i].z)), fabsf(v[i].w)); }
                const unsigned lmax = pack2(mx, 0.f) >> 10;
                unsigned top = wave_max(lmax);
                if (__builtin_popcountll(__ballot(lmax == top)) <= 8) --top;
                const unsigned base = (int)top > 7 ? top - 7u : 0u;
                const unsigned sub = (0x10000u - (base << 10)) * 0x10001u & (base ? 0xffffffffu : 0u);
                unsigned w[16], pl[2] = {0u, 0u};
#pragma unroll
                for (int i = 0; i < 16; ++i) {
                    w[i] = pk_add16(o[i >> 2][i & 3], sub);
                    pl[i >> 3] |= (i & 7) < 7 ? (w[i] & M14) >> (13 - 2 * (i & 7)) : (w[i] & M14) << 1;
                }
                u32x4 e[4];
#pragma unroll
                for (int i = 0; i < 14; ++i) {
                    const unsigned c = w[14 + i / 7], y = (c & 0x1fff1fffu) | ((c >> 2) & 0x20002000u);
                    e[i >> 2][i & 3] = (w[i] & ~M14) | ((y << (13 - 2 * (i % 7))) & M14);
                }
                char* const ed = me + (f1 ? OFF_EXPW + par * WSET_BYTES + (size_t)wu * UNIT : OFF_EXPX + par * XSET_BYTES + (size_t)(s * 16 + rb) * UNIT) + wid * 4096;
                store_wt(ed + lane * 16, e[0]); store_wt(ed + 1024 + lane * 16, e[1]); store_wt(ed + 2048 + lane * 16, e[2]);
                st_sys64(ed + 3072 + lane * 8, e[3].x, e[3].y);
                u64 m = ((u64)pl[1] << 32) | pl[0], has;
                unsigned nexc = 0;
                LAS unsigned char* const list = lds + wid * 256;
                while ((has = __ballot(m != 0ull)) != 0ull && nexc <= MAXEXC) {
                    const unsigned idx = nexc + __builtin_amdgcn_mbcnt_hi((unsigned)(has >> 32), __builtin_amdgcn_mbcnt_lo((unsigned)has, 0u));
                    const unsigned pos = __builtin_ctzll(m | (1ull << 63)) & 62u;
                    if (m != 0ull && idx < MAXEXC)
                        *(volatile LAS unsigned short*)(list + idx * 2u) = (lane << 7) | (pos << 1) | ((unsigned)(m >> pos) & 3u);
                    m &= ~(3ull << pos);
                    nexc += __builtin_popcountll(has);
                }
                if (nexc > MAXEXC) { nexc = 255u; st_sys64(ed + 3584 + lane * 8, pl[0], pl[1]); }
                else if ((unsigned)lane * 8u < nexc) store_wt(ed + 3584 + lane * 16, *(volatile LAS u32x4*)(list + lane * 16));
                if (lane == 0) LDSW(4 + wid) = base | (nexc << 8);
            }
            char* const ldst = me + (f1 ? OFF_OWNB + (size_t)wu * UNIT : OFF_STAGE + par * STAGE_BYTES + (size_t)u * UNIT) + off;
            store_wt(ldst, o[0]); store_wt(ldst + d1, o[1]); store_wt(ldst + d2, o[2]); store_wt(ldst + d2 + d1, o[3]);
            WAIT_V(0);
            __syncthreads();
            if (wid == 0) {
                if (expo && (f1 ? lane < 8 : fd != rank))
                    st_sys64(fws + OFF_EFLAG + (par * 64u * NBLK + u) * 64u + (lane & 7) * 8, call, LDSW(4 + (lane & 7)));
                if (lane == 0) st_sys(me + (f1 ? OFF_OFLAG + wu * 128u : OFF_FLAG + u * 128u), call * 8u);
            }
            TS(1 + (n < 8 ? n : 8));
            if (f1 ? item == 7u : bid < 208 || n == 1u) break;
            item += f1 ? 1u : 8u;
            load_item(bid, item);
            __syncthreads();
        }
    }
    int bid2 = bid;
    asm volatile("" : "+s"(bid2));
    if ((bid2 >> 6) == 1 && (bid2 & 7) != 0) {
        const unsigned par = call & 1u;
        const char* const eflag = me + OFF_EFLAG + (par * 64u + peer * 8u) * (NBLK * 64u);
        auto unit_of = [&](unsigned slice) -> unsigned {
            const unsigned it = min(slice >> 3, NPULL - 1u), e = (P.order[it >> 2] >> ((it & 3u) * 8u)) & 255u, code = e & 31u;
            const int odd = code >= 14u, q = code - (odd ? 14u : 8u);
            const unsigned blk = code < 8u ? code : code >= 20u ? 8u + rank * 2 + (code - 20u)
                                                                 : 8u + ((rank + q + 1 + (q + 1 >= skip)) & (NR - 1)) * 2 + odd;
            return (e >> 5) * NBLK + blk;
        };
        auto src_of = [&](unsigned slice, unsigned su) -> const char* {
            const unsigned s = su / NBLK, blk = su - s * NBLK;
            return pws + (blk < 8u ? OFF_EXPW + par * WSET_BYTES + (size_t)((rank * 8 + s) * 8 + blk) * UNIT
                                   : OFF_EXPX + par * XSET_BYTES + (size_t)(s * 16 + blk - 8) * UNIT) + (slice & 7u) * 4096u;
        };
        auto publish_push = [&]() {
            WAIT_V(0);
            __builtin_amdgcn_s_waitcnt(0x0F70);
            if (lane == 0)
                __hip_atomic_fetch_add((GAS unsigned*)(pws + OFF_FLAG + pu * 128u), 1u, __ATOMIC_RELAXED, __HIP_MEMORY_SCOPE_SYSTEM);
        };
        if (wid == 0) {
            publish_push();
            TS(1);
            __builtin_amdgcn_s_setprio(3);
            const unsigned lbase = (unsigned)(size_t)lds, landed = lbase + (lane ? 98304 + lane * 4 : LDS_STAGE + 68);
            unsigned head = 0, alive = NC, idle = 3;
            int known = -(WIN - 1);
            u32x4 d = *(volatile LAS u32x4*)(lds + LDS_STAGE + 128);
#define PIECE(a, off) "s_waitcnt vmcnt(%7)\n\tds_max_i32 %4, %1\n\tv_add_u32 %1, 1, %1\n\tglobal_load_lds_dwordx4 " a ", off offset:" #off " sc0 sc1\n\t"
            while (alive) {
                if ((unsigned)__builtin_amdgcn_readfirstlane(d.w) != head + 1u) {
                    if (idle == 1u) asm volatile("s_waitcnt vmcnt(3)\n\tds_max_i32 %0, %1" :: "v"(landed), "v"(known + (WIN - 4)) : "memory");
                    if (idle == 2u) asm volatile("s_waitcnt vmcnt(0)\n\tds_max_i32 %0, %1" :: "v"(landed), "v"(known + (WIN - 1)) : "memory");
                    ++idle;
                    d = *(volatile LAS u32x4*)(lds + LDS_STAGE + 128 + (head & 15u) * 16u);
                    continue;
                }
                ++head;
                const unsigned meta = __builtin_amdgcn_readfirstlane(d.z), dst = lbase + (meta >> 10) * 8192u;
                const char* src = (const char*)(((u64)d.y << 32) | d.x) + lane * 16;
                d = *(volatile LAS u32x4*)(lds + LDS_STAGE + 128 + (head & 15u) * 16u);
                unsigned keep;
                if ((meta & 7u) == 4u) {
                    const u64 mask = ~0ull >> (64u - ((meta >> 3) & 127u));
                    asm volatile("s_mov_b32 %0, m0\n\ts_mov_b32 m0, %3\n\t" PIECE("%2", 0) PIECE("%2", 1024) PIECE("%2", 2048)
                                 "s_waitcnt vmcnt(%7)\n\tds_max_i32 %4, %1\n\tv_add_u32 %1, 1, %1\n\ts_mov_b64 exec, %5\n\t"
                                 "global_load_lds_dwordx4 %2, off offset:3072 sc0 sc1\n\ts_mov_b64 exec, -1\n\ts_mov_b32 m0, %0"
                                 : "=&s"(keep), "+v"(known) : "v"(src), "s"(dst), "v"(landed), "s"(mask), "s"(dst), "i"(WIN - 1) : "memory");
                    idle = 0u;
                } else if ((meta & 7u) == 7u) {
                    asm volatile("s_mov_b32 %0, m0\n\ts_mov_b32 m0, %3\n\t" PIECE("%2", 0) PIECE("%2", 1024) PIECE("%2", 2048) PIECE("%2", 3072)
                                 "s_mov_b32 m0, %6\n\t" PIECE("%5", 0) PIECE("%5", 1024) PIECE("%5", 2048) "s_mov_b32 m0, %0"
                                 : "=&s"(keep), "+v"(known) : "v"(src), "s"(dst), "v"(landed), "v"(src + 4096), "s"(dst + 4096u), "i"(WIN - 1) : "memory");
                    idle = 0u;
                } else {
                    --alive;
                }
            }
        } else if (wid <= NC) {
            const unsigned me8 = wid - 1;
            const LAS unsigned char* const buf = lds + me8 * 8192u;
            const float qnan = __builtin_nanf("");
            auto chunk_of = [&](unsigned t) -> unsigned {
                const unsigned i = max(min(t, NWORK - 1u), NSLICE) - NSLICE;
                return CHORDER ? ((i >> 8) * 64u + (i & 63u)) * 4u + ((i >> 6) & 3u) : i;
            };
            auto flag_of = [&](unsigned t, unsigned u) -> const char* {
                const unsigned i = chunk_of(t);
                return t < NSLICE ? eflag + u * 64u + (t & 7u) * 8u
                                  : me + OFF_PFLAG + (peer * 128u + (i >> 2)) * 256u + par * 128u + (i & 3u) * 32u + (lane >> 4) * 8;
            };
            auto rsrc_of = [&](unsigned t, unsigned u) -> const char* {
                return t < NSLICE ? src_of(t, u) : pws + OFF_PART + par * PART_BYTES + (size_t)rank * 16 * TILE_BYTES + (size_t)chunk_of(t) * 8192u;
            };
            int rend = 0;
            auto post = [&](const char* src, unsigned cnt, unsigned last) {
                unsigned k = 0;
                if (lane == 0) {
                    k = __hip_atomic_fetch_add((LAS unsigned*)(lds + LDS_STAGE + 64), 0x10000u + cnt, __ATOMIC_RELAXED, __HIP_MEMORY_SCOPE_WORKGROUP);
                    const unsigned e = 32u + ((k >> 16) & 15u) * 4u;
                    LDSW(e) = (unsigned)(u64)src; LDSW(e + 1) = (unsigned)((u64)src >> 32);
                    LDSW(e + 2) = cnt | (last << 3) | (me8 << 10);
                    LDSW(e + 3) = (k >> 16) + 1u;
                }
                rend = (__builtin_amdgcn_readfirstlane(k) & 0xffffu) + cnt;
            };
            auto post_item = [&](unsigned t2, unsigned u2, unsigned inf) {
                if (t2 < NSLICE) post(src_of(t2, u2), 4u, 32u + ((((inf >> 8) & 255u) + 7u) >> 3)); else post(rsrc_of(t2, u2), 7u, 64u);
            };
            unsigned ta = (slot - 8) * NC + me8, tb = ta + 8u * NC;
            unsigned ua = unit_of(ta), ub = unit_of(tb);
            const u64 f0 = ld_sys64(flag_of(ta, ua));
            unsigned info = (unsigned)(f0 >> 32);
            const bool now = __all((unsigned)f0 >= call);
            if (now) post_item(ta, ua, info);
            publish_push();
            if (!now) { info = wave_wait64<2>(flag_of(ta, ua), call, tmo, call); post_item(ta, ua, info); }
            char* pend = nullptr;
#pragma unroll 1
            for (;;) {
                const bool isp = ta >= NSLICE;
                const unsigned ci = chunk_of(ta), ct = ci >> 5, cw = (ci >> 2) & 7u, cg = ci & 3u;
                const unsigned ooff = (((ct >> 3) * 256 + (cg >> 1) * 128 + (cw >> 2) * 64 + (cg & 1u) * 32 + (lane >> 4) * 4) * NFULL
                                       + peer * NS + (ct & 7u) * 256 + (cw & 3u) * 64 + (lane & 15) * 4) * 4u;
                const char* const fb = flag_of(tb, ub);
                const char* const ofb = me + OFF_OWNF + (peer * 128u + (chunk_of(tb) >> 2)) * 64u;
                u64 fl, ex;
                unsigned ofl, tm, tk;
                u32x4 own[8] = {};
#define SMALL_LOADS(F, O, T, K, X, AF, AO, AT, AQ, ONE) \
                "global_load_dwordx2 " F ", " AF ", off sc0 sc1\n\t" \
                "global_load_dword " O ", " AO ", off sc0 sc1\n\t" \
                "global_load_dword " T ", " AT ", off sc1\n\t" \
                "s_mov_b64 " X ", exec\n\ts_mov_b64 exec, 1\n\t" \
                "global_atomic_add " K ", " AQ ", " ONE ", off sc0\n\t" \
                "s_mov_b64 exec, " X "\n\ts_waitcnt vmcnt(0)"
                if (isp) {
                    unsigned vo = ooff;
                    asm volatile("s_nop 4\n\tglobal_load_dwordx4 %0, %13, %14 sc0 sc1\n\tv_add_u32 %13, 0x10000, %13\n\t"
                                 "global_load_dwordx4 %1, %13, %14 sc0 sc1\n\tv_add_u32 %13, 0x10000, %13\n\t"
                                 "global_load_dwordx4 %2, %13, %14 sc0 sc1\n\tv_add_u32 %13, 0x10000, %13\n\t"
                                 "global_load_dwordx4 %3, %13, %14 sc0 sc1\n\tv_add_u32 %13, 0xd0000, %13\n\t"
                                 "global_load_dwordx4 %4, %13, %14 sc0 sc1\n\tv_add_u32 %13, 0x10000, %13\n\t"
                                 "global_load_dwordx4 %5, %13, %14 sc0 sc1\n\tv_add_u32 %13, 0x10000, %13\n\t"
                                 "global_load_dwordx4 %6, %13, %14 sc0 sc1\n\tv_add_u32 %13, 0x10000, %13\n\t"
                                 "global_load_dwordx4 %7, %13, %14 sc0 sc1\n\t"
                                 SMALL_LOADS("%8", "%9", "%10", "%11", "%12", "%15", "%16", "%17", "%18", "%19")
                                 : "=&v"(own[0]), "=&v"(own[1]), "=&v"(own[2]), "=&v"(own[3]), "=&v"(own[4]), "=&v"(own[5]), "=&v"(own[6]), "=&v"(own[7]),
                                   "=&v"(fl), "=&v"(ofl), "=&v"(tm), "=&v"(tk), "=&s"(ex), "+v"(vo)
                                 : "s"(P.out), "v"(fb), "v"(ofb), "v"(tmo), "v"(me + OFF_QUEUE + peer * 128), "v"(1u) : "memory");
                } else {
                    asm volatile(SMALL_LOADS("%0", "%1", "%2", "%3", "%4", "%5", "%6", "%7", "%8", "%9")
                                 : "=&v"(fl), "=&v"(ofl), "=&v"(tm), "=&v"(tk), "=&s"(ex)
                                 : "v"(fb), "v"(ofb), "v"(tmo), "v"(me + OFF_QUEUE + peer * 128), "v"(1u) : "memory");
                }
                if (pend && lane == 0) add_agent(pend, 1u);
                pend = nullptr;
                const bool bad = tm == call || (info >> 24) != 0u;
                tk = __builtin_amdgcn_readfirstlane(tk) - (call - 1u) * NWORK + 16u * NC;
                const unsigned uc = unit_of(tk);
                while ((int)__builtin_amdgcn_readfirstlane(LDSW(17)) < rend) __builtin_amdgcn_s_sleep(1);
                asm volatile("" ::: "memory");
                u32x4 w[8];
#pragma unroll
                for (int q = 0; q < 7; ++q) w[q] = *(const LAS u32x4*)(buf + q * 1024 + lane * 16);
                if (!isp) {
                    typedef unsigned u32x2 __attribute__((ext_vector_type(2)));
                    const u32x2 t2 = *(const LAS u32x2*)(buf + 3072 + lane * 8), p2 = *(const LAS u32x2*)(buf + 3584 + lane * 8);
                    w[3].x = t2.x; w[3].y = t2.y; w[4].x = p2.x; w[4].y = p2.y;
                    w[4].z = *(const LAS unsigned short*)(buf + 3584 + lane * 2); w[4].w = *(const LAS unsigned short*)(buf + 3712 + lane * 2);
                }
                asm volatile("s_waitcnt lgkmcnt(0)" : "+v"(w[0]), "+v"(w[1]), "+v"(w[2]), "+v"(w[3]), "+v"(w[4]), "+v"(w[5]), "+v"(w[6]) :: "memory");
                const bool more = tb < NWORK, early = more && __all((unsigned)fl >= call);
                if (early) post_item(tb, ub, (unsigned)(fl >> 32));
                if (!isp) {
                    const unsigned nexc = (info >> 8) & 255u;
                    unsigned pl[2] = {w[4].x, w[4].y};
                    if (nexc <= MAXEXC) {
                        volatile LAS unsigned* const sc = (volatile LAS unsigned*)(lds + 65536 + me8 * 512u);
                        sc[lane * 2] = 0u; sc[lane * 2 + 1] = 0u;
#pragma unroll
                        for (int h = 0; h < 2; ++h) {
                            const unsigned en = h ? w[4].w : w[4].z;
                            if ((unsigned)lane + 64u * h < nexc)
                                __hip_atomic_fetch_or((LAS unsigned*)sc + ((en >> 6) & 127u), (en & 3u) << ((en >> 1) & 30u), __ATOMIC_RELAXED, __HIP_MEMORY_SCOPE_WORKGROUP);
                        }
                        pl[0] = sc[lane * 2]; pl[1] = sc[lane * 2 + 1];
                    }
                    unsigned y[2] = {0u, 0u};
#pragma unroll
                    for (int i = 0; i < 14; ++i) y[i / 7] |= (w[i >> 2][i & 3] & M14) >> (13 - 2 * (i % 7));
                    w[3].z = (y[0] & 0x1fff1fffu) | ((y[0] & 0x20002000u) << 2);
                    w[3].w = (y[1] & 0x1fff1fffu) | ((y[1] & 0x20002000u) << 2);
#pragma unroll
                    for (int i = 0; i < 16; ++i)
                        w[i >> 2][i & 3] = (w[i >> 2][i & 3] & ~M14) | (((i & 7) < 7 ? pl[i >> 3] << (13 - 2 * (i & 7)) : pl[i >> 3] >> 1) & M14);
                    const unsigned add = ((info & 31u) << 10) * 0x10001u;
#pragma unroll
                    for (int i = 0; i < 16; ++i) w[i >> 2][i & 3] = pk_add16(w[i >> 2][i & 3], add);
                    const bool isw = ua % NBLK < 8u;
                    const unsigned u = (unsigned)(peer * 8 * NBLK) + ua, w8 = ta & 7u;
                    const unsigned d1 = isw ? 2048u : 1024u, d2 = isw ? 16384u : 2048u;
                    char* const dst = me + OFF_STAGE + par * STAGE_BYTES + (size_t)u * UNIT
                                    + (isw ? ((w8 & 3u) * 4u + (w8 >> 2)) * 1024u : w8 * 4096u) + lane * 16;
                    store_wt(dst, w[0]); store_wt(dst + d1, w[1]); store_wt(dst + d2, w[2]); store_wt(dst + d2 + d1, w[3]);
                    pend = me + OFF_FLAG + u * 128u;
                } else {
                    unpack28(w, info, lane, rsrc_of(ta, ua) + 7168 + lane * 16);
#pragma unroll
                    for (int h = 0; h < 2; ++h)
#pragma unroll
                        for (int r = 0; r < 4; ++r) {
                            const f32x4 a = __builtin_bit_cast(f32x4, own[h * 4 + r]);
                            f32x4 y;
#pragma unroll
                            for (int c = 0; c < 4; ++c) y[c] = gelu(a[c] + __uint_as_float(w[h * 4 + c][r]));
                            if (bad) y = (f32x4){qnan, qnan, qnan, qnan};
                            store_wt((char*)P.out + ooff + (h * 16 + r) * (NFULL * 4), __builtin_bit_cast(u32x4, y));
                        }
                }
                if (!more) break;
                info = (unsigned)(fl >> 32);
                if (!early) {
                    WAIT_V(0);
                    if (pend && lane == 0) add_agent(pend, 1u);
                    pend = nullptr;
                    info = wave_wait64<2>(fb, call, tmo, call);
                    post_item(tb, ub, info);
                }
                if (tb >= NSLICE && !__all(ofl >= call)) wave_wait<2>(ofb, call, tmo, call);
                ta = tb; ua = ub; tb = tk; ub = uc;
            }
            post(nullptr, 0u, 0u);
            WAIT_V(0);
            if (pend && lane == 0) add_agent(pend, 1u);
        } else {
            publish_push();
        }
        WAIT_V(0);
        return;
    }

    const bool ownb = aux && remote;
    int tile = slot;
again:
    const int ti = (aux ? rank : peer) * 2 + (tile >> 3), tj = slot & 7;
    const int nsrc = remote ? 6 : 7;
    const int rfirst = aux && !remote ? 4 : -1, rlast = ownb ? -1 : !aux && !remote ? 3 : 7;
    const int nt = remote ? (aux ? 8 : 56) : (aux ? 28 : 36);
    const int wr = wid >> 2, wc = wid & 3;
    const char* const stage = me + OFF_STAGE + (size_t)(call & 1u) * STAGE_BYTES;
    const char* const abase = stage + (unsigned)(8 + ti) * UNIT;
    const char* const bbase = ownb ? me + OFF_OWNB + (size_t)(((peer - rank) * 64 + tj) * (int)UNIT) : stage + (unsigned)tj * UNIT;
    const unsigned bstep = ownb ? 8u * UNIT : NBLK * UNIT;
    const unsigned voff = (unsigned)tid * 16u;
    const unsigned ldsw = (unsigned)wid * 1024u;
    const unsigned aoff = (unsigned)wr * 8192u + (unsigned)lane * 16u, boff = (unsigned)wc * 4096u + (unsigned)lane * 16u;

#define SA(b, h) (((b) * 2 + (h)) * 16384)
#define SB(b, h) ((4 + (b) * 2 + (h)) * 16384)
#define STAGE(bufoff, gbase) do { _Pragma("unroll") for (int _i = 0; _i < 2; ++_i) \
        __builtin_amdgcn_global_load_lds((const GAS unsigned*)((gbase) + _i * 8192 + voff), (LAS unsigned*)(lds + (bufoff) + ldsw + _i * 8192), 16, 0, 17); } while (0)
#define LDA(dst, b, h) do { _Pragma("unroll") for (int m = 0; m < 4; ++m) _Pragma("unroll") for (int k = 0; k < 2; ++k) \
        dst[m][k] = *(const LAS f16x8*)(lds + SA(b, h) + aoff + m * 2048 + k * 1024); } while (0)
#define LDB(dst, b, h) do { _Pragma("unroll") for (int n = 0; n < 2; ++n) _Pragma("unroll") for (int k = 0; k < 2; ++k) \
        dst[n][k] = *(const LAS f16x8*)(lds + SB(b, h) + boff + n * 2048 + k * 1024); } while (0)
#define MMA(ai, bj, At, Bt) do { __builtin_amdgcn_s_setprio(1); \
        _Pragma("unroll") for (int m = 0; m < 4; ++m) _Pragma("unroll") for (int n = 0; n < 2; ++n) _Pragma("unroll") for (int k = 0; k < 2; ++k) \
        acc[ai][bj][m][n] = __builtin_amdgcn_mfma_f32_16x16x32_f16(At[m][k], Bt[n][k], acc[ai][bj][m][n], 0, 0, 0); \
        __builtin_amdgcn_s_setprio(0); } while (0)

    int cs = rfirst, cq = 0, round3 = rfirst;
    auto next_kt = [&]() -> unsigned {
        const int rel = cq + 1 + (cq + 1 >= skip);
        const unsigned kt = cs < 0 ? (unsigned)(rank * 8 + cq) : (unsigned)(((rank + rel) & (NR - 1)) * 8 + cs);
        round3 = cs;
        if (++cq == (cs < 0 ? 8 : nsrc)) { cq = 0; ++cs; }
        return __builtin_amdgcn_readfirstlane(kt);
    };
    auto ready = [&](int s) -> int {
        if (wid == 0) {
            int lz = lane; asm volatile("" : "+v"(lz));
            const int e = (lz >> 1) & 7, q = e < nsrc ? e : 0, sl = min(s + (lz >> 4), rlast);
            const int kt = sl < 0 ? rank * 8 + e : ((rank + q + 1 + (q + 1 >= skip)) & (NR - 1)) * 8 + sl;
            unsigned fo = OFF_FLAG + (unsigned)(kt * NBLK + tj + (lz & 1) * (8 + ti - tj)) * 128u;
            if (ownb && !(lz & 1)) fo = OFF_OFLAG + (unsigned)((peer * 8 + e) * 8 + tj) * 128u;
            u64 miss = ~__ballot(ld_sys(me + fo) >= call * 8u);
            if (miss & 0xffffull) {
                wave_wait<POLL>(me + __shfl(fo, lz & 15), call * 8u, tmo, call);
                miss = ~0ull;
            }
            miss >>= 16;
            const int got = min(s + (miss ? __builtin_ctzll(miss) >> 4 : 3), rlast);
            LDSW(3) = got;
            if (got == rlast)
                LDSW(1) = (unsigned)__builtin_amdgcn_readfirstlane(ld_agent(tmo)) == call;
            WAIT_L(0);
            TS(s < 0 ? 30 : 13 + s);
        }
        asm volatile("" ::: "memory"); BAR; asm volatile("" ::: "memory");
        return __builtin_amdgcn_readfirstlane(LDSW(3));
    };

    f32x4 acc[2][2][4][2];
#pragma unroll
    for (int a = 0; a < 2; ++a)
#pragma unroll
        for (int b = 0; b < 2; ++b)
#pragma unroll
            for (int m = 0; m < 4; ++m)
#pragma unroll
                for (int n = 0; n < 2; ++n) acc[a][b][m][n] = (f32x4){0.f, 0.f, 0.f, 0.f};
    f16x8 At[4][2], B0[2][2], B1[2][2];

    int have = ready(rfirst);
    unsigned k2 = next_kt(), k3 = next_kt();
    {
        const char* a0 = abase + k2 * (NBLK * UNIT); const char* b0 = bbase + k2 * bstep;
        const char* a1 = abase + k3 * (NBLK * UNIT); const char* b1 = bbase + k3 * bstep;
        WAIT_V(0);
        STAGE(SB(0, 0), b0); STAGE(SA(0, 0), a0); STAGE(SB(0, 1), b0 + 16384); STAGE(SA(0, 1), a0 + 16384);
        if (wr == 1) BAR;
        WAIT_V(4); BAR;
        STAGE(SB(1, 0), b1); STAGE(SA(1, 0), a1); STAGE(SB(1, 1), b1 + 16384);
        WAIT_V(6); BAR;
    }
#pragma unroll 1
    for (int t = 0; t < nt; t += 2) {
        const char* a1 = abase + k3 * (NBLK * UNIT);
        if (t + 2 < nt) {
            k2 = next_kt(); k3 = next_kt();
            if (round3 > have) have = ready(round3);
        }
        const char* a2 = abase + k2 * (NBLK * UNIT); const char* b2 = bbase + k2 * bstep;
        const char* a3 = abase + k3 * (NBLK * UNIT); const char* b3 = bbase + k3 * bstep;
        LDB(B0, 0, 0); SCHED; LDA(At, 0, 0); STAGE(SA(1, 1), a1 + 16384);
        WAIT_L(8); BAR; WAIT_L(0); MMA(0, 0, At, B0); BAR; SCHED;
        LDB(B1, 0, 1); STAGE(SB(0, 0), b2);
        BAR; WAIT_L(0); MMA(0, 1, At, B1); BAR;
        LDA(At, 0, 1); STAGE(SA(0, 0), a2);
        BAR; WAIT_L(0); MMA(1, 0, At, B0); BAR; SCHED;
        STAGE(SB(0, 1), b2 + 16384);
        WAIT_V(6); BAR; MMA(1, 1, At, B1); BAR;
        LDB(B0, 1, 0); SCHED; LDA(At, 1, 0); STAGE(SA(0, 1), a2 + 16384);
        WAIT_L(8); BAR; WAIT_L(0); MMA(0, 0, At, B0); BAR; SCHED;
        LDB(B1, 1, 1); STAGE(SB(1, 0), b3);
        BAR; WAIT_L(0); MMA(0, 1, At, B1); BAR;
        LDA(At, 1, 1); STAGE(SA(1, 0), a3);
        BAR; WAIT_L(0); MMA(1, 0, At, B0); BAR; SCHED;
        STAGE(SB(1, 1), b3 + 16384);
        WAIT_V(6); BAR; MMA(1, 1, At, B1); BAR;
    }
    if (wr == 0) BAR;
    TS(21);
    int tz = threadIdx.x; asm volatile("" : "+v"(tz));
    const int lane2 = tz & 63;
    bool bad = LDSW(1) != 0u;
    const float qnan = __builtin_nanf("");
#define ACC(g, q) acc[(g) >> 1][((q) >> 1) & 1][((g) & 1) * 2 + ((q) >> 2)][(q) & 1]
    const unsigned foff = OFF_PFLAG + (unsigned)wid * 256u + (call & 1u) * 128u;
    const size_t poff = OFF_PART + (size_t)(call & 1u) * PART_BYTES + (size_t)slot * TILE_BYTES + wid * 32768 + lane2 * 16;
    if (!aux) {
        char* const dst = me + poff + (size_t)peer * 16 * TILE_BYTES;
        float mx = 0.f;
#pragma unroll
        for (int g = 0; g < 4; ++g)
#pragma unroll
            for (int q = 0; q < 8; ++q) {
                const f32x4 y = ACC(g, q);
                mx = fmaxf(fmaxf(mx, fabsf(y.x)), fabsf(y.y)); mx = fmaxf(fmaxf(mx, fabsf(y.z)), fabsf(y.w));
            }
        const unsigned emax = (wave_max(__builtin_bit_cast(unsigned, mx)) >> 23) & 255u;
        const unsigned base = emax > 15u ? emax - 15u : 0u;
        unsigned info = base | (bad ? 1u << 24 : 0u);
#pragma unroll
        for (int g = 0; g < 4; ++g) {
            u32x4 w[8], any = {0u, 0u, 0u, 0u};
#pragma unroll
            for (int q = 0; q < 8; ++q) { w[q] = __builtin_bit_cast(u32x4, ACC(g, q)) - (base << 23); any |= w[q]; }
            const u32x4 x7 = (w[7] & 0x07ffffffu) | ((w[7] >> 4) & 0x08000000u);
#pragma unroll
            for (int q = 0; q < 7; ++q) {
                asm volatile("s_waitcnt vmcnt(%0)" :: "i"(EXDEPTH) : "memory");
                store_wt(dst + g * 8192 + q * 1024, (w[q] & 0x87ffffffu) | ((x7 << (27 - 4 * q)) & 0x78000000u));
            }
            const bool need = ((any.x | any.y | any.z | any.w) & 0x78000000u) != 0u;
            const u64 mask = __ballot(need);
            if (need) {
                u32x4 pl = {0u, 0u, 0u, 0u};
#pragma unroll
                for (int q = 0; q < 8; ++q) pl |= ((w[q] >> 27) & 15u) << (4 * q);
                store_wt(dst + g * 8192 + 7168, pl);
            }
            if ((lane2 >> 2) == g) info |= ((unsigned)(mask >> ((lane2 & 3) * 16)) & 0xffffu) << 8;
            if ((g == 0 && CHORDER) || g == 3) {
                WAIT_V(0);
                if (g ? lane2 >= 4 * CHORDER && lane2 < 16 : lane2 < 4) st_sys64(pws + foff + (unsigned)(rank * 16 + slot) * 2048u + lane2 * 8, call, info);
            }
        }
        TS(23);
        return;
    }
    const char* const src = me + poff + (size_t)rank * 16 * TILE_BYTES;
    float* const o = P.out + (size_t)((tile >> 3) * 256 + wr * 64 + (lane2 >> 4) * 4) * NFULL + peer * NS + tj * 256 + wc * 64 + (lane2 & 15) * 4;
    unsigned info16 = 0u;
    if (!remote) {
        info16 = wave_wait64<POLL>(me + foff + (unsigned)(peer * 16 + slot) * 2048u + (lane2 & 15) * 8, call, tmo, call);
        bad = bad || ld_agent(tmo) == call || (info16 >> 24) != 0u;
    }
#pragma unroll
    for (int g = 0; g < 4; ++g) {
        if (!remote) {
            u32x4 w[8];
            asm volatile("global_load_dwordx4 %0, %7, off sc0 sc1\n\t"
                         "global_load_dwordx4 %1, %7, off offset:1024 sc0 sc1\n\t"
                         "global_load_dwordx4 %2, %7, off offset:2048 sc0 sc1\n\t"
                         "global_load_dwordx4 %3, %7, off offset:3072 sc0 sc1\n\t"
                         "global_load_dwordx4 %4, %8, off sc0 sc1\n\t"
                         "global_load_dwordx4 %5, %8, off offset:1024 sc0 sc1\n\t"
                         "global_load_dwordx4 %6, %8, off offset:2048 sc0 sc1\n\ts_waitcnt vmcnt(0)"
                         : "=&v"(w[0]), "=&v"(w[1]), "=&v"(w[2]), "=&v"(w[3]), "=&v"(w[4]), "=&v"(w[5]), "=&v"(w[6])
                         : "v"(src + g * 8192), "v"(src + g * 8192 + 4096) : "memory");
            unpack28(w, __shfl(info16, g * 4 + (lane2 >> 4)), lane2, src + g * 8192 + 7168);
#pragma unroll
            for (int q = 0; q < 8; ++q) {
                f32x4 y = ACC(g, q) + __builtin_bit_cast(f32x4, w[q]);
#pragma unroll
                for (int c = 0; c < 4; ++c) y[c] = gelu(y[c]);
                ACC(g, q) = y;
            }
        }
#pragma unroll
        for (int mm = 0; mm < 2; ++mm)
#pragma unroll
            for (int r = 0; r < 4; ++r) {
                const int ai = g >> 1, m = (g & 1) * 2 + mm;
                f32x4 y = {acc[ai][0][m][0][r], acc[ai][0][m][1][r], acc[ai][1][m][0][r], acc[ai][1][m][1][r]};
                if (bad) y = (f32x4){qnan, qnan, qnan, qnan};
                if (remote) asm volatile("s_waitcnt vmcnt(%0)" :: "i"(F1DEPTH) : "memory");
                store_wt(o + (size_t)(ai * 128 + m * 16 + r) * NFULL, __builtin_bit_cast(u32x4, y));
            }
    }
    WAIT_V(0);
    TS(28);
    if (remote) {
        if (lane2 == 0) st_sys(me + OFF_OWNF + (unsigned)((peer * 16 + tile) * 8 + wid) * 64u, call);
        if (tile < 8) {
            for (int i = 0; i < F1GO; ++i) __builtin_amdgcn_s_sleep(127);
            tile += 8;
            goto again;
        }
    }
}

extern "C" void kernel_launch(int rank, int nranks, ncclComm_t comm,
                              hipStream_t stream,
                              void* const* d_in, const int* in_sizes, int n_in,
                              void* d_out, int out_size,
                              const int* in_dtypes, int out_dtype,
                              void* const* d_out_all,
                              void* const* d_ws_all, size_t ws_bytes) {
    (void)comm; (void)in_dtypes; (void)out_dtype; (void)d_out_all;
    if (nranks != NR || (unsigned)rank >= (unsigned)NR || n_in != 2 || ws_bytes < WS_NEED
        || in_sizes[0] != 4096 * KS || in_sizes[1] != KS * NFULL || out_size != 512 * NFULL) {
        fprintf(stderr, "kernel_launch: unexpected problem shape\n");
        return;
    }
    Params P = {};
    P.x = (const float*)d_in[0];
    P.w = (const float*)d_in[1];
    P.out = (float*)d_out;
    for (int p = 0; p < NR; ++p) P.ws[p] = (char*)d_ws_all[p];
    P.me = P.ws[rank];
    P.rank = rank;
    static const unsigned char port[NR][NR] = {
        {0, 2, 3, 6, 4, 7, 5, 1}, {1, 4, 5, 7, 6, 2, 3, 0}, {2, 0, 6, 3, 7, 1, 5, 4}, {3, 6, 0, 2, 7, 4, 1, 5},
        {4, 1, 7, 5, 6, 3, 0, 2}, {5, 7, 1, 4, 0, 6, 2, 3}, {6, 3, 2, 0, 1, 5, 4, 7}, {7, 5, 4, 1, 2, 0, 3, 6}};
    for (int c = 0; c < NR; ++c) P.peermap |= (unsigned)port[rank][c] << (4 * c);
    unsigned n = 0, sw = 0, sa = 1, sb = 0, so = 1;
    for (const char* c = ORDER; *c; ++c) {
        unsigned& s = *c == 'w' ? sw : *c == 'a' ? sa : *c == 'b' ? sb : so;
        const unsigned first = *c == 'w' ? 0 : *c == 'a' ? 8 : *c == 'b' ? 14 : 20, cnt = *c == 'w' ? 8 : *c == 'o' ? 2 : 6;
        for (unsigned i = 0; i < cnt && n < NPULL; ++i, ++n) P.order[n >> 2] |= (((s & 7u) << 5) | (first + i)) << ((n & 3u) * 8u);
        ++s;
    }
    if (n != NPULL || sw != 8 || sa != 8 || sb != 8 || so != 8) { fprintf(stderr, "kernel_launch: bad ORDER\n"); return; }
    gemm_rs_kernel<<<256, 512, 0, stream>>>(P);
    HIP_CHECK(hipGetLastError());
}
```

```cpp
#include <hip/hip_runtime.h>
#include <rccl/rccl.h>
#include <cstdio>

#define HIP_CHECK(expr) do { hipError_t _e = (expr); if (_e != hipSuccess) { \
    fprintf(stderr, "HIP error %s at %s:%d\n", hipGetErrorString(_e), __FILE__, __LINE__); \
    return; } } while (0)

#define LAS __attribute__((address_space(3)))
#define GAS __attribute__((address_space(1)))
typedef _Float16 f16x8 __attribute__((ext_vector_type(8)));
typedef float f32x4 __attribute__((ext_vector_type(4)));
typedef unsigned u32x4 __attribute__((ext_vector_type(4)));
typedef unsigned u32x2 __attribute__((ext_vector_type(2)));
typedef unsigned long long u64;

#define ORDER "wawbwabwawabwabwawabbbbooooooo"
#define NC 6
#define WIN 7
#define FASTW 2
#define FASTW2 2
#define F1DEPTH 1
#define F1GO 8
#define RAW0 0
#define MAPW 2
#define LSTART 0
#define LSTEP 1
#define POLL 8
#define LATE 80
#define STAMP 0

static_assert(WIN >= 4 && NC <= 7, "loader bookkeeping");

namespace {

constexpr int NR = 8;
constexpr int KS = 512;
constexpr int NFULL = 16384;
constexpr int NS = 2048;
constexpr int NBLK = 24;
constexpr unsigned UNIT = 32768;
constexpr unsigned TILE_BYTES = 262144;
constexpr unsigned NPULL = 168, NSLICE = NPULL * 8, NWORK = NSLICE + 512;
constexpr size_t CTRL_BYTES = 1u << 20;
constexpr size_t STAGE_BYTES = (size_t)64 * NBLK * UNIT;
constexpr size_t PART_BYTES = (size_t)NR * 16 * TILE_BYTES;
constexpr size_t WSET_BYTES = (size_t)NR * 64 * UNIT;
constexpr size_t XSET_BYTES = (size_t)128 * UNIT;
constexpr size_t OFF_STAGE = CTRL_BYTES, OFF_PART = OFF_STAGE + 2 * STAGE_BYTES, OFF_OWNB = OFF_PART + 2 * PART_BYTES,
                 OFF_EXPW = OFF_OWNB + WSET_BYTES, OFF_EXPX = OFF_EXPW + 2 * WSET_BYTES, WS_NEED = OFF_EXPX + 2 * XSET_BYTES;
constexpr unsigned OFF_TMO = 0, OFF_QUEUE = 2048  , OFF_CNT = 4096  ,
                   OFF_FLAG = 65536  , OFF_PFLAG = 262144  ,
                   OFF_OFLAG = 524288  , OFF_OWNF = 589824  ,
                   OFF_EFLAG = 655360  , OFF_STAMP = 851968;
constexpr u64 DEADLINE_TICKS = 50000000ull;
constexpr unsigned LDS_STAGE = 131072, LDS_BYTES = LDS_STAGE + 384 + 8192;
constexpr unsigned LDS_ENC = LDS_STAGE + 384;

struct Params {
    const float* x;
    const float* w;
    char* me;
    int rank;
    unsigned peermap;
    float* out;
    char* ws[NR];
    unsigned order[48];
};

__device__ __forceinline__ void store_wt(void* p, u32x4 v) {
    asm volatile("global_store_dwordx4 %0, %1, off sc0 sc1\n\ts_nop 1" :: "v"(p), "v"(v) : "memory");
}
__device__ __forceinline__ unsigned pack2(float a, float b) {
    typedef _Float16 h2 __attribute__((ext_vector_type(2)));
    h2 h; h.x = (_Float16)a; h.y = (_Float16)b;
    return __builtin_bit_cast(unsigned, h);
}
__device__ __forceinline__ void gelu8(f32x4 (&y)[8]) {
    const float k1 = -2.0f * 0.7978845608028654f * 1.4426950408889634f, k2 = k1 * 0.044715f;
    f32x4 e[8];
#pragma unroll
    for (int q = 0; q < 8; ++q) e[q] = y[q] * __builtin_elementwise_fma(y[q] * y[q], (f32x4)k2, (f32x4)k1);
    __builtin_amdgcn_sched_barrier(0);
#pragma unroll
    for (int i = 0; i < 32; ++i) e[i >> 2][i & 3] = __builtin_amdgcn_exp2f(e[i >> 2][i & 3]);
    __builtin_amdgcn_sched_barrier(0);
#pragma unroll
    for (int q = 0; q < 8; ++q) e[q] += 1.0f;
    __builtin_amdgcn_sched_barrier(0);
#pragma unroll
    for (int i = 0; i < 32; ++i) e[i >> 2][i & 3] = __builtin_amdgcn_rcpf(e[i >> 2][i & 3]);
    __builtin_amdgcn_sched_barrier(0);
#pragma unroll
    for (int q = 0; q < 8; ++q) y[q] *= e[q];
}
__device__ __forceinline__ unsigned wave_scan(unsigned v) {
    v += __builtin_amdgcn_update_dpp(0, (int)v, 0x111, 0xf, 0xf, true);
    v += __builtin_amdgcn_update_dpp(0, (int)v, 0x112, 0xf, 0xf, true);
    v += __builtin_amdgcn_update_dpp(0, (int)v, 0x114, 0xf, 0xf, true);
    v += __builtin_amdgcn_update_dpp(0, (int)v, 0x118, 0xf, 0xf, true);
    v += __builtin_amdgcn_update_dpp(0, (int)v, 0x142, 0xa, 0xf, false);
    v += __builtin_amdgcn_update_dpp(0, (int)v, 0x143, 0xc, 0xf, false);
    return v;
}
__device__ __forceinline__ unsigned ld_sys(const char* p) {
    return __hip_atomic_load((const GAS unsigned*)p, __ATOMIC_RELAXED, __HIP_MEMORY_SCOPE_SYSTEM);
}
__device__ __forceinline__ u64 ld_sys64(const char* p) {
    return __hip_atomic_load((const GAS u64*)p, __ATOMIC_RELAXED, __HIP_MEMORY_SCOPE_SYSTEM);
}
__device__ __forceinline__ unsigned ld_agent(const char* p) {
    return __hip_atomic_load((const GAS unsigned*)p, __ATOMIC_RELAXED, __HIP_MEMORY_SCOPE_AGENT);
}
__device__ __forceinline__ void st_sys(char* p, unsigned v) {
    __hip_atomic_store((GAS unsigned*)p, v, __ATOMIC_RELAXED, __HIP_MEMORY_SCOPE_SYSTEM);
}
__device__ __forceinline__ void st_sys64(char* p, unsigned lo, unsigned hi) {
    __hip_atomic_store((GAS u64*)p, ((u64)hi << 32) | lo, __ATOMIC_RELAXED, __HIP_MEMORY_SCOPE_SYSTEM);
}
__device__ __forceinline__ unsigned add_agent(char* p, unsigned v) {
    return __hip_atomic_fetch_add((GAS unsigned*)p, v, __ATOMIC_RELAXED, __HIP_MEMORY_SCOPE_AGENT);
}
template <int SLEEP> __device__ __forceinline__ bool give_up(unsigned& n, u64 t0, char* tmo, unsigned call) {
    if ((++n & 15u) == 0u
        && ((unsigned)__builtin_amdgcn_readfirstlane(ld_agent(tmo)) == call || __builtin_amdgcn_s_memrealtime() - t0 > DEADLINE_TICKS)) {
        __hip_atomic_store((GAS unsigned*)tmo, call, __ATOMIC_RELAXED, __HIP_MEMORY_SCOPE_AGENT);
        return true;
    }
    __builtin_amdgcn_s_sleep(SLEEP);
    return false;
}
template <int SLEEP> __device__ __forceinline__ void wave_wait(const char* word, unsigned need, char* tmo, unsigned call) {
    const u64 t0 = __builtin_amdgcn_s_memrealtime();
    unsigned n = 0;
    while (!__all(ld_sys(word) >= need) && !give_up<SLEEP>(n, t0, tmo, call)) { }
    asm volatile("" ::: "memory");
}
template <int SLEEP> __device__ __forceinline__ unsigned wave_wait64(const char* word, unsigned need, char* tmo, unsigned call) {
    const u64 t0 = __builtin_amdgcn_s_memrealtime();
    unsigned n = 0;
    u64 v;
    do { v = ld_sys64(word); } while (!__all((unsigned)v >= need) && !give_up<SLEEP>(n, t0, tmo, call));
    asm volatile("" ::: "memory");
    return (unsigned)(v >> 32);
}

constexpr unsigned B01 = 0x01010101u, B03 = 0x03030303u, B07 = 0x07070707u, B80 = 0x80808080u, INFO_RAW = 1u << 23, INFO_BAD = 1u << 24;
__device__ __forceinline__ unsigned sub8(unsigned a, unsigned b) { return ((a | B80) - (b & ~B80)) ^ ((a ^ ~b) & B80); }
__device__ __forceinline__ unsigned perm(unsigned hi, unsigned lo, unsigned sel) { return __builtin_amdgcn_perm(hi, lo, sel); }
__device__ __forceinline__ void lds_or(LAS unsigned char* p, unsigned v) {
    __hip_atomic_fetch_or((LAS unsigned*)p, v, __ATOMIC_RELAXED, __HIP_MEMORY_SCOPE_WORKGROUP);
}
__device__ __forceinline__ unsigned prefix4(unsigned inc) {
    unsigned t = inc + (inc << 8);
    asm("" : "+v"(t));
    return t + (t << 16);
}
__device__ __forceinline__ unsigned exp_encode(unsigned (&e)[8], int lane, LAS unsigned char* tail, unsigned cap) {
    *(volatile LAS u32x4*)(tail + lane * 16) = (u32x4){0u, 0u, 0u, 0u};
    unsigned sum = 0;
#pragma unroll
    for (int k = 0; k < 8; ++k) sum = __builtin_amdgcn_sad_u8(e[k], 0u, sum);
    const unsigned c = ((__builtin_amdgcn_readlane(wave_scan(sum), 63) >> 11) + 3u) & 255u;
    const unsigned c4 = ((c & 15u) | 16u) * B01, r0 = sub8(c * B01, 0x03020100u), r1 = sub8(c * B01, 0x030a0908u), r2 = sub8(c * B01, 0x07060504u);
    unsigned acc[4], len[4], fix[8], any = 0;
#pragma unroll
    for (int k = 0; k < 8; ++k) {
        const unsigned h = c4 - (e[k] & 0x0f0f0f0fu), sel = h & B07, m = perm(0u, 0u, ((h >> 3) & B01) | 0x0c0c0c0cu);
        const unsigned cs = (perm(0x4c3c2c1cu, 0x037c6c5cu, sel) & m) | (perm(0x4c3c2c1cu, 0x0302010cu, sel) & ~m);
        fix[k] = e[k] ^ ((perm(r2, r1, sel) & m) | (perm(r2, r0, sel) & ~m));
        any |= fix[k];
        const unsigned sy = (cs >> 4) & B07, in = prefix4((cs >> 2) & B03);
        const unsigned ch = (sy & 255u) | (((sy >> 8) & 255u) << (in & 255u)) | (((sy >> 16) & 255u) << ((in >> 8) & 255u)) | ((sy >> 24) << ((in >> 16) & 255u));
        if (k & 1) { acc[k >> 1] |= ch << len[k >> 1]; len[k >> 1] += in >> 24; } else { acc[k >> 1] = ch; len[k >> 1] = in >> 24; }
        e[k] = cs & B03;
    }
    const unsigned mine = (len[0] + len[1]) + (len[2] + len[3]), incl = wave_scan(mine);
    unsigned p = incl - mine;
#pragma unroll
    for (int g = 0; g < 4; ++g) {
        const u64 t = (u64)acc[g] << (p & 31u);
        lds_or(tail + (p >> 5) * 4u, (unsigned)t); lds_or(tail + (p >> 5) * 4u + 4u, (unsigned)(t >> 32));
        p += len[g];
    }
    const unsigned sbytes = ((__builtin_amdgcn_readlane(incl, 63) + 31u) >> 5) * 4u;
    unsigned n = 0;
    u64 has;
    while ((has = __ballot(any != 0u)) != 0ull && n <= 64u) {
        unsigned f = 0, kk = 0;
        any = 0;
#pragma unroll
        for (int k = 7; k >= 0; --k) if (fix[k] != 0u) { f = fix[k]; kk = k; }
        const unsigned sh = __builtin_ctz(f | 0x80000000u) & 24u;
        const unsigned idx = n + __builtin_amdgcn_mbcnt_hi((unsigned)(has >> 32), __builtin_amdgcn_mbcnt_lo((unsigned)has, 0u));
        if (f != 0u && idx < 64u)
            *(volatile LAS unsigned short*)(tail + sbytes + idx * 2u) = (lane << 10) | (kk << 7) | (sh << 2) | ((f >> (sh + 3u)) & 31u);
#pragma unroll
        for (int k = 0; k < 8; ++k) { if (kk == (unsigned)k) fix[k] = f & ~(255u << sh); any |= fix[k]; }
        n += __builtin_popcountll(has);
    }
    const unsigned bytes = sbytes + n * 2u;
    return n > 64u || bytes > cap ? INFO_RAW : c | (n << 8) | (((bytes + 15u) >> 4) << 16);
}
__device__ __forceinline__ void exp_decode(unsigned (&e)[8], unsigned info, int lane, const LAS unsigned char* tail, LAS unsigned char* scratch) {
    *(volatile LAS u32x4*)(scratch + lane * 32) = (u32x4){0u, 0u, 0u, 0u};
    *(volatile LAS u32x4*)(scratch + lane * 32 + 16) = (u32x4){0u, 0u, 0u, 0u};
    unsigned m[8], in[8], p[4], ch[8], sy[8], mine = 0;
#pragma unroll
    for (int k = 0; k < 8; ++k) {
        m[k] = perm(0u, 0xffu, e[k]);
        in[k] = prefix4(m[k] & B03);
        if (!(k & 1)) p[k >> 1] = mine;
        mine += in[k] >> 24;
    }
    const unsigned incl = wave_scan(mine), sbytes = ((__builtin_amdgcn_readlane(incl, 63) + 31u) >> 5) * 4u;
    unsigned en = 0;
    if ((unsigned)lane < ((info >> 8) & 127u)) en = *(const volatile LAS unsigned short*)(tail + sbytes + lane * 2);
#pragma unroll
    for (int g = 0; g < 4; ++g) {
        p[g] += incl - mine;
        const LAS unsigned* const q = (const LAS unsigned*)(tail + (p[g] >> 5) * 4u);
        ch[2 * g] = __builtin_amdgcn_alignbit(q[1], q[0], p[g]);
        ch[2 * g + 1] = ch[2 * g] >> (in[2 * g] >> 24);
    }
    if (en & 31u) lds_or(scratch + (en >> 7) * 4u, (en & 31u) << (((en >> 2) & 24u) + 3u));
    const u32x4 f0 = *(volatile LAS u32x4*)(scratch + lane * 32), f1 = *(volatile LAS u32x4*)(scratch + lane * 32 + 16);
#define SPREAD1(d, i, c, j, jm) "v_lshrrev_b32_sdwa " d ", " i ", " c " dst_sel:BYTE_" #j " dst_unused:UNUSED_PRESERVE src0_sel:BYTE_" #jm " src1_sel:DWORD\n\t"
#define SPREAD4(j, jm) SPREAD1("%0", "%4", "%8", j, jm) SPREAD1("%1", "%5", "%9", j, jm) SPREAD1("%2", "%6", "%10", j, jm) SPREAD1("%3", "%7", "%11", j, jm)
#pragma unroll
    for (int k = 0; k < 8; k += 4) {
        sy[k] = ch[k]; sy[k + 1] = ch[k + 1]; sy[k + 2] = ch[k + 2]; sy[k + 3] = ch[k + 3];
        asm(SPREAD4(1, 0) SPREAD4(2, 1) SPREAD4(3, 2) "s_nop 0"
            : "+&v"(sy[k]), "+&v"(sy[k + 1]), "+&v"(sy[k + 2]), "+&v"(sy[k + 3])
            : "v"(in[k]), "v"(in[k + 1]), "v"(in[k + 2]), "v"(in[k + 3]), "v"(ch[k]), "v"(ch[k + 1]), "v"(ch[k + 2]), "v"(ch[k + 3]));
    }
    const unsigned c4 = (info & 255u) * B01, tc = sub8(c4, 0x03020100u), t0 = sub8(c4, 0x06050400u), t1 = sub8(c4, 0x0a090807u);
#pragma unroll
    for (int k = 0; k < 8; ++k)
        e[k] = ((perm(t1, t0, sy[k] & B07) & m[k]) | (perm(0u, tc, e[k]) & ~m[k])) ^ (k < 4 ? f0[k & 3] : f1[k & 3]);
}

#define TS(k) do { } while (0)
#define WAIT_V(n) asm volatile("s_waitcnt vmcnt(" #n ")" ::: "memory")
#define WAIT_L(n) asm volatile("s_waitcnt lgkmcnt(" #n ")" ::: "memory")
#define BAR __builtin_amdgcn_s_barrier()
#define SCHED __builtin_amdgcn_sched_barrier(0)
#define LDSW(i) (*(volatile LAS unsigned*)(lds + LDS_STAGE + 4 * (i)))

}

__global__ __launch_bounds__(512, 2) void gemm_rs_kernel(Params P) {
    __shared__ __attribute__((aligned(16))) unsigned char shm[LDS_BYTES];
    LAS unsigned char* lds = (LAS unsigned char*)shm;
    const int tid = threadIdx.x, lane = tid & 63, wid = __builtin_amdgcn_readfirstlane(tid >> 6);
    const int hb = __builtin_amdgcn_readfirstlane(blockIdx.x);
    const float* px = P.x; const float* pw = P.w; char* me_ = P.me; int rank_ = P.rank; unsigned peermap = P.peermap;
    asm volatile("" : "+s"(px), "+s"(pw), "+s"(me_), "+s"(rank_), "+s"(peermap));
    const int rank = rank_ & (NR - 1);
    char* const me = (char*)(GAS char*)(size_t)me_;
    char* const tmo = me + OFF_TMO;
    int bid = (hb & ~7) | (__builtin_amdgcn_s_getreg((3 << 11) | 20) & 7);
    if (MAPW < 8 && bid >= 128 && (bid & 7) != 0) {
        const int tt = ((bid & 7) - 1) * 8 + ((bid >> 3) & 7), cg = tt / (7 * MAPW), r = tt - cg * (7 * MAPW);
        bid = (bid & 192) | ((cg * MAPW + r % MAPW) << 3) | (r / MAPW + 1);
    }
    f32x4 v[8] = {};
    auto first_item = [&](int b) -> unsigned {
        const int q = ((b >> 3) & 7) + 1, sk = ((peermap >> ((b & 7) * 4)) - rank) & (NR - 1);
        const int pl = q < 7 ? (rank + q + (q >= sk)) & (NR - 1) : ((rank + sk) & (NR - 1)) + (q & 8);
        const int lo = (b & 7) == 0 ? (b >> 6) * 16 + 7 + q : b < 64 ? 0 : pl;
        const int hi = b >= 192 ? (q - 1) * 16 + (b & 7) : b;
        return b >= 128 ? hi : lo;
    };
    auto load_item = [&](int b, unsigned item) {
        const bool f1 = b < 64 && (b & 7) != 0, isw = f1 || item >= 128u;
        const unsigned ws_ = f1 ? item : (item >> 3) & 7u, wj = (f1 ? b >> 3 : item) & 7u, wd = f1 ? (peermap >> ((b & 7) * 4)) & (NR - 1) : rank;
        const unsigned wo = (ws_ * 64u + (wid >> 2) * 32u) * NFULL + wd * NS + wj * 256u + (wid & 3) * 64u;
        const unsigned xo = (((item & 7u) * 2u + ((item >> 3) & 1u)) * 256u + wid * 32u) * KS + ((item >> 4) & 7u) * 64u;
        const float* const base = isw ? pw + wo : px + xo;
        unsigned lo = (lane >> 4) * (isw ? 16u * NFULL : 16u) + (lane & 15) * (isw ? 16u : 4u * KS);
        const unsigned sb = isw ? 4u * NFULL : 64u, sa = (isw ? 64u * NFULL : 64u * KS) - 3u * sb;
#define LD1(i, step) "global_load_dwordx4 %" #i ", %8, %9\n\tv_add_u32 %8, " step ", %8\n\t"
        asm volatile("s_nop 4\n\t" LD1(0, "%10") LD1(1, "%10") LD1(2, "%10") LD1(3, "%11") LD1(4, "%10") LD1(5, "%10") LD1(6, "%10")
                     "global_load_dwordx4 %7, %8, %9"
                     : "+v"(v[0]), "+v"(v[1]), "+v"(v[2]), "+v"(v[3]), "+v"(v[4]), "+v"(v[5]), "+v"(v[6]), "+v"(v[7]), "+v"(lo)
                     : "s"(base), "s"(sb), "s"(sa) : "memory");
    };
#define VWAIT(n) asm volatile("s_waitcnt vmcnt(" #n ")" : "+v"(v[0]), "+v"(v[1]), "+v"(v[2]), "+v"(v[3]), "+v"(v[4]), "+v"(v[5]), "+v"(v[6]), "+v"(v[7]) :: "memory")
    unsigned t = 0, ordw[3];
    char* lws;
    {
        u64 ex;
        asm volatile("s_mov_b64 %5, exec\n\ts_mov_b64 exec, %12\n\t"
                     "global_atomic_add %0, %6, %7, %8 sc0\n\ts_mov_b64 exec, %5\n\t"
                     "global_load_dwordx2 %1, %9, %11 offset:%13\n\t"
                     "global_load_dword %2, %10, %11 offset:%14\n\t"
                     "global_load_dword %3, %10, %11 offset:%14+64\n\t"
                     "global_load_dword %4, %10, %11 offset:%14+128"
                     : "+v"(t), "=&v"(lws), "=&v"(ordw[0]), "=&v"(ordw[1]), "=&v"(ordw[2]), "=&s"(ex)
                     : "v"(OFF_CNT + (lane ? bid * 128 + 4 : hb * 128)), "v"(1u), "s"(me), "v"(lane & 56), "v"(lane & 60),
                       "s"(__builtin_amdgcn_kernarg_segment_ptr()), "s"((u64)(wid == 0 ? 3u : 0u)),
                       "i"(__builtin_offsetof(Params, ws)), "i"(__builtin_offsetof(Params, order)) : "memory");
    }
    if (bid >= 128 ? (bid >> 4) != 12 : (bid & 7) == 0) __builtin_amdgcn_s_sleep(LATE);
    char* pws_;
    unsigned call = 0;
#pragma unroll 1
    for (;;) {
        asm volatile("" ::: "memory");
        load_item(bid, first_item(bid));
        pws_ = P.ws[(peermap >> ((bid & 7) * 4)) & (NR - 1)];
        asm volatile("s_waitcnt vmcnt(8)" : "+v"(t), "+v"(lws), "+v"(ordw[0]), "+v"(ordw[1]), "+v"(ordw[2]) :: "memory");
        if (wid == 0 && call == 0u) {
            const unsigned c = __builtin_amdgcn_readlane(t, 0) + 1u;
            unsigned old = __builtin_amdgcn_readlane(t, 1), r = bid;
            for (int tries = 0; old != c - 1u && tries < 65536; ++tries) {
                if (lane == 0) add_agent(me + OFF_CNT + r * 128u + 4u, ~0u);
                r = (r + 1u) & 255u;
                if (lane == 0) t = add_agent(me + OFF_CNT + r * 128u + 4u, 1u);
                old = __builtin_amdgcn_readfirstlane(t);
            }
            LDSW(32 + lane) = 0u;
            if (lane < 16) LDSW(16 + lane) = 0u;
            if (lane == 0) { LDSW(0) = c; LDSW(12) = r; }
        }
        asm volatile("s_waitcnt lgkmcnt(0)" : "+s"(pws_) :: "memory"); BAR; asm volatile("" ::: "memory");
        call = __builtin_amdgcn_readfirstlane(LDSW(0));
        const int got = __builtin_amdgcn_readfirstlane(LDSW(12));
        if (got == bid) break;
        bid = got;
        VWAIT(0);
    }
    const int peer = (peermap >> ((bid & 7) * 4)) & (NR - 1);
    const int slot = (bid >> 3) & 15;
    const bool aux = bid < 128, remote = (bid & 7) != 0;
    const int skip = remote ? (peer - rank) & (NR - 1) : NR;
    char* const pws = (char*)(GAS char*)(size_t)pws_;

    if (aux && remote && slot >= 8) {
        const unsigned par = call & 1u;
        const unsigned pu = (unsigned)(rank * 8 * NBLK + 8) + (first_item(bid) & 7u) * 2u + ((first_item(bid) >> 3) & 1u);
        char* const rdst = pws + OFF_STAGE + par * STAGE_BYTES + (size_t)pu * UNIT + wid * 4096 + lane * 16;
        auto frag = [&](int f) -> u32x4 {
            return (u32x4){pack2(v[2 * f][0], v[2 * f][1]), pack2(v[2 * f][2], v[2 * f][3]),
                           pack2(v[2 * f + 1][0], v[2 * f + 1][1]), pack2(v[2 * f + 1][2], v[2 * f + 1][3])};
        };
        VWAIT(6); store_wt(rdst, frag(0));
        if (wid > FASTW) WAIT_V(0);
        VWAIT(5); store_wt(rdst + 1024, frag(1));
        if (wid > FASTW2) WAIT_V(0);
        VWAIT(4); store_wt(rdst + 2048, frag(2));
        VWAIT(3);
        const u32x4 o3 = frag(3);
        auto push_last = [&]() { WAIT_V(0); store_wt(rdst + 3072, o3); };
        auto publish_push = [&]() {
            WAIT_V(0);
            __builtin_amdgcn_s_waitcnt(0x0F70);
            if (lane == 0)
                __hip_atomic_fetch_add((GAS unsigned*)(pws + OFF_FLAG + pu * 128u), 1u, __ATOMIC_RELAXED, __HIP_MEMORY_SCOPE_SYSTEM);
        };
        if (wid == 0) {
            push_last();
            publish_push();
            TS(1);
            __builtin_amdgcn_s_setprio(3);
            const unsigned lbase = (unsigned)(size_t)lds, landed = lbase + (lane ? 129024 + lane * 4 : LDS_STAGE + 68);
            unsigned head = 0, alive = NC, idle = 4;
            int issued = 0;
            u32x4 d = *(volatile LAS u32x4*)(lds + LDS_STAGE + 128);
#define PIECE(off) "s_waitcnt vmcnt(%5)\n\tds_max_i32 %4, %1\n\tv_add_u32 %1, 1, %1\n\tglobal_load_lds_dwordx4 %2, off offset:" #off " sc0 sc1\n\t"
#define SHORT(off) "s_waitcnt vmcnt(%5)\n\tds_max_i32 %4, %1\n\tv_add_u32 %1, 1, %1\n\ts_mov_b64 exec, %6\n\t" \
                   "global_load_lds_dwordx4 %2, off offset:" #off " sc0 sc1\n\ts_mov_b64 exec, -1\n\t"
#define ISSUE(body, off) asm volatile("s_mov_b32 %0, m0\n\ts_mov_b32 m0, %3\n\t" body "s_mov_b32 m0, %0" : "=&s"(keep), "+v"(known) \
                                      : "v"(src + off), "s"(dst + off), "v"(landed), "i"(WIN - 1), "s"(mask) : "memory")
            while (alive) {
                if ((unsigned)__builtin_amdgcn_readfirstlane(d.w) != head + 1u) {
                    if (idle == 1u) asm volatile("s_waitcnt vmcnt(6)\n\tds_max_i32 %0, %1" :: "v"(landed), "v"(issued - 6) : "memory");
                    if (idle == 2u) asm volatile("s_waitcnt vmcnt(4)\n\tds_max_i32 %0, %1" :: "v"(landed), "v"(issued - 4) : "memory");
                    if (idle == 3u) asm volatile("s_waitcnt vmcnt(0)\n\tds_max_i32 %0, %1" :: "v"(landed), "v"(issued) : "memory");
                    ++idle;
                    d = *(volatile LAS u32x4*)(lds + LDS_STAGE + 128 + (head & 15u) * 16u);
                    continue;
                }
                ++head;
                const unsigned meta = __builtin_amdgcn_readfirstlane(d.z), cnt = meta & 15u;
                const unsigned dst = lbase + ((meta >> 11) & 15u) * 8192u;
                unsigned keep;
                const char* const src = pws + d.x + lane * 16;
                d = *(volatile LAS u32x4*)(lds + LDS_STAGE + 128 + (head & 15u) * 16u);
                if (cnt == 0u) { --alive; continue; }
                const u64 mask = ~0ull >> (64u - ((meta >> 4) & 127u));
                int known = issued - (WIN - 1);
                if (cnt == 4u) ISSUE(PIECE(0) PIECE(1024) PIECE(2048) SHORT(3072), 0);
                else {
                    ISSUE(PIECE(0) PIECE(1024) PIECE(2048), 0);
                    ISSUE(PIECE(0) PIECE(1024) PIECE(2048) SHORT(3072), 3072);
                    if (cnt == 8u) ISSUE(PIECE(0), 7168);
                }
                issued += cnt;
                idle = 0u;
            }
        } else if (wid <= NC) {
            const unsigned me8 = wid - 1;
            LAS unsigned char* const scratch = lds + 114688 + me8 * 2048u;
            unsigned bi = me8 * 2u + 1u;
            const float qnan = __builtin_nanf("");
            unsigned tbl = 0;
#pragma unroll
            for (int h = 0; h < 3; ++h) {
                const unsigned e = (ordw[h] >> ((lane & 3) * 8)) & 255u, code = e & 31u;
                const int odd = code >= 14u, q = code - (odd ? 14u : 8u);
                const unsigned blk = code < 8u ? code : code >= 20u ? 8u + rank * 2 + (code - 20u)
                                                                     : 8u + ((rank + q + 1 + (q + 1 >= skip)) & (NR - 1)) * 2 + odd;
                tbl |= ((e & 0xe0u) | blk) << (8 * h);
            }
            struct Item { unsigned t, id, flag, src; };
            auto item_of = [&](unsigned t) -> Item {
                const unsigned it = min(t >> 3, NPULL - 1u), e = (unsigned)__builtin_amdgcn_readlane(tbl, it & 63u) >> ((it >> 6) * 8u);
                const unsigned ks = (e >> 5) & 7u, blk = e & 31u, u = (peer * 8u + ks) * NBLK + blk;
                const unsigned so = blk < 8u ? (unsigned)OFF_EXPW + par * (unsigned)WSET_BYTES + ((rank * 8u + ks) * 8u + blk) * UNIT
                                             : (unsigned)OFF_EXPX + par * (unsigned)XSET_BYTES + (ks * 16u + blk - 8u) * UNIT;
                const unsigned i = max(min(t, NWORK - 1u), NSLICE) - NSLICE;
                const unsigned ci = ((i >> 8) * 64u + (i & 63u)) * 4u + ((i >> 6) & 3u);
                Item r;
                r.t = t;
                r.id = t < NSLICE ? u | (blk < 8u ? 0x10000u : 0u) : ci;
                r.flag = t < NSLICE ? OFF_EFLAG + (par * 64u * NBLK + u) * 64u + (t & 7u) * 8u
                                    : OFF_PFLAG + (peer * 128u + (ci >> 2)) * 256u + par * 128u + (ci & 3u) * 8u;
                r.src = t < NSLICE ? so + (t & 7u) * 4096u : (unsigned)OFF_PART + par * (unsigned)PART_BYTES + rank * 16u * TILE_BYTES + ci * 8192u;
                return r;
            };
            auto ownf_of = [&](const Item& x) -> unsigned { return OFF_OWNF + (peer * 128u + (x.t < NSLICE ? 0u : x.id >> 2)) * 64u; };
            auto out_of = [&](const Item& x) -> unsigned {
                const unsigned ct = (x.id >> 5) & 15u, cw = (x.id >> 2) & 7u, cg = x.id & 3u;
                return (((ct >> 3) * 256u + (cg >> 1) * 128u + (cw >> 2) * 64u + (cg & 1u) * 32u) * NFULL + peer * NS + (ct & 7u) * 256u + (cw & 3u) * 64u) * 4u;
            };
            const unsigned olane = ((lane >> 4) * 4u * NFULL + (lane & 15) * 4u) * 4u;
            int rend = 0;
            auto post = [&](unsigned src, unsigned cnt, unsigned last) {
                unsigned k = 0;
                bi ^= 1u;
                if (lane == 0) {
                    asm volatile("ds_add_rtn_u32 %0, %1, %2\n\ts_waitcnt lgkmcnt(0)" : "=&v"(k) : "v"((unsigned)(size_t)lds + LDS_STAGE + 64u), "v"(0x10000u + cnt) : "memory");
                    *(volatile LAS u32x4*)(lds + LDS_STAGE + 128 + ((k >> 16) & 15u) * 16u) = (u32x4){src, 0u, cnt | (last << 4) | (bi << 11), (k >> 16) + 1u};
                }
                rend = (__builtin_amdgcn_readfirstlane(k) & 0xffffu) + cnt;
            };
            auto post_item = [&](const Item& x, unsigned inf) {
                const bool raw = (inf & INFO_RAW) != 0u, op = x.t < NSLICE;
                post(x.src, op ? 4u : raw ? 8u : 7u, raw ? 64u : min((op ? 16u : 32u) + ((inf >> 16) & 127u), 64u));
            };
            Item A = item_of((slot - 8) * NC + me8), B = item_of(A.t + 8u * NC), C = item_of(A.t + 16u * NC);
            const u64 f0 = ld_sys64(me + A.flag);
            unsigned info = __builtin_amdgcn_readfirstlane((unsigned)(f0 >> 32)), ofl = 0, tm = 0;
            const bool now = __all((unsigned)f0 >= call);
            push_last();
            if (now) post_item(A, info);
            publish_push();
            if (!now) { info = __builtin_amdgcn_readfirstlane(wave_wait64<2>(me + A.flag, call, tmo, call)); post_item(A, info); }
            u64 fl;
            do { fl = ld_sys64(me + B.flag); } while ((unsigned)__builtin_amdgcn_readfirstlane((unsigned)fl) < call && (int)__builtin_amdgcn_readfirstlane(LDSW(17)) < rend);
            unsigned pend = 0;
            u32x4 own[8] = {};
#pragma unroll 1
            for (;;) {
                const bool isp = A.t >= NSLICE, bad = tm == call || (info & INFO_BAD) != 0u;
                if ((int)__builtin_amdgcn_readfirstlane(LDSW(17)) < rend) {
                    if (pend) { WAIT_V(0); if (lane == 0) add_agent(me + pend, 1u); pend = 0; }
                    while ((int)__builtin_amdgcn_readfirstlane(LDSW(17)) < rend) __builtin_amdgcn_s_sleep(1);
                }
                asm volatile("" ::: "memory");
                const LAS unsigned char* const buf = lds + bi * 8192u;
                const bool more = B.t < NWORK, early = more && (unsigned)__builtin_amdgcn_readfirstlane((unsigned)fl) >= call;
                unsigned infob = __builtin_amdgcn_readfirstlane((unsigned)(fl >> 32));
                if (early) post_item(B, infob);
                u32x4 nown[8];
                u64 ex;
                unsigned tk;
                const bool nextp = more && B.t >= NSLICE;
                if (nextp && (unsigned)__builtin_amdgcn_readfirstlane(ofl) < call) wave_wait<2>(me + ownf_of(B), call, tmo, call);
                {
                    unsigned vo = out_of(B) + olane;
                    asm volatile("s_mov_b64 %12, exec\n\ts_mov_b32 exec_lo, %20\n\ts_mov_b32 exec_hi, %20\n\ts_nop 2\n\t"
                                 "global_load_dwordx4 %0, %13, %14 sc0 sc1\n\tv_add_u32 %13, 0x10000, %13\n\t"
                                 "global_load_dwordx4 %1, %13, %14 sc0 sc1\n\tv_add_u32 %13, 0x10000, %13\n\t"
                                 "global_load_dwordx4 %2, %13, %14 sc0 sc1\n\tv_add_u32 %13, 0x10000, %13\n\t"
                                 "global_load_dwordx4 %3, %13, %14 sc0 sc1\n\tv_add_u32 %13, 0xd0000, %13\n\t"
                                 "global_load_dwordx4 %4, %13, %14 sc0 sc1\n\tv_add_u32 %13, 0x10000, %13\n\t"
                                 "global_load_dwordx4 %5, %13, %14 sc0 sc1\n\tv_add_u32 %13, 0x10000, %13\n\t"
                                 "global_load_dwordx4 %6, %13, %14 sc0 sc1\n\tv_add_u32 %13, 0x10000, %13\n\t"
                                 "global_load_dwordx4 %7, %13, %14 sc0 sc1\n\t"
                                 "s_mov_b64 exec, 1\n\t"
                                 "global_load_dwordx2 %8, %16, %15 sc0 sc1\n\t"
                                 "global_load_dword %9, %17, %15 sc0 sc1\n\t"
                                 "global_load_dword %10, %21, %15 sc1\n\t"
                                 "global_atomic_add %11, %18, %19, %15 sc0\n\t"
                                 "s_mov_b64 exec, %12"
                                 : "=&v"(nown[0]), "=&v"(nown[1]), "=&v"(nown[2]), "=&v"(nown[3]), "=&v"(nown[4]), "=&v"(nown[5]), "=&v"(nown[6]), "=&v"(nown[7]),
                                   "=&v"(fl), "=&v"(ofl), "=&v"(tm), "=&v"(tk), "=&s"(ex), "+v"(vo)
                                 : "s"(P.out), "s"(me), "v"(C.flag), "v"(ownf_of(C)), "v"(OFF_QUEUE + peer * 128u), "v"(1u),
                                   "s"(__builtin_amdgcn_readfirstlane(nextp ? -1 : 0)), "v"(OFF_TMO) : "memory");
                }
                unsigned e[8];
                if (!isp) {
                    u32x4 w[4];
#pragma unroll
                    for (int q = 0; q < 4; ++q) w[q] = *(const LAS u32x4*)(buf + q * 1024 + lane * 16);
                    if (!(info & INFO_RAW)) {
                        const unsigned sg = *(const LAS unsigned*)(buf + 3072 + lane * 4);
                        const u32x4 l0 = w[0], l1 = w[1], nb = w[2];
#pragma unroll
                        for (int k = 0; k < 8; ++k) e[k] = (nb[k >> 1] >> (2 + 4 * (k & 1))) & B03;
                        exp_decode(e, info, lane, buf + 3328, scratch);
#pragma unroll
                        for (int k = 0; k < 8; ++k) {
                            const unsigned hb = ((nb[k >> 1] >> (4 * (k & 1))) & B03) | (e[k] << 2) | ((sg << k) & B80), lo = k < 4 ? l0[k & 3] : l1[k & 3];
                            w[k >> 1][(k & 1) * 2] = perm(hb, lo, 0x05010400u); w[k >> 1][(k & 1) * 2 + 1] = perm(hb, lo, 0x07030602u);
                        }
                    }
                    const bool isw = (A.id >> 16) != 0u;
                    const unsigned w8 = A.t & 7u, d1 = isw ? 2048u : 1024u, d2 = isw ? 16384u : 2048u;
                    char* const dst = me + ((unsigned)OFF_STAGE + par * (unsigned)STAGE_BYTES + (A.id & 0xffffu) * UNIT
                                            + (isw ? ((w8 & 3u) * 4u + (w8 >> 2)) * 1024u : w8 * 4096u)) + lane * 16;
                    asm volatile("s_waitcnt vmcnt(0)" : "+v"(w[0]), "+v"(w[1]), "+v"(w[2]), "+v"(w[3]) :: "memory");
                    store_wt(dst, w[0]); store_wt(dst + d1, w[1]); store_wt(dst + d2, w[2]); store_wt(dst + d2 + d1, w[3]);
                } else {
                    u32x4 w[8];
#pragma unroll
                    for (int q = 0; q < 8; ++q) w[q] = *(const LAS u32x4*)(buf + q * 1024 + lane * 16);
                    if (!(info & INFO_RAW)) {
                        const u32x2 x2 = *(const LAS u32x2*)(buf + 6144 + lane * 8);
                        unsigned pw_[24];
#pragma unroll
                        for (int k = 0; k < 8; ++k) e[k] = (x2[k >> 2] >> (2 * (k & 3))) & B03;
                        exp_decode(e, info, lane, buf + 6656, scratch);
#pragma unroll
                        for (int i = 0; i < 24; ++i) pw_[i] = w[i >> 2][i & 3];
#pragma unroll
                        for (int q = 0; q < 8; ++q) {
                            const unsigned p0 = pw_[3 * q], p1 = pw_[3 * q + 1], p2 = pw_[3 * q + 2];
                            const u32x4 r = {perm(e[q], p0, 0x04020100u), perm(e[q], p1, 0x05020100u), perm(e[q], p2, 0x06020100u),
                                             perm(p1, p0, 0x0c0c0703u) | perm(e[q], p2, 0x07030c0cu)};
                            w[q] = (r >> 1) | (r << 31);
                        }
                    }
                    char* const o = (char*)P.out + out_of(A) + olane;
                    f32x4 y[8];
#pragma unroll
                    for (int q = 0; q < 8; ++q) y[q] = __builtin_bit_cast(f32x4, own[q]) + __builtin_bit_cast(f32x4, w[q]);
                    if (bad) {
#pragma unroll
                        for (int q = 0; q < 8; ++q) y[q] = (f32x4)qnan;
                    }
                    gelu8(y);
                    asm volatile("s_waitcnt vmcnt(0)" : "+v"(y[0]) :: "memory");
#pragma unroll
                    for (int q = 0; q < 8; ++q) store_wt(o + ((q >> 2) * 16 + (q & 3)) * (NFULL * 4), __builtin_bit_cast(u32x4, y[q]));
                }
                asm volatile(""
                             : "+v"(nown[0]), "+v"(nown[1]), "+v"(nown[2]), "+v"(nown[3]), "+v"(nown[4]), "+v"(nown[5]), "+v"(nown[6]), "+v"(nown[7]),
                               "+v"(fl), "+v"(ofl), "+v"(tm), "+v"(tk) :: "memory");
                if (pend && lane == 0) add_agent(me + pend, 1u);
                pend = isp ? 0u : OFF_FLAG + (A.id & 0xffffu) * 128u;
                tm = __builtin_amdgcn_readfirstlane(tm);
                if (!more) break;
                if (!early) {
                    if (pend) { WAIT_V(0); if (lane == 0) add_agent(me + pend, 1u); pend = 0; }
                    infob = __builtin_amdgcn_readfirstlane(wave_wait64<2>(me + B.flag, call, tmo, call));
                    post_item(B, infob);
                    tm = __builtin_amdgcn_readfirstlane(ld_agent(tmo));
                }
#pragma unroll
                for (int q = 0; q < 8; ++q) own[q] = nown[q];
                info = infob; A = B; B = C;
                C = item_of(__builtin_amdgcn_readfirstlane(tk) - (call - 1u) * NWORK + 24u * NC);
            }
            post(0u, 0u, 0u);
            WAIT_V(0);
            if (pend && lane == 0) add_agent(me + pend, 1u);
        } else {
            push_last();
            publish_push();
        }
        WAIT_V(0);
        asm volatile("s_endpgm" ::: "memory");
    }

    {
        const bool f1 = aux && remote;
        unsigned item = first_item(bid);
        const int fd = f1 ? peer : lane >> 3;
        char* const fws = f1 ? pws : lws;
#pragma unroll 1
        for (unsigned n = 0;; ++n) {
            const bool isw = f1 || item >= 128u;
            const int s = f1 ? item : isw ? (item - 128u) >> 3 : item >> 4;
            const int j = (f1 ? slot : item) & 7, rb = (item & 7) * 2 + ((item >> 3) & 1);
            const unsigned par = call & 1u;
            u32x4 o[4];
            unsigned off, d1, d2;
            VWAIT(0);
            if (isw) {
                off = ((wid & 3) * 4 + (wid >> 2)) * 1024; d1 = 2048; d2 = 16384;
#pragma unroll
                for (int f = 0; f < 4; ++f) {
                    o[f].x = pack2(v[0][f], v[1][f]); o[f].y = pack2(v[2][f], v[3][f]);
                    o[f].z = pack2(v[4][f], v[5][f]); o[f].w = pack2(v[6][f], v[7][f]);
                }
            } else {
                off = wid * 4096; d1 = 1024; d2 = 2048;
#pragma unroll
                for (int f = 0; f < 4; ++f) {
                    o[f].x = pack2(v[2 * f][0], v[2 * f][1]); o[f].y = pack2(v[2 * f][2], v[2 * f][3]);
                    o[f].z = pack2(v[2 * f + 1][0], v[2 * f + 1][1]); o[f].w = pack2(v[2 * f + 1][2], v[2 * f + 1][3]);
                }
            }
            off += lane * 16;
            const bool last = f1 ? item == 7u : bid < 208 || n == 1u;
            const unsigned u = (unsigned)((rank * 8 + s) * NBLK + (isw ? j : 8 + rb));
            const unsigned wu = (unsigned)((peer * 8 + s) * 8 + j);
            const bool expo = f1 || (!isw && (rb >> 1) != rank);
            char* const ldst = me + (f1 ? OFF_OWNB + (size_t)wu * UNIT : OFF_STAGE + par * STAGE_BYTES + (size_t)u * UNIT) + off;
            store_wt(ldst, o[0]); store_wt(ldst + d1, o[1]); store_wt(ldst + d2, o[2]); store_wt(ldst + d2 + d1, o[3]);
            if (!f1) {
                WAIT_V(0);
                if (lane == 0) add_agent(me + OFF_FLAG + u * 128u, 1u);
            }
            if (expo) {
                char* const ed = me + (f1 ? OFF_EXPW + par * WSET_BYTES + (size_t)wu * UNIT : OFF_EXPX + par * XSET_BYTES + (size_t)(s * 16 + rb) * UNIT) + wid * 4096;
                LAS unsigned char* const tail = lds + LDS_ENC + wid * 1024;
                unsigned hb[8], e[8], sg = 0;
                u32x4 lo[2], nb;
#pragma unroll
                for (int k = 0; k < 8; ++k) {
                    const unsigned w0 = o[k >> 1][(k & 1) * 2], w1 = o[k >> 1][(k & 1) * 2 + 1];
                    lo[k >> 2][k & 3] = perm(w1, w0, 0x06040200u); hb[k] = perm(w1, w0, 0x07050301u);
                    e[k] = (hb[k] >> 2) & 0x1f1f1f1fu; sg |= (hb[k] & B80) >> k;
                }
                const unsigned info = f1 && n < RAW0 ? INFO_RAW : exp_encode(e, lane, tail, 768u);
                if (info & INFO_RAW) {
#pragma unroll
                    for (int f = 0; f < 4; ++f) store_wt(ed + f * 1024 + lane * 16, o[f]);
                } else {
#pragma unroll
                    for (int k = 0; k < 4; ++k)
                        nb[k] = (hb[2 * k] & B03) | (e[2 * k] << 2) | ((hb[2 * k + 1] & B03) << 4) | (e[2 * k + 1] << 6);
                    store_wt(ed + lane * 16, lo[0]); store_wt(ed + 1024 + lane * 16, lo[1]); store_wt(ed + 2048 + lane * 16, nb);
                    st_sys(ed + 3072 + lane * 4, sg);
                    if ((unsigned)lane < ((info >> 16) & 127u)) store_wt(ed + 3328 + lane * 16, *(volatile LAS u32x4*)(tail + lane * 16));
                }
                if (lane == 0) LDSW(4 + wid) = info;
            }
            WAIT_V(0);
            WAIT_L(0); BAR; asm volatile("" ::: "memory");
            if (wid == 0) {
                if (expo && (f1 ? lane < 8 : fd != rank))
                    st_sys64(fws + OFF_EFLAG + (par * 64u * NBLK + u) * 64u + (lane & 7) * 8, call, LDSW(4 + (lane & 7)));
                if (f1 && lane == 0) st_sys(me + OFF_OFLAG + wu * 128u, call * 8u);
            }
            TS(1 + (n < 8 ? n : 8));
            if (last) break;
            item += f1 ? 1u : 8u;
            load_item(bid, item);
            WAIT_L(0); BAR;
        }
    }
    const bool ownb = aux && remote;
    int tile = slot;
again:
    const int ti = (aux ? rank : peer) * 2 + (tile >> 3), tj = slot & 7;
    const int nsrc = remote ? 6 : 7;
    const int rfirst = aux && !remote ? 4 : -1, rlast = ownb ? -1 : !aux && !remote ? 3 : 7;
    const int nt = remote ? (aux ? 8 : 56) : (aux ? 28 : 36);
    const int wr = wid >> 2, wc = wid & 3;
    const char* const stage = me + OFF_STAGE + (size_t)(call & 1u) * STAGE_BYTES;
    const char* const abase = stage + (unsigned)(8 + ti) * UNIT;
    const char* const bbase = ownb ? me + OFF_OWNB + (size_t)(((peer - rank) * 64 + tj) * (int)UNIT) : stage + (unsigned)tj * UNIT;
    const unsigned bstep = ownb ? 8u * UNIT : NBLK * UNIT;
    const unsigned voff = (unsigned)tid * 16u;
    const unsigned ldsw = (unsigned)wid * 1024u;
    const unsigned aoff = (unsigned)wr * 8192u + (unsigned)lane * 16u;
    unsigned boff = 65536u + (unsigned)wc * 4096u + (unsigned)lane * 16u;
    asm volatile("" : "+v"(boff));

#define SA(b, h) (((b) * 2 + (h)) * 16384)
#define SB(b, h) ((4 + (b) * 2 + (h)) * 16384)
#define STAGE(bufoff, gbase) do { _Pragma("unroll") for (int _i = 0; _i < 2; ++_i) \
        __builtin_amdgcn_global_load_lds((const GAS unsigned*)((gbase) + _i * 8192 + voff), (LAS unsigned*)(lds + (bufoff) + ldsw + _i * 8192), 16, 0, 17); } while (0)
#define LDA(dst, b, h) do { _Pragma("unroll") for (int m = 0; m < 4; ++m) _Pragma("unroll") for (int k = 0; k < 2; ++k) \
        dst[m][k] = *(const LAS f16x8*)(lds + SA(b, h) + aoff + m * 2048 + k * 1024); } while (0)
#define LDB(dst, b, h) do { _Pragma("unroll") for (int n = 0; n < 2; ++n) _Pragma("unroll") for (int k = 0; k < 2; ++k) \
        dst[n][k] = *(const LAS f16x8*)(lds + boff + (SB(b, h) - 65536 + n * 2048 + k * 1024)); } while (0)
#define MMA(ai, bj, At, Bt) do { __builtin_amdgcn_s_setprio(1); \
        _Pragma("unroll") for (int m = 0; m < 4; ++m) _Pragma("unroll") for (int n = 0; n < 2; ++n) _Pragma("unroll") for (int k = 0; k < 2; ++k) \
        acc[ai][bj][m][n] = __builtin_amdgcn_mfma_f32_16x16x32_f16(At[m][k], Bt[n][k], acc[ai][bj][m][n], 0, 0, 0); \
        __builtin_amdgcn_s_setprio(0); } while (0)

    unsigned ktab;
    {
        const int tt = lane < nt ? lane : nt - 2 + (lane & 1), u = rfirst < 0 ? tt - 8 : tt + rfirst * nsrc;
        const int r = u < 0 ? -1 : remote ? u / 6 : u / 7, q = u - r * nsrc;
        ktab = (u < 0 ? rank * 8 + tt : ((rank + q + 1 + (q + 1 >= skip)) & (NR - 1)) * 8 + r) | ((r + 1) << 8);
    }
    auto flag_of = [&](int s, int lz) -> unsigned {
        const int e = (lz >> 1) & 7, q = e < nsrc ? e : 0, sl = min(s + (lz >> 4), rlast);
        const int kt = sl < 0 ? rank * 8 + e : ((rank + q + 1 + (q + 1 >= skip)) & (NR - 1)) * 8 + sl;
        return ownb && !(lz & 1) ? OFF_OFLAG + (unsigned)((peer * 8 + e) * 8 + tj) * 128u
                                 : OFF_FLAG + (unsigned)(kt * NBLK + tj + (lz & 1) * (8 + ti - tj)) * 128u;
    };
    auto ready = [&](int s) {
        if (wid == 0) {
            int lz = lane; asm volatile("" : "+v"(lz));
            wave_wait<POLL>(me + flag_of(s, lz & 15), call * 8u, tmo, call);
            TS(s < 0 ? 30 : 13 + s);
        }
        asm volatile("" ::: "memory"); BAR; asm volatile("" ::: "memory");
    };

    f32x4 acc[2][2][4][2];
#pragma unroll
    for (int a = 0; a < 2; ++a)
#pragma unroll
        for (int b = 0; b < 2; ++b)
#pragma unroll
            for (int m = 0; m < 4; ++m)
#pragma unroll
                for (int n = 0; n < 2; ++n) acc[a][b][m][n] = (f32x4){0.f, 0.f, 0.f, 0.f};
    f16x8 At[4][2], B0[2][2], B1[2][2];

    int have = rfirst;
    ready(rfirst);
    unsigned k2 = __builtin_amdgcn_readlane(ktab, 0) & 63u, k3 = __builtin_amdgcn_readlane(ktab, 1) & 63u;
    {
        const char* a0 = abase + k2 * (NBLK * UNIT); const char* b0 = bbase + k2 * bstep;
        const char* a1 = abase + k3 * (NBLK * UNIT); const char* b1 = bbase + k3 * bstep;
        WAIT_V(0);
        STAGE(SB(0, 0), b0); STAGE(SA(0, 0), a0); STAGE(SB(0, 1), b0 + 16384); STAGE(SA(0, 1), a0 + 16384);
        if (wr == 1) BAR;
        WAIT_V(4); BAR;
        STAGE(SB(1, 0), b1); STAGE(SA(1, 0), a1); STAGE(SB(1, 1), b1 + 16384);
        WAIT_V(6); BAR;
    }
#pragma unroll 1
    for (int t = 0; t < nt; t += 2) {
        const char* a1 = abase + k3 * (NBLK * UNIT);
        k2 = __builtin_amdgcn_readlane(ktab, t + 2) & 63u; k3 = __builtin_amdgcn_readlane(ktab, t + 3);
        const int round3 = (int)(k3 >> 8) - 1, round5 = (int)((unsigned)__builtin_amdgcn_readlane(ktab, t + 5) >> 8) - 1;
        k3 &= 63u;
        {
            if (round3 > have) {
                int lz = lane; asm volatile("" : "+v"(lz));
                const u64 miss = ~__ballot(*(volatile LAS unsigned*)(lds + LDS_ENC + (t & 2) * 128 + lz * 4) >= call * 8u);
                if (miss & 0xffffull) { ready(round3); have = round3; }
                else have = min(round3 + (miss & 0xffff0000ull ? 0 : 1), rlast);
            }
        }
        const char* a2 = abase + k2 * (NBLK * UNIT); const char* b2 = bbase + k2 * bstep;
        const char* a3 = abase + k3 * (NBLK * UNIT); const char* b3 = bbase + k3 * bstep;
        LDB(B0, 0, 0); SCHED; LDA(At, 0, 0); STAGE(SA(1, 1), a1 + 16384);
        WAIT_L(8); BAR; WAIT_L(0); MMA(0, 0, At, B0); BAR; SCHED;
        LDB(B1, 0, 1); STAGE(SB(0, 0), b2);
        BAR; WAIT_L(0); MMA(0, 1, At, B1); BAR;
        LDA(At, 0, 1); STAGE(SA(0, 0), a2);
        BAR; WAIT_L(0); MMA(1, 0, At, B0); BAR; SCHED;
        STAGE(SB(0, 1), b2 + 16384);
        WAIT_V(6); BAR; MMA(1, 1, At, B1); BAR;
        if (wid == 0 && round5 > have) {
            int lz = lane; asm volatile("" : "+v"(lz));
            if (lz < 32)
                __builtin_amdgcn_global_load_lds((const GAS unsigned*)(me + flag_of(have + 1, lz)), (LAS unsigned*)(lds + LDS_ENC + (~t & 2) * 128), 4, 0, 17);
        }
        LDB(B0, 1, 0); SCHED; LDA(At, 1, 0); STAGE(SA(0, 1), a2 + 16384);
        WAIT_L(8); BAR; WAIT_L(0); MMA(0, 0, At, B0); BAR; SCHED;
        LDB(B1, 1, 1); STAGE(SB(1, 0), b3);
        BAR; WAIT_L(0); MMA(0, 1, At, B1); BAR;
        LDA(At, 1, 1); STAGE(SA(1, 0), a3);
        BAR; WAIT_L(0); MMA(1, 0, At, B0); BAR; SCHED;
        STAGE(SB(1, 1), b3 + 16384);
        WAIT_V(6); BAR; MMA(1, 1, At, B1); BAR;
    }
    if (wr == 0) BAR;
    TS(21);
    int tz = threadIdx.x; asm volatile("" : "+v"(tz));
    const int lane2 = tz & 63;
    const float qnan = __builtin_nanf("");
#define ACC1(g, q, c) acc[(g) >> 1][(c) >> 1][((g) & 1) * 2 + ((q) >> 2)][(c) & 1][(q) & 3]
#define ACC(g, q) ((f32x4){ACC1(g, q, 0), ACC1(g, q, 1), ACC1(g, q, 2), ACC1(g, q, 3)})
    const unsigned foff = OFF_PFLAG + (unsigned)wid * 256u + (call & 1u) * 128u;
    const size_t poff = OFF_PART + (size_t)(call & 1u) * PART_BYTES + (size_t)slot * TILE_BYTES + wid * 32768 + lane2 * 16;
    if (!aux) {
        char* const dst = me + poff + (size_t)peer * 16 * TILE_BYTES;
        LAS unsigned char* const tail = lds + LDS_ENC + wid * 1024;
        char* const flag = pws + foff + (unsigned)(rank * 16 + slot) * 2048u;
        unsigned prev = 0, badbit = 0;
#pragma unroll
        for (int g = 0; g < 4; ++g) {
            unsigned e[8], pw_[24];
#pragma unroll
            for (int q = 0; q < 8; ++q) {
                const u32x4 u = __builtin_bit_cast(u32x4, ACC(g, q)), r = (u << 1) | (u >> 31);
                e[q] = perm(perm(r.w, r.z, 0x0c0c0703u), perm(r.y, r.x, 0x0c0c0703u), 0x05040100u);
                pw_[3 * q] = perm(r.w, r.x, 0x04020100u); pw_[3 * q + 1] = perm(r.w, r.y, 0x05020100u); pw_[3 * q + 2] = perm(r.w, r.z, 0x06020100u);
            }
            const unsigned info = remote ? exp_encode(e, lane2, tail, 512u) : INFO_RAW;
            if (g >= 2) {
                WAIT_V(0);
                if (lane2 == 0) st_sys64(flag + (g - 1) * 8, call, prev);
            }
            if (info & INFO_RAW) {
#pragma unroll
                for (int q = 0; q < 8; ++q) store_wt(dst + g * 8192 + q * 1024, __builtin_bit_cast(u32x4, ACC(g, q)));
            } else {
#pragma unroll
                for (int q = 0; q < 6; ++q) store_wt(dst + g * 8192 + q * 1024, (u32x4){pw_[4 * q], pw_[4 * q + 1], pw_[4 * q + 2], pw_[4 * q + 3]});
                st_sys64(dst + g * 8192 + 6144 - lane2 * 8, e[0] | (e[1] << 2) | (e[2] << 4) | (e[3] << 6), e[4] | (e[5] << 2) | (e[6] << 4) | (e[7] << 6));
                if ((unsigned)lane2 < ((info >> 16) & 127u)) store_wt(dst + g * 8192 + 6656, *(volatile LAS u32x4*)(tail + lane2 * 16));
            }
            if (g == 0) badbit = ld_agent(tmo) == call ? INFO_BAD : 0u;
            prev = info | badbit;
            if (g == 0 || g == 3) {
                WAIT_V(0);
                if (lane2 == 0) st_sys64(flag + g * 8, call, prev);
            }
        }
        TS(23);
        return;
    }
    const char* const src = me + poff + (size_t)rank * 16 * TILE_BYTES;
    float* const o = P.out + (size_t)((tile >> 3) * 256 + wr * 64 + (lane2 >> 4) * 4) * NFULL + peer * NS + tj * 256 + wc * 64 + (lane2 & 15) * 4;
    if (!remote) wave_wait64<POLL>(me + foff + (unsigned)(peer * 16 + slot) * 2048u + (lane2 & 3) * 8, call, tmo, call);
    const bool bad = ld_agent(tmo) == call;
#pragma unroll
    for (int g = 0; g < 4; ++g) {
        f32x4 y[8];
#pragma unroll
        for (int q = 0; q < 8; ++q) y[q] = bad ? (f32x4)qnan : ACC(g, q);
        if (!remote) {
            u32x4 w[8];
            asm volatile("global_load_dwordx4 %0, %8, off sc0 sc1\n\t"
                         "global_load_dwordx4 %1, %8, off offset:1024 sc0 sc1\n\t"
                         "global_load_dwordx4 %2, %8, off offset:2048 sc0 sc1\n\t"
                         "global_load_dwordx4 %3, %8, off offset:3072 sc0 sc1\n\t"
                         "global_load_dwordx4 %4, %9, off sc0 sc1\n\t"
                         "global_load_dwordx4 %5, %9, off offset:1024 sc0 sc1\n\t"
                         "global_load_dwordx4 %6, %9, off offset:2048 sc0 sc1\n\t"
                         "global_load_dwordx4 %7, %9, off offset:3072 sc0 sc1\n\ts_waitcnt vmcnt(0)"
                         : "=&v"(w[0]), "=&v"(w[1]), "=&v"(w[2]), "=&v"(w[3]), "=&v"(w[4]), "=&v"(w[5]), "=&v"(w[6]), "=&v"(w[7])
                         : "v"(src + g * 8192), "v"(src + g * 8192 + 4096) : "memory");
#pragma unroll
            for (int q = 0; q < 8; ++q) y[q] += __builtin_bit_cast(f32x4, w[q]);
            gelu8(y);
        }
#pragma unroll
        for (int q = 0; q < 8; ++q) {
            if (remote) asm volatile("s_waitcnt vmcnt(%0)" :: "i"(F1DEPTH) : "memory");
            store_wt(o + (size_t)((g >> 1) * 128 + (g & 1) * 32 + (q >> 2) * 16 + (q & 3)) * NFULL, __builtin_bit_cast(u32x4, y[q]));
        }
    }
    WAIT_V(0);
    TS(28);
    if (remote) {
        if (lane2 == 0) st_sys(me + OFF_OWNF + (unsigned)((peer * 16 + tile) * 8 + wid) * 64u, call);
        if (tile < 8) {
            for (int i = 0; i < F1GO; ++i) __builtin_amdgcn_s_sleep(127);
            tile += 8;
            goto again;
        }
    }
}

extern "C" void kernel_launch(int rank, int nranks, ncclComm_t comm,
                              hipStream_t stream,
                              void* const* d_in, const int* in_sizes, int n_in,
                              void* d_out, int out_size,
                              const int* in_dtypes, int out_dtype,
                              void* const* d_out_all,
                              void* const* d_ws_all, size_t ws_bytes) {
    (void)comm; (void)in_dtypes; (void)out_dtype; (void)d_out_all;
    if (nranks != NR || (unsigned)rank >= (unsigned)NR || n_in != 2 || ws_bytes < WS_NEED
        || in_sizes[0] != 4096 * KS || in_sizes[1] != KS * NFULL || out_size != 512 * NFULL) {
        fprintf(stderr, "kernel_launch: unexpected problem shape\n");
        return;
    }
    Params P = {};
    P.x = (const float*)d_in[0];
    P.w = (const float*)d_in[1];
    P.out = (float*)d_out;
    for (int p = 0; p < NR; ++p) P.ws[p] = (char*)d_ws_all[p];
    P.me = P.ws[rank];
    P.rank = rank;
    static const unsigned char port[NR][NR] = {
        {0, 2, 3, 6, 4, 7, 5, 1}, {1, 4, 5, 7, 6, 2, 3, 0}, {2, 0, 6, 3, 7, 1, 5, 4}, {3, 6, 0, 2, 7, 4, 1, 5},
        {4, 1, 7, 5, 6, 3, 0, 2}, {5, 7, 1, 4, 0, 6, 2, 3}, {6, 3, 2, 0, 1, 5, 4, 7}, {7, 5, 4, 1, 2, 0, 3, 6}};
    for (int c = 0; c < NR; ++c) P.peermap |= (unsigned)port[rank][c] << (4 * c);
    unsigned n = 0, sw = 0, sa = 1, sb = 0, so = 1;
    for (const char* c = ORDER; *c; ++c) {
        unsigned& s = *c == 'w' ? sw : *c == 'a' ? sa : *c == 'b' ? sb : so;
        const unsigned first = *c == 'w' ? 0 : *c == 'a' ? 8 : *c == 'b' ? 14 : 20, cnt = *c == 'w' ? 8 : *c == 'o' ? 2 : 6;
        for (unsigned i = 0; i < cnt && n < NPULL; ++i, ++n) P.order[n >> 2] |= (((s & 7u) << 5) | (first + i)) << ((n & 3u) * 8u);
        ++s;
    }
    if (n != NPULL || sw != 8 || sa != 8 || sb != 8 || so != 8) { fprintf(stderr, "kernel_launch: bad ORDER\n"); return; }
    gemm_rs_kernel<<<256, 512, 0, stream>>>(P);
    HIP_CHECK(hipGetLastError());
}
```

```cpp
#include <hip/hip_runtime.h>
#include <rccl/rccl.h>
#include <cstdio>

#define HIP_CHECK(expr) do { hipError_t _e = (expr); if (_e != hipSuccess) { \
    fprintf(stderr, "HIP error %s at %s:%d\n", hipGetErrorString(_e), __FILE__, __LINE__); \
    return; } } while (0)

#define LAS __attribute__((address_space(3)))
#define GAS __attribute__((address_space(1)))
typedef _Float16 f16x8 __attribute__((ext_vector_type(8)));
typedef float f32x4 __attribute__((ext_vector_type(4)));
typedef unsigned u32x4 __attribute__((ext_vector_type(4)));
typedef unsigned u32x2 __attribute__((ext_vector_type(2)));
typedef unsigned long long u64;

namespace {

constexpr const char* ORDER = "wawbwabwawabwabwawabbbbooooooo";
constexpr int NC = 6;
constexpr int WIN = 7;
constexpr int FASTW = 2;
constexpr int F1DEPTH = 1;
constexpr int F1GO = 8;
constexpr int MAPW = 2;
constexpr int POLL = 8;
constexpr int LATE = 80;
static_assert(WIN >= 4 && NC <= 7, "loader bookkeeping");

constexpr int NR = 8;
constexpr int KS = 512;
constexpr int NFULL = 16384;
constexpr int NS = 2048;
constexpr int NBLK = 24;
constexpr unsigned UNIT = 32768;
constexpr unsigned TILE_BYTES = 262144;
constexpr unsigned NPULL = 168, NSLICE = NPULL * 8, NWORK = NSLICE + 512;
constexpr size_t CTRL_BYTES = 1u << 20;
constexpr size_t STAGE_BYTES = (size_t)64 * NBLK * UNIT;
constexpr size_t PART_BYTES = (size_t)NR * 16 * TILE_BYTES;
constexpr size_t WSET_BYTES = (size_t)NR * 64 * UNIT;
constexpr size_t XSET_BYTES = (size_t)128 * UNIT;
constexpr size_t OFF_STAGE = CTRL_BYTES, OFF_PART = OFF_STAGE + 2 * STAGE_BYTES, OFF_OWNB = OFF_PART + 2 * PART_BYTES,
                 OFF_EXPW = OFF_OWNB + WSET_BYTES, OFF_EXPX = OFF_EXPW + 2 * WSET_BYTES, WS_NEED = OFF_EXPX + 2 * XSET_BYTES;
constexpr unsigned OFF_TMO = 0, OFF_QUEUE = 2048  , OFF_CNT = 4096  ,
                   OFF_FLAG = 65536  , OFF_PFLAG = 262144  ,
                   OFF_OFLAG = 524288  , OFF_OWNF = 589824  ,
                   OFF_EFLAG = 655360  ;
constexpr u64 DEADLINE_TICKS = 50000000ull;
constexpr unsigned LDS_STAGE = 131072, LDS_BYTES = LDS_STAGE + 384 + 8192;
constexpr unsigned LDS_ENC = LDS_STAGE + 384;

struct Params {
    const float* x;
    const float* w;
    char* me;
    int rank;
    unsigned peermap;
    float* out;
    char* ws[NR];
    unsigned order[48];
};

__device__ __forceinline__ void store_wt(void* p, u32x4 v) {
    asm volatile("global_store_dwordx4 %0, %1, off sc0 sc1\n\ts_nop 1" :: "v"(p), "v"(v) : "memory");
}
__device__ __forceinline__ unsigned pack2(float a, float b) {
    typedef _Float16 h2 __attribute__((ext_vector_type(2)));
    h2 h; h.x = (_Float16)a; h.y = (_Float16)b;
    return __builtin_bit_cast(unsigned, h);
}
__device__ __forceinline__ void gelu8(f32x4 (&y)[8]) {
    const float k1 = -2.0f * 0.7978845608028654f * 1.4426950408889634f, k2 = k1 * 0.044715f;
    f32x4 e[8];
#pragma unroll
    for (int q = 0; q < 8; ++q) e[q] = y[q] * __builtin_elementwise_fma(y[q] * y[q], (f32x4)k2, (f32x4)k1);
    __builtin_amdgcn_sched_barrier(0);
#pragma unroll
    for (int i = 0; i < 32; ++i) e[i >> 2][i & 3] = __builtin_amdgcn_exp2f(e[i >> 2][i & 3]);
    __builtin_amdgcn_sched_barrier(0);
#pragma unroll
    for (int q = 0; q < 8; ++q) e[q] += 1.0f;
    __builtin_amdgcn_sched_barrier(0);
#pragma unroll
    for (int i = 0; i < 32; ++i) e[i >> 2][i & 3] = __builtin_amdgcn_rcpf(e[i >> 2][i & 3]);
    __builtin_amdgcn_sched_barrier(0);
#pragma unroll
    for (int q = 0; q < 8; ++q) y[q] *= e[q];
}
__device__ __forceinline__ unsigned wave_scan(unsigned v) {
    v += __builtin_amdgcn_update_dpp(0, (int)v, 0x111, 0xf, 0xf, true);
    v += __builtin_amdgcn_update_dpp(0, (int)v, 0x112, 0xf, 0xf, true);
    v += __builtin_amdgcn_update_dpp(0, (int)v, 0x114, 0xf, 0xf, true);
    v += __builtin_amdgcn_update_dpp(0, (int)v, 0x118, 0xf, 0xf, true);
    v += __builtin_amdgcn_update_dpp(0, (int)v, 0x142, 0xa, 0xf, false);
    v += __builtin_amdgcn_update_dpp(0, (int)v, 0x143, 0xc, 0xf, false);
    return v;
}
__device__ __forceinline__ unsigned ld_sys(const char* p) {
    return __hip_atomic_load((const GAS unsigned*)p, __ATOMIC_RELAXED, __HIP_MEMORY_SCOPE_SYSTEM);
}
__device__ __forceinline__ u64 ld_sys64(const char* p) {
    return __hip_atomic_load((const GAS u64*)p, __ATOMIC_RELAXED, __HIP_MEMORY_SCOPE_SYSTEM);
}
__device__ __forceinline__ unsigned ld_agent(const char* p) {
    return __hip_atomic_load((const GAS unsigned*)p, __ATOMIC_RELAXED, __HIP_MEMORY_SCOPE_AGENT);
}
__device__ __forceinline__ void st_sys(char* p, unsigned v) {
    __hip_atomic_store((GAS unsigned*)p, v, __ATOMIC_RELAXED, __HIP_MEMORY_SCOPE_SYSTEM);
}
__device__ __forceinline__ void st_sys64(char* p, unsigned lo, unsigned hi) {
    __hip_atomic_store((GAS u64*)p, ((u64)hi << 32) | lo, __ATOMIC_RELAXED, __HIP_MEMORY_SCOPE_SYSTEM);
}
__device__ __forceinline__ unsigned add_agent(char* p, unsigned v) {
    return __hip_atomic_fetch_add((GAS unsigned*)p, v, __ATOMIC_RELAXED, __HIP_MEMORY_SCOPE_AGENT);
}
template <int SLEEP> __device__ __forceinline__ bool give_up(unsigned& n, u64 t0, char* tmo, unsigned call) {
    if ((++n & 15u) == 0u
        && ((unsigned)__builtin_amdgcn_readfirstlane(ld_agent(tmo)) == call || __builtin_amdgcn_s_memrealtime() - t0 > DEADLINE_TICKS)) {
        __hip_atomic_store((GAS unsigned*)tmo, call, __ATOMIC_RELAXED, __HIP_MEMORY_SCOPE_AGENT);
        return true;
    }
    __builtin_amdgcn_s_sleep(SLEEP);
    return false;
}
template <int SLEEP> __device__ __forceinline__ void wave_wait(const char* word, unsigned need, char* tmo, unsigned call) {
    const u64 t0 = __builtin_amdgcn_s_memrealtime();
    unsigned n = 0;
    while (!__all(ld_sys(word) >= need) && !give_up<SLEEP>(n, t0, tmo, call)) { }
    asm volatile("" ::: "memory");
}
template <int SLEEP> __device__ __forceinline__ unsigned wave_wait64(const char* word, unsigned need, char* tmo, unsigned call) {
    const u64 t0 = __builtin_amdgcn_s_memrealtime();
    unsigned n = 0;
    u64 v;
    do { v = ld_sys64(word); } while (!__all((unsigned)v >= need) && !give_up<SLEEP>(n, t0, tmo, call));
    asm volatile("" ::: "memory");
    return (unsigned)(v >> 32);
}

constexpr unsigned B01 = 0x01010101u, B03 = 0x03030303u, B07 = 0x07070707u, B80 = 0x80808080u, INFO_RAW = 1u << 23, INFO_BAD = 1u << 24;
__device__ __forceinline__ unsigned sub8(unsigned a, unsigned b) { return ((a | B80) - (b & ~B80)) ^ ((a ^ ~b) & B80); }
__device__ __forceinline__ unsigned perm(unsigned hi, unsigned lo, unsigned sel) { return __builtin_amdgcn_perm(hi, lo, sel); }
__device__ __forceinline__ void lds_or(LAS unsigned char* p, unsigned v) {
    __hip_atomic_fetch_or((LAS unsigned*)p, v, __ATOMIC_RELAXED, __HIP_MEMORY_SCOPE_WORKGROUP);
}
__device__ __forceinline__ unsigned prefix4(unsigned inc) {
    unsigned t = inc + (inc << 8);
    asm("" : "+v"(t));
    return t + (t << 16);
}
__device__ __forceinline__ unsigned exp_encode(unsigned (&e)[8], int lane, LAS unsigned char* tail, unsigned cap) {
    *(volatile LAS u32x4*)(tail + lane * 16) = (u32x4){0u, 0u, 0u, 0u};
    unsigned sum = 0;
#pragma unroll
    for (int k = 0; k < 8; ++k) sum = __builtin_amdgcn_sad_u8(e[k], 0u, sum);
    const unsigned c = ((__builtin_amdgcn_readlane(wave_scan(sum), 63) >> 11) + 3u) & 255u;
    const unsigned c4 = ((c & 15u) | 16u) * B01, r0 = sub8(c * B01, 0x03020100u), r1 = sub8(c * B01, 0x030a0908u), r2 = sub8(c * B01, 0x07060504u);
    unsigned acc[4], len[4], fix[8], any = 0;
#pragma unroll
    for (int k = 0; k < 8; ++k) {
        const unsigned h = c4 - (e[k] & 0x0f0f0f0fu), sel = h & B07, m = perm(0u, 0u, ((h >> 3) & B01) | 0x0c0c0c0cu);
        const unsigned cs = (perm(0x4c3c2c1cu, 0x037c6c5cu, sel) & m) | (perm(0x4c3c2c1cu, 0x0302010cu, sel) & ~m);
        fix[k] = e[k] ^ ((perm(r2, r1, sel) & m) | (perm(r2, r0, sel) & ~m));
        any |= fix[k];
        const unsigned sy = (cs >> 4) & B07, in = prefix4((cs >> 2) & B03);
        const unsigned ch = (sy & 255u) | (((sy >> 8) & 255u) << (in & 255u)) | (((sy >> 16) & 255u) << ((in >> 8) & 255u)) | ((sy >> 24) << ((in >> 16) & 255u));
        if (k & 1) { acc[k >> 1] |= ch << len[k >> 1]; len[k >> 1] += in >> 24; } else { acc[k >> 1] = ch; len[k >> 1] = in >> 24; }
        e[k] = cs & B03;
    }
    const unsigned mine = (len[0] + len[1]) + (len[2] + len[3]), incl = wave_scan(mine);
    unsigned p = incl - mine;
#pragma unroll
    for (int g = 0; g < 4; ++g) {
        const u64 t = (u64)acc[g] << (p & 31u);
        lds_or(tail + (p >> 5) * 4u, (unsigned)t); lds_or(tail + (p >> 5) * 4u + 4u, (unsigned)(t >> 32));
        p += len[g];
    }
    const unsigned sbytes = ((__builtin_amdgcn_readlane(incl, 63) + 31u) >> 5) * 4u;
    unsigned n = 0;
    u64 has;
    while ((has = __ballot(any != 0u)) != 0ull && n <= 64u) {
        unsigned f = 0, kk = 0;
        any = 0;
#pragma unroll
        for (int k = 7; k >= 0; --k) if (fix[k] != 0u) { f = fix[k]; kk = k; }
        const unsigned sh = __builtin_ctz(f | 0x80000000u) & 24u;
        const unsigned idx = n + __builtin_amdgcn_mbcnt_hi((unsigned)(has >> 32), __builtin_amdgcn_mbcnt_lo((unsigned)has, 0u));
        if (f != 0u && idx < 64u)
            *(volatile LAS unsigned short*)(tail + sbytes + idx * 2u) = (lane << 10) | (kk << 7) | (sh << 2) | ((f >> (sh + 3u)) & 31u);
#pragma unroll
        for (int k = 0; k < 8; ++k) { if (kk == (unsigned)k) fix[k] = f & ~(255u << sh); any |= fix[k]; }
        n += __builtin_popcountll(has);
    }
    const unsigned bytes = sbytes + n * 2u;
    return n > 64u || bytes > cap ? INFO_RAW : c | (n << 8) | (((bytes + 15u) >> 4) << 16);
}
__device__ __forceinline__ void exp_decode(unsigned (&e)[8], unsigned info, int lane, const LAS unsigned char* tail, LAS unsigned char* scratch) {
    *(volatile LAS u32x4*)(scratch + lane * 32) = (u32x4){0u, 0u, 0u, 0u};
    *(volatile LAS u32x4*)(scratch + lane * 32 + 16) = (u32x4){0u, 0u, 0u, 0u};
    unsigned m[8], in[8], p[4], ch[8], sy[8], mine = 0;
#pragma unroll
    for (int k = 0; k < 8; ++k) {
        m[k] = perm(0u, 0xffu, e[k]);
        in[k] = prefix4(m[k] & B03);
        if (!(k & 1)) p[k >> 1] = mine;
        mine += in[k] >> 24;
    }
    const unsigned incl = wave_scan(mine), sbytes = ((__builtin_amdgcn_readlane(incl, 63) + 31u) >> 5) * 4u;
    unsigned en = 0;
    if ((unsigned)lane < ((info >> 8) & 127u)) en = *(const volatile LAS unsigned short*)(tail + sbytes + lane * 2);
#pragma unroll
    for (int g = 0; g < 4; ++g) {
        p[g] += incl - mine;
        const LAS unsigned* const q = (const LAS unsigned*)(tail + (p[g] >> 5) * 4u);
        ch[2 * g] = __builtin_amdgcn_alignbit(q[1], q[0], p[g]);
        ch[2 * g + 1] = ch[2 * g] >> (in[2 * g] >> 24);
    }
    if (en & 31u) lds_or(scratch + (en >> 7) * 4u, (en & 31u) << (((en >> 2) & 24u) + 3u));
    const u32x4 f0 = *(volatile LAS u32x4*)(scratch + lane * 32), f1 = *(volatile LAS u32x4*)(scratch + lane * 32 + 16);
#define SPREAD1(d, i, c, j, jm) "v_lshrrev_b32_sdwa " d ", " i ", " c " dst_sel:BYTE_" #j " dst_unused:UNUSED_PRESERVE src0_sel:BYTE_" #jm " src1_sel:DWORD\n\t"
#define SPREAD4(j, jm) SPREAD1("%0", "%4", "%8", j, jm) SPREAD1("%1", "%5", "%9", j, jm) SPREAD1("%2", "%6", "%10", j, jm) SPREAD1("%3", "%7", "%11", j, jm)
#pragma unroll
    for (int k = 0; k < 8; k += 4) {
        sy[k] = ch[k]; sy[k + 1] = ch[k + 1]; sy[k + 2] = ch[k + 2]; sy[k + 3] = ch[k + 3];
        asm(SPREAD4(1, 0) SPREAD4(2, 1) SPREAD4(3, 2) "s_nop 0"
            : "+&v"(sy[k]), "+&v"(sy[k + 1]), "+&v"(sy[k + 2]), "+&v"(sy[k + 3])
            : "v"(in[k]), "v"(in[k + 1]), "v"(in[k + 2]), "v"(in[k + 3]), "v"(ch[k]), "v"(ch[k + 1]), "v"(ch[k + 2]), "v"(ch[k + 3]));
    }
    const unsigned c4 = (info & 255u) * B01, tc = sub8(c4, 0x03020100u), t0 = sub8(c4, 0x06050400u), t1 = sub8(c4, 0x0a090807u);
#pragma unroll
    for (int k = 0; k < 8; ++k)
        e[k] = ((perm(t1, t0, sy[k] & B07) & m[k]) | (perm(0u, tc, e[k]) & ~m[k])) ^ (k < 4 ? f0[k & 3] : f1[k & 3]);
}

#define WAIT_V(n) asm volatile("s_waitcnt vmcnt(" #n ")" ::: "memory")
#define WAIT_L(n) asm volatile("s_waitcnt lgkmcnt(" #n ")" ::: "memory")
#define BAR __builtin_amdgcn_s_barrier()
#define SCHED __builtin_amdgcn_sched_barrier(0)
#define LDSW(i) (*(volatile LAS unsigned*)(lds + LDS_STAGE + 4 * (i)))

}

__global__ __launch_bounds__(512, 2) void gemm_rs_kernel(Params P) {
    __shared__ __attribute__((aligned(16))) unsigned char shm[LDS_BYTES];
    LAS unsigned char* lds = (LAS unsigned char*)shm;
    const int tid = threadIdx.x, lane = tid & 63, wid = __builtin_amdgcn_readfirstlane(tid >> 6);
    const int hb = __builtin_amdgcn_readfirstlane(blockIdx.x);
    const float* px = P.x; const float* pw = P.w; char* me_ = P.me; int rank_ = P.rank; unsigned peermap = P.peermap;
    asm volatile("" : "+s"(px), "+s"(pw), "+s"(me_), "+s"(rank_), "+s"(peermap));
    const int rank = rank_ & (NR - 1);
    char* const me = (char*)(GAS char*)(size_t)me_;
    char* const tmo = me + OFF_TMO;
    int bid = (hb & ~7) | (__builtin_amdgcn_s_getreg((3 << 11) | 20) & 7);
    if (bid >= 128 && (bid & 7) != 0) {
        const int tt = ((bid & 7) - 1) * 8 + ((bid >> 3) & 7), cg = tt / (7 * MAPW), r = tt - cg * (7 * MAPW);
        bid = (bid & 192) | ((cg * MAPW + r % MAPW) << 3) | (r / MAPW + 1);
    }
    f32x4 v[8] = {};
    auto first_item = [&](int b) -> unsigned {
        const int q = ((b >> 3) & 7) + 1, sk = ((peermap >> ((b & 7) * 4)) - rank) & (NR - 1);
        const int pl = q < 7 ? (rank + q + (q >= sk)) & (NR - 1) : ((rank + sk) & (NR - 1)) + (q & 8);
        const int lo = (b & 7) == 0 ? (b >> 6) * 16 + 7 + q : b < 64 ? 0 : pl;
        const int hi = b >= 192 ? (q - 1) * 16 + (b & 7) : b;
        return b >= 128 ? hi : lo;
    };
    auto load_item = [&](int b, unsigned item) {
        const bool f1 = b < 64 && (b & 7) != 0, isw = f1 || item >= 128u;
        const unsigned ws_ = f1 ? item : (item >> 3) & 7u, wj = (f1 ? b >> 3 : item) & 7u, wd = f1 ? (peermap >> ((b & 7) * 4)) & (NR - 1) : rank;
        const unsigned wo = (ws_ * 64u + (wid >> 2) * 32u) * NFULL + wd * NS + wj * 256u + (wid & 3) * 64u;
        const unsigned xo = (((item & 7u) * 2u + ((item >> 3) & 1u)) * 256u + wid * 32u) * KS + ((item >> 4) & 7u) * 64u;
        const float* const base = isw ? pw + wo : px + xo;
        unsigned lo = (lane >> 4) * (isw ? 16u * NFULL : 16u) + (lane & 15) * (isw ? 16u : 4u * KS);
        const unsigned sb = isw ? 4u * NFULL : 64u, sa = (isw ? 64u * NFULL : 64u * KS) - 3u * sb;
#define LD1(i, step) "global_load_dwordx4 %" #i ", %8, %9\n\tv_add_u32 %8, " step ", %8\n\t"
        asm volatile("s_nop 4\n\t" LD1(0, "%10") LD1(1, "%10") LD1(2, "%10") LD1(3, "%11") LD1(4, "%10") LD1(5, "%10") LD1(6, "%10")
                     "global_load_dwordx4 %7, %8, %9"
                     : "+v"(v[0]), "+v"(v[1]), "+v"(v[2]), "+v"(v[3]), "+v"(v[4]), "+v"(v[5]), "+v"(v[6]), "+v"(v[7]), "+v"(lo)
                     : "s"(base), "s"(sb), "s"(sa) : "memory");
    };
#define VWAIT(n) asm volatile("s_waitcnt vmcnt(" #n ")" : "+v"(v[0]), "+v"(v[1]), "+v"(v[2]), "+v"(v[3]), "+v"(v[4]), "+v"(v[5]), "+v"(v[6]), "+v"(v[7]) :: "memory")
    unsigned t = 0, ordw[3];
    char* lws;
    {
        u64 ex;
        asm volatile("s_mov_b64 %5, exec\n\ts_mov_b64 exec, %12\n\t"
                     "global_atomic_add %0, %6, %7, %8 sc0\n\ts_mov_b64 exec, %5\n\t"
                     "global_load_dwordx2 %1, %9, %11 offset:%13\n\t"
                     "global_load_dword %2, %10, %11 offset:%14\n\t"
                     "global_load_dword %3, %10, %11 offset:%14+64\n\t"
                     "global_load_dword %4, %10, %11 offset:%14+128"
                     : "+v"(t), "=&v"(lws), "=&v"(ordw[0]), "=&v"(ordw[1]), "=&v"(ordw[2]), "=&s"(ex)
                     : "v"(OFF_CNT + (lane ? bid * 128 + 4 : hb * 128)), "v"(1u), "s"(me), "v"(lane & 56), "v"(lane & 60),
                       "s"(__builtin_amdgcn_kernarg_segment_ptr()), "s"((u64)(wid == 0 ? 3u : 0u)),
                       "i"(__builtin_offsetof(Params, ws)), "i"(__builtin_offsetof(Params, order)) : "memory");
    }
    if (bid >= 128 ? (bid >> 4) != 12 : (bid & 7) == 0) __builtin_amdgcn_s_sleep(LATE);
    char* pws_;
    unsigned call = 0;
#pragma unroll 1
    for (;;) {
        asm volatile("" ::: "memory");
        load_item(bid, first_item(bid));
        pws_ = P.ws[(peermap >> ((bid & 7) * 4)) & (NR - 1)];
        asm volatile("s_waitcnt vmcnt(8)" : "+v"(t), "+v"(lws), "+v"(ordw[0]), "+v"(ordw[1]), "+v"(ordw[2]) :: "memory");
        if (wid == 0 && call == 0u) {
            const unsigned c = __builtin_amdgcn_readlane(t, 0) + 1u;
            unsigned old = __builtin_amdgcn_readlane(t, 1), r = bid;
            for (int tries = 0; old != c - 1u && tries < 65536; ++tries) {
                if (lane == 0) add_agent(me + OFF_CNT + r * 128u + 4u, ~0u);
                r = (r + 1u) & 255u;
                if (lane == 0) t = add_agent(me + OFF_CNT + r * 128u + 4u, 1u);
                old = __builtin_amdgcn_readfirstlane(t);
            }
            LDSW(16 + lane) = 0u;
            if (lane == 0) { LDSW(0) = c; LDSW(12) = r; }
        }
        asm volatile("s_waitcnt lgkmcnt(0)" : "+s"(pws_) :: "memory"); BAR; asm volatile("" ::: "memory");
        call = __builtin_amdgcn_readfirstlane(LDSW(0));
        const int got = __builtin_amdgcn_readfirstlane(LDSW(12));
        if (got == bid) break;
        bid = got;
        VWAIT(0);
    }
    const int peer = (peermap >> ((bid & 7) * 4)) & (NR - 1);
    const int slot = (bid >> 3) & 15;
    const bool aux = bid < 128, remote = (bid & 7) != 0;
    const int skip = remote ? (peer - rank) & (NR - 1) : NR;
    char* const pws = (char*)(GAS char*)(size_t)pws_;

    if (aux && remote && slot >= 8) {
        const unsigned par = call & 1u;
        const unsigned pu = (unsigned)(rank * 8 * NBLK + 8) + (first_item(bid) & 7u) * 2u + ((first_item(bid) >> 3) & 1u);
        char* const rdst = pws + OFF_STAGE + par * STAGE_BYTES + (size_t)pu * UNIT + wid * 4096 + lane * 16;
        auto frag = [&](int f) -> u32x4 {
            return (u32x4){pack2(v[2 * f][0], v[2 * f][1]), pack2(v[2 * f][2], v[2 * f][3]),
                           pack2(v[2 * f + 1][0], v[2 * f + 1][1]), pack2(v[2 * f + 1][2], v[2 * f + 1][3])};
        };
        VWAIT(6); store_wt(rdst, frag(0));
        if (wid > FASTW) WAIT_V(0);
        VWAIT(5); store_wt(rdst + 1024, frag(1));
        if (wid > FASTW) WAIT_V(0);
        VWAIT(4); store_wt(rdst + 2048, frag(2));
        VWAIT(3);
        const u32x4 o3 = frag(3);
        auto push_last = [&]() { WAIT_V(0); store_wt(rdst + 3072, o3); };
        auto publish_push = [&]() {
            WAIT_V(0);
            __builtin_amdgcn_s_waitcnt(0x0F70);
            if (lane == 0)
                __hip_atomic_fetch_add((GAS unsigned*)(pws + OFF_FLAG + pu * 128u), 1u, __ATOMIC_RELAXED, __HIP_MEMORY_SCOPE_SYSTEM);
        };
        if (wid == 0) {
            push_last();
            publish_push();
            __builtin_amdgcn_s_setprio(3);
            const unsigned lbase = (unsigned)(size_t)lds, landed = lbase + (lane ? 129024 + lane * 4 : LDS_STAGE + 68);
            unsigned head = 0, alive = NC, idle = 4;
            int issued = 0;
            u32x2 d = *(volatile LAS u32x2*)(lds + LDS_STAGE + 128);
#define PIECE(off) "s_waitcnt vmcnt(%5)\n\tds_max_i32 %4, %1\n\tv_add_u32 %1, 1, %1\n\tglobal_load_lds_dwordx4 %2, off offset:" #off " sc0 sc1\n\t"
#define SHORT(off) "s_waitcnt vmcnt(%5)\n\tds_max_i32 %4, %1\n\tv_add_u32 %1, 1, %1\n\ts_mov_b64 exec, %6\n\t" \
                   "global_load_lds_dwordx4 %2, off offset:" #off " sc0 sc1\n\ts_mov_b64 exec, -1\n\t"
#define ISSUE(body, off) asm volatile("s_mov_b32 %0, m0\n\ts_mov_b32 m0, %3\n\t" body "s_mov_b32 m0, %0" : "=&s"(keep), "+v"(known) \
                                      : "v"(src + off), "s"(dst + off), "v"(landed), "i"(WIN - 1), "s"(mask) : "memory")
            while (alive) {
                if ((unsigned)__builtin_amdgcn_readfirstlane(d.y) != head + 1u) {
                    if (idle == 1u) asm volatile("s_waitcnt vmcnt(6)\n\tds_max_i32 %0, %1" :: "v"(landed), "v"(issued - 6) : "memory");
                    if (idle == 2u) asm volatile("s_waitcnt vmcnt(4)\n\tds_max_i32 %0, %1" :: "v"(landed), "v"(issued - 4) : "memory");
                    if (idle == 3u) asm volatile("s_waitcnt vmcnt(0)\n\tds_max_i32 %0, %1" :: "v"(landed), "v"(issued) : "memory");
                    ++idle;
                    d = *(volatile LAS u32x2*)(lds + LDS_STAGE + 128 + (head & 15u) * 8u);
                    continue;
                }
                ++head;
                const unsigned meta = __builtin_amdgcn_readfirstlane(d.x), cnt = (meta >> 17) & 15u;
                const unsigned dst = lbase + (meta >> 28) * 8192u;
                unsigned keep;
                const char* const src = pws + (meta & 0x1ffffu) * 4096u + lane * 16;
                d = *(volatile LAS u32x2*)(lds + LDS_STAGE + 128 + (head & 15u) * 8u);
                if (cnt == 0u) { --alive; continue; }
                const u64 mask = ~0ull >> (64u - ((meta >> 21) & 127u));
                int known = issued - (WIN - 1);
                if (cnt == 4u) ISSUE(PIECE(0) PIECE(1024) PIECE(2048) SHORT(3072), 0);
                else {
                    ISSUE(PIECE(0) PIECE(1024) PIECE(2048), 0);
                    ISSUE(PIECE(0) PIECE(1024) PIECE(2048) SHORT(3072), 3072);
                    if (cnt == 8u) ISSUE(PIECE(0), 7168);
                }
                issued += cnt;
                idle = 0u;
            }
        } else if (wid <= NC) {
            const unsigned me8 = wid - 1;
            LAS unsigned char* const scratch = lds + 114688 + me8 * 2048u;
            unsigned bi = me8 * 2u + 1u;
            const float qnan = __builtin_nanf("");
            unsigned tbl = 0;
#pragma unroll
            for (int h = 0; h < 3; ++h) {
                const unsigned e = (ordw[h] >> ((lane & 3) * 8)) & 255u, code = e & 31u;
                const int odd = code >= 14u, q = code - (odd ? 14u : 8u);
                const unsigned blk = code < 8u ? code : code >= 20u ? 8u + rank * 2 + (code - 20u)
                                                                     : 8u + ((rank + q + 1 + (q + 1 >= skip)) & (NR - 1)) * 2 + odd;
                tbl |= ((e & 0xe0u) | blk) << (8 * h);
            }
            struct Item { unsigned t, id, flag, src; };
            auto item_of = [&](unsigned t) -> Item {
                const unsigned it = min(t >> 3, NPULL - 1u), e = (unsigned)__builtin_amdgcn_readlane(tbl, it & 63u) >> ((it >> 6) * 8u);
                const unsigned ks = (e >> 5) & 7u, blk = e & 31u, u = (peer * 8u + ks) * NBLK + blk;
                const unsigned so = blk < 8u ? (unsigned)OFF_EXPW + par * (unsigned)WSET_BYTES + ((rank * 8u + ks) * 8u + blk) * UNIT
                                             : (unsigned)OFF_EXPX + par * (unsigned)XSET_BYTES + (ks * 16u + blk - 8u) * UNIT;
                const unsigned i = max(min(t, NWORK - 1u), NSLICE) - NSLICE;
                const unsigned ci = ((i >> 8) * 64u + (i & 63u)) * 4u + ((i >> 6) & 3u);
                Item r;
                r.t = t;
                r.id = t < NSLICE ? u | (blk < 8u ? 0x10000u : 0u) : ci;
                r.flag = t < NSLICE ? OFF_EFLAG + (par * 64u * NBLK + u) * 64u + (t & 7u) * 8u
                                    : OFF_PFLAG + (peer * 128u + (ci >> 2)) * 256u + par * 128u + (ci & 3u) * 8u;
                r.src = t < NSLICE ? so + (t & 7u) * 4096u : (unsigned)OFF_PART + par * (unsigned)PART_BYTES + rank * 16u * TILE_BYTES + ci * 8192u;
                return r;
            };
            auto ownf_of = [&](const Item& x) -> unsigned { return OFF_OWNF + (peer * 128u + (x.t < NSLICE ? 0u : x.id >> 2)) * 64u; };
            auto out_of = [&](const Item& x) -> unsigned {
                const unsigned ct = (x.id >> 5) & 15u, cw = (x.id >> 2) & 7u, cg = x.id & 3u;
                return (((ct >> 3) * 256u + (cg >> 1) * 128u + (cw >> 2) * 64u + (cg & 1u) * 32u) * NFULL + peer * NS + (ct & 7u) * 256u + (cw & 3u) * 64u) * 4u;
            };
            const unsigned olane = ((lane >> 4) * 4u * NFULL + (lane & 15) * 4u) * 4u;
            int rend = 0;
            auto post = [&](unsigned src, unsigned cnt, unsigned last) {
                unsigned k = 0;
                bi ^= 1u;
                if (lane == 0) {
                    asm volatile("ds_add_rtn_u32 %0, %1, %2\n\ts_waitcnt lgkmcnt(0)" : "=&v"(k) : "v"((unsigned)(size_t)lds + LDS_STAGE + 64u), "v"(0x10000u + cnt) : "memory");
                    *(volatile LAS u32x2*)(lds + LDS_STAGE + 128 + ((k >> 16) & 15u) * 8u) = (u32x2){(src >> 12) | (cnt << 17) | (last << 21) | (bi << 28), (k >> 16) + 1u};
                }
                rend = (__builtin_amdgcn_readfirstlane(k) & 0xffffu) + cnt;
            };
            auto post_item = [&](const Item& x, unsigned inf) {
                const bool raw = (inf & INFO_RAW) != 0u, op = x.t < NSLICE;
                post(x.src, op ? 4u : raw ? 8u : 7u, raw ? 64u : min((op ? 16u : 32u) + ((inf >> 16) & 127u), 64u));
            };
            Item A = item_of((slot - 8) * NC + me8), B = item_of(A.t + 8u * NC), C = item_of(A.t + 16u * NC);
            const u64 f0 = ld_sys64(me + A.flag);
            unsigned info = __builtin_amdgcn_readfirstlane((unsigned)(f0 >> 32)), ofl = 0, tm = 0;
            const bool now = __all((unsigned)f0 >= call);
            push_last();
            if (now) post_item(A, info);
            publish_push();
            if (!now) { info = __builtin_amdgcn_readfirstlane(wave_wait64<2>(me + A.flag, call, tmo, call)); post_item(A, info); }
            u64 fl;
            do { fl = ld_sys64(me + B.flag); } while ((unsigned)__builtin_amdgcn_readfirstlane((unsigned)fl) < call && (int)__builtin_amdgcn_readfirstlane(LDSW(17)) < rend);
            unsigned pend = 0;
            u32x4 own[8] = {};
#pragma unroll 1
            for (;;) {
                const bool isp = A.t >= NSLICE, bad = tm == call || (info & INFO_BAD) != 0u;
                if ((int)__builtin_amdgcn_readfirstlane(LDSW(17)) < rend) {
                    if (pend) { WAIT_V(0); if (lane == 0) add_agent(me + pend, 1u); pend = 0; }
                    while ((int)__builtin_amdgcn_readfirstlane(LDSW(17)) < rend) __builtin_amdgcn_s_sleep(1);
                }
                asm volatile("" ::: "memory");
                const LAS unsigned char* const buf = lds + bi * 8192u;
                const bool more = B.t < NWORK, early = more && (unsigned)__builtin_amdgcn_readfirstlane((unsigned)fl) >= call;
                unsigned infob = __builtin_amdgcn_readfirstlane((unsigned)(fl >> 32));
                if (early) post_item(B, infob);
                u32x4 nown[8];
                u64 ex;
                unsigned tk;
                const bool nextp = more && B.t >= NSLICE;
                if (nextp && (unsigned)__builtin_amdgcn_readfirstlane(ofl) < call) wave_wait<2>(me + ownf_of(B), call, tmo, call);
                {
                    unsigned vo = out_of(B) + olane;
                    asm volatile("s_mov_b64 %12, exec\n\ts_mov_b32 exec_lo, %20\n\ts_mov_b32 exec_hi, %20\n\ts_nop 2\n\t"
                                 "global_load_dwordx4 %0, %13, %14 sc0 sc1\n\tv_add_u32 %13, 0x10000, %13\n\t"
                                 "global_load_dwordx4 %1, %13, %14 sc0 sc1\n\tv_add_u32 %13, 0x10000, %13\n\t"
                                 "global_load_dwordx4 %2, %13, %14 sc0 sc1\n\tv_add_u32 %13, 0x10000, %13\n\t"
                                 "global_load_dwordx4 %3, %13, %14 sc0 sc1\n\tv_add_u32 %13, 0xd0000, %13\n\t"
                                 "global_load_dwordx4 %4, %13, %14 sc0 sc1\n\tv_add_u32 %13, 0x10000, %13\n\t"
                                 "global_load_dwordx4 %5, %13, %14 sc0 sc1\n\tv_add_u32 %13, 0x10000, %13\n\t"
                                 "global_load_dwordx4 %6, %13, %14 sc0 sc1\n\tv_add_u32 %13, 0x10000, %13\n\t"
                                 "global_load_dwordx4 %7, %13, %14 sc0 sc1\n\t"
                                 "s_mov_b64 exec, 1\n\t"
                                 "global_load_dwordx2 %8, %16, %15 sc0 sc1\n\t"
                                 "global_load_dword %9, %17, %15 sc0 sc1\n\t"
                                 "global_load_dword %10, %21, %15 sc1\n\t"
                                 "global_atomic_add %11, %18, %19, %15 sc0\n\t"
                                 "s_mov_b64 exec, %12"
                                 : "=&v"(nown[0]), "=&v"(nown[1]), "=&v"(nown[2]), "=&v"(nown[3]), "=&v"(nown[4]), "=&v"(nown[5]), "=&v"(nown[6]), "=&v"(nown[7]),
                                   "=&v"(fl), "=&v"(ofl), "=&v"(tm), "=&v"(tk), "=&s"(ex), "+v"(vo)
                                 : "s"(P.out), "s"(me), "v"(C.flag), "v"(ownf_of(C)), "v"(OFF_QUEUE + peer * 128u), "v"(1u),
                                   "s"(__builtin_amdgcn_readfirstlane(nextp ? -1 : 0)), "v"(OFF_TMO) : "memory");
                }
                unsigned e[8];
                if (!isp) {
                    u32x4 w[4];
#pragma unroll
                    for (int q = 0; q < 4; ++q) w[q] = *(const LAS u32x4*)(buf + q * 1024 + lane * 16);
                    if (!(info & INFO_RAW)) {
                        const unsigned sg = *(const LAS unsigned*)(buf + 3072 + lane * 4);
                        const u32x4 l0 = w[0], l1 = w[1], nb = w[2];
#pragma unroll
                        for (int k = 0; k < 8; ++k) e[k] = (nb[k >> 1] >> (2 + 4 * (k & 1))) & B03;
                        exp_decode(e, info, lane, buf + 3328, scratch);
#pragma unroll
                        for (int k = 0; k < 8; ++k) {
                            const unsigned hb = ((nb[k >> 1] >> (4 * (k & 1))) & B03) | (e[k] << 2) | ((sg << k) & B80), lo = k < 4 ? l0[k & 3] : l1[k & 3];
                            w[k >> 1][(k & 1) * 2] = perm(hb, lo, 0x05010400u); w[k >> 1][(k & 1) * 2 + 1] = perm(hb, lo, 0x07030602u);
                        }
                    }
                    const bool isw = (A.id >> 16) != 0u;
                    const unsigned w8 = A.t & 7u, d1 = isw ? 2048u : 1024u, d2 = isw ? 16384u : 2048u;
                    char* const dst = me + ((unsigned)OFF_STAGE + par * (unsigned)STAGE_BYTES + (A.id & 0xffffu) * UNIT
                                            + (isw ? ((w8 & 3u) * 4u + (w8 >> 2)) * 1024u : w8 * 4096u)) + lane * 16;
                    asm volatile("s_waitcnt vmcnt(0)" : "+v"(w[0]), "+v"(w[1]), "+v"(w[2]), "+v"(w[3]) :: "memory");
                    store_wt(dst, w[0]); store_wt(dst + d1, w[1]); store_wt(dst + d2, w[2]); store_wt(dst + d2 + d1, w[3]);
                } else {
                    u32x4 w[8];
#pragma unroll
                    for (int q = 0; q < 8; ++q) w[q] = *(const LAS u32x4*)(buf + q * 1024 + lane * 16);
                    if (!(info & INFO_RAW)) {
                        const u32x2 x2 = *(const LAS u32x2*)(buf + 6144 + lane * 8);
                        unsigned pw_[24];
#pragma unroll
                        for (int k = 0; k < 8; ++k) e[k] = (x2[k >> 2] >> (2 * (k & 3))) & B03;
                        exp_decode(e, info, lane, buf + 6656, scratch);
#pragma unroll
                        for (int i = 0; i < 24; ++i) pw_[i] = w[i >> 2][i & 3];
#pragma unroll
                        for (int q = 0; q < 8; ++q) {
                            const unsigned p0 = pw_[3 * q], p1 = pw_[3 * q + 1], p2 = pw_[3 * q + 2];
                            const u32x4 r = {perm(e[q], p0, 0x04020100u), perm(e[q], p1, 0x05020100u), perm(e[q], p2, 0x06020100u),
                                             perm(p1, p0, 0x0c0c0703u) | perm(e[q], p2, 0x07030c0cu)};
                            w[q] = (r >> 1) | (r << 31);
                        }
                    }
                    char* const o = (char*)P.out + out_of(A) + olane;
                    f32x4 y[8];
#pragma unroll
                    for (int q = 0; q < 8; ++q) y[q] = __builtin_bit_cast(f32x4, own[q]) + __builtin_bit_cast(f32x4, w[q]);
                    if (bad) {
#pragma unroll
                        for (int q = 0; q < 8; ++q) y[q] = (f32x4)qnan;
                    }
                    gelu8(y);
                    asm volatile("s_waitcnt vmcnt(0)" : "+v"(y[0]) :: "memory");
#pragma unroll
                    for (int q = 0; q < 8; ++q) store_wt(o + ((q >> 2) * 16 + (q & 3)) * (NFULL * 4), __builtin_bit_cast(u32x4, y[q]));
                }
                asm volatile(""
                             : "+v"(nown[0]), "+v"(nown[1]), "+v"(nown[2]), "+v"(nown[3]), "+v"(nown[4]), "+v"(nown[5]), "+v"(nown[6]), "+v"(nown[7]),
                               "+v"(fl), "+v"(ofl), "+v"(tm), "+v"(tk) :: "memory");
                if (pend && lane == 0) add_agent(me + pend, 1u);
                pend = isp ? 0u : OFF_FLAG + (A.id & 0xffffu) * 128u;
                tm = __builtin_amdgcn_readfirstlane(tm);
                if (!more) break;
                if (!early) {
                    if (pend) { WAIT_V(0); if (lane == 0) add_agent(me + pend, 1u); pend = 0; }
                    infob = __builtin_amdgcn_readfirstlane(wave_wait64<2>(me + B.flag, call, tmo, call));
                    post_item(B, infob);
                    tm = __builtin_amdgcn_readfirstlane(ld_agent(tmo));
                }
#pragma unroll
                for (int q = 0; q < 8; ++q) own[q] = nown[q];
                info = infob; A = B; B = C;
                C = item_of(__builtin_amdgcn_readfirstlane(tk) - (call - 1u) * NWORK + 24u * NC);
            }
            post(0u, 0u, 0u);
            WAIT_V(0);
            if (pend && lane == 0) add_agent(me + pend, 1u);
        } else {
            push_last();
            publish_push();
        }
        WAIT_V(0);
        asm volatile("s_endpgm" ::: "memory");
    }

    {
        const bool f1 = aux && remote;
        unsigned item = first_item(bid);
        const int fd = f1 ? peer : lane >> 3;
        char* const fws = f1 ? pws : lws;
#pragma unroll 1
        for (unsigned n = 0;; ++n) {
            const bool isw = f1 || item >= 128u;
            const int s = f1 ? item : isw ? (item - 128u) >> 3 : item >> 4;
            const int j = (f1 ? slot : item) & 7, rb = (item & 7) * 2 + ((item >> 3) & 1);
            const unsigned par = call & 1u;
            u32x4 o[4];
            unsigned off, d1, d2;
            VWAIT(0);
            if (isw) {
                off = ((wid & 3) * 4 + (wid >> 2)) * 1024; d1 = 2048; d2 = 16384;
#pragma unroll
                for (int f = 0; f < 4; ++f) {
                    o[f].x = pack2(v[0][f], v[1][f]); o[f].y = pack2(v[2][f], v[3][f]);
                    o[f].z = pack2(v[4][f], v[5][f]); o[f].w = pack2(v[6][f], v[7][f]);
                }
            } else {
                off = wid * 4096; d1 = 1024; d2 = 2048;
#pragma unroll
                for (int f = 0; f < 4; ++f) {
                    o[f].x = pack2(v[2 * f][0], v[2 * f][1]); o[f].y = pack2(v[2 * f][2], v[2 * f][3]);
                    o[f].z = pack2(v[2 * f + 1][0], v[2 * f + 1][1]); o[f].w = pack2(v[2 * f + 1][2], v[2 * f + 1][3]);
                }
            }
            off += lane * 16;
            const bool last = f1 ? item == 7u : bid < 208 || n == 1u;
            const unsigned u = (unsigned)((rank * 8 + s) * NBLK + (isw ? j : 8 + rb));
            const unsigned wu = (unsigned)((peer * 8 + s) * 8 + j);
            const bool expo = f1 || (!isw && (rb >> 1) != rank);
            char* const ldst = me + (f1 ? OFF_OWNB + (size_t)wu * UNIT : OFF_STAGE + par * STAGE_BYTES + (size_t)u * UNIT) + off;
            store_wt(ldst, o[0]); store_wt(ldst + d1, o[1]); store_wt(ldst + d2, o[2]); store_wt(ldst + d2 + d1, o[3]);
            if (!f1) {
                WAIT_V(0);
                if (lane == 0) add_agent(me + OFF_FLAG + u * 128u, 1u);
            }
            if (expo) {
                char* const ed = me + (f1 ? OFF_EXPW + par * WSET_BYTES + (size_t)wu * UNIT : OFF_EXPX + par * XSET_BYTES + (size_t)(s * 16 + rb) * UNIT) + wid * 4096;
                LAS unsigned char* const tail = lds + LDS_ENC + wid * 1024;
                unsigned hb[8], e[8], sg = 0;
                u32x4 lo[2], nb;
#pragma unroll
                for (int k = 0; k < 8; ++k) {
                    const unsigned w0 = o[k >> 1][(k & 1) * 2], w1 = o[k >> 1][(k & 1) * 2 + 1];
                    lo[k >> 2][k & 3] = perm(w1, w0, 0x06040200u); hb[k] = perm(w1, w0, 0x07050301u);
                    e[k] = (hb[k] >> 2) & 0x1f1f1f1fu; sg |= (hb[k] & B80) >> k;
                }
                const unsigned info = exp_encode(e, lane, tail, 768u);
                if (info & INFO_RAW) {
#pragma unroll
                    for (int f = 0; f < 4; ++f) store_wt(ed + f * 1024 + lane * 16, o[f]);
                } else {
#pragma unroll
                    for (int k = 0; k < 4; ++k)
                        nb[k] = (hb[2 * k] & B03) | (e[2 * k] << 2) | ((hb[2 * k + 1] & B03) << 4) | (e[2 * k + 1] << 6);
                    store_wt(ed + lane * 16, lo[0]); store_wt(ed + 1024 + lane * 16, lo[1]); store_wt(ed + 2048 + lane * 16, nb);
                    st_sys(ed + 3072 + lane * 4, sg);
                    if ((unsigned)lane < ((info >> 16) & 127u)) store_wt(ed + 3328 + lane * 16, *(volatile LAS u32x4*)(tail + lane * 16));
                }
                if (lane == 0) LDSW(4 + wid) = info;
            }
            WAIT_V(0);
            WAIT_L(0); BAR; asm volatile("" ::: "memory");
            if (wid == 0) {
                if (expo && (f1 ? lane < 8 : fd != rank))
                    st_sys64(fws + OFF_EFLAG + (par * 64u * NBLK + u) * 64u + (lane & 7) * 8, call, LDSW(4 + (lane & 7)));
                if (f1 && lane == 0) st_sys(me + OFF_OFLAG + wu * 128u, call * 8u);
            }
            if (last) break;
            item += f1 ? 1u : 8u;
            load_item(bid, item);
            WAIT_L(0); BAR;
        }
    }
    const bool ownb = aux && remote;
    int tile = slot;
again:
    const int ti = (aux ? rank : peer) * 2 + (tile >> 3), tj = slot & 7;
    const int nsrc = remote ? 6 : 7;
    const int rfirst = aux && !remote ? 4 : -1, rlast = ownb ? -1 : !aux && !remote ? 3 : 7;
    const int nt = remote ? (aux ? 8 : 56) : (aux ? 28 : 36);
    const int wr = wid >> 2, wc = wid & 3;
    const char* const stage = me + OFF_STAGE + (size_t)(call & 1u) * STAGE_BYTES;
    const char* const abase = stage + (unsigned)(8 + ti) * UNIT;
    const char* const bbase = ownb ? me + OFF_OWNB + (size_t)(((peer - rank) * 64 + tj) * (int)UNIT) : stage + (unsigned)tj * UNIT;
    const unsigned bstep = ownb ? 8u * UNIT : NBLK * UNIT;
    const unsigned voff = (unsigned)tid * 16u;
    const unsigned ldsw = (unsigned)wid * 1024u;
    const unsigned aoff = (unsigned)wr * 8192u + (unsigned)lane * 16u;
    unsigned boff = 65536u + (unsigned)wc * 4096u + (unsigned)lane * 16u;
    asm volatile("" : "+v"(boff));

#define SA(b, h) (((b) * 2 + (h)) * 16384)
#define SB(b, h) ((4 + (b) * 2 + (h)) * 16384)
#define STAGE(bufoff, gbase) do { _Pragma("unroll") for (int _i = 0; _i < 2; ++_i) \
        __builtin_amdgcn_global_load_lds((const GAS unsigned*)((gbase) + _i * 8192 + voff), (LAS unsigned*)(lds + (bufoff) + ldsw + _i * 8192), 16, 0, 17); } while (0)
#define LDA(dst, b, h) do { _Pragma("unroll") for (int m = 0; m < 4; ++m) _Pragma("unroll") for (int k = 0; k < 2; ++k) \
        dst[m][k] = *(const LAS f16x8*)(lds + SA(b, h) + aoff + m * 2048 + k * 1024); } while (0)
#define LDB(dst, b, h) do { _Pragma("unroll") for (int n = 0; n < 2; ++n) _Pragma("unroll") for (int k = 0; k < 2; ++k) \
        dst[n][k] = *(const LAS f16x8*)(lds + boff + (SB(b, h) - 65536 + n * 2048 + k * 1024)); } while (0)
#define MMA(ai, bj, At, Bt) do { __builtin_amdgcn_s_setprio(1); \
        _Pragma("unroll") for (int m = 0; m < 4; ++m) _Pragma("unroll") for (int n = 0; n < 2; ++n) _Pragma("unroll") for (int k = 0; k < 2; ++k) \
        acc[ai][bj][m][n] = __builtin_amdgcn_mfma_f32_16x16x32_f16(At[m][k], Bt[n][k], acc[ai][bj][m][n], 0, 0, 0); \
        __builtin_amdgcn_s_setprio(0); } while (0)

    unsigned ktab;
    {
        const int tt = lane < nt ? lane : nt - 2 + (lane & 1), u = rfirst < 0 ? tt - 8 : tt + rfirst * nsrc;
        const int r = u < 0 ? -1 : remote ? u / 6 : u / 7, q = u - r * nsrc;
        ktab = (u < 0 ? rank * 8 + tt : ((rank + q + 1 + (q + 1 >= skip)) & (NR - 1)) * 8 + r) | ((r + 1) << 8);
    }
    auto flag_of = [&](int s, int lz) -> unsigned {
        const int e = (lz >> 1) & 7, q = e < nsrc ? e : 0, sl = min(s + (lz >> 4), rlast);
        const int kt = sl < 0 ? rank * 8 + e : ((rank + q + 1 + (q + 1 >= skip)) & (NR - 1)) * 8 + sl;
        return ownb && !(lz & 1) ? OFF_OFLAG + (unsigned)((peer * 8 + e) * 8 + tj) * 128u
                                 : OFF_FLAG + (unsigned)(kt * NBLK + tj + (lz & 1) * (8 + ti - tj)) * 128u;
    };
    auto ready = [&](int s) {
        if (wid == 0) {
            int lz = lane; asm volatile("" : "+v"(lz));
            wave_wait<POLL>(me + flag_of(s, lz & 15), call * 8u, tmo, call);
        }
        asm volatile("" ::: "memory"); BAR; asm volatile("" ::: "memory");
    };

    f32x4 acc[2][2][4][2];
#pragma unroll
    for (int a = 0; a < 2; ++a)
#pragma unroll
        for (int b = 0; b < 2; ++b)
#pragma unroll
            for (int m = 0; m < 4; ++m)
#pragma unroll
                for (int n = 0; n < 2; ++n) acc[a][b][m][n] = (f32x4){0.f, 0.f, 0.f, 0.f};
    f16x8 At[4][2], B0[2][2], B1[2][2];

    int have = rfirst;
    ready(rfirst);
    unsigned k2 = __builtin_amdgcn_readlane(ktab, 0) & 63u, k3 = __builtin_amdgcn_readlane(ktab, 1) & 63u;
    {
        const char* a0 = abase + k2 * (NBLK * UNIT); const char* b0 = bbase + k2 * bstep;
        const char* a1 = abase + k3 * (NBLK * UNIT); const char* b1 = bbase + k3 * bstep;
        WAIT_V(0);
        STAGE(SB(0, 0), b0); STAGE(SA(0, 0), a0); STAGE(SB(0, 1), b0 + 16384); STAGE(SA(0, 1), a0 + 16384);
        if (wr == 1) BAR;
        WAIT_V(4); BAR;
        STAGE(SB(1, 0), b1); STAGE(SA(1, 0), a1); STAGE(SB(1, 1), b1 + 16384);
        WAIT_V(6); BAR;
    }
#pragma unroll 1
    for (int t = 0; t < nt; t += 2) {
        const char* a1 = abase + k3 * (NBLK * UNIT);
        k2 = __builtin_amdgcn_readlane(ktab, t + 2) & 63u; k3 = __builtin_amdgcn_readlane(ktab, t + 3);
        const int round3 = (int)(k3 >> 8) - 1, round5 = (int)((unsigned)__builtin_amdgcn_readlane(ktab, t + 5) >> 8) - 1;
        k3 &= 63u;
        {
            if (round3 > have) {
                int lz = lane; asm volatile("" : "+v"(lz));
                const u64 miss = ~__ballot(*(volatile LAS unsigned*)(lds + LDS_ENC + (t & 2) * 128 + lz * 4) >= call * 8u);
                if (miss & 0xffffull) { ready(round3); have = round3; }
                else have = min(round3 + (miss & 0xffff0000ull ? 0 : 1), rlast);
            }
        }
        const char* a2 = abase + k2 * (NBLK * UNIT); const char* b2 = bbase + k2 * bstep;
        const char* a3 = abase + k3 * (NBLK * UNIT); const char* b3 = bbase + k3 * bstep;
        LDB(B0, 0, 0); SCHED; LDA(At, 0, 0); STAGE(SA(1, 1), a1 + 16384);
        WAIT_L(8); BAR; WAIT_L(0); MMA(0, 0, At, B0); BAR; SCHED;
        LDB(B1, 0, 1); STAGE(SB(0, 0), b2);
        BAR; WAIT_L(0); MMA(0, 1, At, B1); BAR;
        LDA(At, 0, 1); STAGE(SA(0, 0), a2);
        BAR; WAIT_L(0); MMA(1, 0, At, B0); BAR; SCHED;
        STAGE(SB(0, 1), b2 + 16384);
        WAIT_V(6); BAR; MMA(1, 1, At, B1); BAR;
        if (wid == 0 && round5 > have) {
            int lz = lane; asm volatile("" : "+v"(lz));
            if (lz < 32)
                __builtin_amdgcn_global_load_lds((const GAS unsigned*)(me + flag_of(have + 1, lz)), (LAS unsigned*)(lds + LDS_ENC + (~t & 2) * 128), 4, 0, 17);
        }
        LDB(B0, 1, 0); SCHED; LDA(At, 1, 0); STAGE(SA(0, 1), a2 + 16384);
        WAIT_L(8); BAR; WAIT_L(0); MMA(0, 0, At, B0); BAR; SCHED;
        LDB(B1, 1, 1); STAGE(SB(1, 0), b3);
        BAR; WAIT_L(0); MMA(0, 1, At, B1); BAR;
        LDA(At, 1, 1); STAGE(SA(1, 0), a3);
        BAR; WAIT_L(0); MMA(1, 0, At, B0); BAR; SCHED;
        STAGE(SB(1, 1), b3 + 16384);
        WAIT_V(6); BAR; MMA(1, 1, At, B1); BAR;
    }
    if (wr == 0) BAR;
    int tz = threadIdx.x; asm volatile("" : "+v"(tz));
    const int lane2 = tz & 63;
    const float qnan = __builtin_nanf("");
#define ACC1(g, q, c) acc[(g) >> 1][(c) >> 1][((g) & 1) * 2 + ((q) >> 2)][(c) & 1][(q) & 3]
#define ACC(g, q) ((f32x4){ACC1(g, q, 0), ACC1(g, q, 1), ACC1(g, q, 2), ACC1(g, q, 3)})
    const unsigned foff = OFF_PFLAG + (unsigned)wid * 256u + (call & 1u) * 128u;
    const size_t poff = OFF_PART + (size_t)(call & 1u) * PART_BYTES + (size_t)slot * TILE_BYTES + wid * 32768 + lane2 * 16;
    if (!aux) {
        char* const dst = me + poff + (size_t)peer * 16 * TILE_BYTES;
        LAS unsigned char* const tail = lds + LDS_ENC + wid * 1024;
        char* const flag = pws + foff + (unsigned)(rank * 16 + slot) * 2048u;
        unsigned prev = 0, badbit = 0;
#pragma unroll
        for (int g = 0; g < 4; ++g) {
            unsigned e[8], pw_[24];
#pragma unroll
            for (int q = 0; q < 8; ++q) {
                const u32x4 u = __builtin_bit_cast(u32x4, ACC(g, q)), r = (u << 1) | (u >> 31);
                e[q] = perm(perm(r.w, r.z, 0x0c0c0703u), perm(r.y, r.x, 0x0c0c0703u), 0x05040100u);
                pw_[3 * q] = perm(r.w, r.x, 0x04020100u); pw_[3 * q + 1] = perm(r.w, r.y, 0x05020100u); pw_[3 * q + 2] = perm(r.w, r.z, 0x06020100u);
            }
            const unsigned info = remote ? exp_encode(e, lane2, tail, 512u) : INFO_RAW;
            if (g >= 2) {
                WAIT_V(0);
                if (lane2 == 0) st_sys64(flag + (g - 1) * 8, call, prev);
            }
            if (info & INFO_RAW) {
#pragma unroll
                for (int q = 0; q < 8; ++q) store_wt(dst + g * 8192 + q * 1024, __builtin_bit_cast(u32x4, ACC(g, q)));
            } else {
#pragma unroll
                for (int q = 0; q < 6; ++q) store_wt(dst + g * 8192 + q * 1024, (u32x4){pw_[4 * q], pw_[4 * q + 1], pw_[4 * q + 2], pw_[4 * q + 3]});
                st_sys64(dst + g * 8192 + 6144 - lane2 * 8, e[0] | (e[1] << 2) | (e[2] << 4) | (e[3] << 6), e[4] | (e[5] << 2) | (e[6] << 4) | (e[7] << 6));
                if ((unsigned)lane2 < ((info >> 16) & 127u)) store_wt(dst + g * 8192 + 6656, *(volatile LAS u32x4*)(tail + lane2 * 16));
            }
            if (g == 0) badbit = ld_agent(tmo) == call ? INFO_BAD : 0u;
            prev = info | badbit;
            if (g == 0 || g == 3) {
                WAIT_V(0);
                if (lane2 == 0) st_sys64(flag + g * 8, call, prev);
            }
        }
        return;
    }
    const char* const src = me + poff + (size_t)rank * 16 * TILE_BYTES;
    float* const o = P.out + (size_t)((tile >> 3) * 256 + wr * 64 + (lane2 >> 4) * 4) * NFULL + peer * NS + tj * 256 + wc * 64 + (lane2 & 15) * 4;
    if (!remote) wave_wait64<POLL>(me + foff + (unsigned)(peer * 16 + slot) * 2048u + (lane2 & 3) * 8, call, tmo, call);
    const bool bad = ld_agent(tmo) == call;
#pragma unroll
    for (int g = 0; g < 4; ++g) {
        f32x4 y[8];
#pragma unroll
        for (int q = 0; q < 8; ++q) y[q] = bad ? (f32x4)qnan : ACC(g, q);
        if (!remote) {
            u32x4 w[8];
            asm volatile("global_load_dwordx4 %0, %8, off sc0 sc1\n\t"
                         "global_load_dwordx4 %1, %8, off offset:1024 sc0 sc1\n\t"
                         "global_load_dwordx4 %2, %8, off offset:2048 sc0 sc1\n\t"
                         "global_load_dwordx4 %3, %8, off offset:3072 sc0 sc1\n\t"
                         "global_load_dwordx4 %4, %9, off sc0 sc1\n\t"
                         "global_load_dwordx4 %5, %9, off offset:1024 sc0 sc1\n\t"
                         "global_load_dwordx4 %6, %9, off offset:2048 sc0 sc1\n\t"
                         "global_load_dwordx4 %7, %9, off offset:3072 sc0 sc1\n\ts_waitcnt vmcnt(0)"
                         : "=&v"(w[0]), "=&v"(w[1]), "=&v"(w[2]), "=&v"(w[3]), "=&v"(w[4]), "=&v"(w[5]), "=&v"(w[6]), "=&v"(w[7])
                         : "v"(src + g * 8192), "v"(src + g * 8192 + 4096) : "memory");
#pragma unroll
            for (int q = 0; q < 8; ++q) y[q] += __builtin_bit_cast(f32x4, w[q]);
            gelu8(y);
        }
#pragma unroll
        for (int q = 0; q < 8; ++q) {
            if (remote) asm volatile("s_waitcnt vmcnt(%0)" :: "i"(F1DEPTH) : "memory");
            store_wt(o + (size_t)((g >> 1) * 128 + (g & 1) * 32 + (q >> 2) * 16 + (q & 3)) * NFULL, __builtin_bit_cast(u32x4, y[q]));
        }
    }
    WAIT_V(0);
    if (remote) {
        if (lane2 == 0) st_sys(me + OFF_OWNF + (unsigned)((peer * 16 + tile) * 8 + wid) * 64u, call);
        if (tile < 8) {
            for (int i = 0; i < F1GO; ++i) __builtin_amdgcn_s_sleep(127);
            tile += 8;
            goto again;
        }
    }
}

extern "C" void kernel_launch(int rank, int nranks, ncclComm_t comm,
                              hipStream_t stream,
                              void* const* d_in, const int* in_sizes, int n_in,
                              void* d_out, int out_size,
                              const int* in_dtypes, int out_dtype,
                              void* const* d_out_all,
                              void* const* d_ws_all, size_t ws_bytes) {
    (void)comm; (void)in_dtypes; (void)out_dtype; (void)d_out_all;
    if (nranks != NR || (unsigned)rank >= (unsigned)NR || n_in != 2 || ws_bytes < WS_NEED
        || in_sizes[0] != 4096 * KS || in_sizes[1] != KS * NFULL || out_size != 512 * NFULL) {
        fprintf(stderr, "kernel_launch: unexpected problem shape\n");
        return;
    }
    Params P = {};
    P.x = (const float*)d_in[0];
    P.w = (const float*)d_in[1];
    P.out = (float*)d_out;
    for (int p = 0; p < NR; ++p) P.ws[p] = (char*)d_ws_all[p];
    P.me = P.ws[rank];
    P.rank = rank;
    static const unsigned char port[NR][NR] = {
        {0, 2, 3, 6, 4, 7, 5, 1}, {1, 4, 5, 7, 6, 2, 3, 0}, {2, 0, 6, 3, 7, 1, 5, 4}, {3, 6, 0, 2, 7, 4, 1, 5},
        {4, 1, 7, 5, 6, 3, 0, 2}, {5, 7, 1, 4, 0, 6, 2, 3}, {6, 3, 2, 0, 1, 5, 4, 7}, {7, 5, 4, 1, 2, 0, 3, 6}};
    for (int c = 0; c < NR; ++c) P.peermap |= (unsigned)port[rank][c] << (4 * c);
    unsigned n = 0, sw = 0, sa = 1, sb = 0, so = 1;
    for (const char* c = ORDER; *c; ++c) {
        unsigned& s = *c == 'w' ? sw : *c == 'a' ? sa : *c == 'b' ? sb : so;
        const unsigned first = *c == 'w' ? 0 : *c == 'a' ? 8 : *c == 'b' ? 14 : 20, cnt = *c == 'w' ? 8 : *c == 'o' ? 2 : 6;
        for (unsigned i = 0; i < cnt && n < NPULL; ++i, ++n) P.order[n >> 2] |= (((s & 7u) << 5) | (first + i)) << ((n & 3u) * 8u);
        ++s;
    }
    if (n != NPULL || sw != 8 || sa != 8 || sb != 8 || so != 8) { fprintf(stderr, "kernel_launch: bad ORDER\n"); return; }
    gemm_rs_kernel<<<256, 512, 0, stream>>>(P);
    HIP_CHECK(hipGetLastError());
}
```

```cpp
#include <hip/hip_runtime.h>
#include <rccl/rccl.h>
#include <cstdio>

#define HIP_CHECK(expr) do { hipError_t _e = (expr); if (_e != hipSuccess) { \
    fprintf(stderr, "HIP error %s at %s:%d\n", hipGetErrorString(_e), __FILE__, __LINE__); \
    return; } } while (0)

#define LAS __attribute__((address_space(3)))
#define GAS __attribute__((address_space(1)))
typedef _Float16 f16x8 __attribute__((ext_vector_type(8)));
typedef float f32x4 __attribute__((ext_vector_type(4)));
typedef unsigned u32x4 __attribute__((ext_vector_type(4)));
typedef unsigned u32x2 __attribute__((ext_vector_type(2)));
typedef unsigned long long u64;

namespace {

constexpr const char* ORDER = "wawbwabwawabwabwawabbbbooooooo";
constexpr int NC = 6;
constexpr int WIN = 7;
constexpr int FASTW = 2;
constexpr int F1DEPTH = 1;
constexpr int F1GO = 8;
constexpr int MAPW = 2;
constexpr int POLL = 8;
constexpr int LATE = 80;
static_assert(WIN >= 4 && NC <= 7, "loader bookkeeping");

constexpr int NR = 8;
constexpr int KS = 512;
constexpr int NFULL = 16384;
constexpr int NS = 2048;
constexpr int NBLK = 24;
constexpr unsigned UNIT = 32768;
constexpr unsigned TILE_BYTES = 262144;
constexpr unsigned NPULL = 168, NSLICE = NPULL * 8, NWORK = NSLICE + 1024;
constexpr size_t CTRL_BYTES = 1u << 20;
constexpr size_t STAGE_BYTES = (size_t)64 * NBLK * UNIT;
constexpr size_t PART_BYTES = (size_t)NR * 16 * TILE_BYTES;
constexpr size_t WSET_BYTES = (size_t)NR * 64 * UNIT;
constexpr size_t XSET_BYTES = (size_t)128 * UNIT;
constexpr size_t OFF_STAGE = CTRL_BYTES, OFF_PART = OFF_STAGE + 2 * STAGE_BYTES, OFF_OWNB = OFF_PART + 2 * PART_BYTES,
                 OFF_EXPW = OFF_OWNB + WSET_BYTES, OFF_EXPX = OFF_EXPW + 2 * WSET_BYTES, WS_NEED = OFF_EXPX + 2 * XSET_BYTES;
constexpr unsigned OFF_TMO = 0, OFF_QUEUE = 2048  , OFF_CNT = 4096  ,
                   OFF_FLAG = 65536  , OFF_PFLAG = 262144  ,
                   OFF_OFLAG = 524288  , OFF_OWNF = 589824  ,
                   OFF_EFLAG = 655360  ;
constexpr u64 DEADLINE_TICKS = 50000000ull;
constexpr unsigned LDS_STAGE = 131072, LDS_BYTES = LDS_STAGE + 384 + 8192;
constexpr unsigned LDS_ENC = LDS_STAGE + 384;

struct Params {
    const float* x;
    const float* w;
    char* me;
    int rank;
    unsigned peermap;
    float* out;
    char* ws[NR];
    unsigned order[48];
};

__device__ __forceinline__ void store_wt(void* p, u32x4 v) {
    asm volatile("global_store_dwordx4 %0, %1, off sc0 sc1\n\ts_nop 1" :: "v"(p), "v"(v) : "memory");
}
__device__ __forceinline__ unsigned pack2(float a, float b) {
    typedef _Float16 h2 __attribute__((ext_vector_type(2)));
    h2 h; h.x = (_Float16)a; h.y = (_Float16)b;
    return __builtin_bit_cast(unsigned, h);
}
template <int N> __device__ __forceinline__ void gelu(f32x4 (&y)[N]) {
    const float k1 = -2.0f * 0.7978845608028654f * 1.4426950408889634f, k2 = k1 * 0.044715f;
    f32x4 e[N];
#pragma unroll
    for (int q = 0; q < N; ++q) e[q] = y[q] * __builtin_elementwise_fma(y[q] * y[q], (f32x4)k2, (f32x4)k1);
    __builtin_amdgcn_sched_barrier(0);
#pragma unroll
    for (int i = 0; i < 4 * N; ++i) e[i >> 2][i & 3] = __builtin_amdgcn_exp2f(e[i >> 2][i & 3]);
    __builtin_amdgcn_sched_barrier(0);
#pragma unroll
    for (int q = 0; q < N; ++q) e[q] += 1.0f;
    __builtin_amdgcn_sched_barrier(0);
#pragma unroll
    for (int i = 0; i < 4 * N; ++i) e[i >> 2][i & 3] = __builtin_amdgcn_rcpf(e[i >> 2][i & 3]);
    __builtin_amdgcn_sched_barrier(0);
#pragma unroll
    for (int q = 0; q < N; ++q) y[q] *= e[q];
}
__device__ __forceinline__ unsigned wave_scan(unsigned v) {
    v += __builtin_amdgcn_update_dpp(0, (int)v, 0x111, 0xf, 0xf, true);
    v += __builtin_amdgcn_update_dpp(0, (int)v, 0x112, 0xf, 0xf, true);
    v += __builtin_amdgcn_update_dpp(0, (int)v, 0x114, 0xf, 0xf, true);
    v += __builtin_amdgcn_update_dpp(0, (int)v, 0x118, 0xf, 0xf, true);
    v += __builtin_amdgcn_update_dpp(0, (int)v, 0x142, 0xa, 0xf, false);
    v += __builtin_amdgcn_update_dpp(0, (int)v, 0x143, 0xc, 0xf, false);
    return v;
}
__device__ __forceinline__ unsigned ld_sys(const char* p) {
    return __hip_atomic_load((const GAS unsigned*)p, __ATOMIC_RELAXED, __HIP_MEMORY_SCOPE_SYSTEM);
}
__device__ __forceinline__ u64 ld_sys64(const char* p) {
    return __hip_atomic_load((const GAS u64*)p, __ATOMIC_RELAXED, __HIP_MEMORY_SCOPE_SYSTEM);
}
__device__ __forceinline__ unsigned ld_agent(const char* p) {
    return __hip_atomic_load((const GAS unsigned*)p, __ATOMIC_RELAXED, __HIP_MEMORY_SCOPE_AGENT);
}
__device__ __forceinline__ void st_sys(char* p, unsigned v) {
    __hip_atomic_store((GAS unsigned*)p, v, __ATOMIC_RELAXED, __HIP_MEMORY_SCOPE_SYSTEM);
}
__device__ __forceinline__ void st_sys64(char* p, unsigned lo, unsigned hi) {
    __hip_atomic_store((GAS u64*)p, ((u64)hi << 32) | lo, __ATOMIC_RELAXED, __HIP_MEMORY_SCOPE_SYSTEM);
}
__device__ __forceinline__ unsigned add_agent(char* p, unsigned v) {
    return __hip_atomic_fetch_add((GAS unsigned*)p, v, __ATOMIC_RELAXED, __HIP_MEMORY_SCOPE_AGENT);
}
template <int SLEEP> __device__ __forceinline__ bool give_up(unsigned& n, u64 t0, char* tmo, unsigned call) {
    if ((++n & 15u) == 0u
        && ((unsigned)__builtin_amdgcn_readfirstlane(ld_agent(tmo)) == call || __builtin_amdgcn_s_memrealtime() - t0 > DEADLINE_TICKS)) {
        __hip_atomic_store((GAS unsigned*)tmo, call, __ATOMIC_RELAXED, __HIP_MEMORY_SCOPE_AGENT);
        return true;
    }
    __builtin_amdgcn_s_sleep(SLEEP);
    return false;
}
template <int SLEEP> __device__ __forceinline__ void wave_wait(const char* word, unsigned need, char* tmo, unsigned call) {
    const u64 t0 = __builtin_amdgcn_s_memrealtime();
    unsigned n = 0;
    while (!__all(ld_sys(word) >= need) && !give_up<SLEEP>(n, t0, tmo, call)) { }
    asm volatile("" ::: "memory");
}
template <int SLEEP> __device__ __forceinline__ unsigned wave_wait64(const char* word, unsigned need, char* tmo, unsigned call) {
    const u64 t0 = __builtin_amdgcn_s_memrealtime();
    unsigned n = 0;
    u64 v;
    do { v = ld_sys64(word); } while (!__all((unsigned)v >= need) && !give_up<SLEEP>(n, t0, tmo, call));
    asm volatile("" ::: "memory");
    return (unsigned)(v >> 32);
}

constexpr unsigned B01 = 0x01010101u, B03 = 0x03030303u, B07 = 0x07070707u, B80 = 0x80808080u, INFO_RAW = 1u << 23, INFO_BAD = 1u << 24;
__device__ __forceinline__ unsigned sub8(unsigned a, unsigned b) { return ((a | B80) - (b & ~B80)) ^ ((a ^ ~b) & B80); }
__device__ __forceinline__ unsigned perm(unsigned hi, unsigned lo, unsigned sel) { return __builtin_amdgcn_perm(hi, lo, sel); }
__device__ __forceinline__ void lds_or(LAS unsigned char* p, unsigned v) {
    __hip_atomic_fetch_or((LAS unsigned*)p, v, __ATOMIC_RELAXED, __HIP_MEMORY_SCOPE_WORKGROUP);
}
__device__ __forceinline__ unsigned prefix4(unsigned inc) {
    unsigned t = inc + (inc << 8);
    asm("" : "+v"(t));
    return t + (t << 16);
}
template <int NK> __device__ __forceinline__ unsigned exp_encode(unsigned (&e)[NK], int lane, LAS unsigned char* tail) {
    *(volatile LAS u32x4*)(tail + lane * 16) = (u32x4){0u, 0u, 0u, 0u};
    unsigned sum = 0;
#pragma unroll
    for (int k = 0; k < NK; ++k) sum = __builtin_amdgcn_sad_u8(e[k], 0u, sum);
    const unsigned c = (((unsigned)__builtin_amdgcn_readlane(wave_scan(sum), 63) / (NK * 256u)) + 3u) & 255u;
    const unsigned c4 = ((c & 15u) | 16u) * B01, r0 = sub8(c * B01, 0x03020100u), r1 = sub8(c * B01, 0x030a0908u), r2 = sub8(c * B01, 0x07060504u);
    unsigned acc[NK / 2], len[NK / 2], fix[NK], any = 0, mine = 0;
#pragma unroll
    for (int k = 0; k < NK; ++k) {
        const unsigned h = c4 - (e[k] & 0x0f0f0f0fu), sel = h & B07, m = perm(0u, 0u, ((h >> 3) & B01) | 0x0c0c0c0cu);
        const unsigned cs = (perm(0x4c3c2c1cu, 0x037c6c5cu, sel) & m) | (perm(0x4c3c2c1cu, 0x0302010cu, sel) & ~m);
        fix[k] = e[k] ^ ((perm(r2, r1, sel) & m) | (perm(r2, r0, sel) & ~m));
        any |= fix[k];
        const unsigned sy = (cs >> 4) & B07, in = prefix4((cs >> 2) & B03);
        const unsigned ch = (sy & 255u) | (((sy >> 8) & 255u) << (in & 255u)) | (((sy >> 16) & 255u) << ((in >> 8) & 255u)) | ((sy >> 24) << ((in >> 16) & 255u));
        if (k & 1) { acc[k >> 1] |= ch << len[k >> 1]; len[k >> 1] += in >> 24; } else { acc[k >> 1] = ch; len[k >> 1] = in >> 24; }
        mine += in >> 24;
        e[k] = cs & B03;
    }
    const unsigned incl = wave_scan(mine);
    unsigned p = incl - mine;
#pragma unroll
    for (int g = 0; g < NK / 2; ++g) {
        const u64 t = (u64)acc[g] << (p & 31u);
        lds_or(tail + (p >> 5) * 4u, (unsigned)t); lds_or(tail + (p >> 5) * 4u + 4u, (unsigned)(t >> 32));
        p += len[g];
    }
    const unsigned sbytes = ((__builtin_amdgcn_readlane(incl, 63) + 31u) >> 5) * 4u;
    unsigned n = 0;
    u64 has;
    while ((has = __ballot(any != 0u)) != 0ull && n <= 64u) {
        unsigned f = 0, kk = 0;
        any = 0;
#pragma unroll
        for (int k = NK - 1; k >= 0; --k) if (fix[k] != 0u) { f = fix[k]; kk = k; }
        const unsigned sh = __builtin_ctz(f | 0x80000000u) & 24u;
        const unsigned idx = n + __builtin_amdgcn_mbcnt_hi((unsigned)(has >> 32), __builtin_amdgcn_mbcnt_lo((unsigned)has, 0u));
        if (f != 0u && idx < 64u)
            *(volatile LAS unsigned short*)(tail + sbytes + idx * 2u) = (lane << 10) | (kk << 7) | (sh << 2) | ((f >> (sh + 3u)) & 31u);
#pragma unroll
        for (int k = 0; k < NK; ++k) { if (kk == (unsigned)k) fix[k] = f & ~(255u << sh); any |= fix[k]; }
        n += __builtin_popcountll(has);
    }
    const unsigned bytes = sbytes + n * 2u;
    return n > 64u || bytes > 768u ? INFO_RAW : c | (n << 8) | (((bytes + 15u) >> 4) << 16);
}
template <int NK> __device__ __forceinline__ void exp_decode(unsigned (&e)[NK], unsigned info, int lane, const LAS unsigned char* tail, LAS unsigned char* scratch) {
    *(volatile LAS u32x4*)(scratch + lane * 32) = (u32x4){0u, 0u, 0u, 0u};
    if (NK > 4) *(volatile LAS u32x4*)(scratch + lane * 32 + 16) = (u32x4){0u, 0u, 0u, 0u};
    unsigned m[NK], in[NK], p[NK / 2], ch[NK], sy[NK], mine = 0;
#pragma unroll
    for (int k = 0; k < NK; ++k) {
        m[k] = perm(0u, 0xffu, e[k]);
        in[k] = prefix4(m[k] & B03);
        if (!(k & 1)) p[k >> 1] = mine;
        mine += in[k] >> 24;
    }
    const unsigned incl = wave_scan(mine), sbytes = ((__builtin_amdgcn_readlane(incl, 63) + 31u) >> 5) * 4u;
    unsigned en = 0;
    if ((unsigned)lane < ((info >> 8) & 127u)) en = *(const volatile LAS unsigned short*)(tail + sbytes + lane * 2);
#pragma unroll
    for (int g = 0; g < NK / 2; ++g) {
        p[g] += incl - mine;
        const LAS unsigned* const q = (const LAS unsigned*)(tail + (p[g] >> 5) * 4u);
        ch[2 * g] = __builtin_amdgcn_alignbit(q[1], q[0], p[g]);
        ch[2 * g + 1] = ch[2 * g] >> (in[2 * g] >> 24);
    }
    if (en & 31u) lds_or(scratch + (en >> 7) * 4u, (en & 31u) << (((en >> 2) & 24u) + 3u));
    u32x4 fx[NK / 4];
#pragma unroll
    for (int k = 0; k < NK / 4; ++k) fx[k] = *(volatile LAS u32x4*)(scratch + lane * 32 + k * 16);
#define SPREAD1(d, i, c, j, jm) "v_lshrrev_b32_sdwa " d ", " i ", " c " dst_sel:BYTE_" #j " dst_unused:UNUSED_PRESERVE src0_sel:BYTE_" #jm " src1_sel:DWORD\n\t"
#define SPREAD4(j, jm) SPREAD1("%0", "%4", "%8", j, jm) SPREAD1("%1", "%5", "%9", j, jm) SPREAD1("%2", "%6", "%10", j, jm) SPREAD1("%3", "%7", "%11", j, jm)
#pragma unroll
    for (int k = 0; k < NK; k += 4) {
        sy[k] = ch[k]; sy[k + 1] = ch[k + 1]; sy[k + 2] = ch[k + 2]; sy[k + 3] = ch[k + 3];
        asm(SPREAD4(1, 0) SPREAD4(2, 1) SPREAD4(3, 2) "s_nop 0"
            : "+&v"(sy[k]), "+&v"(sy[k + 1]), "+&v"(sy[k + 2]), "+&v"(sy[k + 3])
            : "v"(in[k]), "v"(in[k + 1]), "v"(in[k + 2]), "v"(in[k + 3]), "v"(ch[k]), "v"(ch[k + 1]), "v"(ch[k + 2]), "v"(ch[k + 3]));
    }
    const unsigned c4 = (info & 255u) * B01, tc = sub8(c4, 0x03020100u), t0 = sub8(c4, 0x06050400u), t1 = sub8(c4, 0x0a090807u);
#pragma unroll
    for (int k = 0; k < NK; ++k)
        e[k] = ((perm(t1, t0, sy[k] & B07) & m[k]) | (perm(0u, tc, e[k]) & ~m[k])) ^ fx[k >> 2][k & 3];
}

#define WAIT_V(n) asm volatile("s_waitcnt vmcnt(" #n ")" ::: "memory")
#define WAIT_L(n) asm volatile("s_waitcnt lgkmcnt(" #n ")" ::: "memory")
#define BAR __builtin_amdgcn_s_barrier()
#define SCHED __builtin_amdgcn_sched_barrier(0)
#define LDSW(i) (*(volatile LAS unsigned*)(lds + LDS_STAGE + 4 * (i)))

}

__global__ __launch_bounds__(512, 2) void gemm_rs_kernel(Params P) {
    __shared__ __attribute__((aligned(16))) unsigned char shm[LDS_BYTES];
    LAS unsigned char* lds = (LAS unsigned char*)shm;
    const int tid = threadIdx.x, lane = tid & 63, wid = __builtin_amdgcn_readfirstlane(tid >> 6);
    const int hb = __builtin_amdgcn_readfirstlane(blockIdx.x);
    const float* px = P.x; const float* pw = P.w; char* me_ = P.me; int rank_ = P.rank; unsigned peermap = P.peermap;
    asm volatile("" : "+s"(px), "+s"(pw), "+s"(me_), "+s"(rank_), "+s"(peermap));
    const int rank = rank_ & (NR - 1);
    char* const me = (char*)(GAS char*)(size_t)me_;
    char* const tmo = me + OFF_TMO;
    int bid = (hb & ~7) | (__builtin_amdgcn_s_getreg((3 << 11) | 20) & 7);
    if (bid >= 128 && (bid & 7) != 0) {
        const int tt = ((bid & 7) - 1) * 8 + ((bid >> 3) & 7), cg = tt / (7 * MAPW), r = tt - cg * (7 * MAPW);
        bid = (bid & 192) | ((cg * MAPW + r % MAPW) << 3) | (r / MAPW + 1);
    }
    f32x4 v[8] = {};
    auto first_item = [&](int b) -> unsigned {
        const int q = ((b >> 3) & 7) + 1, sk = ((peermap >> ((b & 7) * 4)) - rank) & (NR - 1);
        const int pl = q < 7 ? (rank + q + (q >= sk)) & (NR - 1) : ((rank + sk) & (NR - 1)) + (q & 8);
        const int lo = (b & 7) == 0 ? (b >> 6) * 16 + 7 + q : b < 64 ? 0 : pl;
        const int hi = b >= 192 ? (q - 1) * 16 + (b & 7) : b;
        return b >= 128 ? hi : lo;
    };
    auto load_item = [&](int b, unsigned item) {
        const bool f1 = b < 64 && (b & 7) != 0, isw = f1 || item >= 128u;
        const unsigned ws_ = f1 ? item : (item >> 3) & 7u, wj = (f1 ? b >> 3 : item) & 7u, wd = f1 ? (peermap >> ((b & 7) * 4)) & (NR - 1) : rank;
        const unsigned wo = (ws_ * 64u + (wid >> 2) * 32u) * NFULL + wd * NS + wj * 256u + (wid & 3) * 64u;
        const unsigned xo = (((item & 7u) * 2u + ((item >> 3) & 1u)) * 256u + wid * 32u) * KS + ((item >> 4) & 7u) * 64u;
        const float* const base = isw ? pw + wo : px + xo;
        unsigned lo = (lane >> 4) * (isw ? 16u * NFULL : 16u) + (lane & 15) * (isw ? 16u : 4u * KS);
        const unsigned sb = isw ? 4u * NFULL : 64u, sa = (isw ? 64u * NFULL : 64u * KS) - 3u * sb;
#define LD1(i, step) "global_load_dwordx4 %" #i ", %8, %9\n\tv_add_u32 %8, " step ", %8\n\t"
        asm volatile("s_nop 4\n\t" LD1(0, "%10") LD1(1, "%10") LD1(2, "%10") LD1(3, "%11") LD1(4, "%10") LD1(5, "%10") LD1(6, "%10")
                     "global_load_dwordx4 %7, %8, %9"
                     : "+v"(v[0]), "+v"(v[1]), "+v"(v[2]), "+v"(v[3]), "+v"(v[4]), "+v"(v[5]), "+v"(v[6]), "+v"(v[7]), "+v"(lo)
                     : "s"(base), "s"(sb), "s"(sa) : "memory");
    };
#define VWAIT(n) asm volatile("s_waitcnt vmcnt(" #n ")" : "+v"(v[0]), "+v"(v[1]), "+v"(v[2]), "+v"(v[3]), "+v"(v[4]), "+v"(v[5]), "+v"(v[6]), "+v"(v[7]) :: "memory")
    unsigned t = 0, ordw[3];
    char* lws;
    {
        u64 ex;
        asm volatile("s_mov_b64 %5, exec\n\ts_mov_b64 exec, %12\n\t"
                     "global_atomic_add %0, %6, %7, %8 sc0\n\ts_mov_b64 exec, %5\n\t"
                     "global_load_dwordx2 %1, %9, %11 offset:%13\n\t"
                     "global_load_dword %2, %10, %11 offset:%14\n\t"
                     "global_load_dword %3, %10, %11 offset:%14+64\n\t"
                     "global_load_dword %4, %10, %11 offset:%14+128"
                     : "+v"(t), "=&v"(lws), "=&v"(ordw[0]), "=&v"(ordw[1]), "=&v"(ordw[2]), "=&s"(ex)
                     : "v"(OFF_CNT + (lane ? bid * 128 + 4 : hb * 128)), "v"(1u), "s"(me), "v"(lane & 56), "v"(lane & 60),
                       "s"(__builtin_amdgcn_kernarg_segment_ptr()), "s"((u64)(wid == 0 ? 3u : 0u)),
                       "i"(__builtin_offsetof(Params, ws)), "i"(__builtin_offsetof(Params, order)) : "memory");
    }
    if (bid >= 128 ? (bid >> 4) != 12 : (bid & 7) == 0) __builtin_amdgcn_s_sleep(LATE);
    char* pws_;
    unsigned call = 0;
#pragma unroll 1
    for (;;) {
        asm volatile("" ::: "memory");
        load_item(bid, first_item(bid));
        pws_ = P.ws[(peermap >> ((bid & 7) * 4)) & (NR - 1)];
        asm volatile("s_waitcnt vmcnt(8)" : "+v"(t), "+v"(lws), "+v"(ordw[0]), "+v"(ordw[1]), "+v"(ordw[2]) :: "memory");
        if (wid == 0 && call == 0u) {
            const unsigned c = __builtin_amdgcn_readlane(t, 0) + 1u;
            unsigned old = __builtin_amdgcn_readlane(t, 1), r = bid;
            for (int tries = 0; old != c - 1u && tries < 65536; ++tries) {
                if (lane == 0) add_agent(me + OFF_CNT + r * 128u + 4u, ~0u);
                r = (r + 1u) & 255u;
                if (lane == 0) t = add_agent(me + OFF_CNT + r * 128u + 4u, 1u);
                old = __builtin_amdgcn_readfirstlane(t);
            }
            LDSW(16 + lane) = 0u;
            if (lane == 0) { LDSW(0) = c; LDSW(12) = r; }
        }
        asm volatile("s_waitcnt lgkmcnt(0)" : "+s"(pws_) :: "memory"); BAR; asm volatile("" ::: "memory");
        call = __builtin_amdgcn_readfirstlane(LDSW(0));
        const int got = __builtin_amdgcn_readfirstlane(LDSW(12));
        if (got == bid) break;
        bid = got;
        VWAIT(0);
    }
    const int peer = (peermap >> ((bid & 7) * 4)) & (NR - 1);
    const int slot = (bid >> 3) & 15;
    const bool aux = bid < 128, remote = (bid & 7) != 0;
    const int skip = remote ? (peer - rank) & (NR - 1) : NR;
    char* const pws = (char*)(GAS char*)(size_t)pws_;

    if (aux && remote && slot >= 8) {
        const unsigned par = call & 1u;
        const unsigned pu = (unsigned)(rank * 8 * NBLK + 8) + (first_item(bid) & 7u) * 2u + ((first_item(bid) >> 3) & 1u);
        char* const rdst = pws + OFF_STAGE + par * STAGE_BYTES + (size_t)pu * UNIT + wid * 4096 + lane * 16;
        auto frag = [&](int f) -> u32x4 {
            return (u32x4){pack2(v[2 * f][0], v[2 * f][1]), pack2(v[2 * f][2], v[2 * f][3]),
                           pack2(v[2 * f + 1][0], v[2 * f + 1][1]), pack2(v[2 * f + 1][2], v[2 * f + 1][3])};
        };
        VWAIT(6); store_wt(rdst, frag(0));
        if (wid > FASTW) WAIT_V(0);
        VWAIT(5); store_wt(rdst + 1024, frag(1));
        if (wid > FASTW) WAIT_V(0);
        VWAIT(4); store_wt(rdst + 2048, frag(2));
        VWAIT(3);
        const u32x4 o3 = frag(3);
        auto push_last = [&]() { WAIT_V(0); store_wt(rdst + 3072, o3); };
        auto publish_push = [&]() {
            WAIT_V(0);
            __builtin_amdgcn_s_waitcnt(0x0F70);
            if (lane == 0)
                __hip_atomic_fetch_add((GAS unsigned*)(pws + OFF_FLAG + pu * 128u), 1u, __ATOMIC_RELAXED, __HIP_MEMORY_SCOPE_SYSTEM);
        };
        if (wid == 0) {
            push_last();
            publish_push();
            __builtin_amdgcn_s_setprio(3);
            const unsigned lbase = (unsigned)(size_t)lds, landed = lbase + (lane ? 129024 + lane * 4 : LDS_STAGE + 68);
            unsigned head = 0, alive = NC, idle = 3;
            int issued = 0;
            u32x2 d = *(volatile LAS u32x2*)(lds + LDS_STAGE + 128);
#define PIECE(off) "s_waitcnt vmcnt(%5)\n\tds_max_i32 %4, %1\n\tv_add_u32 %1, 1, %1\n\tglobal_load_lds_dwordx4 %2, off offset:" #off " sc0 sc1\n\t"
            while (alive) {
                if ((unsigned)__builtin_amdgcn_readfirstlane(d.y) != head + 1u) {
                    if (idle == 1u) asm volatile("s_waitcnt vmcnt(4)\n\tds_max_i32 %0, %1" :: "v"(landed), "v"(issued - 4) : "memory");
                    if (idle == 2u) asm volatile("s_waitcnt vmcnt(0)\n\tds_max_i32 %0, %1" :: "v"(landed), "v"(issued) : "memory");
                    ++idle;
                    d = *(volatile LAS u32x2*)(lds + LDS_STAGE + 128 + (head & 15u) * 8u);
                    continue;
                }
                ++head;
                const unsigned meta = __builtin_amdgcn_readfirstlane(d.x), cnt = (meta >> 17) & 15u;
                const unsigned dst = lbase + (meta >> 28) * 8192u;
                unsigned keep;
                const char* const src = pws + (meta & 0x1ffffu) * 4096u + lane * 16;
                d = *(volatile LAS u32x2*)(lds + LDS_STAGE + 128 + (head & 15u) * 8u);
                if (cnt == 0u) { --alive; continue; }
                const u64 mask = ~0ull >> (64u - ((meta >> 21) & 127u));
                int known = issued - (WIN - 1);
                asm volatile("s_mov_b32 %0, m0\n\ts_mov_b32 m0, %3\n\t" PIECE(0) PIECE(1024) PIECE(2048)
                             "s_waitcnt vmcnt(%5)\n\tds_max_i32 %4, %1\n\ts_mov_b64 exec, %6\n\t"
                             "global_load_lds_dwordx4 %2, off offset:3072 sc0 sc1\n\ts_mov_b64 exec, -1\n\ts_mov_b32 m0, %0"
                             : "=&s"(keep), "+v"(known) : "v"(src), "s"(dst), "v"(landed), "i"(WIN - 1), "s"(mask) : "memory");
                issued += cnt;
                idle = 0u;
            }
        } else if (wid <= NC) {
            const unsigned me8 = wid - 1;
            LAS unsigned char* const scratch = lds + 114688 + me8 * 2048u;
            unsigned bi = me8 * 2u + 1u;
            const float qnan = __builtin_nanf("");
            unsigned tbl = 0;
#pragma unroll
            for (int h = 0; h < 3; ++h) {
                const unsigned e = (ordw[h] >> ((lane & 3) * 8)) & 255u, code = e & 31u;
                const int odd = code >= 14u, q = code - (odd ? 14u : 8u);
                const unsigned blk = code < 8u ? code : code >= 20u ? 8u + rank * 2 + (code - 20u)
                                                                     : 8u + ((rank + q + 1 + (q + 1 >= skip)) & (NR - 1)) * 2 + odd;
                tbl |= ((e & 0xe0u) | blk) << (8 * h);
            }
            struct Item { unsigned t, id, flag, src; };
            auto item_of = [&](unsigned t) -> Item {
                const unsigned it = min(t >> 3, NPULL - 1u), e = (unsigned)__builtin_amdgcn_readlane(tbl, it & 63u) >> ((it >> 6) * 8u);
                const unsigned ks = (e >> 5) & 7u, blk = e & 31u, u = (peer * 8u + ks) * NBLK + blk;
                const unsigned so = blk < 8u ? (unsigned)OFF_EXPW + par * (unsigned)WSET_BYTES + ((rank * 8u + ks) * 8u + blk) * UNIT
                                             : (unsigned)OFF_EXPX + par * (unsigned)XSET_BYTES + (ks * 16u + blk - 8u) * UNIT;
                const unsigned i = max(min(t, NWORK - 1u), NSLICE) - NSLICE;
                const unsigned ci = ((i >> 9) * 64u + (i & 63u)) * 8u + ((i >> 6) & 7u);
                Item r;
                r.t = t;
                r.id = t < NSLICE ? u | (blk < 8u ? 0x10000u : 0u) : ci;
                r.flag = t < NSLICE ? OFF_EFLAG + (par * 64u * NBLK + u) * 64u + (t & 7u) * 8u
                                    : OFF_PFLAG + (peer * 128u + (ci >> 3)) * 256u + par * 128u + (ci & 7u) * 8u;
                r.src = t < NSLICE ? so + (t & 7u) * 4096u : (unsigned)OFF_PART + par * (unsigned)PART_BYTES + rank * 16u * TILE_BYTES + ci * 4096u;
                return r;
            };
            auto ownf_of = [&](const Item& x) -> unsigned { return OFF_OWNF + (peer * 128u + (x.t < NSLICE ? 0u : x.id >> 3)) * 64u; };
            auto out_of = [&](const Item& x) -> unsigned {
                const unsigned ct = (x.id >> 6) & 15u, cw = (x.id >> 3) & 7u, cg = x.id & 7u;
                return (((ct >> 3) * 256u + (cg >> 2) * 128u + (cw >> 2) * 64u + (cg & 3u) * 16u) * NFULL + peer * NS + (ct & 7u) * 256u + (cw & 3u) * 64u) * 4u;
            };
            const unsigned olane = ((lane >> 4) * 4u * NFULL + (lane & 15) * 4u) * 4u;
            int rend = 0;
            auto post = [&](unsigned src, unsigned cnt, unsigned last) {
                unsigned k = 0;
                bi ^= 1u;
                if (lane == 0) {
                    asm volatile("ds_add_rtn_u32 %0, %1, %2\n\ts_waitcnt lgkmcnt(0)" : "=&v"(k) : "v"((unsigned)(size_t)lds + LDS_STAGE + 64u), "v"(0x10000u + cnt) : "memory");
                    *(volatile LAS u32x2*)(lds + LDS_STAGE + 128 + ((k >> 16) & 15u) * 8u) = (u32x2){(src >> 12) | (cnt << 17) | (last << 21) | (bi << 28), (k >> 16) + 1u};
                }
                rend = (__builtin_amdgcn_readfirstlane(k) & 0xffffu) + cnt;
            };
            auto post_item = [&](const Item& x, unsigned inf) {
                post(x.src, 4u, inf & INFO_RAW ? 64u : min(16u + ((inf >> 16) & 127u), 64u));
            };
            Item A = item_of((slot - 8) * NC + me8), B = item_of(A.t + 8u * NC), C = item_of(A.t + 16u * NC);
            const u64 f0 = ld_sys64(me + A.flag);
            unsigned info = __builtin_amdgcn_readfirstlane((unsigned)(f0 >> 32)), ofl = 0, tm = 0;
            const bool now = __all((unsigned)f0 >= call);
            push_last();
            if (now) post_item(A, info);
            publish_push();
            if (!now) { info = __builtin_amdgcn_readfirstlane(wave_wait64<2>(me + A.flag, call, tmo, call)); post_item(A, info); }
            u64 fl;
            do { fl = ld_sys64(me + B.flag); } while ((unsigned)__builtin_amdgcn_readfirstlane((unsigned)fl) < call && (int)__builtin_amdgcn_readfirstlane(LDSW(17)) < rend);
            unsigned pend = 0;
            u32x4 own[4] = {};
#pragma unroll 1
            for (;;) {
                const bool isp = A.t >= NSLICE, bad = tm == call || (info & INFO_BAD) != 0u;
                if ((int)__builtin_amdgcn_readfirstlane(LDSW(17)) < rend) {
                    if (pend) { WAIT_V(0); if (lane == 0) add_agent(me + pend, 1u); pend = 0; }
                    while ((int)__builtin_amdgcn_readfirstlane(LDSW(17)) < rend) __builtin_amdgcn_s_sleep(1);
                }
                asm volatile("" ::: "memory");
                const LAS unsigned char* const buf = lds + bi * 8192u;
                const bool more = B.t < NWORK, early = more && (unsigned)__builtin_amdgcn_readfirstlane((unsigned)fl) >= call;
                unsigned infob = __builtin_amdgcn_readfirstlane((unsigned)(fl >> 32));
                if (early) post_item(B, infob);
                u32x4 nown[4];
                u64 ex;
                unsigned tk;
                const bool nextp = more && B.t >= NSLICE;
                if (nextp && (unsigned)__builtin_amdgcn_readfirstlane(ofl) < call) wave_wait<2>(me + ownf_of(B), call, tmo, call);
                {
                    unsigned vo = out_of(B) + olane;
                    asm volatile("s_mov_b64 %8, exec\n\ts_mov_b32 exec_lo, %16\n\ts_mov_b32 exec_hi, %16\n\ts_nop 2\n\t"
                                 "global_load_dwordx4 %0, %9, %10 sc0 sc1\n\tv_add_u32 %9, 0x10000, %9\n\t"
                                 "global_load_dwordx4 %1, %9, %10 sc0 sc1\n\tv_add_u32 %9, 0x10000, %9\n\t"
                                 "global_load_dwordx4 %2, %9, %10 sc0 sc1\n\tv_add_u32 %9, 0x10000, %9\n\t"
                                 "global_load_dwordx4 %3, %9, %10 sc0 sc1\n\t"
                                 "s_mov_b64 exec, 1\n\t"
                                 "global_load_dwordx2 %4, %12, %11 sc0 sc1\n\t"
                                 "global_load_dword %5, %13, %11 sc0 sc1\n\t"
                                 "global_load_dword %6, %17, %11 sc1\n\t"
                                 "global_atomic_add %7, %14, %15, %11 sc0\n\t"
                                 "s_mov_b64 exec, %8"
                                 : "=&v"(nown[0]), "=&v"(nown[1]), "=&v"(nown[2]), "=&v"(nown[3]),
                                   "=&v"(fl), "=&v"(ofl), "=&v"(tm), "=&v"(tk), "=&s"(ex), "+v"(vo)
                                 : "s"(P.out), "s"(me), "v"(C.flag), "v"(ownf_of(C)), "v"(OFF_QUEUE + peer * 128u), "v"(1u),
                                   "s"(__builtin_amdgcn_readfirstlane(nextp ? -1 : 0)), "v"(OFF_TMO) : "memory");
                }
                u32x4 w[4];
#pragma unroll
                for (int q = 0; q < 4; ++q) w[q] = *(const LAS u32x4*)(buf + q * 1024 + lane * 16);
                const unsigned w13 = *(const LAS unsigned*)(buf + 3072 + lane * 4);
                char* dst;
                unsigned d1, d2;
                if (!isp) {
                    if (!(info & INFO_RAW)) {
                        const u32x4 l0 = w[0], l1 = w[1], nb = w[2];
                        unsigned e[8];
#pragma unroll
                        for (int k = 0; k < 8; ++k) e[k] = (nb[k >> 1] >> (2 + 4 * (k & 1))) & B03;
                        exp_decode(e, info, lane, buf + 3328, scratch);
#pragma unroll
                        for (int k = 0; k < 8; ++k) {
                            const unsigned hb = ((nb[k >> 1] >> (4 * (k & 1))) & B03) | (e[k] << 2) | ((w13 << k) & B80), lo = k < 4 ? l0[k & 3] : l1[k & 3];
                            w[k >> 1][(k & 1) * 2] = perm(hb, lo, 0x05010400u); w[k >> 1][(k & 1) * 2 + 1] = perm(hb, lo, 0x07030602u);
                        }
                    }
                    const bool isw = (A.id >> 16) != 0u;
                    const unsigned w8 = A.t & 7u;
                    d1 = isw ? 2048u : 1024u; d2 = isw ? 16384u : 2048u;
                    dst = me + ((unsigned)OFF_STAGE + par * (unsigned)STAGE_BYTES + (A.id & 0xffffu) * UNIT
                                + (isw ? ((w8 & 3u) * 4u + (w8 >> 2)) * 1024u : w8 * 4096u)) + lane * 16;
                } else {
                    if (!(info & INFO_RAW)) {
                        const u32x4 p0 = w[0], p1 = w[1], p2 = w[2];
                        unsigned e[4];
#pragma unroll
                        for (int k = 0; k < 4; ++k) e[k] = (w13 >> (2 * k)) & B03;
                        exp_decode(e, info, lane, buf + 3328, scratch);
#pragma unroll
                        for (int q = 0; q < 4; ++q) {
                            const u32x4 r = {perm(e[q], p0[q], 0x04020100u), perm(e[q], p1[q], 0x05020100u), perm(e[q], p2[q], 0x06020100u),
                                             perm(p1[q], p0[q], 0x0c0c0703u) | perm(e[q], p2[q], 0x07030c0cu)};
                            w[q] = (r >> 1) | (r << 31);
                        }
                    }
                    f32x4 y[4];
#pragma unroll
                    for (int q = 0; q < 4; ++q) y[q] = __builtin_bit_cast(f32x4, own[q]) + __builtin_bit_cast(f32x4, w[q]);
                    if (bad) {
#pragma unroll
                        for (int q = 0; q < 4; ++q) y[q] = (f32x4)qnan;
                    }
                    gelu(y);
#pragma unroll
                    for (int q = 0; q < 4; ++q) w[q] = __builtin_bit_cast(u32x4, y[q]);
                    d1 = NFULL * 4u; d2 = NFULL * 8u;
                    dst = (char*)P.out + out_of(A) + olane;
                }
                store_wt(dst, w[0]); store_wt(dst + d1, w[1]); store_wt(dst + d2, w[2]); store_wt(dst + d2 + d1, w[3]);
                asm volatile("s_waitcnt vmcnt(4)"
                             : "+v"(nown[0]), "+v"(nown[1]), "+v"(nown[2]), "+v"(nown[3]), "+v"(fl), "+v"(ofl), "+v"(tm), "+v"(tk) :: "memory");
                if (pend && lane == 0) add_agent(me + pend, 1u);
                pend = isp ? 0u : OFF_FLAG + (A.id & 0xffffu) * 128u;
                tm = __builtin_amdgcn_readfirstlane(tm);
                if (!more) break;
                if (!early) {
                    if (pend) { WAIT_V(0); if (lane == 0) add_agent(me + pend, 1u); pend = 0; }
                    infob = __builtin_amdgcn_readfirstlane(wave_wait64<2>(me + B.flag, call, tmo, call));
                    post_item(B, infob);
                    tm = __builtin_amdgcn_readfirstlane(ld_agent(tmo));
                }
#pragma unroll
                for (int q = 0; q < 4; ++q) own[q] = nown[q];
                info = infob; A = B; B = C;
                C = item_of(__builtin_amdgcn_readfirstlane(tk) - (call - 1u) * NWORK + 24u * NC);
            }
            post(0u, 0u, 0u);
            WAIT_V(0);
            if (pend && lane == 0) add_agent(me + pend, 1u);
        } else {
            push_last();
            publish_push();
        }
        WAIT_V(0);
        asm volatile("s_endpgm" ::: "memory");
    }

    {
        const bool f1 = aux && remote;
        unsigned item = first_item(bid);
        const int fd = f1 ? peer : lane >> 3;
        char* const fws = f1 ? pws : lws;
#pragma unroll 1
        for (unsigned n = 0;; ++n) {
            const bool isw = f1 || item >= 128u;
            const int s = f1 ? item : isw ? (item - 128u) >> 3 : item >> 4;
            const int j = (f1 ? slot : item) & 7, rb = (item & 7) * 2 + ((item >> 3) & 1);
            const unsigned par = call & 1u;
            u32x4 o[4];
            unsigned off, d1, d2;
            VWAIT(0);
            if (isw) {
                off = ((wid & 3) * 4 + (wid >> 2)) * 1024; d1 = 2048; d2 = 16384;
#pragma unroll
                for (int f = 0; f < 4; ++f) {
                    o[f].x = pack2(v[0][f], v[1][f]); o[f].y = pack2(v[2][f], v[3][f]);
                    o[f].z = pack2(v[4][f], v[5][f]); o[f].w = pack2(v[6][f], v[7][f]);
                }
            } else {
                off = wid * 4096; d1 = 1024; d2 = 2048;
#pragma unroll
                for (int f = 0; f < 4; ++f) {
                    o[f].x = pack2(v[2 * f][0], v[2 * f][1]); o[f].y = pack2(v[2 * f][2], v[2 * f][3]);
                    o[f].z = pack2(v[2 * f + 1][0], v[2 * f + 1][1]); o[f].w = pack2(v[2 * f + 1][2], v[2 * f + 1][3]);
                }
            }
            off += lane * 16;
            const bool last = f1 ? item == 7u : bid < 208 || n == 1u;
            const unsigned u = (unsigned)((rank * 8 + s) * NBLK + (isw ? j : 8 + rb));
            const unsigned wu = (unsigned)((peer * 8 + s) * 8 + j);
            const bool expo = f1 || (!isw && (rb >> 1) != rank);
            char* const ldst = me + (f1 ? OFF_OWNB + (size_t)wu * UNIT : OFF_STAGE + par * STAGE_BYTES + (size_t)u * UNIT) + off;
            store_wt(ldst, o[0]); store_wt(ldst + d1, o[1]); store_wt(ldst + d2, o[2]); store_wt(ldst + d2 + d1, o[3]);
            if (!f1) {
                WAIT_V(0);
                if (lane == 0) add_agent(me + OFF_FLAG + u * 128u, 1u);
            }
            if (expo) {
                char* const ed = me + (f1 ? OFF_EXPW + par * WSET_BYTES + (size_t)wu * UNIT : OFF_EXPX + par * XSET_BYTES + (size_t)(s * 16 + rb) * UNIT) + wid * 4096;
                LAS unsigned char* const tail = lds + LDS_ENC + wid * 1024;
                unsigned hb[8], e[8], sg = 0;
                u32x4 lo[2], nb;
#pragma unroll
                for (int k = 0; k < 8; ++k) {
                    const unsigned w0 = o[k >> 1][(k & 1) * 2], w1 = o[k >> 1][(k & 1) * 2 + 1];
                    lo[k >> 2][k & 3] = perm(w1, w0, 0x06040200u); hb[k] = perm(w1, w0, 0x07050301u);
                    e[k] = (hb[k] >> 2) & 0x1f1f1f1fu; sg |= (hb[k] & B80) >> k;
                }
                const unsigned info = exp_encode(e, lane, tail);
                if (info & INFO_RAW) {
#pragma unroll
                    for (int f = 0; f < 4; ++f) store_wt(ed + f * 1024 + lane * 16, o[f]);
                } else {
#pragma unroll
                    for (int k = 0; k < 4; ++k)
                        nb[k] = (hb[2 * k] & B03) | (e[2 * k] << 2) | ((hb[2 * k + 1] & B03) << 4) | (e[2 * k + 1] << 6);
                    store_wt(ed + lane * 16, lo[0]); store_wt(ed + 1024 + lane * 16, lo[1]); store_wt(ed + 2048 + lane * 16, nb);
                    st_sys(ed + 3072 + lane * 4, sg);
                    if ((unsigned)lane < ((info >> 16) & 127u)) store_wt(ed + 3328 + lane * 16, *(volatile LAS u32x4*)(tail + lane * 16));
                }
                if (lane == 0) LDSW(4 + wid) = info;
            }
            WAIT_V(0);
            WAIT_L(0); BAR; asm volatile("" ::: "memory");
            if (wid == 0) {
                if (expo && (f1 ? lane < 8 : fd != rank))
                    st_sys64(fws + OFF_EFLAG + (par * 64u * NBLK + u) * 64u + (lane & 7) * 8, call, LDSW(4 + (lane & 7)));
                if (f1 && lane == 0) st_sys(me + OFF_OFLAG + wu * 128u, call * 8u);
            }
            if (last) break;
            item += f1 ? 1u : 8u;
            load_item(bid, item);
            WAIT_L(0); BAR;
        }
    }
    const bool ownb = aux && remote;
    int tile = slot;
again:
    const int ti = (aux ? rank : peer) * 2 + (tile >> 3), tj = slot & 7;
    const int nsrc = remote ? 6 : 7;
    const int rfirst = aux && !remote ? 4 : -1, rlast = ownb ? -1 : !aux && !remote ? 3 : 7;
    const int nt = remote ? (aux ? 8 : 56) : (aux ? 28 : 36);
    const int wr = wid >> 2, wc = wid & 3;
    const char* const stage = me + OFF_STAGE + (size_t)(call & 1u) * STAGE_BYTES;
    const char* const abase = stage + (unsigned)(8 + ti) * UNIT;
    const char* const bbase = ownb ? me + OFF_OWNB + (size_t)(((peer - rank) * 64 + tj) * (int)UNIT) : stage + (unsigned)tj * UNIT;
    const unsigned bstep = ownb ? 8u * UNIT : NBLK * UNIT;
    const unsigned voff = (unsigned)tid * 16u;
    const unsigned ldsw = (unsigned)wid * 1024u;
    const unsigned aoff = (unsigned)wr * 8192u + (unsigned)lane * 16u;
    unsigned boff = 65536u + (unsigned)wc * 4096u + (unsigned)lane * 16u;
    asm volatile("" : "+v"(boff));

#define SA(b, h) (((b) * 2 + (h)) * 16384)
#define SB(b, h) ((4 + (b) * 2 + (h)) * 16384)
#define STAGE(bufoff, gbase) do { _Pragma("unroll") for (int _i = 0; _i < 2; ++_i) \
        __builtin_amdgcn_global_load_lds((const GAS unsigned*)((gbase) + _i * 8192 + voff), (LAS unsigned*)(lds + (bufoff) + ldsw + _i * 8192), 16, 0, 17); } while (0)
#define LDA(dst, b, h) do { _Pragma("unroll") for (int m = 0; m < 4; ++m) _Pragma("unroll") for (int k = 0; k < 2; ++k) \
        dst[m][k] = *(const LAS f16x8*)(lds + SA(b, h) + aoff + m * 2048 + k * 1024); } while (0)
#define LDB(dst, b, h) do { _Pragma("unroll") for (int n = 0; n < 2; ++n) _Pragma("unroll") for (int k = 0; k < 2; ++k) \
        dst[n][k] = *(const LAS f16x8*)(lds + boff + (SB(b, h) - 65536 + n * 2048 + k * 1024)); } while (0)
#define MMA(ai, bj, At, Bt) do { __builtin_amdgcn_s_setprio(1); \
        _Pragma("unroll") for (int m = 0; m < 4; ++m) _Pragma("unroll") for (int n = 0; n < 2; ++n) _Pragma("unroll") for (int k = 0; k < 2; ++k) \
        acc[ai][bj][m][n] = __builtin_amdgcn_mfma_f32_16x16x32_f16(At[m][k], Bt[n][k], acc[ai][bj][m][n], 0, 0, 0); \
        __builtin_amdgcn_s_setprio(0); } while (0)

    unsigned ktab;
    {
        const int tt = lane < nt ? lane : nt - 2 + (lane & 1), u = rfirst < 0 ? tt - 8 : tt + rfirst * nsrc;
        const int r = u < 0 ? -1 : remote ? u / 6 : u / 7, q = u - r * nsrc;
        ktab = (u < 0 ? rank * 8 + tt : ((rank + q + 1 + (q + 1 >= skip)) & (NR - 1)) * 8 + r) | ((r + 1) << 8);
    }
    auto flag_of = [&](int s, int lz) -> unsigned {
        const int e = (lz >> 1) & 7, q = e < nsrc ? e : 0, sl = min(s + (lz >> 4), rlast);
        const int kt = sl < 0 ? rank * 8 + e : ((rank + q + 1 + (q + 1 >= skip)) & (NR - 1)) * 8 + sl;
        return ownb && !(lz & 1) ? OFF_OFLAG + (unsigned)((peer * 8 + e) * 8 + tj) * 128u
                                 : OFF_FLAG + (unsigned)(kt * NBLK + tj + (lz & 1) * (8 + ti - tj)) * 128u;
    };
    auto ready = [&](int s) {
        if (wid == 0) {
            int lz = lane; asm volatile("" : "+v"(lz));
            wave_wait<POLL>(me + flag_of(s, lz & 15), call * 8u, tmo, call);
        }
        asm volatile("" ::: "memory"); BAR; asm volatile("" ::: "memory");
    };

    f32x4 acc[2][2][4][2];
#pragma unroll
    for (int a = 0; a < 2; ++a)
#pragma unroll
        for (int b = 0; b < 2; ++b)
#pragma unroll
            for (int m = 0; m < 4; ++m)
#pragma unroll
                for (int n = 0; n < 2; ++n) acc[a][b][m][n] = (f32x4){0.f, 0.f, 0.f, 0.f};
    f16x8 At[4][2], B0[2][2], B1[2][2];

    int have = rfirst;
    ready(rfirst);
    unsigned k2 = __builtin_amdgcn_readlane(ktab, 0) & 63u, k3 = __builtin_amdgcn_readlane(ktab, 1) & 63u;
    {
        const char* a0 = abase + k2 * (NBLK * UNIT); const char* b0 = bbase + k2 * bstep;
        const char* a1 = abase + k3 * (NBLK * UNIT); const char* b1 = bbase + k3 * bstep;
        WAIT_V(0);
        STAGE(SB(0, 0), b0); STAGE(SA(0, 0), a0); STAGE(SB(0, 1), b0 + 16384); STAGE(SA(0, 1), a0 + 16384);
        if (wr == 1) BAR;
        WAIT_V(4); BAR;
        STAGE(SB(1, 0), b1); STAGE(SA(1, 0), a1); STAGE(SB(1, 1), b1 + 16384);
        WAIT_V(6); BAR;
    }
#pragma unroll 1
    for (int t = 0; t < nt; t += 2) {
        const char* a1 = abase + k3 * (NBLK * UNIT);
        k2 = __builtin_amdgcn_readlane(ktab, t + 2) & 63u; k3 = __builtin_amdgcn_readlane(ktab, t + 3);
        const int round3 = (int)(k3 >> 8) - 1, round5 = (int)((unsigned)__builtin_amdgcn_readlane(ktab, t + 5) >> 8) - 1;
        k3 &= 63u;
        {
            if (round3 > have) {
                int lz = lane; asm volatile("" : "+v"(lz));
                const u64 miss = ~__ballot(*(volatile LAS unsigned*)(lds + LDS_ENC + (t & 2) * 128 + lz * 4) >= call * 8u);
                if (miss & 0xffffull) { ready(round3); have = round3; }
                else have = min(round3 + (miss & 0xffff0000ull ? 0 : 1), rlast);
            }
        }
        const char* a2 = abase + k2 * (NBLK * UNIT); const char* b2 = bbase + k2 * bstep;
        const char* a3 = abase + k3 * (NBLK * UNIT); const char* b3 = bbase + k3 * bstep;
        LDB(B0, 0, 0); SCHED; LDA(At, 0, 0); STAGE(SA(1, 1), a1 + 16384);
        WAIT_L(8); BAR; WAIT_L(0); MMA(0, 0, At, B0); BAR; SCHED;
        LDB(B1, 0, 1); STAGE(SB(0, 0), b2);
        BAR; WAIT_L(0); MMA(0, 1, At, B1); BAR;
        LDA(At, 0, 1); STAGE(SA(0, 0), a2);
        BAR; WAIT_L(0); MMA(1, 0, At, B0); BAR; SCHED;
        STAGE(SB(0, 1), b2 + 16384);
        WAIT_V(6); BAR; MMA(1, 1, At, B1); BAR;
        if (wid == 0 && round5 > have) {
            int lz = lane; asm volatile("" : "+v"(lz));
            if (lz < 32)
                __builtin_amdgcn_global_load_lds((const GAS unsigned*)(me + flag_of(have + 1, lz)), (LAS unsigned*)(lds + LDS_ENC + (~t & 2) * 128), 4, 0, 17);
        }
        LDB(B0, 1, 0); SCHED; LDA(At, 1, 0); STAGE(SA(0, 1), a2 + 16384);
        WAIT_L(8); BAR; WAIT_L(0); MMA(0, 0, At, B0); BAR; SCHED;
        LDB(B1, 1, 1); STAGE(SB(1, 0), b3);
        BAR; WAIT_L(0); MMA(0, 1, At, B1); BAR;
        LDA(At, 1, 1); STAGE(SA(1, 0), a3);
        BAR; WAIT_L(0); MMA(1, 0, At, B0); BAR; SCHED;
        STAGE(SB(1, 1), b3 + 16384);
        WAIT_V(6); BAR; MMA(1, 1, At, B1); BAR;
    }
    if (wr == 0) BAR;
    int tz = threadIdx.x; asm volatile("" : "+v"(tz));
    const int lane2 = tz & 63;
    const float qnan = __builtin_nanf("");
#define ACC1(g, q, c) acc[(g) >> 1][(c) >> 1][((g) & 1) * 2 + ((q) >> 2)][(c) & 1][(q) & 3]
#define ACC(g, q) ((f32x4){ACC1(g, q, 0), ACC1(g, q, 1), ACC1(g, q, 2), ACC1(g, q, 3)})
    const unsigned foff = OFF_PFLAG + (unsigned)wid * 256u + (call & 1u) * 128u;
    const size_t poff = OFF_PART + (size_t)(call & 1u) * PART_BYTES + (size_t)slot * TILE_BYTES + wid * 32768 + lane2 * 16;
    if (!aux) {
        char* const dst = me + poff + (size_t)peer * 16 * TILE_BYTES;
        LAS unsigned char* const tail = lds + LDS_ENC + wid * 1024;
        char* const flag = pws + foff + (unsigned)(rank * 16 + slot) * 2048u;
        unsigned infos = 0, badbit = 0;
#pragma unroll
        for (int h = 0; h < 8; ++h) {
            unsigned e[4];
            u32x4 pw_[3];
#pragma unroll
            for (int q = 0; q < 4; ++q) {
                const u32x4 u = __builtin_bit_cast(u32x4, ACC(h >> 1, (h & 1) * 4 + q)), r = (u << 1) | (u >> 31);
                e[q] = perm(perm(r.w, r.z, 0x0c0c0703u), perm(r.y, r.x, 0x0c0c0703u), 0x05040100u);
                pw_[0][q] = perm(r.w, r.x, 0x04020100u); pw_[1][q] = perm(r.w, r.y, 0x05020100u); pw_[2][q] = perm(r.w, r.z, 0x06020100u);
            }
            const unsigned info = remote ? exp_encode(e, lane2, tail) : INFO_RAW;
            if (h == 3) {
                WAIT_V(0);
                if (lane2 == 1 || lane2 == 2) st_sys64(flag + lane2 * 8, call, infos);
            }
            if (info & INFO_RAW) {
#pragma unroll
                for (int q = 0; q < 4; ++q) store_wt(dst + h * 4096 + q * 1024, __builtin_bit_cast(u32x4, ACC(h >> 1, (h & 1) * 4 + q)));
            } else {
#pragma unroll
                for (int q = 0; q < 3; ++q) store_wt(dst + h * 4096 + q * 1024, pw_[q]);
                st_sys(dst + h * 4096 + 3072 - lane2 * 12, e[0] | (e[1] << 2) | (e[2] << 4) | (e[3] << 6));
                if ((unsigned)lane2 < ((info >> 16) & 127u)) store_wt(dst + h * 4096 + 3328, *(volatile LAS u32x4*)(tail + lane2 * 16));
            }
            if (h == 0) badbit = ld_agent(tmo) == call ? INFO_BAD : 0u;
            if (lane2 == h) infos = info | badbit;
            if (h == 0 || h == 7) {
                WAIT_V(0);
                if (h ? lane2 >= 3 && lane2 < 8 : lane2 == 0) st_sys64(flag + lane2 * 8, call, infos);
            }
        }
        return;
    }
    const char* const src = me + poff + (size_t)rank * 16 * TILE_BYTES;
    float* const o = P.out + (size_t)((tile >> 3) * 256 + wr * 64 + (lane2 >> 4) * 4) * NFULL + peer * NS + tj * 256 + wc * 64 + (lane2 & 15) * 4;
    if (!remote) wave_wait64<POLL>(me + foff + (unsigned)(peer * 16 + slot) * 2048u + (lane2 & 7) * 8, call, tmo, call);
    const bool bad = ld_agent(tmo) == call;
#pragma unroll
    for (int g = 0; g < 4; ++g) {
        f32x4 y[8];
#pragma unroll
        for (int q = 0; q < 8; ++q) y[q] = bad ? (f32x4)qnan : ACC(g, q);
        if (!remote) {
            u32x4 w[8];
            asm volatile("global_load_dwordx4 %0, %8, off sc0 sc1\n\t"
                         "global_load_dwordx4 %1, %8, off offset:1024 sc0 sc1\n\t"
                         "global_load_dwordx4 %2, %8, off offset:2048 sc0 sc1\n\t"
                         "global_load_dwordx4 %3, %8, off offset:3072 sc0 sc1\n\t"
                         "global_load_dwordx4 %4, %9, off sc0 sc1\n\t"
                         "global_load_dwordx4 %5, %9, off offset:1024 sc0 sc1\n\t"
                         "global_load_dwordx4 %6, %9, off offset:2048 sc0 sc1\n\t"
                         "global_load_dwordx4 %7, %9, off offset:3072 sc0 sc1\n\ts_waitcnt vmcnt(0)"
                         : "=&v"(w[0]), "=&v"(w[1]), "=&v"(w[2]), "=&v"(w[3]), "=&v"(w[4]), "=&v"(w[5]), "=&v"(w[6]), "=&v"(w[7])
                         : "v"(src + g * 8192), "v"(src + g * 8192 + 4096) : "memory");
#pragma unroll
            for (int q = 0; q < 8; ++q) y[q] += __builtin_bit_cast(f32x4, w[q]);
            gelu(y);
        }
#pragma unroll
        for (int q = 0; q < 8; ++q) {
            if (remote) asm volatile("s_waitcnt vmcnt(%0)" :: "i"(F1DEPTH) : "memory");
            store_wt(o + (size_t)((g >> 1) * 128 + (g & 1) * 32 + (q >> 2) * 16 + (q & 3)) * NFULL, __builtin_bit_cast(u32x4, y[q]));
        }
    }
    WAIT_V(0);
    if (remote) {
        if (lane2 == 0) st_sys(me + OFF_OWNF + (unsigned)((peer * 16 + tile) * 8 + wid) * 64u, call);
        if (tile < 8) {
            for (int i = 0; i < F1GO; ++i) __builtin_amdgcn_s_sleep(127);
            tile += 8;
            goto again;
        }
    }
}

extern "C" void kernel_launch(int rank, int nranks, ncclComm_t comm,
                              hipStream_t stream,
                              void* const* d_in, const int* in_sizes, int n_in,
                              void* d_out, int out_size,
                              const int* in_dtypes, int out_dtype,
                              void* const* d_out_all,
                              void* const* d_ws_all, size_t ws_bytes) {
    (void)comm; (void)in_dtypes; (void)out_dtype; (void)d_out_all;
    if (nranks != NR || (unsigned)rank >= (unsigned)NR || n_in != 2 || ws_bytes < WS_NEED
        || in_sizes[0] != 4096 * KS || in_sizes[1] != KS * NFULL || out_size != 512 * NFULL) {
        fprintf(stderr, "kernel_launch: unexpected problem shape\n");
        return;
    }
    Params P = {};
    P.x = (const float*)d_in[0];
    P.w = (const float*)d_in[1];
    P.out = (float*)d_out;
    for (int p = 0; p < NR; ++p) P.ws[p] = (char*)d_ws_all[p];
    P.me = P.ws[rank];
    P.rank = rank;
    static const unsigned char port[NR][NR] = {
        {0, 2, 3, 6, 4, 7, 5, 1}, {1, 4, 5, 7, 6, 2, 3, 0}, {2, 0, 6, 3, 7, 1, 5, 4}, {3, 6, 0, 2, 7, 4, 1, 5},
        {4, 1, 7, 5, 6, 3, 0, 2}, {5, 7, 1, 4, 0, 6, 2, 3}, {6, 3, 2, 0, 1, 5, 4, 7}, {7, 5, 4, 1, 2, 0, 3, 6}};
    for (int c = 0; c < NR; ++c) P.peermap |= (unsigned)port[rank][c] << (4 * c);
    unsigned n = 0, sw = 0, sa = 1, sb = 0, so = 1;
    for (const char* c = ORDER; *c; ++c) {
        unsigned& s = *c == 'w' ? sw : *c == 'a' ? sa : *c == 'b' ? sb : so;
        const unsigned first = *c == 'w' ? 0 : *c == 'a' ? 8 : *c == 'b' ? 14 : 20, cnt = *c == 'w' ? 8 : *c == 'o' ? 2 : 6;
        for (unsigned i = 0; i < cnt && n < NPULL; ++i, ++n) P.order[n >> 2] |= (((s & 7u) << 5) | (first + i)) << ((n & 3u) * 8u);
        ++s;
    }
    if (n != NPULL || sw != 8 || sa != 8 || sb != 8 || so != 8) { fprintf(stderr, "kernel_launch: bad ORDER\n"); return; }
    gemm_rs_kernel<<<256, 512, 0, stream>>>(P);
    HIP_CHECK(hipGetLastError());
}
```

```cpp
#include <hip/hip_runtime.h>
#include <rccl/rccl.h>
#include <cstdio>

#define HIP_CHECK(expr) do { hipError_t _e = (expr); if (_e != hipSuccess) { \
    fprintf(stderr, "HIP error %s at %s:%d\n", hipGetErrorString(_e), __FILE__, __LINE__); \
    return; } } while (0)

#define LAS __attribute__((address_space(3)))
#define GAS __attribute__((address_space(1)))
typedef _Float16 f16x8 __attribute__((ext_vector_type(8)));
typedef float f32x4 __attribute__((ext_vector_type(4)));
typedef unsigned u32x4 __attribute__((ext_vector_type(4)));
typedef unsigned u32x2 __attribute__((ext_vector_type(2)));
typedef unsigned long long u64;

namespace {

constexpr const char* ORDER = "wawawbawbawbawawawbbbbbooooooo";
constexpr int NC = 6;
constexpr int WIN = 7;
constexpr int FASTW = 2;
constexpr int F1DEPTH = 1;
constexpr int MAPW = 2;
constexpr int POLL = 8;
constexpr int LATE = 80;
static_assert(WIN >= 4 && NC <= 7, "loader bookkeeping");

constexpr int NR = 8;
constexpr int KS = 512;
constexpr int NFULL = 16384;
constexpr int NS = 2048;
constexpr int NBLK = 24;
constexpr unsigned UNIT = 32768;
constexpr unsigned TILE_BYTES = 262144;
constexpr unsigned NPULL = 168, NSLICE = NPULL * 8, NWORK = NSLICE + 1024;
constexpr size_t CTRL_BYTES = 1u << 20;
constexpr size_t STAGE_BYTES = (size_t)64 * NBLK * UNIT;
constexpr size_t PART_BYTES = (size_t)NR * 16 * TILE_BYTES;
constexpr size_t WSET_BYTES = (size_t)NR * 64 * UNIT;
constexpr size_t XSET_BYTES = (size_t)128 * UNIT;
constexpr size_t OFF_STAGE = CTRL_BYTES, OFF_PART = OFF_STAGE + 2 * STAGE_BYTES, OFF_OWNB = OFF_PART + 2 * PART_BYTES,
                 OFF_EXPW = OFF_OWNB + WSET_BYTES, OFF_EXPX = OFF_EXPW + 2 * WSET_BYTES, WS_NEED = OFF_EXPX + 2 * XSET_BYTES;
constexpr unsigned OFF_TMO = 0, OFF_QUEUE = 2048  , OFF_CNT = 4096  ,
                   OFF_FLAG = 65536  , OFF_PFLAG = 262144  ,
                   OFF_OFLAG = 524288  , OFF_OWNF = 589824  ,
                   OFF_EFLAG = 655360  ;
constexpr u64 DEADLINE_TICKS = 50000000ull;
constexpr unsigned LDS_STAGE = 131072, LDS_BYTES = LDS_STAGE + 384 + 8192;
constexpr unsigned LDS_ENC = LDS_STAGE + 384;

struct Params {
    const float* x;
    const float* w;
    char* me;
    int rank;
    unsigned peermap;
    float* out;
    char* ws[NR];
    unsigned order[48];
};

__device__ __forceinline__ void store_wt(void* p, u32x4 v) {
    asm volatile("global_store_dwordx4 %0, %1, off sc0 sc1\n\ts_nop 1" :: "v"(p), "v"(v) : "memory");
}
__device__ __forceinline__ unsigned pack2(float a, float b) {
    typedef _Float16 h2 __attribute__((ext_vector_type(2)));
    h2 h; h.x = (_Float16)a; h.y = (_Float16)b;
    return __builtin_bit_cast(unsigned, h);
}
template <int N> __device__ __forceinline__ void gelu(f32x4 (&y)[N]) {
    const float k1 = -2.0f * 0.7978845608028654f * 1.4426950408889634f, k2 = k1 * 0.044715f;
    f32x4 e[N];
#pragma unroll
    for (int q = 0; q < N; ++q) e[q] = y[q] * __builtin_elementwise_fma(y[q] * y[q], (f32x4)k2, (f32x4)k1);
    __builtin_amdgcn_sched_barrier(0);
#pragma unroll
    for (int i = 0; i < 4 * N; ++i) e[i >> 2][i & 3] = __builtin_amdgcn_exp2f(e[i >> 2][i & 3]);
    __builtin_amdgcn_sched_barrier(0);
#pragma unroll
    for (int q = 0; q < N; ++q) e[q] += 1.0f;
    __builtin_amdgcn_sched_barrier(0);
#pragma unroll
    for (int i = 0; i < 4 * N; ++i) e[i >> 2][i & 3] = __builtin_amdgcn_rcpf(e[i >> 2][i & 3]);
    __builtin_amdgcn_sched_barrier(0);
#pragma unroll
    for (int q = 0; q < N; ++q) y[q] *= e[q];
}
__device__ __forceinline__ unsigned wave_scan(unsigned v) {
    v += __builtin_amdgcn_update_dpp(0, (int)v, 0x111, 0xf, 0xf, true);
    v += __builtin_amdgcn_update_dpp(0, (int)v, 0x112, 0xf, 0xf, true);
    v += __builtin_amdgcn_update_dpp(0, (int)v, 0x114, 0xf, 0xf, true);
    v += __builtin_amdgcn_update_dpp(0, (int)v, 0x118, 0xf, 0xf, true);
    v += __builtin_amdgcn_update_dpp(0, (int)v, 0x142, 0xa, 0xf, false);
    v += __builtin_amdgcn_update_dpp(0, (int)v, 0x143, 0xc, 0xf, false);
    return v;
}
__device__ __forceinline__ unsigned ld_sys(const char* p) {
    return __hip_atomic_load((const GAS unsigned*)p, __ATOMIC_RELAXED, __HIP_MEMORY_SCOPE_SYSTEM);
}
__device__ __forceinline__ u64 ld_sys64(const char* p) {
    return __hip_atomic_load((const GAS u64*)p, __ATOMIC_RELAXED, __HIP_MEMORY_SCOPE_SYSTEM);
}
__device__ __forceinline__ unsigned ld_agent(const char* p) {
    return __hip_atomic_load((const GAS unsigned*)p, __ATOMIC_RELAXED, __HIP_MEMORY_SCOPE_AGENT);
}
__device__ __forceinline__ void st_sys(char* p, unsigned v) {
    __hip_atomic_store((GAS unsigned*)p, v, __ATOMIC_RELAXED, __HIP_MEMORY_SCOPE_SYSTEM);
}
__device__ __forceinline__ void st_sys64(char* p, unsigned lo, unsigned hi) {
    __hip_atomic_store((GAS u64*)p, ((u64)hi << 32) | lo, __ATOMIC_RELAXED, __HIP_MEMORY_SCOPE_SYSTEM);
}
__device__ __forceinline__ unsigned add_agent(char* p, unsigned v) {
    return __hip_atomic_fetch_add((GAS unsigned*)p, v, __ATOMIC_RELAXED, __HIP_MEMORY_SCOPE_AGENT);
}
template <int SLEEP> __device__ __forceinline__ bool give_up(unsigned& n, u64 t0, char* tmo, unsigned call) {
    if ((++n & 15u) == 0u
        && ((unsigned)__builtin_amdgcn_readfirstlane(ld_agent(tmo)) == call || __builtin_amdgcn_s_memrealtime() - t0 > DEADLINE_TICKS)) {
        __hip_atomic_store((GAS unsigned*)tmo, call, __ATOMIC_RELAXED, __HIP_MEMORY_SCOPE_AGENT);
        return true;
    }
    __builtin_amdgcn_s_sleep(SLEEP);
    return false;
}
template <int SLEEP> __device__ __forceinline__ void wave_wait(const char* word, unsigned need, char* tmo, unsigned call) {
    const u64 t0 = __builtin_amdgcn_s_memrealtime();
    unsigned n = 0;
    while (!__all(ld_sys(word) >= need) && !give_up<SLEEP>(n, t0, tmo, call)) { }
    asm volatile("" ::: "memory");
}
template <int SLEEP> __device__ __forceinline__ unsigned wave_wait64(const char* word, unsigned need, char* tmo, unsigned call) {
    const u64 t0 = __builtin_amdgcn_s_memrealtime();
    unsigned n = 0;
    u64 v;
    do { v = ld_sys64(word); } while (!__all((unsigned)v >= need) && !give_up<SLEEP>(n, t0, tmo, call));
    asm volatile("" ::: "memory");
    return (unsigned)(v >> 32);
}

constexpr unsigned B01 = 0x01010101u, B03 = 0x03030303u, B07 = 0x07070707u, B80 = 0x80808080u, INFO_RAW = 1u << 23, INFO_BAD = 1u << 24;
__device__ __forceinline__ unsigned sub8(unsigned a, unsigned b) { return ((a | B80) - (b & ~B80)) ^ ((a ^ ~b) & B80); }
__device__ __forceinline__ unsigned perm(unsigned hi, unsigned lo, unsigned sel) { return __builtin_amdgcn_perm(hi, lo, sel); }
__device__ __forceinline__ void lds_or(LAS unsigned char* p, unsigned v) {
    __hip_atomic_fetch_or((LAS unsigned*)p, v, __ATOMIC_RELAXED, __HIP_MEMORY_SCOPE_WORKGROUP);
}
__device__ __forceinline__ unsigned prefix4(unsigned inc) {
    unsigned t = inc + (inc << 8);
    asm("" : "+v"(t));
    return t + (t << 16);
}
template <int NK> __device__ __forceinline__ unsigned exp_encode(unsigned (&e)[NK], int lane, LAS unsigned char* tail) {
    *(volatile LAS u32x4*)(tail + lane * 16) = (u32x4){0u, 0u, 0u, 0u};
    unsigned sum = 0;
#pragma unroll
    for (int k = 0; k < NK; ++k) sum = __builtin_amdgcn_sad_u8(e[k], 0u, sum);
    const unsigned c = (((unsigned)__builtin_amdgcn_readlane(wave_scan(sum), 63) / (NK * 256u)) + 3u) & 255u;
    const unsigned c4 = ((c & 15u) | 16u) * B01, r0 = sub8(c * B01, 0x03020100u), r1 = sub8(c * B01, 0x030a0908u), r2 = sub8(c * B01, 0x07060504u);
    unsigned acc[NK / 2], len[NK / 2], fix[NK], any = 0, mine = 0;
#pragma unroll
    for (int k = 0; k < NK; ++k) {
        const unsigned h = c4 - (e[k] & 0x0f0f0f0fu), sel = h & B07, m = perm(0u, 0u, ((h >> 3) & B01) | 0x0c0c0c0cu);
        const unsigned cs = (perm(0x4c3c2c1cu, 0x037c6c5cu, sel) & m) | (perm(0x4c3c2c1cu, 0x0302010cu, sel) & ~m);
        fix[k] = e[k] ^ ((perm(r2, r1, sel) & m) | (perm(r2, r0, sel) & ~m));
        any |= fix[k];
        const unsigned sy = (cs >> 4) & B07, in = prefix4((cs >> 2) & B03);
        const unsigned ch = (sy & 255u) | (((sy >> 8) & 255u) << (in & 255u)) | (((sy >> 16) & 255u) << ((in >> 8) & 255u)) | ((sy >> 24) << ((in >> 16) & 255u));
        if (k & 1) { acc[k >> 1] |= ch << len[k >> 1]; len[k >> 1] += in >> 24; } else { acc[k >> 1] = ch; len[k >> 1] = in >> 24; }
        mine += in >> 24;
        e[k] = cs & B03;
    }
    const unsigned incl = wave_scan(mine);
    unsigned p = incl - mine;
#pragma unroll
    for (int g = 0; g < NK / 2; ++g) {
        const u64 t = (u64)acc[g] << (p & 31u);
        lds_or(tail + (p >> 5) * 4u, (unsigned)t); lds_or(tail + (p >> 5) * 4u + 4u, (unsigned)(t >> 32));
        p += len[g];
    }
    const unsigned sbytes = ((__builtin_amdgcn_readlane(incl, 63) + 31u) >> 5) * 4u;
    unsigned n = 0;
    u64 has;
    while ((has = __ballot(any != 0u)) != 0ull && n <= 64u) {
        unsigned f = 0, kk = 0;
        any = 0;
#pragma unroll
        for (int k = NK - 1; k >= 0; --k) if (fix[k] != 0u) { f = fix[k]; kk = k; }
        const unsigned sh = __builtin_ctz(f | 0x80000000u) & 24u;
        const unsigned idx = n + __builtin_amdgcn_mbcnt_hi((unsigned)(has >> 32), __builtin_amdgcn_mbcnt_lo((unsigned)has, 0u));
        if (f != 0u && idx < 64u)
            *(volatile LAS unsigned short*)(tail + sbytes + idx * 2u) = (lane << 10) | (kk << 7) | (sh << 2) | ((f >> (sh + 3u)) & 31u);
#pragma unroll
        for (int k = 0; k < NK; ++k) { if (kk == (unsigned)k) fix[k] = f & ~(255u << sh); any |= fix[k]; }
        n += __builtin_popcountll(has);
    }
    const unsigned bytes = sbytes + n * 2u;
    return n > 64u || bytes > 768u ? INFO_RAW : c | (n << 8) | (((bytes + 15u) >> 4) << 16);
}
template <int NK> __device__ __forceinline__ void exp_decode(unsigned (&e)[NK], unsigned info, int lane, const LAS unsigned char* tail, LAS unsigned char* scratch) {
    *(volatile LAS u32x4*)(scratch + lane * 32) = (u32x4){0u, 0u, 0u, 0u};
    if (NK > 4) *(volatile LAS u32x4*)(scratch + lane * 32 + 16) = (u32x4){0u, 0u, 0u, 0u};
    unsigned m[NK], in[NK], p[NK / 2], ch[NK], sy[NK], mine = 0;
#pragma unroll
    for (int k = 0; k < NK; ++k) {
        m[k] = perm(0u, 0xffu, e[k]);
        in[k] = prefix4(m[k] & B03);
        if (!(k & 1)) p[k >> 1] = mine;
        mine += in[k] >> 24;
    }
    const unsigned incl = wave_scan(mine), sbytes = ((__builtin_amdgcn_readlane(incl, 63) + 31u) >> 5) * 4u;
    unsigned en = 0;
    if ((unsigned)lane < ((info >> 8) & 127u)) en = *(const volatile LAS unsigned short*)(tail + sbytes + lane * 2);
#pragma unroll
    for (int g = 0; g < NK / 2; ++g) {
        p[g] += incl - mine;
        const LAS unsigned* const q = (const LAS unsigned*)(tail + (p[g] >> 5) * 4u);
        ch[2 * g] = __builtin_amdgcn_alignbit(q[1], q[0], p[g]);
        ch[2 * g + 1] = ch[2 * g] >> (in[2 * g] >> 24);
    }
    if (en & 31u) lds_or(scratch + (en >> 7) * 4u, (en & 31u) << (((en >> 2) & 24u) + 3u));
    u32x4 fx[NK / 4];
#pragma unroll
    for (int k = 0; k < NK / 4; ++k) fx[k] = *(volatile LAS u32x4*)(scratch + lane * 32 + k * 16);
#define SPREAD1(d, i, c, j, jm) "v_lshrrev_b32_sdwa " d ", " i ", " c " dst_sel:BYTE_" #j " dst_unused:UNUSED_PRESERVE src0_sel:BYTE_" #jm " src1_sel:DWORD\n\t"
#define SPREAD4(j, jm) SPREAD1("%0", "%4", "%8", j, jm) SPREAD1("%1", "%5", "%9", j, jm) SPREAD1("%2", "%6", "%10", j, jm) SPREAD1("%3", "%7", "%11", j, jm)
#pragma unroll
    for (int k = 0; k < NK; k += 4) {
        sy[k] = ch[k]; sy[k + 1] = ch[k + 1]; sy[k + 2] = ch[k + 2]; sy[k + 3] = ch[k + 3];
        asm("s_nop 0\n\t" SPREAD4(1, 0) SPREAD4(2, 1) SPREAD4(3, 2) "s_nop 0"
            : "+&v"(sy[k]), "+&v"(sy[k + 1]), "+&v"(sy[k + 2]), "+&v"(sy[k + 3])
            : "v"(in[k]), "v"(in[k + 1]), "v"(in[k + 2]), "v"(in[k + 3]), "v"(ch[k]), "v"(ch[k + 1]), "v"(ch[k + 2]), "v"(ch[k + 3]));
    }
    const unsigned c4 = (info & 255u) * B01, tc = sub8(c4, 0x03020100u), t0 = sub8(c4, 0x06050400u), t1 = sub8(c4, 0x0a090807u);
#pragma unroll
    for (int k = 0; k < NK; ++k)
        e[k] = ((perm(t1, t0, sy[k] & B07) & m[k]) | (perm(0u, tc, e[k]) & ~m[k])) ^ fx[k >> 2][k & 3];
}

#define WAIT_V(n) asm volatile("s_waitcnt vmcnt(" #n ")" ::: "memory")
#define WAIT_L(n) asm volatile("s_waitcnt lgkmcnt(" #n ")" ::: "memory")
#define BAR __builtin_amdgcn_s_barrier()
#define SCHED __builtin_amdgcn_sched_barrier(0)
#define LDSW(i) (*(volatile LAS unsigned*)(lds + LDS_STAGE + 4 * (i)))

}

__global__ __launch_bounds__(512, 2) void gemm_rs_kernel(Params P) {
    __shared__ __attribute__((aligned(16))) unsigned char shm[LDS_BYTES];
    LAS unsigned char* lds = (LAS unsigned char*)shm;
    const int tid = threadIdx.x, lane = tid & 63, wid = __builtin_amdgcn_readfirstlane(tid >> 6);
    const int hb = __builtin_amdgcn_readfirstlane(blockIdx.x);
    const float* px = P.x; const float* pw = P.w; char* me_ = P.me; int rank_ = P.rank; unsigned peermap = P.peermap;
    asm volatile("" : "+s"(px), "+s"(pw), "+s"(me_), "+s"(rank_), "+s"(peermap));
    const int rank = rank_ & (NR - 1);
    char* const me = (char*)(GAS char*)(size_t)me_;
    char* const tmo = me + OFF_TMO;
    int bid = (hb & ~7) | (__builtin_amdgcn_s_getreg((3 << 11) | 20) & 7);
    if (bid >= 128 && (bid & 7) != 0) {
        const int tt = ((bid & 7) - 1) * 8 + ((bid >> 3) & 7), cg = tt / (7 * MAPW), r = tt - cg * (7 * MAPW);
        bid = (bid & 192) | ((cg * MAPW + r % MAPW) << 3) | (r / MAPW + 1);
    }
    f32x4 v[8] = {};
    auto first_item = [&](int b) -> unsigned {
        const int q = ((b >> 3) & 7) + 1, sk = ((peermap >> ((b & 7) * 4)) - rank) & (NR - 1);
        const int pl = q < 7 ? (rank + q + (q >= sk)) & (NR - 1) : ((rank + sk) & (NR - 1)) + (q & 8);
        const int lo = (b & 7) == 0 ? (b >> 6) * 16 + 7 + q : b < 64 ? 0 : pl;
        const int hi = b >= 192 ? (q - 1) * 16 + (b & 7) : b;
        return b >= 128 ? hi : lo;
    };
    auto load_item = [&](int b, unsigned item) {
        const bool f1 = b < 64 && (b & 7) != 0, isw = f1 || item >= 128u;
        const unsigned ws_ = f1 ? item : (item >> 3) & 7u, wj = (f1 ? b >> 3 : item) & 7u, wd = f1 ? (peermap >> ((b & 7) * 4)) & (NR - 1) : rank;
        const unsigned wo = (ws_ * 64u + (wid >> 2) * 32u) * NFULL + wd * NS + wj * 256u + (wid & 3) * 64u;
        const unsigned xo = (((item & 7u) * 2u + ((item >> 3) & 1u)) * 256u + wid * 32u) * KS + ((item >> 4) & 7u) * 64u;
        const float* const base = isw ? pw + wo : px + xo;
        unsigned lo = (lane >> 4) * (isw ? 16u * NFULL : 16u) + (lane & 15) * (isw ? 16u : 4u * KS);
        const unsigned sb = isw ? 4u * NFULL : 64u, sa = (isw ? 64u * NFULL : 64u * KS) - 3u * sb;
#define LD1(i, step) "global_load_dwordx4 %" #i ", %8, %9\n\tv_add_u32 %8, " step ", %8\n\t"
        asm volatile("s_nop 4\n\t" LD1(0, "%10") LD1(1, "%10") LD1(2, "%10") LD1(3, "%11") LD1(4, "%10") LD1(5, "%10") LD1(6, "%10")
                     "global_load_dwordx4 %7, %8, %9"
                     : "+v"(v[0]), "+v"(v[1]), "+v"(v[2]), "+v"(v[3]), "+v"(v[4]), "+v"(v[5]), "+v"(v[6]), "+v"(v[7]), "+v"(lo)
                     : "s"(base), "s"(sb), "s"(sa) : "memory");
    };
#define VWAIT(n) asm volatile("s_waitcnt vmcnt(" #n ")" : "+v"(v[0]), "+v"(v[1]), "+v"(v[2]), "+v"(v[3]), "+v"(v[4]), "+v"(v[5]), "+v"(v[6]), "+v"(v[7]) :: "memory")
    unsigned t = 0, ordw[3];
    char* lws;
    {
        u64 ex;
        asm volatile("s_mov_b64 %5, exec\n\ts_mov_b64 exec, %12\n\t"
                     "global_atomic_add %0, %6, %7, %8 sc0\n\ts_mov_b64 exec, %5\n\t"
                     "global_load_dwordx2 %1, %9, %11 offset:%13\n\t"
                     "global_load_dword %2, %10, %11 offset:%14\n\t"
                     "global_load_dword %3, %10, %11 offset:%14+64\n\t"
                     "global_load_dword %4, %10, %11 offset:%14+128"
                     : "+v"(t), "=&v"(lws), "=&v"(ordw[0]), "=&v"(ordw[1]), "=&v"(ordw[2]), "=&s"(ex)
                     : "v"(OFF_CNT + (lane ? bid * 128 + 4 : hb * 128)), "v"(1u), "s"(me), "v"(lane & 56), "v"(lane & 60),
                       "s"(__builtin_amdgcn_kernarg_segment_ptr()), "s"((u64)(wid == 0 ? 3u : 0u)),
                       "i"(__builtin_offsetof(Params, ws)), "i"(__builtin_offsetof(Params, order)) : "memory");
    }
    if (bid >= 128 ? (bid >> 4) != 12 : (bid & 7) == 0) __builtin_amdgcn_s_sleep(LATE);
    char* pws_;
    unsigned call = 0;
#pragma unroll 1
    for (;;) {
        asm volatile("" ::: "memory");
        load_item(bid, first_item(bid));
        pws_ = P.ws[(peermap >> ((bid & 7) * 4)) & (NR - 1)];
        asm volatile("s_waitcnt vmcnt(8)" : "+v"(t), "+v"(lws), "+v"(ordw[0]), "+v"(ordw[1]), "+v"(ordw[2]) :: "memory");
        if (wid == 0 && call == 0u) {
            const unsigned c = __builtin_amdgcn_readlane(t, 0) + 1u;
            unsigned old = __builtin_amdgcn_readlane(t, 1), r = bid;
            for (int tries = 0; old != c - 1u && tries < 65536; ++tries) {
                if (lane == 0) add_agent(me + OFF_CNT + r * 128u + 4u, ~0u);
                r = (r + 1u) & 255u;
                if (lane == 0) t = add_agent(me + OFF_CNT + r * 128u + 4u, 1u);
                old = __builtin_amdgcn_readfirstlane(t);
            }
            LDSW(16 + lane) = 0u;
            if (lane == 0) { LDSW(0) = c; LDSW(12) = r; }
        }
        asm volatile("s_waitcnt lgkmcnt(0)" : "+s"(pws_) :: "memory"); BAR; asm volatile("" ::: "memory");
        call = __builtin_amdgcn_readfirstlane(LDSW(0));
        const int got = __builtin_amdgcn_readfirstlane(LDSW(12));
        if (got == bid) break;
        bid = got;
        VWAIT(0);
    }
    const int peer = (peermap >> ((bid & 7) * 4)) & (NR - 1);
    const int slot = (bid >> 3) & 15;
    const bool aux = bid < 128, remote = (bid & 7) != 0;
    const int skip = remote ? (peer - rank) & (NR - 1) : NR;
    char* const pws = (char*)(GAS char*)(size_t)pws_;

    if (aux && remote && slot >= 8) {
        const unsigned par = call & 1u;
        const unsigned pu = (unsigned)(rank * 8 * NBLK + 8) + (first_item(bid) & 7u) * 2u + ((first_item(bid) >> 3) & 1u);
        char* const rdst = pws + OFF_STAGE + par * STAGE_BYTES + (size_t)pu * UNIT + wid * 4096 + lane * 16;
        auto frag = [&](int f) -> u32x4 {
            return (u32x4){pack2(v[2 * f][0], v[2 * f][1]), pack2(v[2 * f][2], v[2 * f][3]),
                           pack2(v[2 * f + 1][0], v[2 * f + 1][1]), pack2(v[2 * f + 1][2], v[2 * f + 1][3])};
        };
        VWAIT(6); store_wt(rdst, frag(0));
        if (wid > FASTW) WAIT_V(0);
        VWAIT(5); store_wt(rdst + 1024, frag(1));
        if (wid > FASTW) WAIT_V(0);
        VWAIT(4); store_wt(rdst + 2048, frag(2));
        VWAIT(3);
        const u32x4 o3 = frag(3);
        auto push_last = [&]() { WAIT_V(0); store_wt(rdst + 3072, o3); };
        auto publish_push = [&]() {
            WAIT_V(0);
            __builtin_amdgcn_s_waitcnt(0x0F70);
            if (lane == 0)
                __hip_atomic_fetch_add((GAS unsigned*)(pws + OFF_FLAG + pu * 128u), 1u, __ATOMIC_RELAXED, __HIP_MEMORY_SCOPE_SYSTEM);
        };
        if (wid == 0) {
            push_last();
            publish_push();
            __builtin_amdgcn_s_setprio(3);
            const unsigned lbase = (unsigned)(size_t)lds, landed = lbase + (lane ? 129024 + lane * 4 : LDS_STAGE + 68);
            unsigned head = 0, alive = NC, idle = 3;
            int issued = 0;
            u32x2 d = *(volatile LAS u32x2*)(lds + LDS_STAGE + 128);
#define PIECE(off) "s_waitcnt vmcnt(%5)\n\tds_max_i32 %4, %1\n\tv_add_u32 %1, 1, %1\n\tglobal_load_lds_dwordx4 %2, off offset:" #off " sc0 sc1\n\t"
            while (alive) {
                if ((unsigned)__builtin_amdgcn_readfirstlane(d.y) != head + 1u) {
                    if (idle == 1u) asm volatile("s_waitcnt vmcnt(4)\n\tds_max_i32 %0, %1" :: "v"(landed), "v"(issued - 4) : "memory");
                    if (idle == 2u) asm volatile("s_waitcnt vmcnt(0)\n\tds_max_i32 %0, %1" :: "v"(landed), "v"(issued) : "memory");
                    ++idle;
                    d = *(volatile LAS u32x2*)(lds + LDS_STAGE + 128 + (head & 15u) * 8u);
                    continue;
                }
                ++head;
                const unsigned meta = __builtin_amdgcn_readfirstlane(d.x), cnt = (meta >> 17) & 15u;
                const unsigned dst = lbase + (meta >> 28) * 8192u;
                unsigned keep;
                const char* const src = pws + (meta & 0x1ffffu) * 4096u + lane * 16;
                d = *(volatile LAS u32x2*)(lds + LDS_STAGE + 128 + (head & 15u) * 8u);
                if (cnt == 0u) { --alive; continue; }
                const u64 mask = ~0ull >> (64u - ((meta >> 21) & 127u));
                int known = issued - (WIN - 1);
                asm volatile("s_mov_b32 %0, m0\n\ts_mov_b32 m0, %3\n\t" PIECE(0) PIECE(1024) PIECE(2048)
                             "s_waitcnt vmcnt(%5)\n\tds_max_i32 %4, %1\n\ts_mov_b64 exec, %6\n\t"
                             "global_load_lds_dwordx4 %2, off offset:3072 sc0 sc1\n\ts_mov_b64 exec, -1\n\ts_mov_b32 m0, %0"
                             : "=&s"(keep), "+v"(known) : "v"(src), "s"(dst), "v"(landed), "i"(WIN - 1), "s"(mask) : "memory");
                issued += cnt;
                idle = 0u;
            }
        } else if (wid <= NC) {
            const unsigned me8 = wid - 1;
            LAS unsigned char* const scratch = lds + 114688 + me8 * 2048u;
            unsigned bi = me8 * 2u + 1u;
            const float qnan = __builtin_nanf("");
            unsigned tbl = 0;
#pragma unroll
            for (int h = 0; h < 3; ++h) {
                const unsigned e = (ordw[h] >> ((lane & 3) * 8)) & 255u, code = e & 31u;
                const int odd = code >= 14u, q = code - (odd ? 14u : 8u);
                const unsigned blk = code < 8u ? code : code >= 20u ? 8u + rank * 2 + (code - 20u)
                                                                     : 8u + ((rank + q + 1 + (q + 1 >= skip)) & (NR - 1)) * 2 + odd;
                tbl |= ((e & 0xe0u) | blk) << (8 * h);
            }
            struct Item { unsigned t, id, flag, src; };
            auto item_of = [&](unsigned t) -> Item {
                const unsigned it = min(t >> 3, NPULL - 1u), e = (unsigned)__builtin_amdgcn_readlane(tbl, it & 63u) >> ((it >> 6) * 8u);
                const unsigned ks = (e >> 5) & 7u, blk = e & 31u, u = (peer * 8u + ks) * NBLK + blk;
                const unsigned so = blk < 8u ? (unsigned)OFF_EXPW + par * (unsigned)WSET_BYTES + ((rank * 8u + ks) * 8u + blk) * UNIT
                                             : (unsigned)OFF_EXPX + par * (unsigned)XSET_BYTES + (ks * 16u + blk - 8u) * UNIT;
                const unsigned i = max(min(t, NWORK - 1u), NSLICE) - NSLICE;
                const unsigned ci = ((i >> 9) * 64u + (i & 63u)) * 8u + ((i >> 6) & 7u);
                Item r;
                r.t = t;
                r.id = t < NSLICE ? u | (blk < 8u ? 0x10000u : 0u) : ci;
                r.flag = t < NSLICE ? OFF_EFLAG + (par * 64u * NBLK + u) * 64u + (t & 7u) * 8u
                                    : OFF_PFLAG + (peer * 128u + (ci >> 3)) * 256u + par * 128u + (ci & 7u) * 8u;
                r.src = t < NSLICE ? so + (t & 7u) * 4096u : (unsigned)OFF_PART + par * (unsigned)PART_BYTES + rank * 16u * TILE_BYTES + ci * 4096u;
                return r;
            };
            auto ownf_of = [&](const Item& x) -> unsigned { return OFF_OWNF + (peer * 128u + (x.t < NSLICE ? 0u : x.id >> 3)) * 64u; };
            auto out_of = [&](const Item& x) -> unsigned {
                const unsigned ct = (x.id >> 6) & 15u, cw = (x.id >> 3) & 7u, cg = x.id & 7u;
                return (((ct >> 3) * 256u + (cg >> 2) * 128u + (cw >> 2) * 64u + (cg & 3u) * 16u) * NFULL + peer * NS + (ct & 7u) * 256u + (cw & 3u) * 64u) * 4u;
            };
            const unsigned olane = ((lane >> 4) * 4u * NFULL + (lane & 15) * 4u) * 4u;
            int rend = 0;
            auto post = [&](unsigned src, unsigned cnt, unsigned last) {
                unsigned k = 0;
                bi ^= 1u;
                if (lane == 0) {
                    asm volatile("ds_add_rtn_u32 %0, %1, %2\n\ts_waitcnt lgkmcnt(0)" : "=&v"(k) : "v"((unsigned)(size_t)lds + LDS_STAGE + 64u), "v"(0x10000u + cnt) : "memory");
                    *(volatile LAS u32x2*)(lds + LDS_STAGE + 128 + ((k >> 16) & 15u) * 8u) = (u32x2){(src >> 12) | (cnt << 17) | (last << 21) | (bi << 28), (k >> 16) + 1u};
                }
                rend = (__builtin_amdgcn_readfirstlane(k) & 0xffffu) + cnt;
            };
            auto post_item = [&](const Item& x, unsigned inf) {
                post(x.src, 4u, inf & INFO_RAW ? 64u : min(16u + ((inf >> 16) & 127u), 64u));
            };
            Item A = item_of((slot - 8) * NC + me8), B = item_of(A.t + 8u * NC), C = item_of(A.t + 16u * NC);
            const u64 f0 = ld_sys64(me + A.flag);
            unsigned info = __builtin_amdgcn_readfirstlane((unsigned)(f0 >> 32)), ofl = 0, tm = 0;
            const bool now = __all((unsigned)f0 >= call);
            push_last();
            if (now) post_item(A, info);
            publish_push();
            if (!now) { info = __builtin_amdgcn_readfirstlane(wave_wait64<2>(me + A.flag, call, tmo, call)); post_item(A, info); }
            u64 fl;
            do { fl = ld_sys64(me + B.flag); } while ((unsigned)__builtin_amdgcn_readfirstlane((unsigned)fl) < call && (int)__builtin_amdgcn_readfirstlane(LDSW(17)) < rend);
            unsigned pend = 0;
            u32x4 own[4] = {};
#pragma unroll 1
            for (;;) {
                const bool isp = A.t >= NSLICE, bad = tm == call || (info & INFO_BAD) != 0u;
                if ((int)__builtin_amdgcn_readfirstlane(LDSW(17)) < rend) {
                    if (pend) { WAIT_V(0); if (lane == 0) add_agent(me + pend, 1u); pend = 0; }
                    while ((int)__builtin_amdgcn_readfirstlane(LDSW(17)) < rend) __builtin_amdgcn_s_sleep(1);
                }
                asm volatile("" ::: "memory");
                const LAS unsigned char* const buf = lds + bi * 8192u;
                const bool more = B.t < NWORK, early = more && (unsigned)__builtin_amdgcn_readfirstlane((unsigned)fl) >= call;
                unsigned infob = __builtin_amdgcn_readfirstlane((unsigned)(fl >> 32));
                if (early) post_item(B, infob);
                u32x4 nown[4];
                u64 ex;
                unsigned tk;
                const bool nextp = more && B.t >= NSLICE;
                if (nextp && (unsigned)__builtin_amdgcn_readfirstlane(ofl) < call) wave_wait<2>(me + ownf_of(B), call, tmo, call);
                {
                    unsigned vo = out_of(B) + olane;
                    asm volatile("s_mov_b64 %8, exec\n\ts_mov_b32 exec_lo, %16\n\ts_mov_b32 exec_hi, %16\n\ts_nop 2\n\t"
                                 "global_load_dwordx4 %0, %9, %10 sc0 sc1\n\tv_add_u32 %9, 0x10000, %9\n\t"
                                 "global_load_dwordx4 %1, %9, %10 sc0 sc1\n\tv_add_u32 %9, 0x10000, %9\n\t"
                                 "global_load_dwordx4 %2, %9, %10 sc0 sc1\n\tv_add_u32 %9, 0x10000, %9\n\t"
                                 "global_load_dwordx4 %3, %9, %10 sc0 sc1\n\t"
                                 "s_mov_b64 exec, 1\n\t"
                                 "global_load_dwordx2 %4, %12, %11 sc0 sc1\n\t"
                                 "global_load_dword %5, %13, %11 sc0 sc1\n\t"
                                 "global_load_dword %6, %17, %11 sc1\n\t"
                                 "global_atomic_add %7, %14, %15, %11 sc0\n\t"
                                 "s_mov_b64 exec, %8"
                                 : "=&v"(nown[0]), "=&v"(nown[1]), "=&v"(nown[2]), "=&v"(nown[3]),
                                   "=&v"(fl), "=&v"(ofl), "=&v"(tm), "=&v"(tk), "=&s"(ex), "+v"(vo)
                                 : "s"(P.out), "s"(me), "v"(C.flag), "v"(ownf_of(C)), "v"(OFF_QUEUE + peer * 128u), "v"(1u),
                                   "s"(__builtin_amdgcn_readfirstlane(nextp ? -1 : 0)), "v"(OFF_TMO) : "memory");
                }
                u32x4 w[4];
#pragma unroll
                for (int q = 0; q < 4; ++q) w[q] = *(const LAS u32x4*)(buf + q * 1024 + lane * 16);
                const unsigned w13 = *(const LAS unsigned*)(buf + 3072 + lane * 4);
                char* dst;
                unsigned d1, d2;
                if (!isp) {
                    if (!(info & INFO_RAW)) {
                        const u32x4 l0 = w[0], l1 = w[1], nb = w[2];
                        unsigned e[8];
#pragma unroll
                        for (int k = 0; k < 8; ++k) e[k] = (nb[k >> 1] >> (2 + 4 * (k & 1))) & B03;
                        exp_decode(e, info, lane, buf + 3328, scratch);
#pragma unroll
                        for (int k = 0; k < 8; ++k) {
                            const unsigned hb = ((nb[k >> 1] >> (4 * (k & 1))) & B03) | (e[k] << 2) | ((w13 << k) & B80), lo = k < 4 ? l0[k & 3] : l1[k & 3];
                            w[k >> 1][(k & 1) * 2] = perm(hb, lo, 0x05010400u); w[k >> 1][(k & 1) * 2 + 1] = perm(hb, lo, 0x07030602u);
                        }
                    }
                    const bool isw = (A.id >> 16) != 0u;
                    const unsigned w8 = A.t & 7u;
                    d1 = isw ? 2048u : 1024u; d2 = isw ? 16384u : 2048u;
                    dst = me + ((unsigned)OFF_STAGE + par * (unsigned)STAGE_BYTES + (A.id & 0xffffu) * UNIT
                                + (isw ? ((w8 & 3u) * 4u + (w8 >> 2)) * 1024u : w8 * 4096u)) + lane * 16;
                } else {
                    if (!(info & INFO_RAW)) {
                        const u32x4 p0 = w[0], p1 = w[1], p2 = w[2];
                        unsigned e[4];
#pragma unroll
                        for (int k = 0; k < 4; ++k) e[k] = (w13 >> (2 * k)) & B03;
                        exp_decode(e, info, lane, buf + 3328, scratch);
#pragma unroll
                        for (int q = 0; q < 4; ++q) {
                            const u32x4 r = {perm(e[q], p0[q], 0x04020100u), perm(e[q], p1[q], 0x05020100u), perm(e[q], p2[q], 0x06020100u),
                                             perm(p1[q], p0[q], 0x0c0c0703u) | perm(e[q], p2[q], 0x07030c0cu)};
                            w[q] = (r >> 1) | (r << 31);
                        }
                    }
                    f32x4 y[4];
#pragma unroll
                    for (int q = 0; q < 4; ++q) y[q] = __builtin_bit_cast(f32x4, own[q]) + __builtin_bit_cast(f32x4, w[q]);
                    if (bad) {
#pragma unroll
                        for (int q = 0; q < 4; ++q) y[q] = (f32x4)qnan;
                    }
                    gelu(y);
#pragma unroll
                    for (int q = 0; q < 4; ++q) w[q] = __builtin_bit_cast(u32x4, y[q]);
                    d1 = NFULL * 4u; d2 = NFULL * 8u;
                    dst = (char*)P.out + out_of(A) + olane;
                }
                store_wt(dst, w[0]); store_wt(dst + d1, w[1]); store_wt(dst + d2, w[2]); store_wt(dst + d2 + d1, w[3]);
                asm volatile("s_waitcnt vmcnt(4)"
                             : "+v"(nown[0]), "+v"(nown[1]), "+v"(nown[2]), "+v"(nown[3]), "+v"(fl), "+v"(ofl), "+v"(tm), "+v"(tk) :: "memory");
                if (pend && lane == 0) add_agent(me + pend, 1u);
                pend = isp ? 0u : OFF_FLAG + (A.id & 0xffffu) * 128u;
                tm = __builtin_amdgcn_readfirstlane(tm);
                if (!more) break;
                if (!early) {
                    if (pend) { WAIT_V(0); if (lane == 0) add_agent(me + pend, 1u); pend = 0; }
                    infob = __builtin_amdgcn_readfirstlane(wave_wait64<2>(me + B.flag, call, tmo, call));
                    post_item(B, infob);
                    tm = __builtin_amdgcn_readfirstlane(ld_agent(tmo));
                }
#pragma unroll
                for (int q = 0; q < 4; ++q) own[q] = nown[q];
                info = infob; A = B; B = C;
                C = item_of(__builtin_amdgcn_readfirstlane(tk) - (call - 1u) * NWORK + 24u * NC);
            }
            post(0u, 0u, 0u);
            WAIT_V(0);
            if (pend && lane == 0) add_agent(me + pend, 1u);
        } else {
            push_last();
            publish_push();
        }
        WAIT_V(0);
        asm volatile("s_endpgm" ::: "memory");
    }

    {
        const bool f1 = aux && remote;
        unsigned item = first_item(bid);
        const int fd = f1 ? peer : lane >> 3;
        char* const fws = f1 ? pws : lws;
#pragma unroll 1
        for (unsigned n = 0;; ++n) {
            const bool isw = f1 || item >= 128u;
            const int s = f1 ? item : isw ? (item - 128u) >> 3 : item >> 4;
            const int j = (f1 ? slot : item) & 7, rb = (item & 7) * 2 + ((item >> 3) & 1);
            const unsigned par = call & 1u;
            u32x4 o[4];
            unsigned off, d1, d2;
            VWAIT(0);
            if (isw) {
                off = ((wid & 3) * 4 + (wid >> 2)) * 1024; d1 = 2048; d2 = 16384;
#pragma unroll
                for (int f = 0; f < 4; ++f) {
                    o[f].x = pack2(v[0][f], v[1][f]); o[f].y = pack2(v[2][f], v[3][f]);
                    o[f].z = pack2(v[4][f], v[5][f]); o[f].w = pack2(v[6][f], v[7][f]);
                }
            } else {
                off = wid * 4096; d1 = 1024; d2 = 2048;
#pragma unroll
                for (int f = 0; f < 4; ++f) {
                    o[f].x = pack2(v[2 * f][0], v[2 * f][1]); o[f].y = pack2(v[2 * f][2], v[2 * f][3]);
                    o[f].z = pack2(v[2 * f + 1][0], v[2 * f + 1][1]); o[f].w = pack2(v[2 * f + 1][2], v[2 * f + 1][3]);
                }
            }
            off += lane * 16;
            const bool last = f1 ? item == 7u : bid < 208 || n == 1u;
            const unsigned u = (unsigned)((rank * 8 + s) * NBLK + (isw ? j : 8 + rb));
            const unsigned wu = (unsigned)((peer * 8 + s) * 8 + j);
            const bool expo = f1 || (!isw && (rb >> 1) != rank && item >= 8u);
            char* const ldst = me + (f1 ? OFF_OWNB + (size_t)wu * UNIT : OFF_STAGE + par * STAGE_BYTES + (size_t)u * UNIT) + off;
            store_wt(ldst, o[0]); store_wt(ldst + d1, o[1]); store_wt(ldst + d2, o[2]); store_wt(ldst + d2 + d1, o[3]);
            if (!f1) {
                WAIT_V(0);
                if (lane == 0) add_agent(me + OFF_FLAG + u * 128u, 1u);
            }
            if (expo) {
                char* const ed = me + (f1 ? OFF_EXPW + par * WSET_BYTES + (size_t)wu * UNIT : OFF_EXPX + par * XSET_BYTES + (size_t)(s * 16 + rb) * UNIT) + wid * 4096;
                LAS unsigned char* const tail = lds + LDS_ENC + wid * 1024;
                unsigned hb[8], e[8], sg = 0;
                u32x4 lo[2], nb;
#pragma unroll
                for (int k = 0; k < 8; ++k) {
                    const unsigned w0 = o[k >> 1][(k & 1) * 2], w1 = o[k >> 1][(k & 1) * 2 + 1];
                    lo[k >> 2][k & 3] = perm(w1, w0, 0x06040200u); hb[k] = perm(w1, w0, 0x07050301u);
                    e[k] = (hb[k] >> 2) & 0x1f1f1f1fu; sg |= (hb[k] & B80) >> k;
                }
                const unsigned info = exp_encode(e, lane, tail);
                if (info & INFO_RAW) {
#pragma unroll
                    for (int f = 0; f < 4; ++f) store_wt(ed + f * 1024 + lane * 16, o[f]);
                } else {
#pragma unroll
                    for (int k = 0; k < 4; ++k)
                        nb[k] = (hb[2 * k] & B03) | (e[2 * k] << 2) | ((hb[2 * k + 1] & B03) << 4) | (e[2 * k + 1] << 6);
                    store_wt(ed + lane * 16, lo[0]); store_wt(ed + 1024 + lane * 16, lo[1]); store_wt(ed + 2048 + lane * 16, nb);
                    st_sys(ed + 3072 + lane * 4, sg);
                    if ((unsigned)lane < ((info >> 16) & 127u)) store_wt(ed + 3328 + lane * 16, *(volatile LAS u32x4*)(tail + lane * 16));
                }
                if (lane == 0) LDSW(4 + wid) = info;
            }
            WAIT_V(0);
            WAIT_L(0); BAR; asm volatile("" ::: "memory");
            if (wid == 0) {
                if (expo && (f1 ? lane < 8 : fd != rank))
                    st_sys64(fws + OFF_EFLAG + (par * 64u * NBLK + u) * 64u + (lane & 7) * 8, call, LDSW(4 + (lane & 7)));
                if (f1 && lane == 0) st_sys(me + OFF_OFLAG + wu * 128u, call * 8u);
            }
            if (last) break;
            item += f1 ? 1u : 8u;
            load_item(bid, item);
            WAIT_L(0); BAR;
        }
    }
    const bool ownb = aux && remote;
    int tile = slot;
again:
    const int ti = (aux ? rank : peer) * 2 + (tile >> 3), tj = slot & 7;
    const int nsrc = remote ? 6 : 7;
    const int rfirst = aux && !remote ? 4 : -1, rlast = ownb ? -1 : !aux && !remote ? 3 : 7;
    const int nt = remote ? (aux ? 8 : 56) : (aux ? 28 : 36);
    const int wr = wid >> 2, wc = wid & 3;
    const char* const stage = me + OFF_STAGE + (size_t)(call & 1u) * STAGE_BYTES;
    const char* const abase = stage + (unsigned)(8 + ti) * UNIT;
    const char* const bbase = ownb ? me + OFF_OWNB + (size_t)(((peer - rank) * 64 + tj) * (int)UNIT) : stage + (unsigned)tj * UNIT;
    const unsigned bstep = ownb ? 8u * UNIT : NBLK * UNIT;
    const unsigned voff = (unsigned)tid * 16u;
    const unsigned ldsw = (unsigned)wid * 1024u;
    const unsigned aoff = (unsigned)wr * 8192u + (unsigned)lane * 16u;
    unsigned boff = 65536u + (unsigned)wc * 4096u + (unsigned)lane * 16u;
    asm volatile("" : "+v"(boff));

#define SA(b, h) (((b) * 2 + (h)) * 16384)
#define SB(b, h) ((4 + (b) * 2 + (h)) * 16384)
#define STAGE(bufoff, gbase) do { _Pragma("unroll") for (int _i = 0; _i < 2; ++_i) \
        __builtin_amdgcn_global_load_lds((const GAS unsigned*)((gbase) + _i * 8192 + voff), (LAS unsigned*)(lds + (bufoff) + ldsw + _i * 8192), 16, 0, 17); } while (0)
#define LDA(dst, b, h) do { _Pragma("unroll") for (int m = 0; m < 4; ++m) _Pragma("unroll") for (int k = 0; k < 2; ++k) \
        dst[m][k] = *(const LAS f16x8*)(lds + SA(b, h) + aoff + m * 2048 + k * 1024); } while (0)
#define LDB(dst, b, h) do { _Pragma("unroll") for (int n = 0; n < 2; ++n) _Pragma("unroll") for (int k = 0; k < 2; ++k) \
        dst[n][k] = *(const LAS f16x8*)(lds + boff + (SB(b, h) - 65536 + n * 2048 + k * 1024)); } while (0)
#define MMA(ai, bj, At, Bt) do { __builtin_amdgcn_s_setprio(1); \
        _Pragma("unroll") for (int m = 0; m < 4; ++m) _Pragma("unroll") for (int n = 0; n < 2; ++n) _Pragma("unroll") for (int k = 0; k < 2; ++k) \
        acc[ai][bj][m][n] = __builtin_amdgcn_mfma_f32_16x16x32_f16(At[m][k], Bt[n][k], acc[ai][bj][m][n], 0, 0, 0); \
        __builtin_amdgcn_s_setprio(0); } while (0)

    unsigned ktab;
    {
        const int tt = lane < nt ? lane : nt - 2 + (lane & 1), u = rfirst < 0 ? tt - 8 : tt + rfirst * nsrc;
        const int r = u < 0 ? -1 : remote ? u / 6 : u / 7, q = u - r * nsrc;
        ktab = (u < 0 ? rank * 8 + tt : ((rank + q + 1 + (q + 1 >= skip)) & (NR - 1)) * 8 + r) | ((r + 1) << 8);
    }
    auto flag_of = [&](int s, int lz) -> unsigned {
        const int e = (lz >> 1) & 7, q = e < nsrc ? e : 0, sl = min(s + (lz >> 4), rlast);
        const int kt = sl < 0 ? rank * 8 + e : ((rank + q + 1 + (q + 1 >= skip)) & (NR - 1)) * 8 + sl;
        return ownb && !(lz & 1) ? OFF_OFLAG + (unsigned)((peer * 8 + e) * 8 + tj) * 128u
                                 : OFF_FLAG + (unsigned)(kt * NBLK + tj + (lz & 1) * (8 + ti - tj)) * 128u;
    };
    auto ready = [&](int s) {
        if (wid == 0) {
            int lz = lane; asm volatile("" : "+v"(lz));
            wave_wait<POLL>(me + flag_of(s, lz & 15), call * 8u, tmo, call);
        }
        asm volatile("" ::: "memory"); BAR; asm volatile("" ::: "memory");
    };

    f32x4 acc[2][2][4][2];
#pragma unroll
    for (int a = 0; a < 2; ++a)
#pragma unroll
        for (int b = 0; b < 2; ++b)
#pragma unroll
            for (int m = 0; m < 4; ++m)
#pragma unroll
                for (int n = 0; n < 2; ++n) acc[a][b][m][n] = (f32x4){0.f, 0.f, 0.f, 0.f};
    f16x8 At[4][2], B0[2][2], B1[2][2];

    int have = rfirst;
    ready(rfirst);
    unsigned k2 = __builtin_amdgcn_readlane(ktab, 0) & 63u, k3 = __builtin_amdgcn_readlane(ktab, 1) & 63u;
    {
        const char* a0 = abase + k2 * (NBLK * UNIT); const char* b0 = bbase + k2 * bstep;
        const char* a1 = abase + k3 * (NBLK * UNIT); const char* b1 = bbase + k3 * bstep;
        WAIT_V(0);
        STAGE(SB(0, 0), b0); STAGE(SA(0, 0), a0); STAGE(SB(0, 1), b0 + 16384); STAGE(SA(0, 1), a0 + 16384);
        if (wr == 1) BAR;
        WAIT_V(4); BAR;
        STAGE(SB(1, 0), b1); STAGE(SA(1, 0), a1); STAGE(SB(1, 1), b1 + 16384);
        WAIT_V(6); BAR;
    }
#pragma unroll 1
    for (int t = 0; t < nt; t += 2) {
        const char* a1 = abase + k3 * (NBLK * UNIT);
        k2 = __builtin_amdgcn_readlane(ktab, t + 2) & 63u; k3 = __builtin_amdgcn_readlane(ktab, t + 3);
        const int round3 = (int)(k3 >> 8) - 1, round5 = (int)((unsigned)__builtin_amdgcn_readlane(ktab, t + 5) >> 8) - 1;
        k3 &= 63u;
        if (round3 > have) {
            int lz = lane; asm volatile("" : "+v"(lz) :: "memory");
            const u64 miss = ~__ballot(*(volatile LAS unsigned*)(lds + LDS_ENC + (t & 2) * 128 + lz * 4) >= call * 8u);
            if (miss & 0xffffull) { ready(round3); have = round3; }
            else have = min(round3 + (miss & 0xffff0000ull ? 0 : 1), rlast);
            asm volatile("" ::: "memory");
        }
        const char* a2 = abase + k2 * (NBLK * UNIT); const char* b2 = bbase + k2 * bstep;
        const char* a3 = abase + k3 * (NBLK * UNIT); const char* b3 = bbase + k3 * bstep;
        LDB(B0, 0, 0); SCHED; LDA(At, 0, 0); STAGE(SA(1, 1), a1 + 16384);
        WAIT_L(8); BAR; WAIT_L(0); MMA(0, 0, At, B0); BAR; SCHED;
        LDB(B1, 0, 1); STAGE(SB(0, 0), b2);
        BAR; WAIT_L(0); MMA(0, 1, At, B1); BAR;
        LDA(At, 0, 1); STAGE(SA(0, 0), a2);
        BAR; WAIT_L(0); MMA(1, 0, At, B0); BAR; SCHED;
        STAGE(SB(0, 1), b2 + 16384);
        WAIT_V(6); BAR; MMA(1, 1, At, B1); BAR;
        if (wid == 0 && round5 > have) {
            int lz = lane; asm volatile("" : "+v"(lz));
            if (lz < 32)
                __builtin_amdgcn_global_load_lds((const GAS unsigned*)(me + flag_of(have + 1, lz)), (LAS unsigned*)(lds + LDS_ENC + (~t & 2) * 128), 4, 0, 17);
        }
        LDB(B0, 1, 0); SCHED; LDA(At, 1, 0); STAGE(SA(0, 1), a2 + 16384);
        WAIT_L(8); BAR; WAIT_L(0); MMA(0, 0, At, B0); BAR; SCHED;
        LDB(B1, 1, 1); STAGE(SB(1, 0), b3);
        BAR; WAIT_L(0); MMA(0, 1, At, B1); BAR;
        LDA(At, 1, 1); STAGE(SA(1, 0), a3);
        BAR; WAIT_L(0); MMA(1, 0, At, B0); BAR; SCHED;
        STAGE(SB(1, 1), b3 + 16384);
        WAIT_V(6); BAR; MMA(1, 1, At, B1); BAR;
    }
    if (wr == 0) BAR;
    int tz = threadIdx.x; asm volatile("" : "+v"(tz));
    const int lane2 = tz & 63;
    const float qnan = __builtin_nanf("");
#define ACC1(g, q, c) acc[(g) >> 1][(c) >> 1][((g) & 1) * 2 + ((q) >> 2)][(c) & 1][(q) & 3]
#define ACC(g, q) ((f32x4){ACC1(g, q, 0), ACC1(g, q, 1), ACC1(g, q, 2), ACC1(g, q, 3)})
    const unsigned foff = OFF_PFLAG + (unsigned)wid * 256u + (call & 1u) * 128u;
    const size_t poff = OFF_PART + (size_t)(call & 1u) * PART_BYTES + (size_t)slot * TILE_BYTES + wid * 32768 + lane2 * 16;
    if (!aux) {
        char* const dst = me + poff + (size_t)peer * 16 * TILE_BYTES;
        LAS unsigned char* const tail = lds + LDS_ENC + wid * 1024;
        char* const flag = pws + foff + (unsigned)(rank * 16 + slot) * 2048u;
        unsigned infos = 0, badbit = 0;
#pragma unroll
        for (int h = 0; h < 8; ++h) {
            unsigned e[4];
            u32x4 pw_[3];
#pragma unroll
            for (int q = 0; q < 4; ++q) {
                const u32x4 u = __builtin_bit_cast(u32x4, ACC(h >> 1, (h & 1) * 4 + q)), r = (u << 1) | (u >> 31);
                e[q] = perm(perm(r.w, r.z, 0x0c0c0703u), perm(r.y, r.x, 0x0c0c0703u), 0x05040100u);
                pw_[0][q] = perm(r.w, r.x, 0x04020100u); pw_[1][q] = perm(r.w, r.y, 0x05020100u); pw_[2][q] = perm(r.w, r.z, 0x06020100u);
            }
            const unsigned info = remote ? exp_encode(e, lane2, tail) : INFO_RAW;
            if (h == 3) {
                WAIT_V(0);
                if (lane2 == 1 || lane2 == 2) st_sys64(flag + lane2 * 8, call, infos);
            }
            if (info & INFO_RAW) {
#pragma unroll
                for (int q = 0; q < 4; ++q) store_wt(dst + h * 4096 + q * 1024, __builtin_bit_cast(u32x4, ACC(h >> 1, (h & 1) * 4 + q)));
            } else {
#pragma unroll
                for (int q = 0; q < 3; ++q) store_wt(dst + h * 4096 + q * 1024, pw_[q]);
                st_sys(dst + h * 4096 + 3072 - lane2 * 12, e[0] | (e[1] << 2) | (e[2] << 4) | (e[3] << 6));
                if ((unsigned)lane2 < ((info >> 16) & 127u)) store_wt(dst + h * 4096 + 3328, *(volatile LAS u32x4*)(tail + lane2 * 16));
            }
            if (h == 0) badbit = ld_agent(tmo) == call ? INFO_BAD : 0u;
            if (lane2 == h) infos = info | badbit;
            if (h == 0 || h == 7) {
                WAIT_V(0);
                if (h ? lane2 >= 3 && lane2 < 8 : lane2 == 0) st_sys64(flag + lane2 * 8, call, infos);
            }
        }
        return;
    }
    const char* const src = me + poff + (size_t)rank * 16 * TILE_BYTES;
    float* const o = P.out + (size_t)((tile >> 3) * 256 + wr * 64 + (lane2 >> 4) * 4) * NFULL + peer * NS + tj * 256 + wc * 64 + (lane2 & 15) * 4;
    if (!remote) wave_wait64<POLL>(me + foff + (unsigned)(peer * 16 + slot) * 2048u + (lane2 & 7) * 8, call, tmo, call);
    const bool bad = ld_agent(tmo) == call;
#pragma unroll
    for (int g = 0; g < 4; ++g) {
        f32x4 y[8];
#pragma unroll
        for (int q = 0; q < 8; ++q) y[q] = bad ? (f32x4)qnan : ACC(g, q);
        if (!remote) {
            u32x4 w[8];
            asm volatile("global_load_dwordx4 %0, %8, off sc0 sc1\n\t"
                         "global_load_dwordx4 %1, %8, off offset:1024 sc0 sc1\n\t"
                         "global_load_dwordx4 %2, %8, off offset:2048 sc0 sc1\n\t"
                         "global_load_dwordx4 %3, %8, off offset:3072 sc0 sc1\n\t"
                         "global_load_dwordx4 %4, %9, off sc0 sc1\n\t"
                         "global_load_dwordx4 %5, %9, off offset:1024 sc0 sc1\n\t"
                         "global_load_dwordx4 %6, %9, off offset:2048 sc0 sc1\n\t"
                         "global_load_dwordx4 %7, %9, off offset:3072 sc0 sc1\n\ts_waitcnt vmcnt(0)"
                         : "=&v"(w[0]), "=&v"(w[1]), "=&v"(w[2]), "=&v"(w[3]), "=&v"(w[4]), "=&v"(w[5]), "=&v"(w[6]), "=&v"(w[7])
                         : "v"(src + g * 8192), "v"(src + g * 8192 + 4096) : "memory");
#pragma unroll
            for (int q = 0; q < 8; ++q) y[q] += __builtin_bit_cast(f32x4, w[q]);
            gelu(y);
        }
#pragma unroll
        for (int q = 0; q < 8; ++q) {
            if (remote) asm volatile("s_waitcnt vmcnt(%0)" :: "i"(F1DEPTH) : "memory");
            store_wt(o + (size_t)((g >> 1) * 128 + (g & 1) * 32 + (q >> 2) * 16 + (q & 3)) * NFULL, __builtin_bit_cast(u32x4, y[q]));
        }
    }
    WAIT_V(0);
    if (remote) {
        if (lane2 == 0) st_sys(me + OFF_OWNF + (unsigned)((peer * 16 + tile) * 8 + wid) * 64u, call);
        if (tile < 8) { tile += 8; goto again; }
    }
}

extern "C" void kernel_launch(int rank, int nranks, ncclComm_t comm,
                              hipStream_t stream,
                              void* const* d_in, const int* in_sizes, int n_in,
                              void* d_out, int out_size,
                              const int* in_dtypes, int out_dtype,
                              void* const* d_out_all,
                              void* const* d_ws_all, size_t ws_bytes) {
    (void)comm; (void)in_dtypes; (void)out_dtype; (void)d_out_all;
    if (nranks != NR || (unsigned)rank >= (unsigned)NR || n_in != 2 || ws_bytes < WS_NEED
        || in_sizes[0] != 4096 * KS || in_sizes[1] != KS * NFULL || out_size != 512 * NFULL) {
        fprintf(stderr, "kernel_launch: unexpected problem shape\n");
        return;
    }
    Params P = {};
    P.x = (const float*)d_in[0];
    P.w = (const float*)d_in[1];
    P.out = (float*)d_out;
    for (int p = 0; p < NR; ++p) P.ws[p] = (char*)d_ws_all[p];
    P.me = P.ws[rank];
    P.rank = rank;
    static const unsigned char port[NR][NR] = {
        {0, 2, 3, 6, 4, 7, 5, 1}, {1, 4, 5, 7, 6, 2, 3, 0}, {2, 0, 6, 3, 7, 1, 5, 4}, {3, 6, 0, 2, 7, 4, 1, 5},
        {4, 1, 7, 5, 6, 3, 0, 2}, {5, 7, 1, 4, 0, 6, 2, 3}, {6, 3, 2, 0, 1, 5, 4, 7}, {7, 5, 4, 1, 2, 0, 3, 6}};
    for (int c = 0; c < NR; ++c) P.peermap |= (unsigned)port[rank][c] << (4 * c);
    unsigned n = 0, sw = 0, sa = 1, sb = 0, so = 1;
    for (const char* c = ORDER; *c; ++c) {
        unsigned& s = *c == 'w' ? sw : *c == 'a' ? sa : *c == 'b' ? sb : so;
        const unsigned first = *c == 'w' ? 0 : *c == 'a' ? 8 : *c == 'b' ? 14 : 20, cnt = *c == 'w' ? 8 : *c == 'o' ? 2 : 6;
        for (unsigned i = 0; i < cnt && n < NPULL; ++i, ++n) P.order[n >> 2] |= (((s & 7u) << 5) | (first + i)) << ((n & 3u) * 8u);
        ++s;
    }
    if (n != NPULL || sw != 8 || sa != 8 || sb != 8 || so != 8) { fprintf(stderr, "kernel_launch: bad ORDER\n"); return; }
    gemm_rs_kernel<<<256, 512, 0, stream>>>(P);
    HIP_CHECK(hipGetLastError());
}
```

```cpp
#include <hip/hip_runtime.h>
#include <rccl/rccl.h>
#include <cstdio>

#define HIP_CHECK(expr) do { hipError_t _e = (expr); if (_e != hipSuccess) { \
    fprintf(stderr, "HIP error %s at %s:%d\n", hipGetErrorString(_e), __FILE__, __LINE__); \
    return; } } while (0)

#define LAS __attribute__((address_space(3)))
#define GAS __attribute__((address_space(1)))
typedef _Float16 f16x8 __attribute__((ext_vector_type(8)));
typedef float f32x4 __attribute__((ext_vector_type(4)));
typedef unsigned u32x4 __attribute__((ext_vector_type(4)));
typedef unsigned u32x2 __attribute__((ext_vector_type(2)));
typedef unsigned long long u64;

namespace {

constexpr const char* ORDER = "wawawbawbawbawawawbbbbb";
constexpr int NC = 6;
constexpr int WIN = 7;
constexpr int FASTW = 2;
constexpr int F1DEPTH = 1;
constexpr int MAPW = 2;
constexpr int POLL = 8;
constexpr int LATE = 80;
constexpr int PLAN1 = 1000, PLAND = 185, PLANN = 12;
constexpr int PLANB = 20, PLANMAX = 5, PLANGAP = 13;
static_assert(WIN >= 4 && NC <= 6, "loader bookkeeping, one idle wave");

constexpr int NR = 8;
constexpr int KS = 512;
constexpr int NFULL = 16384;
constexpr int NS = 2048;
constexpr int NBLK = 24;
constexpr unsigned UNIT = 32768;
constexpr unsigned TILE_BYTES = 262144;
constexpr unsigned NPULL = 168, NFIX = NPULL - 7, NCHUNK = 1024;
constexpr size_t CTRL_BYTES = 1u << 20;
constexpr size_t STAGE_BYTES = (size_t)64 * NBLK * UNIT;
constexpr size_t PART_BYTES = (size_t)NR * 16 * TILE_BYTES;
constexpr size_t WSET_BYTES = (size_t)NR * 64 * UNIT;
constexpr size_t XSET_BYTES = (size_t)128 * UNIT;
constexpr size_t OFF_STAGE = CTRL_BYTES, OFF_PART = OFF_STAGE + 2 * STAGE_BYTES, OFF_OWNB = OFF_PART + 2 * PART_BYTES,
                 OFF_EXPW = OFF_OWNB + WSET_BYTES, OFF_EXPX = OFF_EXPW + 2 * WSET_BYTES, WS_NEED = OFF_EXPX + 2 * XSET_BYTES;
constexpr unsigned OFF_TMO = 0, OFF_PLAN = 1024  , OFF_QUEUE = 2048  , OFF_CNT = 4096  ,
                   OFF_FLAG = 65536  , OFF_PFLAG = 262144  ,
                   OFF_OFLAG = 524288  , OFF_OWNF = 589824  ,
                   OFF_EFLAG = 655360  ;
constexpr u64 DEADLINE_TICKS = 50000000ull;
constexpr unsigned LDS_STAGE = 131072, LDS_BYTES = LDS_STAGE + 384 + 8192;
constexpr unsigned LDS_ENC = LDS_STAGE + 384;

struct Params {
    const float* x;
    const float* w;
    char* me;
    int rank;
    unsigned peermap;
    float* out;
    char* ws[NR];
    unsigned order[48];
};

__device__ __forceinline__ void store_wt(void* p, u32x4 v) {
    asm volatile("global_store_dwordx4 %0, %1, off sc0 sc1\n\ts_nop 1" :: "v"(p), "v"(v) : "memory");
}
__device__ __forceinline__ unsigned pack2(float a, float b) {
    typedef _Float16 h2 __attribute__((ext_vector_type(2)));
    h2 h; h.x = (_Float16)a; h.y = (_Float16)b;
    return __builtin_bit_cast(unsigned, h);
}
template <int N> __device__ __forceinline__ void gelu(f32x4 (&y)[N]) {
    const float k1 = -2.0f * 0.7978845608028654f * 1.4426950408889634f, k2 = k1 * 0.044715f;
    f32x4 e[N];
#pragma unroll
    for (int q = 0; q < N; ++q) e[q] = y[q] * __builtin_elementwise_fma(y[q] * y[q], (f32x4)k2, (f32x4)k1);
    __builtin_amdgcn_sched_barrier(0);
#pragma unroll
    for (int i = 0; i < 4 * N; ++i) e[i >> 2][i & 3] = __builtin_amdgcn_exp2f(e[i >> 2][i & 3]);
    __builtin_amdgcn_sched_barrier(0);
#pragma unroll
    for (int q = 0; q < N; ++q) e[q] += 1.0f;
    __builtin_amdgcn_sched_barrier(0);
#pragma unroll
    for (int i = 0; i < 4 * N; ++i) e[i >> 2][i & 3] = __builtin_amdgcn_rcpf(e[i >> 2][i & 3]);
    __builtin_amdgcn_sched_barrier(0);
#pragma unroll
    for (int q = 0; q < N; ++q) y[q] *= e[q];
}
__device__ __forceinline__ unsigned wave_scan(unsigned v) {
    v += __builtin_amdgcn_update_dpp(0, (int)v, 0x111, 0xf, 0xf, true);
    v += __builtin_amdgcn_update_dpp(0, (int)v, 0x112, 0xf, 0xf, true);
    v += __builtin_amdgcn_update_dpp(0, (int)v, 0x114, 0xf, 0xf, true);
    v += __builtin_amdgcn_update_dpp(0, (int)v, 0x118, 0xf, 0xf, true);
    v += __builtin_amdgcn_update_dpp(0, (int)v, 0x142, 0xa, 0xf, false);
    v += __builtin_amdgcn_update_dpp(0, (int)v, 0x143, 0xc, 0xf, false);
    return v;
}
__device__ __forceinline__ unsigned ld_sys(const char* p) {
    return __hip_atomic_load((const GAS unsigned*)p, __ATOMIC_RELAXED, __HIP_MEMORY_SCOPE_SYSTEM);
}
__device__ __forceinline__ u64 ld_sys64(const char* p) {
    return __hip_atomic_load((const GAS u64*)p, __ATOMIC_RELAXED, __HIP_MEMORY_SCOPE_SYSTEM);
}
__device__ __forceinline__ unsigned ld_agent(const char* p) {
    return __hip_atomic_load((const GAS unsigned*)p, __ATOMIC_RELAXED, __HIP_MEMORY_SCOPE_AGENT);
}
__device__ __forceinline__ void st_sys(char* p, unsigned v) {
    __hip_atomic_store((GAS unsigned*)p, v, __ATOMIC_RELAXED, __HIP_MEMORY_SCOPE_SYSTEM);
}
__device__ __forceinline__ void st_sys64(char* p, unsigned lo, unsigned hi) {
    __hip_atomic_store((GAS u64*)p, ((u64)hi << 32) | lo, __ATOMIC_RELAXED, __HIP_MEMORY_SCOPE_SYSTEM);
}
__device__ __forceinline__ void add_sys(char* p) {
    __hip_atomic_fetch_add((GAS unsigned*)p, 1u, __ATOMIC_RELAXED, __HIP_MEMORY_SCOPE_SYSTEM);
}
__device__ __forceinline__ unsigned add_agent(char* p, unsigned v) {
    return __hip_atomic_fetch_add((GAS unsigned*)p, v, __ATOMIC_RELAXED, __HIP_MEMORY_SCOPE_AGENT);
}
template <int SLEEP> __device__ __forceinline__ bool give_up(unsigned& n, u64 t0, char* tmo, unsigned call) {
    if ((++n & 15u) == 0u
        && ((unsigned)__builtin_amdgcn_readfirstlane(ld_agent(tmo)) == call || __builtin_amdgcn_s_memrealtime() - t0 > DEADLINE_TICKS)) {
        __hip_atomic_store((GAS unsigned*)tmo, call, __ATOMIC_RELAXED, __HIP_MEMORY_SCOPE_AGENT);
        return true;
    }
    __builtin_amdgcn_s_sleep(SLEEP);
    return false;
}
template <int SLEEP> __device__ __forceinline__ void wave_wait(const char* word, unsigned need, char* tmo, unsigned call) {
    const u64 t0 = __builtin_amdgcn_s_memrealtime();
    unsigned n = 0;
    while (!__all(ld_sys(word) >= need) && !give_up<SLEEP>(n, t0, tmo, call)) { }
    asm volatile("" ::: "memory");
}
template <int SLEEP> __device__ __forceinline__ unsigned wave_wait64(const char* word, unsigned need, char* tmo, unsigned call) {
    const u64 t0 = __builtin_amdgcn_s_memrealtime();
    unsigned n = 0;
    u64 v;
    do { v = ld_sys64(word); } while (!__all((unsigned)v >= need) && !give_up<SLEEP>(n, t0, tmo, call));
    asm volatile("" ::: "memory");
    return (unsigned)(v >> 32);
}

constexpr unsigned B01 = 0x01010101u, B03 = 0x03030303u, B07 = 0x07070707u, B80 = 0x80808080u, INFO_RAW = 1u << 23, INFO_BAD = 1u << 24;
__device__ __forceinline__ unsigned sub8(unsigned a, unsigned b) { return ((a | B80) - (b & ~B80)) ^ ((a ^ ~b) & B80); }
__device__ __forceinline__ unsigned perm(unsigned hi, unsigned lo, unsigned sel) { return __builtin_amdgcn_perm(hi, lo, sel); }
__device__ __forceinline__ void lds_or(LAS unsigned char* p, unsigned v) {
    __hip_atomic_fetch_or((LAS unsigned*)p, v, __ATOMIC_RELAXED, __HIP_MEMORY_SCOPE_WORKGROUP);
}
__device__ __forceinline__ unsigned prefix4(unsigned inc) {
    unsigned t = inc + (inc << 8);
    asm("" : "+v"(t));
    return t + (t << 16);
}
template <int NK> __device__ __forceinline__ unsigned exp_encode(unsigned (&e)[NK], int lane, LAS unsigned char* tail) {
    *(volatile LAS u32x4*)(tail + lane * 16) = (u32x4){0u, 0u, 0u, 0u};
    unsigned sum = 0;
#pragma unroll
    for (int k = 0; k < NK; ++k) sum = __builtin_amdgcn_sad_u8(e[k], 0u, sum);
    const unsigned c = (((unsigned)__builtin_amdgcn_readlane(wave_scan(sum), 63) / (NK * 256u)) + 3u) & 255u;
    const unsigned c4 = ((c & 15u) | 16u) * B01, r0 = sub8(c * B01, 0x03020100u), r1 = sub8(c * B01, 0x030a0908u), r2 = sub8(c * B01, 0x07060504u);
    unsigned acc[NK / 2], len[NK / 2], fix[NK], any = 0, mine = 0;
#pragma unroll
    for (int k = 0; k < NK; ++k) {
        const unsigned h = c4 - (e[k] & 0x0f0f0f0fu), sel = h & B07, m = perm(0u, 0u, ((h >> 3) & B01) | 0x0c0c0c0cu);
        const unsigned cs = (perm(0x4c3c2c1cu, 0x037c6c5cu, sel) & m) | (perm(0x4c3c2c1cu, 0x0302010cu, sel) & ~m);
        fix[k] = e[k] ^ ((perm(r2, r1, sel) & m) | (perm(r2, r0, sel) & ~m));
        any |= fix[k];
        const unsigned sy = (cs >> 4) & B07, in = prefix4((cs >> 2) & B03);
        const unsigned ch = (sy & 255u) | (((sy >> 8) & 255u) << (in & 255u)) | (((sy >> 16) & 255u) << ((in >> 8) & 255u)) | ((sy >> 24) << ((in >> 16) & 255u));
        if (k & 1) { acc[k >> 1] |= ch << len[k >> 1]; len[k >> 1] += in >> 24; } else { acc[k >> 1] = ch; len[k >> 1] = in >> 24; }
        mine += in >> 24;
        e[k] = cs & B03;
    }
    const unsigned incl = wave_scan(mine);
    unsigned p = incl - mine;
#pragma unroll
    for (int g = 0; g < NK / 2; ++g) {
        const u64 t = (u64)acc[g] << (p & 31u);
        lds_or(tail + (p >> 5) * 4u, (unsigned)t); lds_or(tail + (p >> 5) * 4u + 4u, (unsigned)(t >> 32));
        p += len[g];
    }
    const unsigned sbytes = ((__builtin_amdgcn_readlane(incl, 63) + 31u) >> 5) * 4u;
    unsigned n = 0;
    u64 has;
    while ((has = __ballot(any != 0u)) != 0ull && n <= 64u) {
        unsigned f = 0, kk = 0;
        any = 0;
#pragma unroll
        for (int k = NK - 1; k >= 0; --k) if (fix[k] != 0u) { f = fix[k]; kk = k; }
        const unsigned sh = __builtin_ctz(f | 0x80000000u) & 24u;
        const unsigned idx = n + __builtin_amdgcn_mbcnt_hi((unsigned)(has >> 32), __builtin_amdgcn_mbcnt_lo((unsigned)has, 0u));
        if (f != 0u && idx < 64u)
            *(volatile LAS unsigned short*)(tail + sbytes + idx * 2u) = (lane << 10) | (kk << 7) | (sh << 2) | ((f >> (sh + 3u)) & 31u);
#pragma unroll
        for (int k = 0; k < NK; ++k) { if (kk == (unsigned)k) fix[k] = f & ~(255u << sh); any |= fix[k]; }
        n += __builtin_popcountll(has);
    }
    const unsigned bytes = sbytes + n * 2u;
    return n > 64u || bytes > 768u ? INFO_RAW : c | (n << 8) | (((bytes + 15u) >> 4) << 16);
}
template <int NK> __device__ __forceinline__ void exp_decode(unsigned (&e)[NK], unsigned info, int lane, const LAS unsigned char* tail, LAS unsigned char* scratch) {
    *(volatile LAS u32x4*)(scratch + lane * 32) = (u32x4){0u, 0u, 0u, 0u};
    if (NK > 4) *(volatile LAS u32x4*)(scratch + lane * 32 + 16) = (u32x4){0u, 0u, 0u, 0u};
    unsigned m[NK], in[NK], p[NK / 2], ch[NK], sy[NK], mine = 0;
#pragma unroll
    for (int k = 0; k < NK; ++k) {
        m[k] = perm(0u, 0xffu, e[k]);
        in[k] = prefix4(m[k] & B03);
        if (!(k & 1)) p[k >> 1] = mine;
        mine += in[k] >> 24;
    }
    const unsigned incl = wave_scan(mine), sbytes = ((__builtin_amdgcn_readlane(incl, 63) + 31u) >> 5) * 4u;
    unsigned en = 0;
    if ((unsigned)lane < ((info >> 8) & 127u)) en = *(const volatile LAS unsigned short*)(tail + sbytes + lane * 2);
#pragma unroll
    for (int g = 0; g < NK / 2; ++g) {
        p[g] += incl - mine;
        const LAS unsigned* const q = (const LAS unsigned*)(tail + (p[g] >> 5) * 4u);
        ch[2 * g] = __builtin_amdgcn_alignbit(q[1], q[0], p[g]);
        ch[2 * g + 1] = ch[2 * g] >> (in[2 * g] >> 24);
    }
    if (en & 31u) lds_or(scratch + (en >> 7) * 4u, (en & 31u) << (((en >> 2) & 24u) + 3u));
    u32x4 fx[NK / 4];
#pragma unroll
    for (int k = 0; k < NK / 4; ++k) fx[k] = *(volatile LAS u32x4*)(scratch + lane * 32 + k * 16);
#define SPREAD1(d, i, c, j, jm) "v_lshrrev_b32_sdwa " d ", " i ", " c " dst_sel:BYTE_" #j " dst_unused:UNUSED_PRESERVE src0_sel:BYTE_" #jm " src1_sel:DWORD\n\t"
#define SPREAD4(j, jm) SPREAD1("%0", "%4", "%8", j, jm) SPREAD1("%1", "%5", "%9", j, jm) SPREAD1("%2", "%6", "%10", j, jm) SPREAD1("%3", "%7", "%11", j, jm)
#pragma unroll
    for (int k = 0; k < NK; k += 4) {
        sy[k] = ch[k]; sy[k + 1] = ch[k + 1]; sy[k + 2] = ch[k + 2]; sy[k + 3] = ch[k + 3];
        asm("s_nop 0\n\t" SPREAD4(1, 0) SPREAD4(2, 1) SPREAD4(3, 2) "s_nop 0"
            : "+&v"(sy[k]), "+&v"(sy[k + 1]), "+&v"(sy[k + 2]), "+&v"(sy[k + 3])
            : "v"(in[k]), "v"(in[k + 1]), "v"(in[k + 2]), "v"(in[k + 3]), "v"(ch[k]), "v"(ch[k + 1]), "v"(ch[k + 2]), "v"(ch[k + 3]));
    }
    const unsigned c4 = (info & 255u) * B01, tc = sub8(c4, 0x03020100u), t0 = sub8(c4, 0x06050400u), t1 = sub8(c4, 0x0a090807u);
#pragma unroll
    for (int k = 0; k < NK; ++k)
        e[k] = ((perm(t1, t0, sy[k] & B07) & m[k]) | (perm(0u, tc, e[k]) & ~m[k])) ^ fx[k >> 2][k & 3];
}

#define WAIT_V(n) asm volatile("s_waitcnt vmcnt(" #n ")" ::: "memory")
#define WAIT_L(n) asm volatile("s_waitcnt lgkmcnt(" #n ")" ::: "memory")
#define BAR __builtin_amdgcn_s_barrier()
#define SCHED __builtin_amdgcn_sched_barrier(0)
#define LDSW(i) (*(volatile LAS unsigned*)(lds + LDS_STAGE + 4 * (i)))

}

__global__ __launch_bounds__(512, 2) void gemm_rs_kernel(Params P) {
    __shared__ __attribute__((aligned(16))) unsigned char shm[LDS_BYTES];
    LAS unsigned char* lds = (LAS unsigned char*)shm;
    const int tid = threadIdx.x, lane = tid & 63, wid = __builtin_amdgcn_readfirstlane(tid >> 6);
    const int hb = __builtin_amdgcn_readfirstlane(blockIdx.x);
    const float* px = P.x; const float* pw = P.w; char* me_ = P.me; int rank_ = P.rank; unsigned peermap = P.peermap;
    asm volatile("" : "+s"(px), "+s"(pw), "+s"(me_), "+s"(rank_), "+s"(peermap));
    const int rank = rank_ & (NR - 1);
    char* const me = (char*)(GAS char*)(size_t)me_;
    char* const tmo = me + OFF_TMO;
    int bid = (hb & ~7) | (__builtin_amdgcn_s_getreg((3 << 11) | 20) & 7);
    if (bid >= 128 && (bid & 7) != 0) {
        const int tt = ((bid & 7) - 1) * 8 + ((bid >> 3) & 7), cg = tt / (7 * MAPW), r = tt - cg * (7 * MAPW);
        bid = (bid & 192) | ((cg * MAPW + r % MAPW) << 3) | (r / MAPW + 1);
    }
    f32x4 v[8] = {};
    auto first_item = [&](int b) -> unsigned {
        const int q = ((b >> 3) & 7) + 1, sk = ((peermap >> ((b & 7) * 4)) - rank) & (NR - 1);
        const int pl = q < 7 ? (rank + q + (q >= sk)) & (NR - 1) : ((rank + sk) & (NR - 1)) + (q & 8);
        const int lo = (b & 7) == 0 ? (b >> 6) * 16 + 7 + q : b < 64 ? 0 : pl;
        const int hi = b >= 192 ? (q - 1) * 16 + (b & 7) : b;
        return b >= 128 ? hi : lo;
    };
    auto load_item = [&](int b, unsigned item) {
        const bool f1 = b < 64 && (b & 7) != 0, isw = f1 || item >= 128u;
        const unsigned ws_ = f1 ? item : (item >> 3) & 7u, wj = (f1 ? b >> 3 : item) & 7u, wd = f1 ? (peermap >> ((b & 7) * 4)) & (NR - 1) : rank;
        const unsigned wo = (ws_ * 64u + (wid >> 2) * 32u) * NFULL + wd * NS + wj * 256u + (wid & 3) * 64u;
        const unsigned xo = (((item & 7u) * 2u + ((item >> 3) & 1u)) * 256u + wid * 32u) * KS + ((item >> 4) & 7u) * 64u;
        const float* const base = isw ? pw + wo : px + xo;
        unsigned lo = (lane >> 4) * (isw ? 16u * NFULL : 16u) + (lane & 15) * (isw ? 16u : 4u * KS);
        const unsigned sb = isw ? 4u * NFULL : 64u, sa = (isw ? 64u * NFULL : 64u * KS) - 3u * sb;
#define LD1(i, step) "global_load_dwordx4 %" #i ", %8, %9\n\tv_add_u32 %8, " step ", %8\n\t"
        asm volatile("s_nop 4\n\t" LD1(0, "%10") LD1(1, "%10") LD1(2, "%10") LD1(3, "%11") LD1(4, "%10") LD1(5, "%10") LD1(6, "%10")
                     "global_load_dwordx4 %7, %8, %9"
                     : "+v"(v[0]), "+v"(v[1]), "+v"(v[2]), "+v"(v[3]), "+v"(v[4]), "+v"(v[5]), "+v"(v[6]), "+v"(v[7]), "+v"(lo)
                     : "s"(base), "s"(sb), "s"(sa) : "memory");
    };
#define VWAIT(n) asm volatile("s_waitcnt vmcnt(" #n ")" : "+v"(v[0]), "+v"(v[1]), "+v"(v[2]), "+v"(v[3]), "+v"(v[4]), "+v"(v[5]), "+v"(v[6]), "+v"(v[7]) :: "memory")
    unsigned t = 0, ordw[3];
    char* lws;
    {
        u64 ex;
        asm volatile("s_mov_b64 %5, exec\n\ts_mov_b64 exec, %12\n\t"
                     "global_atomic_add %0, %6, %7, %8 sc0\n\ts_mov_b64 exec, %5\n\t"
                     "global_load_dwordx2 %1, %9, %11 offset:%13\n\t"
                     "global_load_dword %2, %10, %11 offset:%14\n\t"
                     "global_load_dword %3, %10, %11 offset:%14+64\n\t"
                     "global_load_dword %4, %10, %11 offset:%14+128"
                     : "+v"(t), "=&v"(lws), "=&v"(ordw[0]), "=&v"(ordw[1]), "=&v"(ordw[2]), "=&s"(ex)
                     : "v"(OFF_CNT + (lane ? bid * 128 + 4 : hb * 128)), "v"(1u), "s"(me), "v"(lane & 56), "v"(lane & 60),
                       "s"(__builtin_amdgcn_kernarg_segment_ptr()), "s"((u64)(wid == 0 ? 3u : 0u)),
                       "i"(__builtin_offsetof(Params, ws)), "i"(__builtin_offsetof(Params, order)) : "memory");
    }
    if (bid >= 128 ? (bid >> 4) != 12 : (bid & 7) == 0) __builtin_amdgcn_s_sleep(LATE);
    char* pws_;
    unsigned call = 0;
#pragma unroll 1
    for (;;) {
        asm volatile("" ::: "memory");
        load_item(bid, first_item(bid));
        pws_ = P.ws[(peermap >> ((bid & 7) * 4)) & (NR - 1)];
        asm volatile("s_waitcnt vmcnt(8)" : "+v"(t), "+v"(lws), "+v"(ordw[0]), "+v"(ordw[1]), "+v"(ordw[2]) :: "memory");
        if (wid == 0 && call == 0u) {
            const unsigned c = __builtin_amdgcn_readlane(t, 0) + 1u;
            unsigned old = __builtin_amdgcn_readlane(t, 1), r = bid;
            for (int tries = 0; old != c - 1u && tries < 65536; ++tries) {
                if (lane == 0) add_agent(me + OFF_CNT + r * 128u + 4u, ~0u);
                r = (r + 1u) & 255u;
                if (lane == 0) t = add_agent(me + OFF_CNT + r * 128u + 4u, 1u);
                old = __builtin_amdgcn_readfirstlane(t);
            }
            LDSW(16 + lane) = 0u;
            if (lane == 0) { LDSW(0) = c; LDSW(12) = r; }
        }
        asm volatile("s_waitcnt lgkmcnt(0)" : "+s"(pws_) :: "memory"); BAR; asm volatile("" ::: "memory");
        call = __builtin_amdgcn_readfirstlane(LDSW(0));
        const int got = __builtin_amdgcn_readfirstlane(LDSW(12));
        if (got == bid) break;
        bid = got;
        VWAIT(0);
    }
    const int peer = (peermap >> ((bid & 7) * 4)) & (NR - 1);
    const int slot = (bid >> 3) & 15;
    const bool aux = bid < 128, remote = (bid & 7) != 0;
    const int skip = remote ? (peer - rank) & (NR - 1) : NR;
    char* const pws = (char*)(GAS char*)(size_t)pws_;

    if (aux && remote && slot >= 8) {
        const unsigned par = call & 1u;
        const unsigned pu = (unsigned)(rank * 8 * NBLK + 8) + (first_item(bid) & 7u) * 2u + ((first_item(bid) >> 3) & 1u);
        char* const rdst = pws + OFF_STAGE + par * STAGE_BYTES + (size_t)pu * UNIT + wid * 4096 + lane * 16;
        auto frag = [&](int f) -> u32x4 {
            return (u32x4){pack2(v[2 * f][0], v[2 * f][1]), pack2(v[2 * f][2], v[2 * f][3]),
                           pack2(v[2 * f + 1][0], v[2 * f + 1][1]), pack2(v[2 * f + 1][2], v[2 * f + 1][3])};
        };
        VWAIT(6); store_wt(rdst, frag(0));
        if (wid > FASTW) WAIT_V(0);
        VWAIT(5); store_wt(rdst + 1024, frag(1));
        if (wid > FASTW) WAIT_V(0);
        VWAIT(4); store_wt(rdst + 2048, frag(2));
        VWAIT(3);
        const u32x4 o3 = frag(3);
        auto push_last = [&]() { WAIT_V(0); store_wt(rdst + 3072, o3); };
        auto publish_push = [&]() {
            WAIT_V(0);
            __builtin_amdgcn_s_waitcnt(0x0F70);
            if (lane == 0)
                add_sys(pws + OFF_FLAG + pu * 128u);
        };
        if (wid == 0) {
            push_last();
            publish_push();
            __builtin_amdgcn_s_setprio(3);
            const unsigned lbase = (unsigned)(size_t)lds, landed = lbase + (lane ? 129024 + lane * 4 : LDS_STAGE + 68);
            unsigned head = 0, alive = NC, idle = 3;
            int issued = 0;
            u32x2 d = *(volatile LAS u32x2*)(lds + LDS_STAGE + 128);
#define PIECE(off) "s_waitcnt vmcnt(%5)\n\tds_max_i32 %4, %1\n\tv_add_u32 %1, 1, %1\n\tglobal_load_lds_dwordx4 %2, off offset:" #off " sc0 sc1\n\t"
            while (alive) {
                if ((unsigned)__builtin_amdgcn_readfirstlane(d.y) != head + 1u) {
                    if (idle == 1u) asm volatile("s_waitcnt vmcnt(4)\n\tds_max_i32 %0, %1" :: "v"(landed), "v"(issued - 4) : "memory");
                    if (idle == 2u) asm volatile("s_waitcnt vmcnt(0)\n\tds_max_i32 %0, %1" :: "v"(landed), "v"(issued) : "memory");
                    if (idle > 2u) __builtin_amdgcn_s_sleep(1);
                    ++idle;
                    d = *(volatile LAS u32x2*)(lds + LDS_STAGE + 128 + (head & 15u) * 8u);
                    continue;
                }
                ++head;
                const unsigned meta = __builtin_amdgcn_readfirstlane(d.x), cnt = (meta >> 17) & 15u;
                const unsigned dst = lbase + (meta >> 28) * 8192u;
                unsigned keep;
                const char* const src = pws + (meta & 0x1ffffu) * 4096u + lane * 16;
                d = *(volatile LAS u32x2*)(lds + LDS_STAGE + 128 + (head & 15u) * 8u);
                if (cnt == 0u) { --alive; continue; }
                const u64 mask = ~0ull >> (64u - ((meta >> 21) & 127u));
                int known = issued - (WIN - 1);
                asm volatile("s_mov_b32 %0, m0\n\ts_mov_b32 m0, %3\n\t" PIECE(0) PIECE(1024) PIECE(2048)
                             "s_waitcnt vmcnt(%5)\n\tds_max_i32 %4, %1\n\ts_mov_b64 exec, %6\n\t"
                             "global_load_lds_dwordx4 %2, off offset:3072 sc0 sc1\n\ts_mov_b64 exec, -1\n\ts_mov_b32 m0, %0"
                             : "=&s"(keep), "+v"(known) : "v"(src), "s"(dst), "v"(landed), "i"(WIN - 1), "s"(mask) : "memory");
                issued += cnt;
                idle = 0u;
            }
        } else if (wid <= NC) {
            const unsigned me8 = wid - 1;
            LAS unsigned char* const scratch = lds + 114688 + me8 * 2048u;
            unsigned bi = me8 * 2u + 1u;
            const float qnan = __builtin_nanf("");
            unsigned tbl = 0;
#pragma unroll
            for (int h = 0; h < 3; ++h) {
                const unsigned e = (ordw[h] >> ((lane & 3) * 8)) & 255u, code = e & 31u;
                const int odd = code >= 14u, q = code - (odd ? 14u : 8u);
                const unsigned blk = code < 8u ? code : code >= 20u ? 8u + rank * 2 + (code - 20u)
                                                                     : 8u + ((rank + q + 1 + (q + 1 >= skip)) & (NR - 1)) * 2 + odd;
                tbl |= ((e & 0xe0u) | blk) << (8 * h);
            }
            struct Item { unsigned t, id, flag, src; };
            unsigned ns = ~0u - NCHUNK;
            auto item_of = [&](unsigned t) -> Item {
                if (t >= NFIX * 8u && ns > 4096u) {
                    const unsigned pl = __builtin_amdgcn_readfirstlane(wave_wait64<2>(me + OFF_PLAN + peer * 128u, call, tmo, call));
                    const unsigned first = NFIX + (pl & 7u), x = (unsigned)lane + 128u - first, f = (pl >> (6u + 6u * (x & 3u))) & 63u;
                    if (x < 4u) tbl = (tbl & 0xffffu) | (((f & 7u) << 5 | 24u | (f >> 3)) << 16);
                    ns = (first + min((pl >> 3) & 7u, 4u)) * 8u;
                }
                const unsigned it = min(t >> 3, 191u), e = (unsigned)__builtin_amdgcn_readlane(tbl, it & 63u) >> ((it >> 6) * 8u);
                const unsigned ks = (e >> 5) & 7u, blk = e & 31u;
                const bool relay = blk >= 24u;
                const unsigned u = relay ? ((blk & 7u) * 8u + ks) * NBLK + 8u + rank * 2u : (peer * 8u + ks) * NBLK + blk;
                const unsigned so = relay ? (unsigned)OFF_STAGE + par * (unsigned)STAGE_BYTES + u * UNIT
                                  : blk < 8u ? (unsigned)OFF_EXPW + par * (unsigned)WSET_BYTES + ((rank * 8u + ks) * 8u + blk) * UNIT
                                             : (unsigned)OFF_EXPX + par * (unsigned)XSET_BYTES + (ks * 16u + blk - 8u) * UNIT;
                const unsigned i = max(min(t, ns + NCHUNK - 1u), ns) - ns;
                const unsigned ci = ((i >> 9) * 64u + (i & 63u)) * 8u + ((i >> 6) & 7u);
                Item r;
                r.t = t;
                r.id = t < ns ? u | (blk < 8u ? 0x10000u : relay ? 0x20000u : 0u) : ci;
                r.flag = t >= ns ? OFF_PFLAG + (peer * 128u + (ci >> 3)) * 256u + par * 128u + (ci & 7u) * 8u
                       : relay ? OFF_PLAN + peer * 128u : OFF_EFLAG + (par * 64u * NBLK + u) * 64u + (t & 7u) * 8u;
                r.src = t < ns ? so + (t & 7u) * 4096u : (unsigned)OFF_PART + par * (unsigned)PART_BYTES + rank * 16u * TILE_BYTES + ci * 4096u;
                return r;
            };
            auto ownf_of = [&](const Item& x) -> unsigned { return OFF_OWNF + (peer * 128u + (x.t < ns ? 0u : x.id >> 3)) * 64u; };
            auto out_of = [&](const Item& x) -> unsigned {
                const unsigned ct = (x.id >> 6) & 15u, cw = (x.id >> 3) & 7u, cg = x.id & 7u;
                return (((ct >> 3) * 256u + (cg >> 2) * 128u + (cw >> 2) * 64u + (cg & 3u) * 16u) * NFULL + peer * NS + (ct & 7u) * 256u + (cw & 3u) * 64u) * 4u;
            };
            const unsigned olane = ((lane >> 4) * 4u * NFULL + (lane & 15) * 4u) * 4u;
            int rend = 0;
            auto post = [&](unsigned src, unsigned cnt, unsigned last) {
                unsigned k = 0;
                bi ^= 1u;
                if (lane == 0) {
                    asm volatile("ds_add_rtn_u32 %0, %1, %2\n\ts_waitcnt lgkmcnt(0)" : "=&v"(k) : "v"((unsigned)(size_t)lds + LDS_STAGE + 64u), "v"(0x10000u + cnt) : "memory");
                    *(volatile LAS u32x2*)(lds + LDS_STAGE + 128 + ((k >> 16) & 15u) * 8u) = (u32x2){(src >> 12) | (cnt << 17) | (last << 21) | (bi << 28), (k >> 16) + 1u};
                }
                rend = (__builtin_amdgcn_readfirstlane(k) & 0xffffu) + cnt;
            };
            auto post_item = [&](const Item& x, unsigned inf) {
                post(x.src, 4u, inf & INFO_RAW ? 64u : min(16u + ((inf >> 16) & 127u), 64u));
            };
            Item A = item_of((slot - 8) * NC + me8), B = item_of(A.t + 8u * NC), C = item_of(A.t + 16u * NC);
            const u64 f0 = ld_sys64(me + A.flag);
            const unsigned qbase = __builtin_amdgcn_readfirstlane(ld_agent(me + OFF_PLAN + peer * 128u + 32u + par * 4u));
            unsigned info = __builtin_amdgcn_readfirstlane((unsigned)(f0 >> 32)), ofl = 0, tm = 0;
            const bool now = __all((unsigned)f0 >= call);
            push_last();
            if (now) post_item(A, info);
            publish_push();
            if (!now) { info = __builtin_amdgcn_readfirstlane(wave_wait64<2>(me + A.flag, call, tmo, call)); post_item(A, info); }
            u64 fl;
            do { fl = ld_sys64(me + B.flag); } while ((unsigned)__builtin_amdgcn_readfirstlane((unsigned)fl) < call && (int)__builtin_amdgcn_readfirstlane(LDSW(17)) < rend);
            unsigned pend = 0;
            u32x4 own[4] = {};
#pragma unroll 1
            for (;;) {
                const bool isp = A.t >= ns, bad = tm == call || (info & INFO_BAD) != 0u;
                if ((int)__builtin_amdgcn_readfirstlane(LDSW(17)) < rend) {
                    if (pend) { WAIT_V(0); if (lane == 0) add_sys(me + pend); pend = 0; }
                    while ((int)__builtin_amdgcn_readfirstlane(LDSW(17)) < rend) __builtin_amdgcn_s_sleep(1);
                }
                asm volatile("" ::: "memory");
                const LAS unsigned char* const buf = lds + bi * 8192u;
                const bool brelay = (B.id >> 17) != 0u, more = B.t < ns + NCHUNK;
                const bool early = more && (brelay || (unsigned)__builtin_amdgcn_readfirstlane((unsigned)fl) >= call);
                unsigned infob = brelay ? INFO_RAW : __builtin_amdgcn_readfirstlane((unsigned)(fl >> 32));
                if (early) post_item(B, infob);
                u32x4 nown[4];
                u64 ex;
                unsigned tk;
                const bool nextp = more && B.t >= ns;
                if (nextp && (unsigned)__builtin_amdgcn_readfirstlane(ofl) < call) wave_wait<2>(me + ownf_of(B), call, tmo, call);
                {
                    unsigned vo = out_of(B) + olane;
                    asm volatile("s_mov_b64 %8, exec\n\ts_mov_b32 exec_lo, %16\n\ts_mov_b32 exec_hi, %16\n\ts_nop 2\n\t"
                                 "global_load_dwordx4 %0, %9, %10 sc0 sc1\n\tv_add_u32 %9, 0x10000, %9\n\t"
                                 "global_load_dwordx4 %1, %9, %10 sc0 sc1\n\tv_add_u32 %9, 0x10000, %9\n\t"
                                 "global_load_dwordx4 %2, %9, %10 sc0 sc1\n\tv_add_u32 %9, 0x10000, %9\n\t"
                                 "global_load_dwordx4 %3, %9, %10 sc0 sc1\n\t"
                                 "s_mov_b64 exec, 1\n\t"
                                 "global_load_dwordx2 %4, %12, %11 sc0 sc1\n\t"
                                 "global_load_dword %5, %13, %11 sc0 sc1\n\t"
                                 "global_load_dword %6, %17, %11 sc1\n\t"
                                 "global_atomic_add %7, %14, %15, %11 sc0\n\t"
                                 "s_mov_b64 exec, %8"
                                 : "=&v"(nown[0]), "=&v"(nown[1]), "=&v"(nown[2]), "=&v"(nown[3]),
                                   "=&v"(fl), "=&v"(ofl), "=&v"(tm), "=&v"(tk), "=&s"(ex), "+v"(vo)
                                 : "s"(P.out), "s"(me), "v"(C.flag), "v"(ownf_of(C)), "v"(OFF_QUEUE + peer * 128u), "v"(1u),
                                   "s"(__builtin_amdgcn_readfirstlane(nextp ? -1 : 0)), "v"(OFF_TMO) : "memory");
                }
                u32x4 w[4];
#pragma unroll
                for (int q = 0; q < 4; ++q) w[q] = *(const LAS u32x4*)(buf + q * 1024 + lane * 16);
                const unsigned w13 = *(const LAS unsigned*)(buf + 3072 + lane * 4);
                char* dst;
                unsigned d1, d2;
                if (!isp) {
                    if (!(info & INFO_RAW)) {
                        const u32x4 l0 = w[0], l1 = w[1], nb = w[2];
                        unsigned e[8];
#pragma unroll
                        for (int k = 0; k < 8; ++k) e[k] = (nb[k >> 1] >> (2 + 4 * (k & 1))) & B03;
                        exp_decode(e, info, lane, buf + 3328, scratch);
#pragma unroll
                        for (int k = 0; k < 8; ++k) {
                            const unsigned hb = ((nb[k >> 1] >> (4 * (k & 1))) & B03) | (e[k] << 2) | ((w13 << k) & B80), lo = k < 4 ? l0[k & 3] : l1[k & 3];
                            w[k >> 1][(k & 1) * 2] = perm(hb, lo, 0x05010400u); w[k >> 1][(k & 1) * 2 + 1] = perm(hb, lo, 0x07030602u);
                        }
                    }
                    const bool isw = (A.id & 0x10000u) != 0u;
                    const unsigned w8 = A.t & 7u;
                    d1 = isw ? 2048u : 1024u; d2 = isw ? 16384u : 2048u;
                    dst = me + ((unsigned)OFF_STAGE + par * (unsigned)STAGE_BYTES + (A.id & 0xffffu) * UNIT
                                + (isw ? ((w8 & 3u) * 4u + (w8 >> 2)) * 1024u : w8 * 4096u)) + lane * 16;
                } else {
                    if (!(info & INFO_RAW)) {
                        const u32x4 p0 = w[0], p1 = w[1], p2 = w[2];
                        unsigned e[4];
#pragma unroll
                        for (int k = 0; k < 4; ++k) e[k] = (w13 >> (2 * k)) & B03;
                        exp_decode(e, info, lane, buf + 3328, scratch);
#pragma unroll
                        for (int q = 0; q < 4; ++q) {
                            const u32x4 r = {perm(e[q], p0[q], 0x04020100u), perm(e[q], p1[q], 0x05020100u), perm(e[q], p2[q], 0x06020100u),
                                             perm(p1[q], p0[q], 0x0c0c0703u) | perm(e[q], p2[q], 0x07030c0cu)};
                            w[q] = (r >> 1) | (r << 31);
                        }
                    }
                    f32x4 y[4];
#pragma unroll
                    for (int q = 0; q < 4; ++q) y[q] = __builtin_bit_cast(f32x4, own[q]) + __builtin_bit_cast(f32x4, w[q]);
                    if (bad) {
#pragma unroll
                        for (int q = 0; q < 4; ++q) y[q] = (f32x4)qnan;
                    }
                    gelu(y);
#pragma unroll
                    for (int q = 0; q < 4; ++q) w[q] = __builtin_bit_cast(u32x4, y[q]);
                    d1 = NFULL * 4u; d2 = NFULL * 8u;
                    dst = (char*)P.out + out_of(A) + olane;
                }
                store_wt(dst, w[0]); store_wt(dst + d1, w[1]); store_wt(dst + d2, w[2]); store_wt(dst + d2 + d1, w[3]);
                asm volatile("s_waitcnt vmcnt(4)"
                             : "+v"(nown[0]), "+v"(nown[1]), "+v"(nown[2]), "+v"(nown[3]), "+v"(fl), "+v"(ofl), "+v"(tm), "+v"(tk) :: "memory");
                if (pend && lane == 0) add_sys(me + pend);
                pend = isp ? 0u : OFF_FLAG + (A.id & 0xffffu) * 128u;
                tm = __builtin_amdgcn_readfirstlane(tm);
                if (!more) break;
                if (!early) {
                    if (pend) { WAIT_V(0); if (lane == 0) add_sys(me + pend); pend = 0; }
                    infob = __builtin_amdgcn_readfirstlane(wave_wait64<2>(me + B.flag, call, tmo, call));
                    post_item(B, infob);
                    tm = __builtin_amdgcn_readfirstlane(ld_agent(tmo));
                }
#pragma unroll
                for (int q = 0; q < 4; ++q) own[q] = nown[q];
                info = infob; A = B; B = C;
                C = item_of(__builtin_amdgcn_readfirstlane(tk) - qbase + 24u * NC);
            }
            post(0u, 0u, 0u);
            WAIT_V(0);
            if (pend && lane == 0) add_sys(me + pend);
        } else {
            push_last();
            publish_push();
            if (bid == 65 && wid == 7) {
                const u64 t0 = __builtin_amdgcn_s_memrealtime();
                char* const pa = me + OFF_PLAN + (lane & 7) * 128u;
                const unsigned base = ld_agent(pa + 32u + par * 4u);
                unsigned m1 = 0, m2 = 0, p2 = 0;
                for (int k = 0; k < 2 * PLANN; ++k) {
                    while (__builtin_amdgcn_s_memrealtime() - t0 < (unsigned)(PLAN1 + k * PLAND)) __builtin_amdgcn_s_sleep(8);
                    p2 = ld_agent(me + OFF_QUEUE + (lane & 7) * 128u) - base;
                    if (k < PLANN) m1 += p2; else m2 += p2;
                }
                const bool link = lane < 8 && lane != rank;
                int end = -(int)((m2 * 64u + (m2 - m1) * (8u * PLANB)) / PLANN);
                unsigned plan = 7u;
                if (__all(!link || (m2 - m1 > PLANN * 200u && p2 < NFIX * 8u)))
                    for (int mv = 0; mv < 16; ++mv) {
                        int hi = -1, lo = -1, ehi = 0, elo = 0;
                        for (int l = 0; l < 8; ++l) {
                            const int el = __builtin_amdgcn_readlane(end, l);
                            const unsigned pl = __builtin_amdgcn_readlane(plan, l);
                            if (l == rank) continue;
                            if ((pl & 7u) > 7u - PLANMAX && (pl & 56u) == 0u && (hi < 0 || el > ehi)) { hi = l; ehi = el; }
                            if ((pl & 7u) == 7u && (pl & 56u) < 32u && (lo < 0 || el < elo)) { lo = l; elo = el; }
                        }
                        if (hi < 0 || lo < 0 || ehi - elo < PLANGAP * 64) break;
                        const unsigned ks = 8u - (__builtin_amdgcn_readlane(plan, hi) & 7u);
                        if (lane == hi) { plan -= 1u; end -= 512; }
                        if (lane == lo) { plan = (plan | ((unsigned)(hi << 3 | ks) << (6u + 6u * ((plan >> 3) & 7u)))) + 8u; end += 585; }
                    }
                {
                    const unsigned pl = __shfl(plan, lane >> 3), x = lane & 7, f = (pl >> (6u + 6u * (x & 3u))) & 63u;
                    bool there = true;
                    if (x < ((pl >> 3) & 7u)) there = ld_sys(lws + OFF_FLAG + (f * NBLK + 8u + rank * 2u) * 128u) >= call * 8u;
                    if (!__all(there)) plan = 7u;
                }
                if (link) {
                    st_sys64(pa, call, plan);
                    st_sys(pa + 32u + (par ^ 1u) * 4u, base + (NFIX + (plan & 7u) + ((plan >> 3) & 7u)) * 8u + NCHUNK);
                }
            }
        }
        WAIT_V(0);
        asm volatile("s_endpgm" ::: "memory");
    }

    {
        const bool f1 = aux && remote;
        unsigned item = first_item(bid);
        const int fd = f1 ? peer : lane >> 3;
        char* const fws = f1 ? pws : lws;
#pragma unroll 1
        for (unsigned n = 0;; ++n) {
            const bool isw = f1 || item >= 128u;
            const int s = f1 ? item : isw ? (item - 128u) >> 3 : item >> 4;
            const int j = (f1 ? slot : item) & 7, rb = (item & 7) * 2 + ((item >> 3) & 1);
            const unsigned par = call & 1u;
            u32x4 o[4];
            unsigned off, d1, d2;
            VWAIT(0);
            if (isw) {
                off = ((wid & 3) * 4 + (wid >> 2)) * 1024; d1 = 2048; d2 = 16384;
#pragma unroll
                for (int f = 0; f < 4; ++f) {
                    o[f].x = pack2(v[0][f], v[1][f]); o[f].y = pack2(v[2][f], v[3][f]);
                    o[f].z = pack2(v[4][f], v[5][f]); o[f].w = pack2(v[6][f], v[7][f]);
                }
            } else {
                off = wid * 4096; d1 = 1024; d2 = 2048;
#pragma unroll
                for (int f = 0; f < 4; ++f) {
                    o[f].x = pack2(v[2 * f][0], v[2 * f][1]); o[f].y = pack2(v[2 * f][2], v[2 * f][3]);
                    o[f].z = pack2(v[2 * f + 1][0], v[2 * f + 1][1]); o[f].w = pack2(v[2 * f + 1][2], v[2 * f + 1][3]);
                }
            }
            off += lane * 16;
            const bool last = f1 ? item == 7u : bid < 208 || n == 1u;
            const unsigned u = (unsigned)((rank * 8 + s) * NBLK + (isw ? j : 8 + rb));
            const unsigned wu = (unsigned)((peer * 8 + s) * 8 + j);
            const bool expo = f1 || (!isw && (rb >> 1) != rank && item >= 8u);
            char* const ldst = me + (f1 ? OFF_OWNB + (size_t)wu * UNIT : OFF_STAGE + par * STAGE_BYTES + (size_t)u * UNIT) + off;
            store_wt(ldst, o[0]); store_wt(ldst + d1, o[1]); store_wt(ldst + d2, o[2]); store_wt(ldst + d2 + d1, o[3]);
            if (!f1) {
                WAIT_V(0);
                if (lane == 0) add_agent(me + OFF_FLAG + u * 128u, 1u);
            }
            if (expo) {
                char* const ed = me + (f1 ? OFF_EXPW + par * WSET_BYTES + (size_t)wu * UNIT : OFF_EXPX + par * XSET_BYTES + (size_t)(s * 16 + rb) * UNIT) + wid * 4096;
                LAS unsigned char* const tail = lds + LDS_ENC + wid * 1024;
                unsigned hb[8], e[8], sg = 0;
                u32x4 lo[2], nb;
#pragma unroll
                for (int k = 0; k < 8; ++k) {
                    const unsigned w0 = o[k >> 1][(k & 1) * 2], w1 = o[k >> 1][(k & 1) * 2 + 1];
                    lo[k >> 2][k & 3] = perm(w1, w0, 0x06040200u); hb[k] = perm(w1, w0, 0x07050301u);
                    e[k] = (hb[k] >> 2) & 0x1f1f1f1fu; sg |= (hb[k] & B80) >> k;
                }
                const unsigned info = exp_encode(e, lane, tail);
                if (info & INFO_RAW) {
#pragma unroll
                    for (int f = 0; f < 4; ++f) store_wt(ed + f * 1024 + lane * 16, o[f]);
                } else {
#pragma unroll
                    for (int k = 0; k < 4; ++k)
                        nb[k] = (hb[2 * k] & B03) | (e[2 * k] << 2) | ((hb[2 * k + 1] & B03) << 4) | (e[2 * k + 1] << 6);
                    store_wt(ed + lane * 16, lo[0]); store_wt(ed + 1024 + lane * 16, lo[1]); store_wt(ed + 2048 + lane * 16, nb);
                    st_sys(ed + 3072 + lane * 4, sg);
                    if ((unsigned)lane < ((info >> 16) & 127u)) store_wt(ed + 3328 + lane * 16, *(volatile LAS u32x4*)(tail + lane * 16));
                }
                if (lane == 0) LDSW(4 + wid) = info;
            }
            WAIT_V(0);
            WAIT_L(0); BAR; asm volatile("" ::: "memory");
            if (wid == 0) {
                if (expo && (f1 ? lane < 8 : fd != rank))
                    st_sys64(fws + OFF_EFLAG + (par * 64u * NBLK + u) * 64u + (lane & 7) * 8, call, LDSW(4 + (lane & 7)));
                if (f1 && lane == 0) st_sys(me + OFF_OFLAG + wu * 128u, call * 8u);
            }
            if (last) break;
            item += f1 ? 1u : 8u;
            load_item(bid, item);
            WAIT_L(0); BAR;
        }
    }
    const bool ownb = aux && remote;
    int tile = slot;
again:
    const int ti = (aux ? rank : peer) * 2 + (tile >> 3), tj = slot & 7;
    const int nsrc = remote ? 6 : 7;
    const int rfirst = aux && !remote ? 4 : -1, rlast = ownb ? -1 : !aux && !remote ? 3 : 7;
    const int nt = remote ? (aux ? 8 : 56) : (aux ? 28 : 36);
    const int wr = wid >> 2, wc = wid & 3;
    const char* const stage = me + OFF_STAGE + (size_t)(call & 1u) * STAGE_BYTES;
    const char* const abase = stage + (unsigned)(8 + ti) * UNIT;
    const char* const bbase = ownb ? me + OFF_OWNB + (size_t)(((peer - rank) * 64 + tj) * (int)UNIT) : stage + (unsigned)tj * UNIT;
    const unsigned bstep = ownb ? 8u * UNIT : NBLK * UNIT;
    const unsigned voff = (unsigned)tid * 16u;
    const unsigned ldsw = (unsigned)wid * 1024u;
    const unsigned aoff = (unsigned)wr * 8192u + (unsigned)lane * 16u;
    unsigned boff = 65536u + (unsigned)wc * 4096u + (unsigned)lane * 16u;
    asm volatile("" : "+v"(boff));

#define SA(b, h) (((b) * 2 + (h)) * 16384)
#define SB(b, h) ((4 + (b) * 2 + (h)) * 16384)
#define STAGE(bufoff, gbase) do { _Pragma("unroll") for (int _i = 0; _i < 2; ++_i) \
        __builtin_amdgcn_global_load_lds((const GAS unsigned*)((gbase) + _i * 8192 + voff), (LAS unsigned*)(lds + (bufoff) + ldsw + _i * 8192), 16, 0, 17); } while (0)
#define LDA(dst, b, h) do { _Pragma("unroll") for (int m = 0; m < 4; ++m) _Pragma("unroll") for (int k = 0; k < 2; ++k) \
        dst[m][k] = *(const LAS f16x8*)(lds + SA(b, h) + aoff + m * 2048 + k * 1024); } while (0)
#define LDB(dst, b, h) do { _Pragma("unroll") for (int n = 0; n < 2; ++n) _Pragma("unroll") for (int k = 0; k < 2; ++k) \
        dst[n][k] = *(const LAS f16x8*)(lds + boff + (SB(b, h) - 65536 + n * 2048 + k * 1024)); } while (0)
#define MMA(ai, bj, At, Bt) do { __builtin_amdgcn_s_setprio(1); \
        _Pragma("unroll") for (int m = 0; m < 4; ++m) _Pragma("unroll") for (int n = 0; n < 2; ++n) _Pragma("unroll") for (int k = 0; k < 2; ++k) \
        acc[ai][bj][m][n] = __builtin_amdgcn_mfma_f32_16x16x32_f16(At[m][k], Bt[n][k], acc[ai][bj][m][n], 0, 0, 0); \
        __builtin_amdgcn_s_setprio(0); } while (0)

    unsigned ktab;
    {
        const int tt = lane < nt ? lane : nt - 2 + (lane & 1), u = rfirst < 0 ? tt - 8 : tt + rfirst * nsrc;
        const int r = u < 0 ? -1 : remote ? u / 6 : u / 7, q = u - r * nsrc;
        ktab = (u < 0 ? rank * 8 + tt : ((rank + q + 1 + (q + 1 >= skip)) & (NR - 1)) * 8 + r) | ((r + 1) << 8);
    }
    auto flag_of = [&](int s, int lz) -> unsigned {
        const int e = (lz >> 1) & 7, q = e < nsrc ? e : 0, sl = min(s + (lz >> 4), rlast);
        const int kt = sl < 0 ? rank * 8 + e : ((rank + q + 1 + (q + 1 >= skip)) & (NR - 1)) * 8 + sl;
        return ownb && !(lz & 1) ? OFF_OFLAG + (unsigned)((peer * 8 + e) * 8 + tj) * 128u
                                 : OFF_FLAG + (unsigned)(kt * NBLK + tj + (lz & 1) * (8 + ti - tj)) * 128u;
    };
    auto ready = [&](int s) {
        if (wid == 0) {
            int lz = lane; asm volatile("" : "+v"(lz));
            wave_wait<POLL>(me + flag_of(s, lz & 15), call * 8u, tmo, call);
        }
        asm volatile("" ::: "memory"); BAR; asm volatile("" ::: "memory");
    };

    f32x4 acc[2][2][4][2];
#pragma unroll
    for (int a = 0; a < 2; ++a)
#pragma unroll
        for (int b = 0; b < 2; ++b)
#pragma unroll
            for (int m = 0; m < 4; ++m)
#pragma unroll
                for (int n = 0; n < 2; ++n) acc[a][b][m][n] = (f32x4){0.f, 0.f, 0.f, 0.f};
    f16x8 At[4][2], B0[2][2], B1[2][2];

    int have = rfirst;
    ready(rfirst);
    unsigned k2 = __builtin_amdgcn_readlane(ktab, 0) & 63u, k3 = __builtin_amdgcn_readlane(ktab, 1) & 63u;
    {
        const char* a0 = abase + k2 * (NBLK * UNIT); const char* b0 = bbase + k2 * bstep;
        const char* a1 = abase + k3 * (NBLK * UNIT); const char* b1 = bbase + k3 * bstep;
        WAIT_V(0);
        STAGE(SB(0, 0), b0); STAGE(SA(0, 0), a0); STAGE(SB(0, 1), b0 + 16384); STAGE(SA(0, 1), a0 + 16384);
        if (wr == 1) BAR;
        WAIT_V(4); BAR;
        STAGE(SB(1, 0), b1); STAGE(SA(1, 0), a1); STAGE(SB(1, 1), b1 + 16384);
        WAIT_V(6); BAR;
    }
#pragma unroll 1
    for (int t = 0; t < nt; t += 2) {
        const char* a1 = abase + k3 * (NBLK * UNIT);
        k2 = __builtin_amdgcn_readlane(ktab, t + 2) & 63u; k3 = __builtin_amdgcn_readlane(ktab, t + 3);
        const int round3 = (int)(k3 >> 8) - 1, round5 = (int)((unsigned)__builtin_amdgcn_readlane(ktab, t + 5) >> 8) - 1;
        k3 &= 63u;
        if (round3 > have) {
            int lz = lane; asm volatile("" : "+v"(lz) :: "memory");
            const u64 miss = ~__ballot(*(volatile LAS unsigned*)(lds + LDS_ENC + (t & 2) * 128 + lz * 4) >= call * 8u);
            if (miss & 0xffffull) { ready(round3); have = round3; }
            else have = min(round3 + (miss & 0xffff0000ull ? 0 : 1), rlast);
            asm volatile("" ::: "memory");
        }
        const char* a2 = abase + k2 * (NBLK * UNIT); const char* b2 = bbase + k2 * bstep;
        const char* a3 = abase + k3 * (NBLK * UNIT); const char* b3 = bbase + k3 * bstep;
        LDB(B0, 0, 0); SCHED; LDA(At, 0, 0); STAGE(SA(1, 1), a1 + 16384);
        WAIT_L(8); BAR; WAIT_L(0); MMA(0, 0, At, B0); BAR; SCHED;
        LDB(B1, 0, 1); STAGE(SB(0, 0), b2);
        BAR; WAIT_L(0); MMA(0, 1, At, B1); BAR;
        LDA(At, 0, 1); STAGE(SA(0, 0), a2);
        BAR; WAIT_L(0); MMA(1, 0, At, B0); BAR; SCHED;
        STAGE(SB(0, 1), b2 + 16384);
        WAIT_V(6); BAR; MMA(1, 1, At, B1); BAR;
        if (wid == 0 && round5 > have) {
            int lz = lane; asm volatile("" : "+v"(lz));
            if (lz < 32)
                __builtin_amdgcn_global_load_lds((const GAS unsigned*)(me + flag_of(have + 1, lz)), (LAS unsigned*)(lds + LDS_ENC + (~t & 2) * 128), 4, 0, 17);
        }
        LDB(B0, 1, 0); SCHED; LDA(At, 1, 0); STAGE(SA(0, 1), a2 + 16384);
        WAIT_L(8); BAR; WAIT_L(0); MMA(0, 0, At, B0); BAR; SCHED;
        LDB(B1, 1, 1); STAGE(SB(1, 0), b3);
        BAR; WAIT_L(0); MMA(0, 1, At, B1); BAR;
        LDA(At, 1, 1); STAGE(SA(1, 0), a3);
        BAR; WAIT_L(0); MMA(1, 0, At, B0); BAR; SCHED;
        STAGE(SB(1, 1), b3 + 16384);
        WAIT_V(6); BAR; MMA(1, 1, At, B1); BAR;
    }
    if (wr == 0) BAR;
    int tz = threadIdx.x; asm volatile("" : "+v"(tz));
    const int lane2 = tz & 63;
    const float qnan = __builtin_nanf("");
#define ACC1(g, q, c) acc[(g) >> 1][(c) >> 1][((g) & 1) * 2 + ((q) >> 2)][(c) & 1][(q) & 3]
#define ACC(g, q) ((f32x4){ACC1(g, q, 0), ACC1(g, q, 1), ACC1(g, q, 2), ACC1(g, q, 3)})
    const unsigned foff = OFF_PFLAG + (unsigned)wid * 256u + (call & 1u) * 128u;
    const size_t poff = OFF_PART + (size_t)(call & 1u) * PART_BYTES + (size_t)slot * TILE_BYTES + wid * 32768 + lane2 * 16;
    if (!aux) {
        char* const dst = me + poff + (size_t)peer * 16 * TILE_BYTES;
        LAS unsigned char* const tail = lds + LDS_ENC + wid * 1024;
        char* const flag = pws + foff + (unsigned)(rank * 16 + slot) * 2048u;
        unsigned infos = 0, badbit = 0;
#pragma unroll
        for (int h = 0; h < 8; ++h) {
            unsigned e[4];
            u32x4 pw_[3];
#pragma unroll
            for (int q = 0; q < 4; ++q) {
                const u32x4 u = __builtin_bit_cast(u32x4, ACC(h >> 1, (h & 1) * 4 + q)), r = (u << 1) | (u >> 31);
                e[q] = perm(perm(r.w, r.z, 0x0c0c0703u), perm(r.y, r.x, 0x0c0c0703u), 0x05040100u);
                pw_[0][q] = perm(r.w, r.x, 0x04020100u); pw_[1][q] = perm(r.w, r.y, 0x05020100u); pw_[2][q] = perm(r.w, r.z, 0x06020100u);
            }
            const unsigned info = remote ? exp_encode(e, lane2, tail) : INFO_RAW;
            if (h == 3) {
                WAIT_V(0);
                if (lane2 == 1 || lane2 == 2) st_sys64(flag + lane2 * 8, call, infos);
            }
            if (info & INFO_RAW) {
#pragma unroll
                for (int q = 0; q < 4; ++q) store_wt(dst + h * 4096 + q * 1024, __builtin_bit_cast(u32x4, ACC(h >> 1, (h & 1) * 4 + q)));
            } else {
#pragma unroll
                for (int q = 0; q < 3; ++q) store_wt(dst + h * 4096 + q * 1024, pw_[q]);
                st_sys(dst + h * 4096 + 3072 - lane2 * 12, e[0] | (e[1] << 2) | (e[2] << 4) | (e[3] << 6));
                if ((unsigned)lane2 < ((info >> 16) & 127u)) store_wt(dst + h * 4096 + 3328, *(volatile LAS u32x4*)(tail + lane2 * 16));
            }
            if (h == 0) badbit = ld_agent(tmo) == call ? INFO_BAD : 0u;
            if (lane2 == h) infos = info | badbit;
            if (h == 0 || h == 7) {
                WAIT_V(0);
                if (h ? lane2 >= 3 && lane2 < 8 : lane2 == 0) st_sys64(flag + lane2 * 8, call, infos);
            }
        }
        return;
    }
    const char* const src = me + poff + (size_t)rank * 16 * TILE_BYTES;
    float* const o = P.out + (size_t)((tile >> 3) * 256 + wr * 64 + (lane2 >> 4) * 4) * NFULL + peer * NS + tj * 256 + wc * 64 + (lane2 & 15) * 4;
    if (!remote) wave_wait64<POLL>(me + foff + (unsigned)(peer * 16 + slot) * 2048u + (lane2 & 7) * 8, call, tmo, call);
    const bool bad = ld_agent(tmo) == call;
#pragma unroll
    for (int g = 0; g < 4; ++g) {
        f32x4 y[8];
#pragma unroll
        for (int q = 0; q < 8; ++q) y[q] = bad ? (f32x4)qnan : ACC(g, q);
        if (!remote) {
            u32x4 w[8];
            asm volatile("global_load_dwordx4 %0, %8, off sc0 sc1\n\t"
                         "global_load_dwordx4 %1, %8, off offset:1024 sc0 sc1\n\t"
                         "global_load_dwordx4 %2, %8, off offset:2048 sc0 sc1\n\t"
                         "global_load_dwordx4 %3, %8, off offset:3072 sc0 sc1\n\t"
                         "global_load_dwordx4 %4, %9, off sc0 sc1\n\t"
                         "global_load_dwordx4 %5, %9, off offset:1024 sc0 sc1\n\t"
                         "global_load_dwordx4 %6, %9, off offset:2048 sc0 sc1\n\t"
                         "global_load_dwordx4 %7, %9, off offset:3072 sc0 sc1\n\ts_waitcnt vmcnt(0)"
                         : "=&v"(w[0]), "=&v"(w[1]), "=&v"(w[2]), "=&v"(w[3]), "=&v"(w[4]), "=&v"(w[5]), "=&v"(w[6]), "=&v"(w[7])
                         : "v"(src + g * 8192), "v"(src + g * 8192 + 4096) : "memory");
#pragma unroll
            for (int q = 0; q < 8; ++q) y[q] += __builtin_bit_cast(f32x4, w[q]);
            gelu(y);
        }
#pragma unroll
        for (int q = 0; q < 8; ++q) {
            if (remote) asm volatile("s_waitcnt vmcnt(%0)" :: "i"(F1DEPTH) : "memory");
            store_wt(o + (size_t)((g >> 1) * 128 + (g & 1) * 32 + (q >> 2) * 16 + (q & 3)) * NFULL, __builtin_bit_cast(u32x4, y[q]));
        }
    }
    WAIT_V(0);
    if (remote) {
        if (lane2 == 0) st_sys(me + OFF_OWNF + (unsigned)((peer * 16 + tile) * 8 + wid) * 64u, call);
        if (tile < 8) { tile += 8; goto again; }
    }
}

extern "C" void kernel_launch(int rank, int nranks, ncclComm_t comm,
                              hipStream_t stream,
                              void* const* d_in, const int* in_sizes, int n_in,
                              void* d_out, int out_size,
                              const int* in_dtypes, int out_dtype,
                              void* const* d_out_all,
                              void* const* d_ws_all, size_t ws_bytes) {
    (void)comm; (void)in_dtypes; (void)out_dtype; (void)d_out_all;
    if (nranks != NR || (unsigned)rank >= (unsigned)NR || n_in != 2 || ws_bytes < WS_NEED
        || in_sizes[0] != 4096 * KS || in_sizes[1] != KS * NFULL || out_size != 512 * NFULL) {
        fprintf(stderr, "kernel_launch: unexpected problem shape\n");
        return;
    }
    Params P = {};
    P.x = (const float*)d_in[0];
    P.w = (const float*)d_in[1];
    P.out = (float*)d_out;
    for (int p = 0; p < NR; ++p) P.ws[p] = (char*)d_ws_all[p];
    P.me = P.ws[rank];
    P.rank = rank;
    static const unsigned char port[NR][NR] = {
        {0, 2, 3, 6, 4, 7, 5, 1}, {1, 4, 5, 7, 6, 2, 3, 0}, {2, 0, 6, 3, 7, 1, 5, 4}, {3, 6, 0, 2, 7, 4, 1, 5},
        {4, 1, 7, 5, 6, 3, 0, 2}, {5, 7, 1, 4, 0, 6, 2, 3}, {6, 3, 2, 0, 1, 5, 4, 7}, {7, 5, 4, 1, 2, 0, 3, 6}};
    for (int c = 0; c < NR; ++c) P.peermap |= (unsigned)port[rank][c] << (4 * c);
    unsigned n = 0, sw = 0, sa = 1, sb = 0;
    auto put = [&](unsigned s, unsigned code) { if (n < NPULL) P.order[n >> 2] |= (((s & 7u) << 5) | code) << ((n & 3u) * 8u); ++n; };
    for (const char* c = ORDER; *c; ++c) {
        unsigned& s = *c == 'w' ? sw : *c == 'a' ? sa : sb;
        const unsigned first = *c == 'w' ? 0 : *c == 'a' ? 8 : 14, cnt = *c == 'w' ? 8 : 6;
        for (unsigned i = 0; i < cnt; ++i) put(s, first + i);
        ++s;
    }
    for (unsigned s = 1; s < 8; ++s) put(s, 21);
    for (unsigned s = 7; s >= 1; --s) put(s, 20);
    if (n != NPULL || sw != 8 || sa != 8 || sb != 8) { fprintf(stderr, "kernel_launch: bad ORDER\n"); return; }
    gemm_rs_kernel<<<256, 512, 0, stream>>>(P);
    HIP_CHECK(hipGetLastError());
}
```
